```python
import numpy as np

kernel_name = "gpu_gemm_ar_m8192_k8192_n8192_bf16_relu"


ABSMAX_RTOL = 0.008

COMPUTE_DTYPE = "bfloat16"
EPILOGUE = "relu"
IN_QUANT = None
OUT_QUANT = None
IN_DTYPES = {"x": "bfloat16", "w_mat": "bfloat16"}
WORLD_SIZE = 8

def _to_bf16(x):
    u = np.ascontiguousarray(x, dtype=np.float32).view(np.uint32)
    return ((u + 0x7FFF + ((u >> 16) & 1)) & 0xFFFF0000).view(np.float32)

def setup_inputs():

    x = _to_bf16(np.random.randn(8192, 8192).astype(np.float32))
    w_mat = _to_bf16((np.random.randn(8192, 8192) / np.sqrt(8192)).astype(np.float32))
    return {"x": x, "w_mat": w_mat}

def _epilogue(y):
    return np.maximum(y, 0.0).astype(np.float32)

def reference(x, w_mat):
    y = (x @ w_mat).astype(np.float32)
    return _epilogue(y)
```

```cpp
#include <hip/hip_runtime.h>
#include <rccl/rccl.h>
#include <cstdio>
#include <cstdint>

#define HIP_CHECK(expr) do { hipError_t _e = (expr); if (_e != hipSuccess) { \
    fprintf(stderr, "HIP error %s at %s:%d\n", hipGetErrorString(_e), __FILE__, __LINE__); \
    return; } } while (0)

typedef short bf16x8 __attribute__((ext_vector_type(8)));
typedef float f32x4 __attribute__((ext_vector_type(4)));
typedef unsigned u32x4 __attribute__((ext_vector_type(4)));
typedef unsigned u32x2 __attribute__((ext_vector_type(2)));
typedef unsigned long long u64;
#define LAS __attribute__((address_space(3)))
#define GAS __attribute__((address_space(1)))

namespace {

constexpr int NR = 8;
constexpr int NG = 8192;
constexpr int KSH = 1024;
constexpr int NPOS = 128;
constexpr int NWG = 256;
constexpr int NTHR = 768;
constexpr int LAG = 4;
constexpr int NSEG = 2048;
constexpr int SEG_BYTES = 512 + 8192;
constexpr u64 DEADLINE_TICKS = 50000000ull;

constexpr size_t WS_TICKET = 0;
constexpr size_t WS_TMO = 128;
constexpr size_t WS_XF = 4096;
constexpr size_t WS_WF = WS_XF + 16 * 128 * 16;
constexpr size_t WS_SEGRDY = WS_WF + 16 * 128 * 8;
constexpr size_t WS_RF = WS_SEGRDY + NSEG * 4;
constexpr size_t WS_DBG = WS_RF + NR * NSEG * 4;
constexpr size_t WS_XA = 1u << 20;
constexpr size_t WS_WB = WS_XA + (size_t)NPOS * 16 * 32768;
constexpr size_t WS_OSTG = WS_WB + (size_t)NPOS * 16 * 16384;
constexpr size_t WS_ORECV = WS_OSTG + (size_t)NSEG * SEG_BYTES;
constexpr size_t WS_END = WS_ORECV + (size_t)NR * NSEG * SEG_BYTES;
static_assert(WS_DBG + 256 * 1024 <= WS_XA, "control region");

constexpr int STAGE_BYTES = 49152;
constexpr int LDS_CTL = 3 * STAGE_BYTES;
constexpr int LDS_BYTES = LDS_CTL + 64;
constexpr int WAVE_LDS = 9216;

struct Params {
    const char* x;
    const char* w;
    char* out;
    char* ws[NR];
    int rank;
};

constexpr int AUX_SYS = 17;
constexpr int AUX_AGT = 16;

__device__ __forceinline__ __amdgpu_buffer_rsrc_t mk_rsrc(const void* p, unsigned bytes) {
    return __builtin_amdgcn_make_buffer_rsrc((void*)p, 0, bytes, 0x00020000);
}
__device__ __forceinline__ unsigned ld_sys(const unsigned* p) {
    return __hip_atomic_load(p, __ATOMIC_RELAXED, __HIP_MEMORY_SCOPE_SYSTEM);
}
__device__ __forceinline__ void st_sys(unsigned* p, unsigned v) {
    __hip_atomic_store(p, v, __ATOMIC_RELAXED, __HIP_MEMORY_SCOPE_SYSTEM);
}
__device__ __forceinline__ void wait_vm0() { asm volatile("s_waitcnt vmcnt(0)" ::: "memory"); }

struct Ctx {
    unsigned call;
    unsigned* tmo;
    u64 t0;
};
__device__ __forceinline__ bool give_up(const Ctx& c) {
    if (__hip_atomic_load(c.tmo, __ATOMIC_RELAXED, __HIP_MEMORY_SCOPE_AGENT) == c.call) return true;
    if (__builtin_amdgcn_s_memrealtime() - c.t0 > DEADLINE_TICKS) {
        __hip_atomic_store(c.tmo, c.call, __ATOMIC_RELAXED, __HIP_MEMORY_SCOPE_AGENT);
        return true;
    }
    return false;
}
__device__ __forceinline__ bool timed_out(const Ctx& c) {
    return __hip_atomic_load(c.tmo, __ATOMIC_RELAXED, __HIP_MEMORY_SCOPE_AGENT) == c.call;
}

__device__ __forceinline__ void send_input(const Params& p, int dst, int part, int c, int v, int lane, unsigned call,
                                           unsigned*& pend) {
    const int rank = p.rank;
    char* wsd = p.ws[dst];
    const int pos = (dst == rank) ? c : 16 + c * 7 + ((rank - dst - 1) & 7);
    u32x4 r[8];
    unsigned* flag;
    char* dbase;
    if (part == 0) {
        const int tj = v >> 1, h = v & 1;
        const int col0 = (dst & 3) * 2048 + tj * 128 + h * 64;
        const char* src = p.w + (size_t)(c * 64 + (lane >> 3) * 8) * (NG * 2) + (size_t)(col0 + (lane & 7) * 8) * 2;
#pragma unroll
        for (int j = 0; j < 8; ++j) r[j] = *(const u32x4*)(src + (size_t)j * (NG * 2));
        flag = (unsigned*)(wsd + WS_WF) + (tj * NPOS + pos) * 2 + h;
        dbase = wsd + WS_WB + (size_t)pos * (16 * 16384) + tj * 16384 + h * 8192;
    } else {
        const int ti = (part - 1) * 8 + (v >> 2), q = v & 3;
        const int row0 = (dst >> 2) * 4096 + ti * 256 + q * 64;
        const int rr = lane >> 3, pp = lane & 7;
        const char* src = p.x + (size_t)(row0 + rr) * (KSH * 2) + c * 128;
        const int o0 = (pp ^ (rr >> 1)) * 16, o1 = (pp ^ (4 + (rr >> 1))) * 16;
#pragma unroll
        for (int j = 0; j < 8; ++j) r[j] = *(const u32x4*)(src + (size_t)j * 8 * (KSH * 2) + ((j & 1) ? o1 : o0));
        flag = (unsigned*)(wsd + WS_XF) + (ti * NPOS + pos) * 4 + q;
        dbase = wsd + WS_XA + (size_t)pos * (16 * 32768) + ti * 32768 + q * 8192;
    }
    wait_vm0();
    if (pend && lane == 0) st_sys(pend, call);
    pend = flag;
    const __amdgpu_buffer_rsrc_t rs = mk_rsrc(dbase, 8192);
    if (part == 0) {
#pragma unroll
        for (int cc = 0; cc < 8; ++cc) {
            u32x4 o;
#pragma unroll
            for (int d = 0; d < 4; ++d)
                o[d] = __builtin_amdgcn_perm(r[2 * d + 1][cc >> 1], r[2 * d][cc >> 1], (cc & 1) ? 0x07060302u : 0x05040100u);
            __builtin_amdgcn_raw_buffer_store_b128(o, rs, lane * 16 + cc * 1024, 0, AUX_SYS);
        }
    } else {
#pragma unroll
        for (int j = 0; j < 8; ++j) __builtin_amdgcn_raw_buffer_store_b128(r[j], rs, lane * 16 + j * 1024, 0, AUX_SYS);
    }
}

__device__ __forceinline__ void flush_pending(unsigned*& pend, unsigned val, int lane) {
    wait_vm0();
    if (pend && lane == 0) st_sys(pend, val);
    pend = nullptr;
}

__device__ __forceinline__ void role_helper(const Params& p, const Ctx& cx, int wg, int hw, int lane) {
    unsigned* pend = nullptr;
    const int h = wg * 2 + hw;
#pragma unroll 1
    for (int i = 0; i < 3; ++i) {
        const int j = h + 512 * i, c = j / 96, v3 = j % 96;
        send_input(p, p.rank, v3 >> 5, c, v3 & 31, lane, cx.call, pend);
    }
    flush_pending(pend, cx.call, lane);
}

__device__ __forceinline__ void role_sender(const Params& p, const Ctx& cx, int wg, int lane) {
    if (wg >= 224) return;
    const int rank = p.rank;
    const int link = wg % 7, v = wg / 7;
    const int dst = (rank + 1 + link) & 7;
    unsigned* pend = nullptr;
#pragma unroll 1
    for (int d = 0; d < 16 + LAG; ++d) {
#pragma unroll 1
        for (int part = 0; part < 3; ++part) {
            const int c = (part == 2) ? d - LAG : d;
            if (c < 0 || c >= 16) continue;
            send_input(p, dst, part, c, v, lane, cx.call, pend);
        }
    }
    const char* ostg = p.ws[rank] + WS_OSTG;
    const unsigned* segrdy = (const unsigned*)(p.ws[rank] + WS_SEGRDY);
    char* orecv = p.ws[dst] + WS_ORECV + (size_t)rank * NSEG * SEG_BYTES;
    unsigned* rf = (unsigned*)(p.ws[dst] + WS_RF) + rank * NSEG;
    const unsigned tag = cx.call << 13;
    unsigned pval0 = cx.call, pval1 = 0;
    bool pend2 = false;
    bool dead = false;
#pragma unroll 1
    for (int i = 0; i < 32; ++i) {
        const int s0 = 2 * (v + 32 * i);
        unsigned f0, f1;
        for (;;) {
            const u64 f = __hip_atomic_load((const u64*)(segrdy + s0), __ATOMIC_RELAXED, __HIP_MEMORY_SCOPE_AGENT);
            f0 = __builtin_amdgcn_readfirstlane((unsigned)f);
            f1 = __builtin_amdgcn_readfirstlane((unsigned)(f >> 32));
            if ((f0 >> 13) == (tag >> 13) && (f1 >> 13) == (tag >> 13)) break;
            if (pend) {
                wait_vm0();
                if (lane == 0) { st_sys(pend, pval0); if (pend2) st_sys(pend + 1, pval1); }
                pend = nullptr;
            }
            if (dead || give_up(cx)) { dead = true; break; }
            __builtin_amdgcn_s_sleep(8);
        }
        if (dead) break;
        const unsigned len0 = (512 + (f0 & 8191) * 2 + 63) & ~63u, len1 = (512 + (f1 & 8191) * 2 + 63) & ~63u;
        const __amdgpu_buffer_rsrc_t a0 = mk_rsrc(ostg + (size_t)s0 * SEG_BYTES, len0);
        const __amdgpu_buffer_rsrc_t a1 = mk_rsrc(ostg + (size_t)(s0 + 1) * SEG_BYTES, len1);
        u32x4 r0[9], r1[9];
#pragma unroll
        for (int j = 0; j < 9; ++j) if (j * 1024u < len0) r0[j] = __builtin_amdgcn_raw_buffer_load_b128(a0, lane * 16 + j * 1024, 0, AUX_AGT);
#pragma unroll
        for (int j = 0; j < 9; ++j) if (j * 1024u < len1) r1[j] = __builtin_amdgcn_raw_buffer_load_b128(a1, lane * 16 + j * 1024, 0, AUX_AGT);
        wait_vm0();
        if (pend && lane == 0) { st_sys(pend, pval0); if (pend2) st_sys(pend + 1, pval1); }
        pend = rf + s0; pend2 = true; pval0 = f0; pval1 = f1;
        const __amdgpu_buffer_rsrc_t b0 = mk_rsrc(orecv + (size_t)s0 * SEG_BYTES, len0);
        const __amdgpu_buffer_rsrc_t b1 = mk_rsrc(orecv + (size_t)(s0 + 1) * SEG_BYTES, len1);
#pragma unroll
        for (int j = 0; j < 9; ++j) if (j * 1024u < len0) __builtin_amdgcn_raw_buffer_store_b128(r0[j], b0, lane * 16 + j * 1024, 0, AUX_SYS);
#pragma unroll
        for (int j = 0; j < 9; ++j) if (j * 1024u < len1) __builtin_amdgcn_raw_buffer_store_b128(r1[j], b1, lane * 16 + j * 1024, 0, AUX_SYS);
    }
    wait_vm0();
    if (pend && lane == 0) { st_sys(pend, pval0); if (pend2) st_sys(pend + 1, pval1); }
}

__device__ __forceinline__ void role_poller(const Params& p, const Ctx& cx, int ti, int tj, int lane, char* lds) {
    const char* ws = p.ws[p.rank];
    const __amdgpu_buffer_rsrc_t rx = mk_rsrc(ws + WS_XF + ti * (NPOS * 16), NPOS * 16);
    const __amdgpu_buffer_rsrc_t rw = mk_rsrc(ws + WS_WF + tj * (NPOS * 8), NPOS * 8);
    volatile LAS unsigned* ready = (volatile LAS unsigned*)(LAS char*)(lds + LDS_CTL + 4);
    int pos = 0;
    while (pos < NPOS) {
        const int q = pos + (lane & 15);
        bool ok = false;
        if (lane < 16) {
            const u32x4 a = __builtin_amdgcn_raw_buffer_load_b128(rx, q * 16, 0, AUX_SYS);
            const u32x2 b = __builtin_amdgcn_raw_buffer_load_b64(rw, q * 8, 0, AUX_SYS);
            ok = a[0] == cx.call && a[1] == cx.call && a[2] == cx.call && a[3] == cx.call && b[0] == cx.call && b[1] == cx.call;
        }
        const u64 m = __ballot(ok);
        const int n = __builtin_ctzll(~m);
        if (n > 0) {
            pos += n;
            if (lane == 0) *ready = pos;
        } else {
            if (give_up(cx)) { pos = NPOS; if (lane == 0) *ready = NPOS; break; }
            __builtin_amdgcn_s_sleep(4);
        }
    }
}

__device__ __forceinline__ void expand_segment(const Params& p, const Ctx& cx, int src, int seg, int lane, char* wl) {
    const char* ws = p.ws[p.rank];
    const unsigned* rf = (const unsigned*)(ws + WS_RF) + src * NSEG + seg;
    unsigned f;
    bool dead = false;
    for (;;) {
        f = __builtin_amdgcn_readfirstlane(ld_sys(rf));
        if ((f >> 13) == ((cx.call << 13) >> 13)) break;
        if (give_up(cx)) { dead = true; break; }
        __builtin_amdgcn_s_sleep(16);
    }
    const int tile = seg >> 3, wv = seg & 7;
    const int row0 = (src >> 2) * 4096 + (tile >> 4) * 256 + (wv >> 1) * 64 + (lane >> 4) * 4;
    const int col0 = (src & 3) * 2048 + (tile & 15) * 128 + (wv & 1) * 64 + (lane & 7) * 8 + ((lane >> 3) & 1) * 4;
    char* o = p.out + ((size_t)row0 * NG + col0) * 2;
    if (dead) {
        const u32x2 nan2 = {0x7FC07FC0u, 0x7FC07FC0u};
#pragma unroll 1
        for (int i = 0; i < 16; ++i) *(u32x2*)(o + (size_t)((i >> 2) * 16 + (i & 3)) * (NG * 2)) = nan2;
        return;
    }
    const unsigned cnt = f & 8191;
    const unsigned vbytes = (cnt * 2 + 63) & ~63u;
    const char* sp = ws + WS_ORECV + ((size_t)src * NSEG + seg) * SEG_BYTES;
    const __amdgpu_buffer_rsrc_t rs = mk_rsrc(sp, 512 + vbytes);
    const u32x2 mk = __builtin_amdgcn_raw_buffer_load_b64(rs, lane * 8, 0, AUX_AGT);
    u32x4 r[8];
#pragma unroll
    for (int j = 0; j < 8; ++j) if (j * 1024u < vbytes) r[j] = __builtin_amdgcn_raw_buffer_load_b128(rs, 512 + lane * 16 + j * 1024, 0, AUX_AGT);
#pragma unroll
    for (int j = 0; j < 8; ++j) if (j * 1024u < vbytes) *(u32x4*)(wl + lane * 16 + j * 1024) = r[j];
    __builtin_amdgcn_fence(__ATOMIC_RELEASE, "wavefront");
    __builtin_amdgcn_wave_barrier();
    const unsigned short* vals = (const unsigned short*)wl;
    unsigned base = 0;
#pragma unroll
    for (int i = 0; i < 16; ++i) {
        unsigned e[4];
#pragma unroll
        for (int f4 = 0; f4 < 4; ++f4) {
            const unsigned mlo = __builtin_amdgcn_readlane(mk[0], i * 4 + f4), mhi = __builtin_amdgcn_readlane(mk[1], i * 4 + f4);
            const unsigned idx = __builtin_amdgcn_mbcnt_hi(mhi, __builtin_amdgcn_mbcnt_lo(mlo, 0)) + base;
            const u64 m = ((u64)mhi << 32) | mlo;
            const unsigned val = vals[idx];
            e[f4] = __builtin_amdgcn_inverse_ballot_w64(m) ? val : 0u;
            base += __builtin_popcountll(m);
        }
        const u32x2 ov = {e[0] | (e[1] << 16), e[2] | (e[3] << 16)};
        *(u32x2*)(o + (size_t)((i >> 2) * 16 + (i & 3)) * (NG * 2)) = ov;
    }
    __builtin_amdgcn_fence(__ATOMIC_RELEASE, "wavefront");
    __builtin_amdgcn_wave_barrier();
}

__device__ __forceinline__ unsigned bf16_rne(float f) {
    const unsigned u = __float_as_uint(f);
    return (u + 0x7FFFu + ((u >> 16) & 1u)) >> 16;
}

__device__ __forceinline__ void role_gemm(const Params& p, const Ctx& cx, int ti, int tj, int wv, int lane, char* lds) {
    const char* ws = p.ws[p.rank];
    const unsigned ldsbase = (unsigned)(uintptr_t)(LAS char*)lds;
    const unsigned ctl = ldsbase + LDS_CTL;
    const int q = wv >> 1, h = wv & 1;
    const char* ga = ws + WS_XA + ti * 32768 + wv * 4096 + lane * 16;
    const char* gb = ws + WS_WB + tj * 16384 + wv * 2048 + lane * 16;
    const int fr = lane & 15, kq = lane >> 4;
    const int offA0 = q * 8192 + fr * 128 + ((kq ^ (fr >> 1)) * 16);
    const int offA1 = q * 8192 + fr * 128 + (((4 + kq) ^ (fr >> 1)) * 16);
    const int offB = 32768 + h * 8192 + (fr >> 3) * 4096 + kq * 128 + (fr & 7) * 16;

    f32x4 acc[4][4];
#pragma unroll
    for (int m = 0; m < 4; ++m)
#pragma unroll
        for (int n = 0; n < 4; ++n) acc[m][n] = (f32x4){0.f, 0.f, 0.f, 0.f};

    int issued = 0, islot = 0;
    unsigned rdy = 0, cnt = 0;
    auto issue = [&]() {
        const char* a = ga + (size_t)issued * (16 * 32768);
        const char* b = gb + (size_t)issued * (16 * 16384);
        char* la = lds + islot * STAGE_BYTES + wv * 4096;
        char* lb = lds + islot * STAGE_BYTES + 32768 + wv * 2048;
        __builtin_amdgcn_global_load_lds((const GAS void*)a, (LAS void*)la, 16, 0, AUX_AGT);
        __builtin_amdgcn_global_load_lds((const GAS void*)a, (LAS void*)la, 16, 1024, AUX_AGT);
        __builtin_amdgcn_global_load_lds((const GAS void*)a, (LAS void*)la, 16, 2048, AUX_AGT);
        __builtin_amdgcn_global_load_lds((const GAS void*)a, (LAS void*)la, 16, 3072, AUX_AGT);
        __builtin_amdgcn_global_load_lds((const GAS void*)b, (LAS void*)lb, 16, 0, AUX_AGT);
        __builtin_amdgcn_global_load_lds((const GAS void*)b, (LAS void*)lb, 16, 1024, AUX_AGT);
        ++issued;
        islot = (islot == 2) ? 0 : islot + 1;
    };
    auto poll = [&]() {
        u32x2 v;
        asm volatile("ds_read_b64 %0, %1\n\ts_waitcnt lgkmcnt(0)" : "=v"(v) : "v"(ctl) : "memory");
        cnt = __builtin_amdgcn_readfirstlane(v[0]);
        rdy = __builtin_amdgcn_readfirstlane(v[1]);
    };
    auto arrive = [&]() {
        if (lane == 0) asm volatile("ds_add_u32 %0, %1" :: "v"(ctl), "v"(1u) : "memory");
    };
    auto wait_bar = [&](unsigned target) {
        poll();
        while (cnt < target) { __builtin_amdgcn_s_sleep(1); poll(); }
    };
    auto need = [&](int upto) {
        while (issued < upto) {
            while ((int)rdy <= issued) { __builtin_amdgcn_s_sleep(2); poll(); }
            issue();
        }
    };

    need(1);
    if ((int)rdy > issued) issue();
    if (issued == 2) asm volatile("s_waitcnt vmcnt(6)" ::: "memory"); else wait_vm0();
    arrive();
    wait_bar(8);

    bf16x8 a0[4], b0[4], a1[4], b1[4];
    int cslot = 0;
    {
        const char* s = lds;
#pragma unroll
        for (int m = 0; m < 4; ++m) a0[m] = *(const bf16x8*)(s + offA0 + m * 2048);
#pragma unroll
        for (int n = 0; n < 4; ++n) b0[n] = *(const bf16x8*)(s + offB + n * 1024);
    }
#pragma unroll 1
    for (int t = 0; t < NPOS; ++t) {
        const char* s = lds + cslot * STAGE_BYTES;
        cslot = (cslot == 2) ? 0 : cslot + 1;
        const char* sn = lds + cslot * STAGE_BYTES;
        {
            const int lim = (t + 3 < NPOS) ? t + 3 : NPOS;
            if (issued < lim && issued < (int)rdy) issue();
            if (issued < lim && issued < (int)rdy) issue();
        }
#pragma unroll
        for (int m = 0; m < 4; ++m) a1[m] = *(const bf16x8*)(s + offA1 + m * 2048);
#pragma unroll
        for (int n = 0; n < 4; ++n) b1[n] = *(const bf16x8*)(s + offB + 512 + n * 1024);
        __builtin_amdgcn_s_setprio(1);
#pragma unroll
        for (int m = 0; m < 4; ++m)
#pragma unroll
            for (int n = 0; n < 4; ++n) acc[m][n] = __builtin_amdgcn_mfma_f32_16x16x32_bf16(a0[m], b0[n], acc[m][n], 0, 0, 0);
        __builtin_amdgcn_s_setprio(0);
        __builtin_amdgcn_sched_barrier(0);
        if (t + 1 < NPOS) {
            need(t + 2);
            if (issued == t + 3) asm volatile("s_waitcnt vmcnt(6) lgkmcnt(0)" ::: "memory");
            else asm volatile("s_waitcnt vmcnt(0) lgkmcnt(0)" ::: "memory");
        } else {
            asm volatile("s_waitcnt lgkmcnt(0)" ::: "memory");
        }
        arrive();
        __builtin_amdgcn_sched_barrier(0);
        __builtin_amdgcn_s_setprio(1);
#pragma unroll
        for (int m = 0; m < 2; ++m)
#pragma unroll
            for (int n = 0; n < 4; ++n) acc[m][n] = __builtin_amdgcn_mfma_f32_16x16x32_bf16(a1[m], b1[n], acc[m][n], 0, 0, 0);
        __builtin_amdgcn_s_setprio(0);
        __builtin_amdgcn_sched_barrier(0);
        wait_bar(8u * (t + 2));
        if (t + 1 < NPOS) {
#pragma unroll
            for (int m = 0; m < 4; ++m) a0[m] = *(const bf16x8*)(sn + offA0 + m * 2048);
#pragma unroll
            for (int n = 0; n < 4; ++n) b0[n] = *(const bf16x8*)(sn + offB + n * 1024);
        }
        __builtin_amdgcn_s_setprio(1);
#pragma unroll
        for (int m = 2; m < 4; ++m)
#pragma unroll
            for (int n = 0; n < 4; ++n) acc[m][n] = __builtin_amdgcn_mfma_f32_16x16x32_bf16(a1[m], b1[n], acc[m][n], 0, 0, 0);
        __builtin_amdgcn_s_setprio(0);
        __builtin_amdgcn_sched_barrier(0);
    }

    char* wl = lds + wv * WAVE_LDS;
    unsigned short* vals = (unsigned short*)wl;
    const bool bad = timed_out(cx);
    const int seg = ((ti * 16 + tj) << 3) + wv;
    u32x2 mk = {0u, 0u};
    u32x2 ov[16];
    unsigned base = 0;
#pragma unroll
    for (int i = 0; i < 16; ++i) {
        unsigned e[4];
#pragma unroll
        for (int f4 = 0; f4 < 4; ++f4) {
            const float v = acc[i >> 2][f4][i & 3];
            unsigned b = bf16_rne(v > 0.f ? v : 0.f);
            if (bad) b = 0x7FC0u;
            e[f4] = b;
            const u64 m = __ballot(b != 0u);
            const unsigned idx = __builtin_amdgcn_mbcnt_hi((unsigned)(m >> 32), __builtin_amdgcn_mbcnt_lo((unsigned)m, 0)) + base;
            if (b != 0u) vals[idx] = (unsigned short)b;
            base += __builtin_popcountll(m);
            if (lane == i * 4 + f4) { mk[0] = (unsigned)m; mk[1] = (unsigned)(m >> 32); }
        }
        ov[i] = (u32x2){e[0] | (e[1] << 16), e[2] | (e[3] << 16)};
    }
    __builtin_amdgcn_fence(__ATOMIC_RELEASE, "wavefront");
    __builtin_amdgcn_wave_barrier();
    {
        const unsigned vbytes = (base * 2 + 63) & ~63u;
        const __amdgpu_buffer_rsrc_t rs = mk_rsrc(ws + WS_OSTG + (size_t)seg * SEG_BYTES, 512 + vbytes);
        __builtin_amdgcn_raw_buffer_store_b64(mk, rs, lane * 8, 0, AUX_SYS);
#pragma unroll
        for (int j = 0; j < 8; ++j)
            if (j * 1024u < vbytes)
                __builtin_amdgcn_raw_buffer_store_b128(*(const u32x4*)(wl + lane * 16 + j * 1024), rs, 512 + lane * 16 + j * 1024, 0, AUX_SYS);
        wait_vm0();
        if (lane == 0)
            __hip_atomic_store((unsigned*)(ws + WS_SEGRDY) + seg, (cx.call << 13) | base, __ATOMIC_RELAXED, __HIP_MEMORY_SCOPE_AGENT);
    }
    {
        const int row0 = (p.rank >> 2) * 4096 + ti * 256 + q * 64 + (lane >> 4) * 4;
        const int col0 = (p.rank & 3) * 2048 + tj * 128 + h * 64 + (lane & 7) * 8 + ((lane >> 3) & 1) * 4;
        char* o = p.out + ((size_t)row0 * NG + col0) * 2;
#pragma unroll
        for (int i = 0; i < 16; ++i) *(u32x2*)(o + (size_t)((i >> 2) * 16 + (i & 3)) * (NG * 2)) = ov[i];
    }
    __builtin_amdgcn_fence(__ATOMIC_RELEASE, "wavefront");
    __builtin_amdgcn_wave_barrier();
#pragma unroll 1
    for (int i = 0; i < 7; ++i) {
        const int e = seg + NSEG * i;
        expand_segment(p, cx, (p.rank + 1 + e % 7) & 7, e / 7, lane, wl);
    }
}

}

__global__ void __launch_bounds__(NTHR) fused_gemm_ar(Params p) {
    extern __shared__ __attribute__((aligned(16))) char lds[];
    const int lane = threadIdx.x & 63;
    const int wave = __builtin_amdgcn_readfirstlane(threadIdx.x >> 6);
    const int wg = blockIdx.x;
    volatile unsigned* ctl = (volatile unsigned*)(lds + LDS_CTL);
    char* ws = p.ws[p.rank];
    if (threadIdx.x == 0) {
        const unsigned ticket = __hip_atomic_fetch_add((unsigned*)(ws + WS_TICKET), 1u, __ATOMIC_RELAXED, __HIP_MEMORY_SCOPE_AGENT);
        ctl[0] = 0u; ctl[1] = 0u; ctl[2] = ticket / NWG + 1u;
    }
    __syncthreads();
    Ctx cx;
    cx.call = __builtin_amdgcn_readfirstlane(ctl[2]);
    cx.tmo = (unsigned*)(ws + WS_TMO);
    cx.t0 = __builtin_amdgcn_s_memrealtime();
    const int xcd = wg & 7, idx = wg >> 3;
    const int ti = (xcd >> 1) * 4 + (idx & 3), tj = (xcd & 1) * 8 + (idx >> 2);
    if (wave < 8) role_gemm(p, cx, ti, tj, wave, lane, lds);
    else if (wave == 8) role_poller(p, cx, ti, tj, lane, lds);
    else if (wave == 9) role_sender(p, cx, wg, lane);
    else role_helper(p, cx, wg, wave - 10, lane);
}

extern "C" void kernel_launch(int rank, int nranks, ncclComm_t comm, hipStream_t stream,
                              void* const* d_in, const int* in_sizes, int n_in,
                              void* d_out, int out_size,
                              const int* in_dtypes, int out_dtype,
                              void* const* d_out_all, void* const* d_ws_all, size_t ws_bytes) {
    if (nranks != NR || n_in < 2 || ws_bytes < WS_END || (unsigned)rank >= (unsigned)NR) {
        fprintf(stderr, "unsupported configuration\n");
        return;
    }
    static bool attr_set[NR] = {};
    if (!attr_set[rank]) {
        HIP_CHECK(hipFuncSetAttribute((const void*)fused_gemm_ar, hipFuncAttributeMaxDynamicSharedMemorySize, LDS_BYTES));
        attr_set[rank] = true;
    }
    Params p;
    p.x = (const char*)d_in[0];
    p.w = (const char*)d_in[1];
    p.out = (char*)d_out;
    for (int i = 0; i < NR; ++i) p.ws[i] = (char*)d_ws_all[i];
    p.rank = rank;
    fused_gemm_ar<<<NWG, NTHR, LDS_BYTES, stream>>>(p);
    HIP_CHECK(hipGetLastError());
}
```

```cpp
#include <hip/hip_runtime.h>
#include <rccl/rccl.h>
#include <cstdio>
#include <cstdint>

#define HIP_CHECK(expr) do { hipError_t _e = (expr); if (_e != hipSuccess) { \
    fprintf(stderr, "HIP error %s at %s:%d\n", hipGetErrorString(_e), __FILE__, __LINE__); \
    return; } } while (0)

typedef short bf16x8 __attribute__((ext_vector_type(8)));
typedef float f32x4 __attribute__((ext_vector_type(4)));
typedef unsigned u32x4 __attribute__((ext_vector_type(4)));
typedef unsigned u32x2 __attribute__((ext_vector_type(2)));
typedef unsigned long long u64;
#define LAS __attribute__((address_space(3)))
#define GAS __attribute__((address_space(1)))

namespace {

constexpr int NR = 8;
constexpr int NG = 8192;
constexpr int KSH = 1024;
constexpr int NPOS = 128;
constexpr int NWG = 256;
constexpr int NTHR = 768;
#define CFG_LAG 6
#define CFG_NP 35
#define CFG_NLD 3
constexpr int LAG = CFG_LAG;
constexpr int NP = CFG_NP;
static_assert(NP * 7 <= 249, "relay waves live in workgroups 249..255");
constexpr int U_IN = 1536;
constexpr int U_ALL = U_IN + 2048;
constexpr int NSEG = 2048;
constexpr int SEG_BYTES = 512 + 8192;
constexpr u64 DEADLINE_TICKS = 50000000ull;

constexpr size_t WS_TICKET = 0;
constexpr size_t WS_TMO = 128;
constexpr size_t WS_LINKCTR = 256;
constexpr size_t WS_PKCNT = 4096;
constexpr size_t WS_XF = 8192;
constexpr size_t WS_WF = WS_XF + 16 * 128 * 16;
constexpr size_t WS_SEGRDY = WS_WF + 16 * 128 * 8;
constexpr size_t WS_RF = WS_SEGRDY + NSEG * 4;
constexpr size_t WS_MPK = WS_RF + NR * NSEG * 4;
constexpr size_t WS_MSEG = WS_MPK + 1024;
constexpr size_t WS_DBG = WS_MSEG + 8 * NSEG * 4;
constexpr size_t WS_PKX = 1u << 20;
constexpr size_t WS_PKW = WS_PKX + (16u << 20);
constexpr size_t WS_XA = WS_PKW + (16u << 20);
constexpr size_t WS_WB = WS_XA + (size_t)NPOS * 16 * 32768;
constexpr size_t WS_OSTG = WS_WB + (size_t)NPOS * 16 * 16384;
constexpr size_t WS_ORECV = WS_OSTG + (size_t)NSEG * SEG_BYTES;
constexpr size_t WS_END = WS_ORECV + (size_t)NR * NSEG * SEG_BYTES;
static_assert(WS_DBG + 256 * 1024 <= WS_PKX, "control region");

constexpr int STAGE_BYTES = 49152;
constexpr int LDS_CTL = 3 * STAGE_BYTES;
constexpr int LDS_BYTES = LDS_CTL + 64;
constexpr int WAVE_LDS = 9216;

struct Params {
    const char* x;
    const char* w;
    char* out;
    char* ws[NR];
    int rank;
};

constexpr int AUX_SYS = 17;
constexpr int AUX_AGT = 16;

__device__ __forceinline__ __amdgpu_buffer_rsrc_t mk_rsrc(const void* p, unsigned bytes) {
    return __builtin_amdgcn_make_buffer_rsrc((void*)p, 0, bytes, 0x00020000);
}
__device__ __forceinline__ unsigned ld_sys(const unsigned* p) {
    return __hip_atomic_load(p, __ATOMIC_RELAXED, __HIP_MEMORY_SCOPE_SYSTEM);
}
__device__ __forceinline__ void st_sys(unsigned* p, unsigned v) {
    __hip_atomic_store(p, v, __ATOMIC_RELAXED, __HIP_MEMORY_SCOPE_SYSTEM);
}
__device__ __forceinline__ void wait_vm0() { asm volatile("s_waitcnt vmcnt(0)" ::: "memory"); }

struct Ctx {
    unsigned call;
    unsigned* tmo;
    u64 t0;
};
__device__ __forceinline__ bool give_up(const Ctx& c) {
    if (__hip_atomic_load(c.tmo, __ATOMIC_RELAXED, __HIP_MEMORY_SCOPE_AGENT) == c.call) return true;
    if (__builtin_amdgcn_s_memrealtime() - c.t0 > DEADLINE_TICKS) {
        __hip_atomic_store(c.tmo, c.call, __ATOMIC_RELAXED, __HIP_MEMORY_SCOPE_AGENT);
        return true;
    }
    return false;
}
__device__ __forceinline__ bool timed_out(const Ctx& c) {
    return __hip_atomic_load(c.tmo, __ATOMIC_RELAXED, __HIP_MEMORY_SCOPE_AGENT) == c.call;
}

#define STAMP(p, wg, k, lane) do {} while (0)

__device__ __forceinline__ void st_agt(unsigned* p, unsigned v) {
    __hip_atomic_store(p, v, __ATOMIC_RELAXED, __HIP_MEMORY_SCOPE_AGENT);
}

__device__ __forceinline__ void role_packer(const Params& p, const Ctx& cx, int h, int lane) {
    const int rank = p.rank;
    char* ws = p.ws[rank];
    unsigned* pend_cnt = nullptr;
    unsigned* pend_flag = nullptr;
#pragma unroll 1
    for (int i = 0; i < 8; ++i) {
        const int j = h + 512 * i, c = j >> 8, r = j & 255;
        u32x4 v[8];
        char* dbase;
        unsigned* flag = nullptr;
        if (r < 128) {
            const int cb = r >> 1, hh = r & 1;
            const char* src = p.w + (size_t)(c * 64 + (lane >> 3) * 8) * (NG * 2) + (size_t)(cb * 128 + hh * 64 + (lane & 7) * 8) * 2;
#pragma unroll
            for (int k = 0; k < 8; ++k) v[k] = *(const u32x4*)(src + (size_t)k * (NG * 2));
            dbase = ws + WS_PKW + (size_t)(c * 64 + cb) * 16384 + hh * 8192;
            if ((cb >> 4) == (rank & 3)) flag = (unsigned*)(ws + WS_WF) + ((cb & 15) * NPOS + c) * 2 + hh;
        } else {
            const int rb = (r - 128) >> 2, q = r & 3;
            const int rr = lane >> 3, pp = lane & 7;
            const char* src = p.x + (size_t)(rb * 256 + q * 64 + rr) * (KSH * 2) + c * 128;
            const int o0 = (pp ^ (rr >> 1)) * 16, o1 = (pp ^ (4 + (rr >> 1))) * 16;
#pragma unroll
            for (int k = 0; k < 8; ++k) v[k] = *(const u32x4*)(src + (size_t)k * 8 * (KSH * 2) + ((k & 1) ? o1 : o0));
            dbase = ws + WS_PKX + (size_t)(c * 32 + rb) * 32768 + q * 8192;
            if ((rb >> 4) == (rank >> 2)) flag = (unsigned*)(ws + WS_XF) + ((rb & 15) * NPOS + c) * 4 + q;
        }
        wait_vm0();
        if (pend_cnt && lane == 0) {
            __hip_atomic_fetch_add(pend_cnt, 1u, __ATOMIC_RELAXED, __HIP_MEMORY_SCOPE_SYSTEM);
            if (pend_flag) st_agt(pend_flag, cx.call);
        }
        pend_cnt = (unsigned*)(ws + WS_PKCNT + c * 256);
        pend_flag = flag;
        const __amdgpu_buffer_rsrc_t rs = mk_rsrc(dbase, 8192);
        if (r < 128) {
#pragma unroll
            for (int cc = 0; cc < 8; ++cc) {
                u32x4 o;
#pragma unroll
                for (int d = 0; d < 4; ++d)
                    o[d] = __builtin_amdgcn_perm(v[2 * d + 1][cc >> 1], v[2 * d][cc >> 1], (cc & 1) ? 0x07060302u : 0x05040100u);
                __builtin_amdgcn_raw_buffer_store_b128(o, rs, lane * 16 + cc * 1024, 0, AUX_SYS);
            }
        } else {
#pragma unroll
            for (int k = 0; k < 8; ++k) __builtin_amdgcn_raw_buffer_store_b128(v[k], rs, lane * 16 + k * 1024, 0, AUX_SYS);
        }
    }
    wait_vm0();
    if (lane == 0) {
        __hip_atomic_fetch_add(pend_cnt, 1u, __ATOMIC_RELAXED, __HIP_MEMORY_SCOPE_SYSTEM);
        if (pend_flag) st_agt(pend_flag, cx.call);
    }
}

constexpr int NLD = CFG_NLD;
constexpr unsigned OOB = 0x7FFFFF00u;
struct Unit {
    const char* src;
    char* dst;
    unsigned flag;
    unsigned info;
    unsigned len, fval;
    int need;
};

__device__ __forceinline__ void role_puller(const Params& p, const Ctx& cx, int link, int lane) {
    const int rank = p.rank;
    const int src = (rank + 1 + link) & 7;
    char* ws = p.ws[rank];
    const char* rws = p.ws[src];
    const __amdgpu_buffer_rsrc_t rctl = mk_rsrc(ws, 1u << 20);
    const unsigned ctr_off = (lane == 0) ? (unsigned)WS_LINKCTR + link * 128 : OOB;
    const unsigned base = (cx.call - 1) * (unsigned)(U_ALL + NP);
    const unsigned tag = (cx.call << 13) >> 13;
    const unsigned pk_target = cx.call * 256u;
    unsigned packed = 0;
    bool dead = false;

    auto make_unit = [&](int n, Unit& u) {
        u.len = 8192; u.fval = cx.call;
        if (n >= U_ALL) { u.need = -2; u.src = rws; u.dst = ws; u.flag = OOB; u.info = OOB; u.len = 0; return; }
        if (n < U_IN) {
            int d, r;
            if (n < 64 * LAG) { d = n >> 6; r = n & 63; }
            else if (n < 64 * LAG + 96 * (16 - LAG)) { const int m = n - 64 * LAG; d = LAG + m / 96; r = m % 96; }
            else { const int m = n - 64 * LAG - 96 * (16 - LAG); d = 16 + (m >> 5); r = 64 + (m & 31); }
            const int part = r >> 5, v = r & 31;
            const int c = (part == 2) ? d - LAG : d;
            const int pos = 16 + c * 7 + link;
            u.need = c;
            u.info = ((packed >> c) & 1u) ? OOB : (unsigned)WS_MPK + link * 128 + c * 4;
            if (part == 0) {
                u.src = rws + WS_PKW + (size_t)(c * 64 + (rank & 3) * 16) * 16384 + v * 8192;
                u.dst = ws + WS_WB + (size_t)pos * (16 * 16384) + v * 8192;
                u.flag = (unsigned)WS_WF + ((v >> 1) * NPOS + pos) * 8 + (v & 1) * 4;
            } else {
                const int vv = (part - 1) * 32 + v;
                u.src = rws + WS_PKX + (size_t)(c * 32 + (rank >> 2) * 16) * 32768 + vv * 8192;
                u.dst = ws + WS_XA + (size_t)pos * (16 * 32768) + vv * 8192;
                u.flag = (unsigned)WS_XF + ((vv >> 2) * NPOS + pos) * 16 + (vv & 3) * 4;
            }
        } else {
            const int sg = n - U_IN;
            u.need = -1;
            u.info = (unsigned)WS_MSEG + (link * NSEG + sg) * 4;
            u.src = rws + WS_OSTG + (size_t)sg * SEG_BYTES;
            u.dst = ws + WS_ORECV + ((size_t)src * NSEG + sg) * SEG_BYTES;
            u.flag = (unsigned)WS_RF + (src * NSEG + sg) * 4;
        }
    };
    auto check_info = [&](Unit& u, unsigned info_v) -> bool {
        const unsigned info = __builtin_amdgcn_readfirstlane(info_v);
        if (u.need >= 0) {
            if (u.info == OOB) return true;
            if ((int)(info - pk_target) < 0) return false;
            packed |= 1u << u.need;
            return true;
        }
        if (u.need == -1) {
            u.fval = info;
            u.len = (512 + (info & 8191) * 2 + 63) & ~63u;
            return (info >> 13) == tag;
        }
        return true;
    };
    auto load_info = [&](const Unit& u) -> unsigned {
        return __builtin_amdgcn_raw_buffer_load_b32(rctl, (lane == 0) ? u.info : OOB, 0, AUX_AGT);
    };
    auto wait_ready = [&](Unit& u) {
        for (;;) {
            if (check_info(u, load_info(u))) return;
            if (dead || give_up(cx)) { dead = true; u.len = 0; u.flag = OOB; return; }
            __builtin_amdgcn_s_sleep(8);
        }
    };
#define PL_PUBLISH() do { if (pf != OOB) { __builtin_amdgcn_raw_buffer_store_b32(pv, rctl, (lane == 0) ? pf : OOB, 0, AUX_AGT); pf = OOB; } } while (0)
#define PL_TICKET() ((unsigned)__builtin_amdgcn_raw_ptr_buffer_atomic_add_i32(1, rctl, ctr_off, 0, AUX_AGT))
#define PL_PIECE(k) do { u32x4 b_[NLD]; \
        _Pragma("unroll") for (int j_ = 0; j_ < NLD; ++j_) b_[j_] = __builtin_amdgcn_raw_buffer_load_b128(rs, lane * 16 + (k) * (NLD * 1024) + j_ * 1024, 0, AUX_SYS); \
        wait_vm0(); \
        PL_PUBLISH(); \
        _Pragma("unroll") for (int j_ = 0; j_ < NLD; ++j_) __builtin_amdgcn_raw_buffer_store_b128(b_[j_], rd, lane * 16 + (k) * (NLD * 1024) + j_ * 1024, 0, AUX_AGT); } while (0)

    unsigned pf = OOB, pv = 0;
    Unit u, un;
    make_unit((int)(__builtin_amdgcn_readfirstlane(PL_TICKET()) - base), u);
    if (u.need == -2) return;
    wait_ready(u);
    bool stamped = false;
    for (;;) {
        const __amdgpu_buffer_rsrc_t rs = mk_rsrc(u.src, u.len), rd = mk_rsrc(u.dst, u.len);
        const unsigned t = PL_TICKET();
        PL_PIECE(0);
        make_unit((int)(__builtin_amdgcn_readfirstlane(t) - base), un);
        const unsigned info = load_info(un);
        PL_PIECE(1);
#pragma unroll 1
        for (unsigned k = 2; k * (NLD * 1024) < u.len; ++k) PL_PIECE(k);
        pf = u.flag; pv = u.fval;
        if (un.need == -2 || !check_info(un, info)) {
            wait_vm0();
            PL_PUBLISH();
            if (un.need == -2) break;
            wait_ready(un);
        }
        if (!stamped && un.need == -1) { STAMP(p, blockIdx.x, 1, lane); stamped = true; }
        u = un;
    }
}

__device__ __forceinline__ void role_relay(const Params& p, const Ctx& cx, int link, int lane) {
    const int src = (p.rank + 1 + link) & 7;
    const __amdgpu_buffer_rsrc_t rctl = mk_rsrc(p.ws[p.rank], 1u << 20);
    const __amdgpu_buffer_rsrc_t rinfo = mk_rsrc(p.ws[src], 1u << 20);
    const unsigned pk_target = cx.call * 256u;
    const unsigned tag = (cx.call << 13) >> 13;
    bool done = lane >= 16;
    for (;;) {
        const unsigned v = __builtin_amdgcn_raw_buffer_load_b32(rinfo, done ? OOB : (unsigned)WS_PKCNT + lane * 256, 0, AUX_SYS);
        const bool ok = !done && (int)(v - pk_target) >= 0;
        __builtin_amdgcn_raw_buffer_store_b32(v, rctl, ok ? (unsigned)WS_MPK + link * 128 + lane * 4 : OOB, 0, AUX_AGT);
        done = done || ok;
        if (__ballot(!done) == 0ull) break;
        if (give_up(cx)) return;
        __builtin_amdgcn_s_sleep(4);
    }
    int pos = 0;
    while (pos < NSEG) {
        const unsigned q = pos + lane * 4;
        const u32x4 v = __builtin_amdgcn_raw_buffer_load_b128(rinfo, q < NSEG ? (unsigned)WS_SEGRDY + q * 4 : OOB, 0, AUX_SYS);
        const bool any = (v[0] >> 13) == tag || (v[1] >> 13) == tag || (v[2] >> 13) == tag || (v[3] >> 13) == tag;
        const bool all = (v[0] >> 13) == tag && (v[1] >> 13) == tag && (v[2] >> 13) == tag && (v[3] >> 13) == tag;
        __builtin_amdgcn_raw_buffer_store_b128(v, rctl, (any && q < NSEG) ? (unsigned)WS_MSEG + (link * NSEG + q) * 4 : OOB, 0, AUX_AGT);
        const u64 m = __ballot(all && q < NSEG);
        const int n = (m == ~0ull) ? 64 : __builtin_ctzll(~m);
        pos += n * 4;
        if (n < 64) {
            if (give_up(cx)) return;
            __builtin_amdgcn_s_sleep(32);
        }
    }
}

__device__ __forceinline__ void role_worker(const Params& p, const Ctx& cx, int wg, int k, int lane) {
    __builtin_amdgcn_s_setprio(3);
    if (k == 0) {
        STAMP(p, wg, 0, lane);
        if (wg < NP * 7) role_puller(p, cx, wg % 7, lane);
        else if (wg >= 249) role_relay(p, cx, wg - 249, lane);
        STAMP(p, wg, 2, lane);
    } else {
        role_packer(p, cx, wg * 2 + k - 1, lane);
        if (k == 1) STAMP(p, wg, 7, lane);
    }
}

__device__ __forceinline__ void role_poller(const Params& p, const Ctx& cx, int ti, int tj, int lane, char* lds) {
    const char* ws = p.ws[p.rank];
    const __amdgpu_buffer_rsrc_t rx = mk_rsrc(ws + WS_XF + ti * (NPOS * 16), NPOS * 16);
    const __amdgpu_buffer_rsrc_t rw = mk_rsrc(ws + WS_WF + tj * (NPOS * 8), NPOS * 8);
    volatile LAS unsigned* ready = (volatile LAS unsigned*)(LAS char*)(lds + LDS_CTL + 4);
    int pos = 0;
    while (pos < NPOS) {
        const int q = pos + (lane & 15);
        bool ok = false;
        if (lane < 16) {
            const u32x4 a = __builtin_amdgcn_raw_buffer_load_b128(rx, q * 16, 0, AUX_AGT);
            const u32x2 b = __builtin_amdgcn_raw_buffer_load_b64(rw, q * 8, 0, AUX_AGT);
            ok = a[0] == cx.call && a[1] == cx.call && a[2] == cx.call && a[3] == cx.call && b[0] == cx.call && b[1] == cx.call;
        }
        const u64 m = __ballot(ok);
        const int n = __builtin_ctzll(~m);
        if (n > 0) {
            if (pos <= 16 && pos + n > 16) STAMP(p, blockIdx.x, 8, lane);
            pos += n;
            if (lane == 0) *ready = pos;
        } else {
            if (give_up(cx)) { pos = NPOS; if (lane == 0) *ready = NPOS; break; }
            __builtin_amdgcn_s_sleep(4);
        }
    }
    STAMP(p, blockIdx.x, 6, lane);
}

__device__ __forceinline__ void expand_segment(const Params& p, const Ctx& cx, int src, int seg, int lane, char* wl) {
    const char* ws = p.ws[p.rank];
    const unsigned* rf = (const unsigned*)(ws + WS_RF) + src * NSEG + seg;
    unsigned f;
    bool dead = false;
    for (;;) {
        f = __builtin_amdgcn_readfirstlane(__hip_atomic_load(rf, __ATOMIC_RELAXED, __HIP_MEMORY_SCOPE_AGENT));
        if ((f >> 13) == ((cx.call << 13) >> 13)) break;
        if (give_up(cx)) { dead = true; break; }
        __builtin_amdgcn_s_sleep(16);
    }
    const int tile = seg >> 3, wv = seg & 7;
    const int row0 = (src >> 2) * 4096 + (tile >> 4) * 256 + (wv >> 1) * 64 + (lane >> 4) * 4;
    const int col0 = (src & 3) * 2048 + (tile & 15) * 128 + (wv & 1) * 64 + (lane & 7) * 8 + ((lane >> 3) & 1) * 4;
    char* o = p.out + ((size_t)row0 * NG + col0) * 2;
    if (dead) {
        const u32x2 nan2 = {0x7FC07FC0u, 0x7FC07FC0u};
#pragma unroll 1
        for (int i = 0; i < 16; ++i) *(u32x2*)(o + (size_t)((i >> 2) * 16 + (i & 3)) * (NG * 2)) = nan2;
        return;
    }
    const unsigned cnt = f & 8191;
    const unsigned vbytes = (cnt * 2 + 63) & ~63u;
    const char* sp = ws + WS_ORECV + ((size_t)src * NSEG + seg) * SEG_BYTES;
    const __amdgpu_buffer_rsrc_t rs = mk_rsrc(sp, 512 + vbytes);
    const u32x2 mk = __builtin_amdgcn_raw_buffer_load_b64(rs, lane * 8, 0, AUX_AGT);
    u32x4 r[8];
#pragma unroll
    for (int j = 0; j < 8; ++j) if (j * 1024u < vbytes) r[j] = __builtin_amdgcn_raw_buffer_load_b128(rs, 512 + lane * 16 + j * 1024, 0, AUX_AGT);
#pragma unroll
    for (int j = 0; j < 8; ++j) if (j * 1024u < vbytes) *(u32x4*)(wl + lane * 16 + j * 1024) = r[j];
    __builtin_amdgcn_fence(__ATOMIC_RELEASE, "wavefront");
    __builtin_amdgcn_wave_barrier();
    const unsigned short* vals = (const unsigned short*)wl;
    unsigned base = 0;
#pragma unroll
    for (int i = 0; i < 16; ++i) {
        unsigned e[4];
#pragma unroll
        for (int f4 = 0; f4 < 4; ++f4) {
            const unsigned mlo = __builtin_amdgcn_readlane(mk[0], i * 4 + f4), mhi = __builtin_amdgcn_readlane(mk[1], i * 4 + f4);
            const unsigned idx = __builtin_amdgcn_mbcnt_hi(mhi, __builtin_amdgcn_mbcnt_lo(mlo, 0)) + base;
            const u64 m = ((u64)mhi << 32) | mlo;
            const unsigned val = vals[idx];
            e[f4] = __builtin_amdgcn_inverse_ballot_w64(m) ? val : 0u;
            base += __builtin_popcountll(m);
        }
        const u32x2 ov = {e[0] | (e[1] << 16), e[2] | (e[3] << 16)};
        *(u32x2*)(o + (size_t)((i >> 2) * 16 + (i & 3)) * (NG * 2)) = ov;
    }
    __builtin_amdgcn_fence(__ATOMIC_RELEASE, "wavefront");
    __builtin_amdgcn_wave_barrier();
}

__device__ __forceinline__ unsigned bf16_rne(float f) {
    const unsigned u = __float_as_uint(f);
    return (u + 0x7FFFu + ((u >> 16) & 1u)) >> 16;
}

__device__ __forceinline__ void role_gemm(const Params& p, const Ctx& cx, int ti, int tj, int wv, int lane, char* lds) {
    const char* ws = p.ws[p.rank];
    const unsigned ldsbase = (unsigned)(uintptr_t)(LAS char*)lds;
    const unsigned ctl = ldsbase + LDS_CTL;
    const int q = wv >> 1, h = wv & 1;
    const unsigned lo_a = wv * 4096 + lane * 16, lo_b = wv * 2048 + lane * 16;
    const char* ga = ws + WS_XA + ti * 32768;
    const char* gb = ws + WS_WB + tj * 16384;
    const char* ga_own = ws + WS_PKX + (size_t)((p.rank >> 2) * 16 + ti) * 32768;
    const char* gb_own = ws + WS_PKW + (size_t)((p.rank & 3) * 16 + tj) * 16384;
    const int fr = lane & 15, kq = lane >> 4;
    const int offA0 = q * 8192 + fr * 128 + ((kq ^ (fr >> 1)) * 16);
    const int offA1 = q * 8192 + fr * 128 + (((4 + kq) ^ (fr >> 1)) * 16);
    const int offB = 32768 + h * 8192 + (fr >> 3) * 4096 + kq * 128 + (fr & 7) * 16;

    f32x4 acc[4][4];
#pragma unroll
    for (int m = 0; m < 4; ++m)
#pragma unroll
        for (int n = 0; n < 4; ++n) acc[m][n] = (f32x4){0.f, 0.f, 0.f, 0.f};

    int issued = 0, islot = 0;
    unsigned rdy = 0, cnt = 0;
    auto issue = [&]() {
        const char* a = ((issued < 16) ? ga_own + (size_t)issued * (32 * 32768) : ga + (size_t)issued * (16 * 32768)) + lo_a;
        const char* b = ((issued < 16) ? gb_own + (size_t)issued * (64 * 16384) : gb + (size_t)issued * (16 * 16384)) + lo_b;
        char* la = lds + islot * STAGE_BYTES + wv * 4096;
        char* lb = lds + islot * STAGE_BYTES + 32768 + wv * 2048;
        __builtin_amdgcn_global_load_lds((const GAS void*)a, (LAS void*)la, 16, 0, AUX_AGT);
        __builtin_amdgcn_global_load_lds((const GAS void*)a, (LAS void*)la, 16, 1024, AUX_AGT);
        __builtin_amdgcn_global_load_lds((const GAS void*)a, (LAS void*)la, 16, 2048, AUX_AGT);
        __builtin_amdgcn_global_load_lds((const GAS void*)a, (LAS void*)la, 16, 3072, AUX_AGT);
        __builtin_amdgcn_global_load_lds((const GAS void*)b, (LAS void*)lb, 16, 0, AUX_AGT);
        __builtin_amdgcn_global_load_lds((const GAS void*)b, (LAS void*)lb, 16, 1024, AUX_AGT);
        ++issued;
        islot = (islot == 2) ? 0 : islot + 1;
    };
    auto poll = [&]() {
        u32x2 v;
        asm volatile("ds_read_b64 %0, %1\n\ts_waitcnt lgkmcnt(0)" : "=v"(v) : "v"(ctl) : "memory");
        cnt = __builtin_amdgcn_readfirstlane(v[0]);
        rdy = __builtin_amdgcn_readfirstlane(v[1]);
    };
    auto arrive = [&]() {
        if (lane == 0) asm volatile("ds_add_u32 %0, %1" :: "v"(ctl), "v"(1u) : "memory");
    };
    auto wait_bar = [&](unsigned target) {
        poll();
        while (cnt < target) { __builtin_amdgcn_s_sleep(1); poll(); }
    };
    auto need = [&](int upto) {
        while (issued < upto) {
            while ((int)rdy <= issued) { __builtin_amdgcn_s_sleep(2); poll(); }
            issue();
        }
    };

    need(1);
    if ((int)rdy > issued) issue();
    if (issued == 2) asm volatile("s_waitcnt vmcnt(6)" ::: "memory"); else wait_vm0();
    arrive();
    wait_bar(8);

    bf16x8 a0[4], b0[4], a1[4], b1[4];
    int cslot = 0;
    {
        const char* s = lds;
#pragma unroll
        for (int m = 0; m < 4; ++m) a0[m] = *(const bf16x8*)(s + offA0 + m * 2048);
#pragma unroll
        for (int n = 0; n < 4; ++n) b0[n] = *(const bf16x8*)(s + offB + n * 1024);
    }
#pragma unroll 1
    for (int t = 0; t < NPOS; ++t) {
        const char* s = lds + cslot * STAGE_BYTES;
        cslot = (cslot == 2) ? 0 : cslot + 1;
        const char* sn = lds + cslot * STAGE_BYTES;
        {
            const int lim = (t + 3 < NPOS) ? t + 3 : NPOS;
            if (issued < lim && issued < (int)rdy) issue();
            if (issued < lim && issued < (int)rdy) issue();
        }
#pragma unroll
        for (int m = 0; m < 4; ++m) a1[m] = *(const bf16x8*)(s + offA1 + m * 2048);
#pragma unroll
        for (int n = 0; n < 4; ++n) b1[n] = *(const bf16x8*)(s + offB + 512 + n * 1024);
        __builtin_amdgcn_s_setprio(1);
#pragma unroll
        for (int m = 0; m < 4; ++m)
#pragma unroll
            for (int n = 0; n < 4; ++n) acc[m][n] = __builtin_amdgcn_mfma_f32_16x16x32_bf16(a0[m], b0[n], acc[m][n], 0, 0, 0);
        __builtin_amdgcn_s_setprio(0);
        __builtin_amdgcn_sched_barrier(0);
        if (t + 1 < NPOS) {
            need(t + 2);
            if (issued == t + 3) asm volatile("s_waitcnt vmcnt(6) lgkmcnt(0)" ::: "memory");
            else asm volatile("s_waitcnt vmcnt(0) lgkmcnt(0)" ::: "memory");
        } else {
            asm volatile("s_waitcnt lgkmcnt(0)" ::: "memory");
        }
        arrive();
        __builtin_amdgcn_sched_barrier(0);
        __builtin_amdgcn_s_setprio(1);
#pragma unroll
        for (int m = 0; m < 2; ++m)
#pragma unroll
            for (int n = 0; n < 4; ++n) acc[m][n] = __builtin_amdgcn_mfma_f32_16x16x32_bf16(a1[m], b1[n], acc[m][n], 0, 0, 0);
        __builtin_amdgcn_s_setprio(0);
        __builtin_amdgcn_sched_barrier(0);
        wait_bar(8u * (t + 2));
        if (t + 1 < NPOS) {
#pragma unroll
            for (int m = 0; m < 4; ++m) a0[m] = *(const bf16x8*)(sn + offA0 + m * 2048);
#pragma unroll
            for (int n = 0; n < 4; ++n) b0[n] = *(const bf16x8*)(sn + offB + n * 1024);
        }
        __builtin_amdgcn_s_setprio(1);
#pragma unroll
        for (int m = 2; m < 4; ++m)
#pragma unroll
            for (int n = 0; n < 4; ++n) acc[m][n] = __builtin_amdgcn_mfma_f32_16x16x32_bf16(a1[m], b1[n], acc[m][n], 0, 0, 0);
        __builtin_amdgcn_s_setprio(0);
        __builtin_amdgcn_sched_barrier(0);
    }
    if (wv == 0) STAMP(p, blockIdx.x, 3, lane);

    char* wl = lds + wv * WAVE_LDS;
    unsigned short* vals = (unsigned short*)wl;
    const bool bad = timed_out(cx);
    const int seg = ((ti * 16 + tj) << 3) + wv;
    u32x2 mk = {0u, 0u};
    u32x2 ov[16];
    unsigned base = 0;
#pragma unroll
    for (int i = 0; i < 16; ++i) {
        unsigned e[4];
#pragma unroll
        for (int f4 = 0; f4 < 4; ++f4) {
            const float v = acc[i >> 2][f4][i & 3];
            unsigned b = bf16_rne(v > 0.f ? v : 0.f);
            if (bad) b = 0x7FC0u;
            e[f4] = b;
            const u64 m = __ballot(b != 0u);
            const unsigned idx = __builtin_amdgcn_mbcnt_hi((unsigned)(m >> 32), __builtin_amdgcn_mbcnt_lo((unsigned)m, 0)) + base;
            if (b != 0u) vals[idx] = (unsigned short)b;
            base += __builtin_popcountll(m);
            if (lane == i * 4 + f4) { mk[0] = (unsigned)m; mk[1] = (unsigned)(m >> 32); }
        }
        ov[i] = (u32x2){e[0] | (e[1] << 16), e[2] | (e[3] << 16)};
    }
    __builtin_amdgcn_fence(__ATOMIC_RELEASE, "wavefront");
    __builtin_amdgcn_wave_barrier();
    {
        const unsigned vbytes = (base * 2 + 63) & ~63u;
        const __amdgpu_buffer_rsrc_t rs = mk_rsrc(ws + WS_OSTG + (size_t)seg * SEG_BYTES, 512 + vbytes);
        __builtin_amdgcn_raw_buffer_store_b64(mk, rs, lane * 8, 0, AUX_SYS);
#pragma unroll
        for (int j = 0; j < 8; ++j)
            if (j * 1024u < vbytes)
                __builtin_amdgcn_raw_buffer_store_b128(*(const u32x4*)(wl + lane * 16 + j * 1024), rs, 512 + lane * 16 + j * 1024, 0, AUX_SYS);
        wait_vm0();
        if (lane == 0)
            st_sys((unsigned*)(ws + WS_SEGRDY) + seg, (cx.call << 13) | base);
    }
    {
        const int row0 = (p.rank >> 2) * 4096 + ti * 256 + q * 64 + (lane >> 4) * 4;
        const int col0 = (p.rank & 3) * 2048 + tj * 128 + h * 64 + (lane & 7) * 8 + ((lane >> 3) & 1) * 4;
        char* o = p.out + ((size_t)row0 * NG + col0) * 2;
#pragma unroll
        for (int i = 0; i < 16; ++i) *(u32x2*)(o + (size_t)((i >> 2) * 16 + (i & 3)) * (NG * 2)) = ov[i];
    }
    __builtin_amdgcn_fence(__ATOMIC_RELEASE, "wavefront");
    __builtin_amdgcn_wave_barrier();
    if (wv == 0) STAMP(p, blockIdx.x, 4, lane);
#pragma unroll 1
    for (int i = 0; i < 7; ++i) {
        const int e = seg + NSEG * i;
        expand_segment(p, cx, (p.rank + 1 + e % 7) & 7, e / 7, lane, wl);
        if (wv == 0 && i == 0) STAMP(p, blockIdx.x, 9, lane);
    }
    if (wv == 7) STAMP(p, blockIdx.x, 5, lane);
}

}

__global__ void __launch_bounds__(NTHR) fused_gemm_ar(Params p) {
    extern __shared__ __attribute__((aligned(16))) char lds[];
    const int lane = threadIdx.x & 63;
    const int wave = __builtin_amdgcn_readfirstlane(threadIdx.x >> 6);
    const int wg = blockIdx.x;
    volatile unsigned* ctl = (volatile unsigned*)(lds + LDS_CTL);
    char* ws = p.ws[p.rank];
    if (threadIdx.x == 0) {
        const unsigned ticket = __hip_atomic_fetch_add((unsigned*)(ws + WS_TICKET), 1u, __ATOMIC_RELAXED, __HIP_MEMORY_SCOPE_AGENT);
        ctl[0] = 0u; ctl[1] = 0u; ctl[2] = ticket / NWG + 1u;
    }
    __syncthreads();
    Ctx cx;
    cx.call = __builtin_amdgcn_readfirstlane(ctl[2]);
    cx.tmo = (unsigned*)(ws + WS_TMO);
    cx.t0 = __builtin_amdgcn_s_memrealtime();
    const int xcd = wg & 7, idx = wg >> 3;
    const int ti = (xcd >> 1) * 4 + (idx & 3), tj = (xcd & 1) * 8 + (idx >> 2);
    if (wave < 8) role_gemm(p, cx, ti, tj, wave, lane, lds);
    else if (wave == 8) role_poller(p, cx, ti, tj, lane, lds);
    else role_worker(p, cx, wg, wave - 9, lane);
}

extern "C" void kernel_launch(int rank, int nranks, ncclComm_t comm, hipStream_t stream,
                              void* const* d_in, const int* in_sizes, int n_in,
                              void* d_out, int out_size,
                              const int* in_dtypes, int out_dtype,
                              void* const* d_out_all, void* const* d_ws_all, size_t ws_bytes) {
    if (nranks != NR || n_in < 2 || ws_bytes < WS_END || (unsigned)rank >= (unsigned)NR) {
        fprintf(stderr, "unsupported configuration\n");
        return;
    }
    static bool attr_set[NR] = {};
    if (!attr_set[rank]) {
        HIP_CHECK(hipFuncSetAttribute((const void*)fused_gemm_ar, hipFuncAttributeMaxDynamicSharedMemorySize, LDS_BYTES));
        attr_set[rank] = true;
    }
    Params p;
    p.x = (const char*)d_in[0];
    p.w = (const char*)d_in[1];
    p.out = (char*)d_out;
    for (int i = 0; i < NR; ++i) p.ws[i] = (char*)d_ws_all[i];
    p.rank = rank;
    fused_gemm_ar<<<NWG, NTHR, LDS_BYTES, stream>>>(p);
    HIP_CHECK(hipGetLastError());
}
```

```cpp
#include <hip/hip_runtime.h>
#include <rccl/rccl.h>
#include <cstdio>
#include <cstdint>

#define HIP_CHECK(expr) do { hipError_t _e = (expr); if (_e != hipSuccess) { \
    fprintf(stderr, "HIP error %s at %s:%d\n", hipGetErrorString(_e), __FILE__, __LINE__); \
    return; } } while (0)

typedef short bf16x8 __attribute__((ext_vector_type(8)));
typedef float f32x4 __attribute__((ext_vector_type(4)));
typedef unsigned u32x4 __attribute__((ext_vector_type(4)));
typedef unsigned u32x2 __attribute__((ext_vector_type(2)));
typedef unsigned short us2 __attribute__((ext_vector_type(2)));
typedef unsigned long long u64;
#define LAS __attribute__((address_space(3)))
#define GAS __attribute__((address_space(1)))

namespace {

constexpr int NR = 8;
constexpr int NG = 8192;
constexpr int KSH = 1024;
constexpr int NPOS = 128;
constexpr int NWG = 256;
constexpr int NTHR = 768;
#define CFG_LAG 6
#define CFG_NPI 24
#define CFG_NPO 10
constexpr int LAG = CFG_LAG;
constexpr int NPI = CFG_NPI;
constexpr int NPO = CFG_NPO;
static_assert(NPI <= 34 && NPO <= 18, "comm waves: two per CU of group 1, one per CU of group 0");
constexpr int U_IN = 1536;
constexpr int U_OUT = 1024;
#define CFG_EARLY 0
constexpr int NSEG = 2048;
constexpr int SEG_BYTES = 512 + 8192;
constexpr int CU_BYTES = 6144;
constexpr u64 DEADLINE_TICKS = 50000000ull;

constexpr size_t WS_TICKET = 0;
constexpr size_t WS_TMO = 128;
constexpr size_t WS_LINKCTR = 256;
constexpr size_t WS_PKCNT = 4096;
constexpr size_t WS_XF = 8192;
constexpr size_t WS_WF = WS_XF + 16 * 128 * 16;
constexpr size_t WS_SEGRDY = WS_WF + 16 * 128 * 8;
constexpr size_t WS_RF = WS_SEGRDY + NSEG * 4;
constexpr size_t WS_MPK = WS_RF + NR * NSEG * 4;
constexpr size_t WS_MSEG = WS_MPK + 1024;
constexpr size_t WS_HDR = WS_MSEG + 8 * NSEG * 4;
constexpr size_t WS_MHDR = WS_HDR + 16 * 256 * 64;
constexpr size_t WS_DBG = WS_MHDR + 7 * 16 * 96 * 64;
constexpr size_t WS_CTL_END = 2u << 20;
constexpr size_t WS_PKX = WS_CTL_END;
constexpr size_t WS_PKW = WS_PKX + (16u << 20);
constexpr size_t WS_CX = WS_PKW + (16u << 20);
constexpr size_t WS_CW = WS_CX + (12u << 20);
constexpr size_t WS_XA = WS_CW + (12u << 20);
constexpr size_t WS_WB = WS_XA + (size_t)NPOS * 16 * 32768;
constexpr size_t WS_OSTG = WS_WB + (size_t)NPOS * 16 * 16384;
constexpr size_t WS_ORECV = WS_OSTG + (size_t)NSEG * SEG_BYTES;
constexpr size_t WS_END = WS_ORECV + (size_t)NR * NSEG * SEG_BYTES;
static_assert(WS_DBG + 64 * 1024 <= WS_CTL_END, "control region");
static_assert(WS_END < 0x7FFF0000u, "32-bit workspace offsets");

constexpr int STAGE_BYTES = 49152;
constexpr int LDS_CTL = 3 * STAGE_BYTES;
constexpr int LDS_BYTES = LDS_CTL + 64;
constexpr int WAVE_LDS = 9216;

struct Params {
    const char* x;
    const char* w;
    char* out;
    char* ws[NR];
    int rank;
};

constexpr int AUX_SYS = 17;
constexpr int AUX_AGT = 16;
constexpr unsigned OOB = 0x7FFFFF00u;

__device__ __forceinline__ __amdgpu_buffer_rsrc_t mk_rsrc(const void* p, unsigned bytes) {
    return __builtin_amdgcn_make_buffer_rsrc((void*)p, 0, bytes, 0x00020000);
}
__device__ __forceinline__ void st_sys(unsigned* p, unsigned v) {
    __hip_atomic_store(p, v, __ATOMIC_RELAXED, __HIP_MEMORY_SCOPE_SYSTEM);
}
__device__ __forceinline__ void st_agt(unsigned* p, unsigned v) {
    __hip_atomic_store(p, v, __ATOMIC_RELAXED, __HIP_MEMORY_SCOPE_AGENT);
}
__device__ __forceinline__ void wait_vm0() { asm volatile("s_waitcnt vmcnt(0)" ::: "memory"); __builtin_amdgcn_s_waitcnt(0x0F70); }
__device__ __forceinline__ void wait_vm4() { asm volatile("s_waitcnt vmcnt(4)" ::: "memory"); __builtin_amdgcn_s_waitcnt(0x0F74); }
__device__ __forceinline__ void wait_vm8() { asm volatile("s_waitcnt vmcnt(8)" ::: "memory"); __builtin_amdgcn_s_waitcnt(0x0F78); }
__device__ __forceinline__ unsigned mbcnt64(u64 m) {
    return __builtin_amdgcn_mbcnt_hi((unsigned)(m >> 32), __builtin_amdgcn_mbcnt_lo((unsigned)m, 0));
}
__device__ __forceinline__ unsigned pk_max(unsigned a, unsigned b) {
    return __builtin_bit_cast(unsigned, __builtin_elementwise_max(__builtin_bit_cast(us2, a), __builtin_bit_cast(us2, b)));
}
__device__ __forceinline__ unsigned pk_sub(unsigned a, unsigned b) {
    return __builtin_bit_cast(unsigned, (us2)(__builtin_bit_cast(us2, a) - __builtin_bit_cast(us2, b)));
}
__device__ __forceinline__ unsigned pk_sub_sat(unsigned a, unsigned b) {
    return __builtin_bit_cast(unsigned, __builtin_elementwise_sub_sat(__builtin_bit_cast(us2, a), __builtin_bit_cast(us2, b)));
}
__device__ __forceinline__ unsigned wave_pk_max(unsigned v) {
#pragma unroll
    for (int o = 32; o > 0; o >>= 1) v = pk_max(v, (unsigned)__shfl_xor((int)v, o, 64));
    return __builtin_amdgcn_readfirstlane(v);
}

struct Ctx {
    unsigned call;
    unsigned* tmo;
    u64 t0;
};
__device__ __forceinline__ bool give_up(const Ctx& c) {
    if (__hip_atomic_load(c.tmo, __ATOMIC_RELAXED, __HIP_MEMORY_SCOPE_AGENT) == c.call) return true;
    if (__builtin_amdgcn_s_memrealtime() - c.t0 > DEADLINE_TICKS) {
        __hip_atomic_store(c.tmo, c.call, __ATOMIC_RELAXED, __HIP_MEMORY_SCOPE_AGENT);
        return true;
    }
    return false;
}
__device__ __forceinline__ bool timed_out(const Ctx& c) {
    return __hip_atomic_load(c.tmo, __ATOMIC_RELAXED, __HIP_MEMORY_SCOPE_AGENT) == c.call;
}

#define STAMP(p, wg, k, lane) do {} while (0)

__device__ __forceinline__ int seq_slot(int seq, bool g1) {
    if (g1) {
        if (seq < 16) return seq * 8;
        const int m = seq - 16;
        return (m / 7) * 8 + m % 7 + 1;
    }
    if (seq < 8) return seq * 8;
    const int m = seq - 8, pr = m / 15, r = m % 15;
    return r < 7 ? pr * 16 + r + 1 : (r == 7 ? (8 + pr) * 8 : pr * 16 + 8 + r - 7);
}

__device__ __forceinline__ unsigned seg_tag(unsigned call) { return call & 2047u; }
__device__ __forceinline__ unsigned seg_len(unsigned f) {
    const unsigned cnt = f & 8191u, lb = (cnt + 15u) & ~15u;
    return (512u + ((f >> 20) & 1u ? 2u * lb : lb + (lb >> 1)) + 63u) & ~63u;
}

__device__ __forceinline__ void role_packer(const Params& p, const Ctx& cx, int h, int lane) {
    const int rank = p.rank;
    char* ws = p.ws[rank];
    unsigned* pend_cnt = nullptr;
    unsigned* pend_flag = nullptr;
#pragma unroll 1
    for (int i = 0; i < 8; ++i) {
        const int j = h + 512 * i, c = j >> 8, r = j & 255;
        u32x4 v[8];
        unsigned* flag = nullptr;
        const int un = r & 127;
        if (r < 128) {
            const char* src = p.w + (size_t)(c * 64 + (lane >> 3) * 8) * (NG * 2) + (size_t)(un * 64 + (lane & 7) * 8) * 2;
#pragma unroll
            for (int k = 0; k < 8; ++k) v[k] = *(const u32x4*)(src + (size_t)k * (NG * 2));
            if ((un >> 5) == (rank & 3)) flag = (unsigned*)(ws + WS_WF) + (((un >> 1) & 15) * NPOS + c * 8) * 2 + (un & 1);
        } else {
            const int rr = lane >> 3, pp = lane & 7;
            const char* src = p.x + (size_t)(un * 64 + rr) * (KSH * 2) + c * 128;
            const int o0 = (pp ^ (rr >> 1)) * 16, o1 = (pp ^ (4 + (rr >> 1))) * 16;
#pragma unroll
            for (int k = 0; k < 8; ++k) v[k] = *(const u32x4*)(src + (size_t)k * 8 * (KSH * 2) + ((k & 1) ? o1 : o0));
            if ((un >> 6) == (rank >> 2)) flag = (unsigned*)(ws + WS_XF) + (((un >> 2) & 15) * NPOS + c * 8) * 4 + (un & 3);
        }
        wait_vm0();
        if (pend_cnt && lane == 0) {
            __hip_atomic_fetch_add(pend_cnt, 1u, __ATOMIC_RELAXED, __HIP_MEMORY_SCOPE_SYSTEM);
            if (pend_flag) st_agt(pend_flag, cx.call);
        }
        pend_cnt = (unsigned*)(ws + WS_PKCNT + c * 256);
        pend_flag = flag;
        if (r < 128) {
            u32x4 o[8];
#pragma unroll
            for (int cc = 0; cc < 8; ++cc)
#pragma unroll
                for (int d = 0; d < 4; ++d)
                    o[cc][d] = __builtin_amdgcn_perm(v[2 * d + 1][cc >> 1], v[2 * d][cc >> 1], (cc & 1) ? 0x07060302u : 0x05040100u);
#pragma unroll
            for (int cc = 0; cc < 8; ++cc) v[cc] = o[cc];
        }
        unsigned mx = 0, mn = 0;
#pragma unroll
        for (int k = 0; k < 8; ++k)
#pragma unroll
            for (int d = 0; d < 4; ++d) {
                const unsigned t = v[k][d] & 0x7F007F00u;
                mx = pk_max(mx, t);
                mn = pk_max(mn, pk_sub(0u, t));
            }
        const unsigned red = wave_pk_max((max(mx >> 16, mx & 0xFFFFu) << 16) | max(mn >> 16, mn & 0xFFFFu));
        const unsigned top = red >> 24;
        const unsigned base = top > 7u ? top - 7u : 0u;
        const unsigned lowest = ((0x10000u - (red & 0xFFFFu)) >> 8);
        const unsigned base2 = base * 0x01000100u;
        unsigned nesc = 0;
        bool raw = false;
        const __amdgpu_buffer_rsrc_t rh = mk_rsrc(ws + WS_HDR + (size_t)(c * 256 + r) * 64, 64);
        if (lowest <= base) {
            unsigned r0 = 0, r1 = 0, n = 0;
#pragma unroll
            for (int k = 0; k < 8; ++k)
#pragma unroll
                for (int e = 0; e < 8; ++e) {
                    const unsigned hw = (e & 1) ? (v[k][e >> 1] >> 16) : (v[k][e >> 1] & 0xFFFFu);
                    const unsigned u7 = (hw >> 8) & 0x7Fu;
                    if (u7 - 1u < base) { r1 = r0; r0 = (((unsigned)(k << 9 | e) | (unsigned)(lane << 3)) << 16) | hw; ++n; }
                }
            const u64 m1 = __ballot(n >= 1), m2 = __ballot(n >= 2);
            nesc = __builtin_popcountll(m1) + __builtin_popcountll(m2);
            raw = __ballot(n > 2) != 0ull || nesc > 15u;
            if (!raw) {
                __builtin_amdgcn_raw_buffer_store_b32(r0, rh, n >= 1 ? 4 + mbcnt64(m1) * 4 : OOB, 0, AUX_SYS);
                __builtin_amdgcn_raw_buffer_store_b32(r1, rh, n >= 2 ? 4 + ((unsigned)__builtin_popcountll(m1) + mbcnt64(m2)) * 4 : OOB, 0, AUX_SYS);
            }
        }
        __builtin_amdgcn_raw_buffer_store_b32(base | ((nesc & 15u) << 8) | (raw ? 0x8000u : 0u), rh, lane == 0 ? 0u : OOB, 0, AUX_SYS);
        if (raw || flag) {
            const __amdgpu_buffer_rsrc_t rs = mk_rsrc(ws + (r < 128 ? WS_PKW : WS_PKX) + (size_t)(c * 128 + un) * 8192, 8192);
#pragma unroll
            for (int k = 0; k < 8; ++k) __builtin_amdgcn_raw_buffer_store_b128(v[k], rs, lane * 16 + k * 1024, 0, AUX_SYS);
        }
        if (!raw) {
            const __amdgpu_buffer_rsrc_t rc = mk_rsrc(ws + (r < 128 ? WS_CW : WS_CX) + (size_t)(c * 128 + un) * CU_BYTES, CU_BYTES);
#pragma unroll
            for (int hf = 0; hf < 2; ++hf) {
                u32x4 L[2], N;
#pragma unroll
                for (int jj = 0; jj < 4; ++jj) {
                    const u32x4 d = v[hf * 4 + jj];
                    unsigned nb[4];
#pragma unroll
                    for (int q = 0; q < 4; ++q) nb[q] = pk_sub_sat(d[q] & 0x7F007F00u, base2) | ((d[q] >> 4) & 0x08000800u);
                    L[jj >> 1][(jj & 1) * 2] = __builtin_amdgcn_perm(d[1], d[0], 0x06040200u);
                    L[jj >> 1][(jj & 1) * 2 + 1] = __builtin_amdgcn_perm(d[3], d[2], 0x06040200u);
                    N[jj] = __builtin_amdgcn_perm(nb[1], nb[0], 0x07050301u) | (__builtin_amdgcn_perm(nb[3], nb[2], 0x07050301u) << 4);
                }
                __builtin_amdgcn_raw_buffer_store_b128(L[0], rc, lane * 16 + hf * 3072, 0, AUX_SYS);
                __builtin_amdgcn_raw_buffer_store_b128(L[1], rc, lane * 16 + hf * 3072 + 1024, 0, AUX_SYS);
                __builtin_amdgcn_raw_buffer_store_b128(N, rc, lane * 16 + hf * 3072 + 2048, 0, AUX_SYS);
            }
        }
    }
    wait_vm0();
    if (lane == 0) {
        __hip_atomic_fetch_add(pend_cnt, 1u, __ATOMIC_RELAXED, __HIP_MEMORY_SCOPE_SYSTEM);
        if (pend_flag) st_agt(pend_flag, cx.call);
    }
}

#define UNI(x) __builtin_amdgcn_readfirstlane(x)
#define LD_REMOTE(off, imm) __builtin_amdgcn_raw_buffer_load_b128(rrem, (off) + (imm), 0, AUX_SYS)
#define ST_LOCAL(v, off, imm) __builtin_amdgcn_raw_buffer_store_b128(v, rloc, (off) + (imm), 0, AUX_AGT)

struct InUnit { unsigned idx, dst, flag, info, c; };

__device__ __forceinline__ void decode_half(const u32x4 l0, const u32x4 l1, const u32x4 nn, unsigned hdr, unsigned h0, unsigned k,
                                            int lane, u32x4 (&o)[4]) {
    const unsigned bs = (h0 & 0x7Fu) * 0x01010101u;
    const unsigned tlo = (bs + 0x03020100u) & 0xFFFFFF00u, thi = bs + 0x07060504u;
    unsigned H[8];
#pragma unroll
    for (int jj = 0; jj < 4; ++jj) {
        const unsigned n = nn[jj];
        H[2 * jj] = __builtin_amdgcn_perm(thi, tlo, n & 0x07070707u) | ((n << 4) & 0x80808080u);
        H[2 * jj + 1] = __builtin_amdgcn_perm(thi, tlo, (n >> 4) & 0x07070707u) | (n & 0x80808080u);
    }
    const int nesc = (h0 >> 8) & 15;
#pragma unroll 1
    for (int i = 1; i <= nesc; ++i) {
        const unsigned e = __builtin_amdgcn_readlane(hdr, i);
        if (((e >> 27) & 1u) != k) continue;
        const unsigned key = ((unsigned)lane == ((e >> 19) & 63u)) ? ((e >> 25) & 3u) * 2 + ((e >> 18) & 1u) : 8u;
        const unsigned sh = ((e >> 16) & 3u) * 8, msk = 0xFFu << sh, val = ((e >> 8) & 0xFFu) << sh;
#pragma unroll
        for (int q = 0; q < 8; ++q) H[q] = (key == (unsigned)q) ? ((H[q] & ~msk) | val) : H[q];
    }
#pragma unroll
    for (int jj = 0; jj < 4; ++jj) {
        const unsigned la = (jj >> 1 ? l1 : l0)[(jj & 1) * 2], lb = (jj >> 1 ? l1 : l0)[(jj & 1) * 2 + 1];
        o[jj][0] = __builtin_amdgcn_perm(H[2 * jj], la, 0x05010400u);
        o[jj][1] = __builtin_amdgcn_perm(H[2 * jj], la, 0x07030602u);
        o[jj][2] = __builtin_amdgcn_perm(H[2 * jj + 1], lb, 0x05010400u);
        o[jj][3] = __builtin_amdgcn_perm(H[2 * jj + 1], lb, 0x07030602u);
    }
}

__device__ __forceinline__ void role_pull_in(const Params& p, const Ctx& cx, int link, int lane) {
    const int rank = p.rank;
    const __amdgpu_buffer_rsrc_t rloc = mk_rsrc(p.ws[rank], (unsigned)WS_END);
    const __amdgpu_buffer_rsrc_t rrem = mk_rsrc(p.ws[(rank + 1 + link) & 7], (unsigned)WS_END);
    const unsigned ctr_off = (lane == 0) ? (unsigned)WS_LINKCTR + link * 128 : OOB;
    const unsigned base = (cx.call - 1) * (unsigned)(U_IN + NPI);
    const unsigned l16 = lane * 16;
    unsigned packed = 0;

    auto ticket = [&]() -> unsigned { return (unsigned)__builtin_amdgcn_raw_ptr_buffer_atomic_add_i32(1, rloc, ctr_off, 0, AUX_AGT); };
    auto make = [&](unsigned tv) -> InUnit {
        const int n = (int)(UNI(tv) - base);
        InUnit u;
        int d, r;
        if (n < 64 * LAG) { d = n >> 6; r = n & 63; }
        else if (n < 64 * LAG + 96 * (16 - LAG)) { const int m = n - 64 * LAG; d = LAG + m / 96; r = m % 96; }
        else { const int m = n - 64 * LAG - 96 * (16 - LAG); d = 16 + (m >> 5); r = 64 + (m & 31); }
        const int c = (r >= 64) ? d - LAG : d;
        const int pos = c * 8 + 1 + link;
        const bool isw = r < 32;
        const int vv = isw ? r : r - 32;
        u.c = (n >= U_IN) ? 16u : (unsigned)c;
        u.info = (unsigned)WS_MHDR + ((link * 16 + c) * 96 + r) * 64;
        u.idx = isw ? (unsigned)(c * 128 + (rank & 3) * 32 + vv) : (unsigned)(c * 128 + (rank >> 2) * 64 + vv) | 0x80000000u;
        u.dst = (isw ? (unsigned)WS_WB + pos * (16 * 16384) : (unsigned)WS_XA + pos * (16 * 32768)) + vv * 8192;
        u.flag = isw ? (unsigned)WS_WF + ((vv >> 1) * NPOS + pos) * 8 + (vv & 1) * 4
                     : (unsigned)WS_XF + ((vv >> 2) * NPOS + pos) * 16 + (vv & 3) * 4;
        return u;
    };
    auto load_hdr = [&](const InUnit& u) -> unsigned {
        return __builtin_amdgcn_raw_buffer_load_b32(rloc, (lane < 16 && ((packed >> u.c) & 1u)) ? u.info + lane * 4 : OOB, 0, AUX_AGT);
    };
    auto publish = [&](unsigned flag) {
        __builtin_amdgcn_raw_buffer_store_b32(cx.call, rloc, lane == 0 ? flag : OOB, 0, AUX_AGT);
    };
    auto store_half = [&](u32x4 (&C)[4], unsigned hdr, unsigned h0, unsigned dst, unsigned k) {
        if (!(h0 & 0x8000u)) {
            u32x4 o[4];
            decode_half(C[0], C[1], C[2], hdr, h0, k, lane, o);
#pragma unroll
            for (int j = 0; j < 4; ++j) C[j] = o[j];
        }
        const unsigned a0 = dst + l16 + k * 4096u;
#pragma unroll
        for (int j = 0; j < 4; ++j) ST_LOCAL(C[j], a0, j * 1024);
    };
    auto src_of = [&](const InUnit& u, unsigned h0) -> unsigned {
        const bool x = (u.idx >> 31) != 0u;
        const unsigned ix = u.idx & 0x7FFFFFFFu;
        return ((h0 & 0x8000u) ? (x ? (unsigned)WS_PKX : (unsigned)WS_PKW) + ix * 8192u
                               : (x ? (unsigned)WS_CX : (unsigned)WS_CW) + ix * (unsigned)CU_BYTES) + l16;
    };
    auto fetch = [&](u32x4 (&N)[4], unsigned so, bool raw) {
        N[0] = LD_REMOTE(so, 0); N[1] = LD_REMOTE(so, 1024); N[2] = LD_REMOTE(so, 2048); N[3] = LD_REMOTE(raw ? so : OOB, 3072);
    };
    auto wait_hdr = [&](const InUnit& u) -> unsigned {
        for (;;) {
            const unsigned f = __builtin_amdgcn_raw_buffer_load_b32(rloc, lane < 16 ? (unsigned)WS_MPK + link * 128 + lane * 4 : OOB, 0, AUX_AGT);
            packed |= (unsigned)__ballot(lane < 16 && f == cx.call);
            if ((packed >> u.c) & 1u) break;
            if (give_up(cx)) { packed = 0xFFFFu; break; }
            __builtin_amdgcn_s_sleep(4);
        }
        return load_hdr(u);
    };

    InUnit u = make(ticket());
    if (u.c == 16u) return;
    unsigned t = ticket();
    unsigned hdr = wait_hdr(u);
    unsigned h0 = UNI(hdr);
    unsigned so = src_of(u, h0);
    u32x4 A[4], B[4];
    fetch(A, so, (h0 & 0x8000u) != 0u);
    wait_vm0();
    unsigned pflag = OOB;
    for (;;) {
        asm volatile("" : "+v"(A[0]), "+v"(A[1]), "+v"(A[2]), "+v"(A[3]), "+v"(t));
        const bool raw = (h0 & 0x8000u) != 0u;
        fetch(B, so + (raw ? 4096u : 3072u), raw);
        __builtin_amdgcn_sched_barrier(0);
        const InUnit un = make(t);
        const bool infok = ((packed >> un.c) & 1u) != 0u;
        unsigned info = load_hdr(un);
        store_half(A, hdr, h0, u.dst, 0);
        wait_vm4();
        asm volatile("" : "+v"(B[0]), "+v"(B[1]), "+v"(B[2]), "+v"(B[3]), "+v"(info));
        if (!infok) {
            publish(pflag);
            store_half(B, hdr, h0, u.dst, 1);
            wait_vm0();
            publish(u.flag);
            pflag = OOB;
            if (un.c == 16u) return;
            t = ticket();
            hdr = wait_hdr(un);
            h0 = UNI(hdr);
            so = src_of(un, h0);
            fetch(A, so, (h0 & 0x8000u) != 0u);
            wait_vm0();
        } else {
            const unsigned hn0 = UNI(info);
            so = src_of(un, hn0);
            fetch(A, so, (hn0 & 0x8000u) != 0u);
            __builtin_amdgcn_sched_barrier(0);
            publish(pflag);
            t = ticket();
            store_half(B, hdr, h0, u.dst, 1);
            wait_vm4();
            pflag = u.flag;
            hdr = info; h0 = hn0;
        }
        u = un;
    }
}

__device__ __forceinline__ void role_pull_out(const Params& p, const Ctx& cx, int link, int lane) {
    const int rank = p.rank, src = (rank + 1 + link) & 7;
    const __amdgpu_buffer_rsrc_t rloc = mk_rsrc(p.ws[rank], (unsigned)WS_END);
    const __amdgpu_buffer_rsrc_t rrem = mk_rsrc(p.ws[src], (unsigned)WS_END);
    const unsigned ctr_off = (lane == 0) ? (unsigned)WS_LINKCTR + (7 + link) * 128 : OOB;
    const unsigned base = (cx.call - 1) * (unsigned)(U_OUT + NPO);
    const unsigned tag = seg_tag(cx.call);
    const unsigned l16 = lane * 16;
    bool dead = false;
    {
        const unsigned ibase = (cx.call - 1) * (unsigned)(U_IN + NPI);
        for (;;) {
            const unsigned v = UNI(__builtin_amdgcn_raw_buffer_load_b32(rloc, lane == 0 ? (unsigned)WS_LINKCTR + link * 128 : OOB, 0, AUX_AGT));
            if ((int)(v - ibase) >= U_IN - CFG_EARLY) break;
            if (give_up(cx)) { dead = true; break; }
            __builtin_amdgcn_s_sleep(32);
        }
        STAMP(p, blockIdx.x, 1, lane);
    }
    auto ticket = [&]() -> unsigned { return (unsigned)__builtin_amdgcn_raw_ptr_buffer_atomic_add_i32(1, rloc, ctr_off, 0, AUX_AGT); };
    auto load_words = [&](unsigned n) -> unsigned {
        return __builtin_amdgcn_raw_buffer_load_b32(rloc, (lane < 2 && n < (unsigned)U_OUT) ? (unsigned)WS_MSEG + (link * NSEG + 2 * n + lane) * 4 : OOB, 0, AUX_AGT);
    };
    unsigned pn = U_OUT, qn = U_OUT, pw = 0, qw = 0;
    auto publish = [&]() {
        __builtin_amdgcn_raw_buffer_store_b32(pw, rloc, (lane < 2 && pn < (unsigned)U_OUT) ? (unsigned)WS_RF + (src * NSEG + 2 * pn + lane) * 4 : OOB, 0, AUX_AGT);
        pn = qn; pw = qw; qn = U_OUT;
    };
    auto store_unit = [&](u32x4 (&C)[8], unsigned n, unsigned w, unsigned len0, unsigned len1) {
        const unsigned d0 = (unsigned)WS_ORECV + (src * NSEG + 2 * n) * (unsigned)SEG_BYTES + l16, d1 = d0 + SEG_BYTES;
#pragma unroll
        for (int j = 0; j < 4; ++j) ST_LOCAL(C[j], l16 + j * 1024u < len0 ? d0 : OOB, j * 1024);
#pragma unroll
        for (int j = 0; j < 4; ++j) ST_LOCAL(C[4 + j], l16 + j * 1024u < len1 ? d1 : OOB, j * 1024);
        qn = n; qw = w;
    };
    auto rest = [&](unsigned sg, unsigned len) {
#pragma unroll 1
        for (unsigned o = 4096; o < len; o += 1024) {
            const bool in = o + l16 < len;
            const u32x4 v = LD_REMOTE(in ? (unsigned)WS_OSTG + sg * (unsigned)SEG_BYTES + o + l16 : OOB, 0);
            ST_LOCAL(v, in ? (unsigned)WS_ORECV + (src * NSEG + sg) * (unsigned)SEG_BYTES + o + l16 : OOB, 0);
        }
    };
    unsigned nn = UNI(ticket()) - base;
    if (nn >= (unsigned)U_OUT) return;
    unsigned t = ticket();
    unsigned info = load_words(nn), cn = U_OUT, cw = 0, cl0 = 0, cl1 = 0;
    wait_vm0();
    u32x4 A[8], B[8];
#pragma unroll
    for (int j = 0; j < 8; ++j) A[j] = (u32x4){0u, 0u, 0u, 0u};
#define OUT_STEP(C, N) do { \
        asm volatile("" : "+v"(C[0]), "+v"(C[1]), "+v"(C[2]), "+v"(C[3]), "+v"(C[4]), "+v"(C[5]), "+v"(C[6]), "+v"(C[7]), "+v"(t), "+v"(info)); \
        unsigned w0_ = UNI(info), w1_ = __builtin_amdgcn_readlane(info, 1); \
        if ((w0_ >> 21) != tag || (w1_ >> 21) != tag || nn >= (unsigned)U_OUT) {        \
            publish(); \
            store_unit(C, cn, cw, cl0, cl1); cn = U_OUT; cl0 = cl1 = 0; \
            wait_vm0(); publish(); publish(); \
            if (nn >= (unsigned)U_OUT) return; \
            for (;;) { \
                info = load_words(nn); \
                w0_ = UNI(info); w1_ = __builtin_amdgcn_readlane(info, 1); \
                if ((w0_ >> 21) == tag && (w1_ >> 21) == tag) break; \
                if (dead || give_up(cx)) { dead = true; w0_ = w1_ = 0u; info = 0u; break; } \
                __builtin_amdgcn_s_sleep(8); \
            } \
        } \
        const unsigned wv_ = info, l0_ = seg_len(w0_), l1_ = seg_len(w1_); \
        const unsigned s0_ = (unsigned)WS_OSTG + 2 * nn * (unsigned)SEG_BYTES + l16, s1_ = s0_ + SEG_BYTES; \
        _Pragma("unroll") for (int j = 0; j < 4; ++j) N[j] = LD_REMOTE(l16 + j * 1024u < l0_ ? s0_ : OOB, j * 1024); \
        _Pragma("unroll") for (int j = 0; j < 4; ++j) N[4 + j] = LD_REMOTE(l16 + j * 1024u < l1_ ? s1_ : OOB, j * 1024); \
        __builtin_amdgcn_sched_barrier(0); \
        publish(); \
        if (l0_ > 4096u) rest(2 * nn, l0_); \
        if (l1_ > 4096u) rest(2 * nn + 1, l1_); \
        const unsigned n2_ = UNI(t) - base; \
        if (n2_ < (unsigned)U_OUT) t = ticket(); \
        info = load_words(n2_); \
        store_unit(C, cn, cw, cl0, cl1); \
        wait_vm8(); \
        cn = nn; cw = wv_; cl0 = l0_; cl1 = l1_; nn = n2_; \
    } while (0)
    for (;;) {
        OUT_STEP(A, B);
        OUT_STEP(B, A);
    }
}

__device__ __forceinline__ void role_relay(const Params& p, const Ctx& cx, int link, int lane) {
    const int rank = p.rank;
    const int src = (rank + 1 + link) & 7;
    const __amdgpu_buffer_rsrc_t rctl = mk_rsrc(p.ws[rank], (unsigned)WS_CTL_END);
    const __amdgpu_buffer_rsrc_t rinfo = mk_rsrc(p.ws[src], (unsigned)WS_CTL_END);
    const unsigned pk_target = cx.call * 256u;
    const unsigned tag = seg_tag(cx.call);
    unsigned done = 0;
    while (done != 0xFFFFu) {
        const bool ask = lane < 16 && !((done >> lane) & 1u);
        const unsigned v = __builtin_amdgcn_raw_buffer_load_b32(rinfo, ask ? (unsigned)WS_PKCNT + lane * 256 : OOB, 0, AUX_SYS);
        const unsigned fresh = (unsigned)__ballot(ask && (int)(v - pk_target) >= 0);
        if (fresh == 0u) {
            if (give_up(cx)) return;
            __builtin_amdgcn_s_sleep(4);
            continue;
        }
        const int c = __builtin_ctz(fresh);
        const unsigned rest = fresh & (fresh - 1u);
        const int c2 = rest ? __builtin_ctz(rest) : c;
        u32x4 hd[6], he[6];
#pragma unroll
        for (int j = 0; j < 6; ++j) {
            const unsigned off = (j < 2) ? (unsigned)((rank & 3) * 32) * 64 + j * 1024 : (unsigned)(128 + (rank >> 2) * 64) * 64 + (j - 2) * 1024;
            hd[j] = __builtin_amdgcn_raw_buffer_load_b128(rinfo, (unsigned)WS_HDR + c * 16384 + off + lane * 16, 0, AUX_SYS);
            he[j] = __builtin_amdgcn_raw_buffer_load_b128(rinfo, rest ? (unsigned)WS_HDR + c2 * 16384 + off + lane * 16 : OOB, 0, AUX_SYS);
        }
#pragma unroll
        for (int j = 0; j < 6; ++j) {
            __builtin_amdgcn_raw_buffer_store_b128(hd[j], rctl, (unsigned)WS_MHDR + (link * 16 + c) * 6144 + j * 1024 + lane * 16, 0, AUX_AGT);
            __builtin_amdgcn_raw_buffer_store_b128(he[j], rctl, rest ? (unsigned)WS_MHDR + (link * 16 + c2) * 6144 + j * 1024 + lane * 16 : OOB, 0, AUX_AGT);
        }
        wait_vm0();
        __builtin_amdgcn_raw_buffer_store_b32(cx.call, rctl, (lane == c || lane == c2) ? (unsigned)WS_MPK + link * 128 + lane * 4 : OOB, 0, AUX_AGT);
        done |= (1u << c) | (1u << c2);
    }
    int pos = 0;
    while (pos < NSEG) {
        const unsigned q = pos + lane * 4;
        const u32x4 v = __builtin_amdgcn_raw_buffer_load_b128(rinfo, q < NSEG ? (unsigned)WS_SEGRDY + q * 4 : OOB, 0, AUX_SYS);
        const bool any = (v[0] >> 21) == tag || (v[1] >> 21) == tag || (v[2] >> 21) == tag || (v[3] >> 21) == tag;
        const bool all = (v[0] >> 21) == tag && (v[1] >> 21) == tag && (v[2] >> 21) == tag && (v[3] >> 21) == tag;
        __builtin_amdgcn_raw_buffer_store_b128(v, rctl, (any && q < NSEG) ? (unsigned)WS_MSEG + (link * NSEG + q) * 4 : OOB, 0, AUX_AGT);
        const u64 m = __ballot(all && q < NSEG);
        const int n = (m == ~0ull) ? 64 : __builtin_ctzll(~m);
        pos += n * 4;
        if (n < 64) {
            if (give_up(cx)) return;
            __builtin_amdgcn_s_sleep(32);
        }
    }
}

#define CFG_SPREAD 0
__device__ __forceinline__ void role_worker(const Params& p, const Ctx& cx, int wg, int k, int lane) {
    __builtin_amdgcn_s_setprio(3);
    const int gi = (wg >> 4) * 8 + (wg & 7);
    const bool g1 = (wg & 8) != 0;
    if (k == 0) STAMP(p, wg, 0, lane);
    const bool second = k == 1 && (CFG_SPREAD ? !g1 : g1);
    if ((g1 && k == 0) || second) {
        if (gi < 119) { if (gi / 7 + 17 * k < NPI) role_pull_in(p, cx, gi % 7, lane); }
        else if (gi >= 121 && k == 0) role_relay(p, cx, gi - 121, lane);
    } else {
        role_packer(p, cx, gi * 4 + (k == 1 ? 1 : (g1 ? 3 : k)), lane);
        if (k == 2) STAMP(p, wg, 7, lane);
        if (k == 0 && gi < NPO * 7) role_pull_out(p, cx, gi % 7, lane);
    }
    if (k == 0) STAMP(p, wg, 2, lane);
}

__device__ __forceinline__ void role_poller(const Params& p, const Ctx& cx, int ti, int tj, int lane, char* lds) {
    const char* ws = p.ws[p.rank];
    const __amdgpu_buffer_rsrc_t rx = mk_rsrc(ws + WS_XF + ti * (NPOS * 16), NPOS * 16);
    const __amdgpu_buffer_rsrc_t rw = mk_rsrc(ws + WS_WF + tj * (NPOS * 8), NPOS * 8);
    volatile LAS unsigned* ready = (volatile LAS unsigned*)(LAS char*)(lds + LDS_CTL + 4);
    int pos = 0;
    while (pos < NPOS) {
        const int sq = pos + (lane & 15);
        const int q = sq < NPOS ? seq_slot(sq, ti >= 8) : NPOS;
        bool ok = false;
        if (lane < 16) {
            const u32x4 a = __builtin_amdgcn_raw_buffer_load_b128(rx, q * 16, 0, AUX_AGT);
            const u32x2 b = __builtin_amdgcn_raw_buffer_load_b64(rw, q * 8, 0, AUX_AGT);
            ok = a[0] == cx.call && a[1] == cx.call && a[2] == cx.call && a[3] == cx.call && b[0] == cx.call && b[1] == cx.call;
        }
        const u64 m = __ballot(ok);
        const int n = __builtin_ctzll(~m);
        if (n > 0) {
            if (pos <= (ti >= 8 ? 16 : 8) && pos + n > (ti >= 8 ? 16 : 8)) STAMP(p, blockIdx.x, 8, lane);
            pos += n;
            if (lane == 0) *ready = pos;
        } else {
            if (give_up(cx)) { pos = NPOS; if (lane == 0) *ready = NPOS; break; }
            __builtin_amdgcn_s_sleep(4);
        }
    }
    STAMP(p, blockIdx.x, 6, lane);
}

__device__ __forceinline__ void expand_segment(const Params& p, const Ctx& cx, int src, int seg, int lane, char* wl) {
    const char* ws = p.ws[p.rank];
    const unsigned* rf = (const unsigned*)(ws + WS_RF) + src * NSEG + seg;
    unsigned f;
    bool dead = false;
    for (;;) {
        f = __builtin_amdgcn_readfirstlane(__hip_atomic_load(rf, __ATOMIC_RELAXED, __HIP_MEMORY_SCOPE_AGENT));
        if ((f >> 21) == seg_tag(cx.call)) break;
        if (give_up(cx)) { dead = true; break; }
        __builtin_amdgcn_s_sleep(16);
    }
    const int tile = seg >> 3, wv = seg & 7;
    const int row0 = (src >> 2) * 4096 + (tile >> 4) * 256 + (wv >> 1) * 64 + (lane >> 4) * 4;
    const int col0 = (src & 3) * 2048 + (tile & 15) * 128 + (wv & 1) * 64 + (lane & 7) * 8 + ((lane >> 3) & 1) * 4;
    char* o = p.out + ((size_t)row0 * NG + col0) * 2;
    if (dead) {
        const u32x2 nan2 = {0x7FC07FC0u, 0x7FC07FC0u};
#pragma unroll 1
        for (int i = 0; i < 16; ++i) *(u32x2*)(o + (size_t)((i >> 2) * 16 + (i & 3)) * (NG * 2)) = nan2;
        return;
    }
    const unsigned cnt = f & 8191u, lb = (cnt + 15u) & ~15u;
    const char* sp = ws + WS_ORECV + ((size_t)src * NSEG + seg) * SEG_BYTES;
    const __amdgpu_buffer_rsrc_t rs = mk_rsrc(sp, seg_len(f));
    const u32x2 mk = __builtin_amdgcn_raw_buffer_load_b64(rs, lane * 8, 0, AUX_AGT);
    if ((f >> 20) & 1u) {
        u32x4 r[8];
#pragma unroll
        for (int j = 0; j < 8; ++j) if (j * 512u < lb) r[j] = __builtin_amdgcn_raw_buffer_load_b128(rs, 512 + lane * 16 + j * 1024, 0, AUX_AGT);
#pragma unroll
        for (int j = 0; j < 8; ++j) if (j * 512u < lb) *(u32x4*)(wl + lane * 16 + j * 1024) = r[j];
    } else {
        u32x4 L[4];
        u32x2 N[4];
#pragma unroll
        for (int j = 0; j < 4; ++j)
            if (j * 1024u < lb) {
                L[j] = __builtin_amdgcn_raw_buffer_load_b128(rs, 512 + lane * 16 + j * 1024, 0, AUX_AGT);
                N[j] = __builtin_amdgcn_raw_buffer_load_b64(rs, 512 + lb + lane * 8 + j * 512, 0, AUX_AGT);
            }
        const unsigned bs = ((f >> 13) & 0x7Fu) * 0x01010101u;
#pragma unroll
        for (int j = 0; j < 4; ++j)
            if (j * 1024u < lb) {
#pragma unroll
                for (int g = 0; g < 2; ++g) {
                    const unsigned ha = (N[j][g] & 0x0F0F0F0Fu) + bs, hb = ((N[j][g] >> 4) & 0x0F0F0F0Fu) + bs;
                    u32x4 w4;
                    w4[0] = __builtin_amdgcn_perm(ha, L[j][2 * g], 0x05010400u);
                    w4[1] = __builtin_amdgcn_perm(ha, L[j][2 * g], 0x07030602u);
                    w4[2] = __builtin_amdgcn_perm(hb, L[j][2 * g + 1], 0x05010400u);
                    w4[3] = __builtin_amdgcn_perm(hb, L[j][2 * g + 1], 0x07030602u);
                    *(u32x4*)(wl + j * 2048 + lane * 32 + g * 16) = w4;
                }
            }
    }
    __builtin_amdgcn_fence(__ATOMIC_RELEASE, "wavefront");
    __builtin_amdgcn_wave_barrier();
    const unsigned short* vals = (const unsigned short*)wl;
    unsigned base = 0;
#pragma unroll
    for (int i = 0; i < 16; ++i) {
        unsigned e[4];
#pragma unroll
        for (int f4 = 0; f4 < 4; ++f4) {
            const unsigned mlo = __builtin_amdgcn_readlane(mk[0], i * 4 + f4), mhi = __builtin_amdgcn_readlane(mk[1], i * 4 + f4);
            const unsigned idx = __builtin_amdgcn_mbcnt_hi(mhi, __builtin_amdgcn_mbcnt_lo(mlo, 0)) + base;
            const u64 m = ((u64)mhi << 32) | mlo;
            const unsigned val = vals[idx];
            e[f4] = __builtin_amdgcn_inverse_ballot_w64(m) ? val : 0u;
            base += __builtin_popcountll(m);
        }
        const u32x2 ov = {e[0] | (e[1] << 16), e[2] | (e[3] << 16)};
        *(u32x2*)(o + (size_t)((i >> 2) * 16 + (i & 3)) * (NG * 2)) = ov;
    }
    __builtin_amdgcn_fence(__ATOMIC_RELEASE, "wavefront");
    __builtin_amdgcn_wave_barrier();
}

__device__ __forceinline__ unsigned bf16_rne(float f) {
    const unsigned u = __float_as_uint(f);
    return (u + 0x7FFFu + ((u >> 16) & 1u)) >> 16;
}

__device__ __forceinline__ void role_gemm(const Params& p, const Ctx& cx, int ti, int tj, int wv, int lane, char* lds) {
    const char* ws = p.ws[p.rank];
    const unsigned ldsbase = (unsigned)(uintptr_t)(LAS char*)lds;
    const unsigned ctl = ldsbase + LDS_CTL;
    const int q = wv >> 1, h = wv & 1;
    const unsigned lo_a = wv * 4096 + lane * 16, lo_b = wv * 2048 + lane * 16;
    const char* ga = ws + WS_XA + ti * 32768;
    const char* gb = ws + WS_WB + tj * 16384;
    const char* ga_own = ws + WS_PKX + (size_t)((p.rank >> 2) * 16 + ti) * 32768;
    const char* gb_own = ws + WS_PKW + (size_t)((p.rank & 3) * 16 + tj) * 16384;
    const int fr = lane & 15, kq = lane >> 4;
    const int offA0 = q * 8192 + fr * 128 + ((kq ^ (fr >> 1)) * 16);
    const int offA1 = q * 8192 + fr * 128 + (((4 + kq) ^ (fr >> 1)) * 16);
    const int offB = 32768 + h * 8192 + (fr >> 3) * 4096 + kq * 128 + (fr & 7) * 16;

    f32x4 acc[4][4];
#pragma unroll
    for (int m = 0; m < 4; ++m)
#pragma unroll
        for (int n = 0; n < 4; ++n) acc[m][n] = (f32x4){0.f, 0.f, 0.f, 0.f};

    int issued = 0, islot = 0;
    unsigned rdy = 0, cnt = 0;
    auto issue = [&]() {
        const int slot = seq_slot(issued, ti >= 8);
        const bool own = (slot & 7) == 0;
        const char* a = (own ? ga_own + (size_t)slot * (16 * 8192) : ga + (size_t)slot * (16 * 32768)) + lo_a;
        const char* b = (own ? gb_own + (size_t)slot * (16 * 8192) : gb + (size_t)slot * (16 * 16384)) + lo_b;
        char* la = lds + islot * STAGE_BYTES + wv * 4096;
        char* lb = lds + islot * STAGE_BYTES + 32768 + wv * 2048;
        __builtin_amdgcn_global_load_lds((const GAS void*)a, (LAS void*)la, 16, 0, AUX_AGT);
        __builtin_amdgcn_global_load_lds((const GAS void*)a, (LAS void*)la, 16, 1024, AUX_AGT);
        __builtin_amdgcn_global_load_lds((const GAS void*)a, (LAS void*)la, 16, 2048, AUX_AGT);
        __builtin_amdgcn_global_load_lds((const GAS void*)a, (LAS void*)la, 16, 3072, AUX_AGT);
        __builtin_amdgcn_global_load_lds((const GAS void*)b, (LAS void*)lb, 16, 0, AUX_AGT);
        __builtin_amdgcn_global_load_lds((const GAS void*)b, (LAS void*)lb, 16, 1024, AUX_AGT);
        ++issued;
        islot = (islot == 2) ? 0 : islot + 1;
    };
    auto poll = [&]() {
        u32x2 v;
        asm volatile("ds_read_b64 %0, %1\n\ts_waitcnt lgkmcnt(0)" : "=v"(v) : "v"(ctl) : "memory");
        cnt = __builtin_amdgcn_readfirstlane(v[0]);
        rdy = __builtin_amdgcn_readfirstlane(v[1]);
    };
    auto arrive = [&]() {
        if (lane == 0) asm volatile("ds_add_u32 %0, %1" :: "v"(ctl), "v"(1u) : "memory");
    };
    auto wait_bar = [&](unsigned target) {
        poll();
        while (cnt < target) { __builtin_amdgcn_s_sleep(1); poll(); }
    };
    auto need = [&](int upto) {
        while (issued < upto) {
            while ((int)rdy <= issued) { __builtin_amdgcn_s_sleep(2); poll(); }
            issue();
        }
    };

    need(1);
    if ((int)rdy > issued) issue();
    if (issued == 2) asm volatile("s_waitcnt vmcnt(6)" ::: "memory"); else wait_vm0();
    arrive();
    wait_bar(8);

    bf16x8 a0[4], b0[4], a1[4], b1[4];
    int cslot = 0;
    {
        const char* s = lds;
#pragma unroll
        for (int m = 0; m < 4; ++m) a0[m] = *(const bf16x8*)(s + offA0 + m * 2048);
#pragma unroll
        for (int n = 0; n < 4; ++n) b0[n] = *(const bf16x8*)(s + offB + n * 1024);
    }
#pragma unroll 1
    for (int t = 0; t < NPOS; ++t) {
        const char* s = lds + cslot * STAGE_BYTES;
        cslot = (cslot == 2) ? 0 : cslot + 1;
        const char* sn = lds + cslot * STAGE_BYTES;
        {
            const int lim = (t + 3 < NPOS) ? t + 3 : NPOS;
            if (issued < lim && issued < (int)rdy) issue();
            if (issued < lim && issued < (int)rdy) issue();
        }
        __builtin_amdgcn_s_setprio(1);
#pragma unroll
        for (int n = 0; n < 4; ++n) acc[0][n] = __builtin_amdgcn_mfma_f32_16x16x32_bf16(a0[0], b0[n], acc[0][n], 0, 0, 0);
        __builtin_amdgcn_sched_barrier(0);
#pragma unroll
        for (int m = 0; m < 4; ++m) a1[m] = *(const bf16x8*)(s + offA1 + m * 2048);
#pragma unroll
        for (int n = 0; n < 4; ++n) b1[n] = *(const bf16x8*)(s + offB + 512 + n * 1024);
        __builtin_amdgcn_sched_barrier(0);
#pragma unroll
        for (int m = 1; m < 4; ++m)
#pragma unroll
            for (int n = 0; n < 4; ++n) acc[m][n] = __builtin_amdgcn_mfma_f32_16x16x32_bf16(a0[m], b0[n], acc[m][n], 0, 0, 0);
        __builtin_amdgcn_s_setprio(0);
        __builtin_amdgcn_sched_barrier(0);
        if (t + 1 < NPOS) {
            need(t + 2);
            if (issued == t + 3) asm volatile("s_waitcnt vmcnt(6) lgkmcnt(0)" ::: "memory");
            else asm volatile("s_waitcnt vmcnt(0) lgkmcnt(0)" ::: "memory");
        } else {
            asm volatile("s_waitcnt lgkmcnt(0)" ::: "memory");
        }
        arrive();
        __builtin_amdgcn_sched_barrier(0);
        __builtin_amdgcn_s_setprio(1);
#pragma unroll
        for (int m = 0; m < 2; ++m)
#pragma unroll
            for (int n = 0; n < 4; ++n) acc[m][n] = __builtin_amdgcn_mfma_f32_16x16x32_bf16(a1[m], b1[n], acc[m][n], 0, 0, 0);
        __builtin_amdgcn_s_setprio(0);
        __builtin_amdgcn_sched_barrier(0);
        wait_bar(8u * (t + 2));
        if (t + 1 < NPOS) {
#pragma unroll
            for (int m = 0; m < 4; ++m) a0[m] = *(const bf16x8*)(sn + offA0 + m * 2048);
#pragma unroll
            for (int n = 0; n < 4; ++n) b0[n] = *(const bf16x8*)(sn + offB + n * 1024);
        }
        __builtin_amdgcn_s_setprio(1);
#pragma unroll
        for (int m = 2; m < 4; ++m)
#pragma unroll
            for (int n = 0; n < 4; ++n) acc[m][n] = __builtin_amdgcn_mfma_f32_16x16x32_bf16(a1[m], b1[n], acc[m][n], 0, 0, 0);
        __builtin_amdgcn_s_setprio(0);
        __builtin_amdgcn_sched_barrier(0);
    }
    if (wv == 0) STAMP(p, blockIdx.x, 3, lane);

    char* wl = lds + wv * WAVE_LDS;
    unsigned short* vals = (unsigned short*)wl;
    const bool bad = timed_out(cx);
    const int seg = ((ti * 16 + tj) << 3) + wv;
    u32x2 mk = {0u, 0u};
    u32x2 ov[16];
    unsigned base = 0, mx = 0, mn = 0;
#pragma unroll
    for (int i = 0; i < 16; ++i) {
        unsigned e[4];
#pragma unroll
        for (int f4 = 0; f4 < 4; ++f4) {
            const float v = acc[i >> 2][f4][i & 3];
            unsigned b = bf16_rne(v > 0.f ? v : 0.f);
            if (bad) b = 0x7FC0u;
            e[f4] = b;
            const u64 m = __ballot(b != 0u);
            const unsigned idx = mbcnt64(m) + base;
            if (b != 0u) vals[idx] = (unsigned short)b;
            base += __builtin_popcountll(m);
            if (lane == i * 4 + f4) { mk[0] = (unsigned)m; mk[1] = (unsigned)(m >> 32); }
        }
        ov[i] = (u32x2){e[0] | (e[1] << 16), e[2] | (e[3] << 16)};
        mx = pk_max(mx, pk_max(ov[i][0], ov[i][1]));
        mn = pk_max(mn, pk_max(pk_sub(0u, ov[i][0]), pk_sub(0u, ov[i][1])));
    }
    __builtin_amdgcn_fence(__ATOMIC_RELEASE, "wavefront");
    __builtin_amdgcn_wave_barrier();
    {
        const unsigned red = wave_pk_max((max(mx >> 16, mx & 0xFFFFu) << 16) | max(mn >> 16, mn & 0xFFFFu));
        const unsigned top = red >> 24;
        const unsigned hb = top > 15u ? top - 15u : 0u;
        const unsigned lowest = (0x10000u - (red & 0xFFFFu)) >> 8;
        const bool raw = lowest < hb;
        const unsigned f = (seg_tag(cx.call) << 21) | (raw ? 1u << 20 : 0u) | (hb << 13) | base;
        const unsigned lb = (base + 15u) & ~15u;
        const __amdgpu_buffer_rsrc_t rs = mk_rsrc(ws + WS_OSTG + (size_t)seg * SEG_BYTES, seg_len(f));
        __builtin_amdgcn_raw_buffer_store_b64(mk, rs, lane * 8, 0, AUX_SYS);
        if (raw) {
#pragma unroll
            for (int j = 0; j < 8; ++j)
                if (j * 512u < lb)
                    __builtin_amdgcn_raw_buffer_store_b128(*(const u32x4*)(wl + lane * 16 + j * 1024), rs, 512 + lane * 16 + j * 1024, 0, AUX_SYS);
        } else {
            const unsigned bs = hb * 0x01010101u;
#pragma unroll
            for (int j = 0; j < 4; ++j)
                if (j * 1024u < lb) {
                    u32x4 L;
                    u32x2 N;
#pragma unroll
                    for (int g = 0; g < 2; ++g) {
                        const u32x4 d = *(const u32x4*)(wl + j * 2048 + lane * 32 + g * 16);
                        L[2 * g] = __builtin_amdgcn_perm(d[1], d[0], 0x06040200u);
                        L[2 * g + 1] = __builtin_amdgcn_perm(d[3], d[2], 0x06040200u);
                        const unsigned ha = (__builtin_amdgcn_perm(d[1], d[0], 0x07050301u) | 0x80808080u) - bs;
                        const unsigned hc = (__builtin_amdgcn_perm(d[3], d[2], 0x07050301u) | 0x80808080u) - bs;
                        N[g] = (ha & 0x0F0F0F0Fu) | ((hc & 0x0F0F0F0Fu) << 4);
                    }
                    const bool in = j * 1024u + lane * 16u < lb;
                    __builtin_amdgcn_raw_buffer_store_b128(L, rs, in ? 512 + lane * 16 + j * 1024 : OOB, 0, AUX_SYS);
                    __builtin_amdgcn_raw_buffer_store_b64(N, rs, in ? 512 + lb + lane * 8 + j * 512 : OOB, 0, AUX_SYS);
                }
        }
        wait_vm0();
        if (lane == 0) st_sys((unsigned*)(ws + WS_SEGRDY) + seg, f);
    }
    {
        const int row0 = (p.rank >> 2) * 4096 + ti * 256 + q * 64 + (lane >> 4) * 4;
        const int col0 = (p.rank & 3) * 2048 + tj * 128 + h * 64 + (lane & 7) * 8 + ((lane >> 3) & 1) * 4;
        char* o = p.out + ((size_t)row0 * NG + col0) * 2;
#pragma unroll
        for (int i = 0; i < 16; ++i) *(u32x2*)(o + (size_t)((i >> 2) * 16 + (i & 3)) * (NG * 2)) = ov[i];
    }
    __builtin_amdgcn_fence(__ATOMIC_RELEASE, "wavefront");
    __builtin_amdgcn_wave_barrier();
    if (wv == 0) STAMP(p, blockIdx.x, 4, lane);
#pragma unroll 1
    for (int i = 0; i < 7; ++i) {
        const int e = seg + NSEG * i;
        expand_segment(p, cx, (p.rank + 1 + e % 7) & 7, e / 7, lane, wl);
        if (wv == 0 && i == 0) STAMP(p, blockIdx.x, 9, lane);
    }
    if (wv == 7) STAMP(p, blockIdx.x, 5, lane);
}

}

__global__ void __launch_bounds__(NTHR) fused_gemm_ar(Params p) {
    extern __shared__ __attribute__((aligned(16))) char lds[];
    const int lane = threadIdx.x & 63;
    const int wave = __builtin_amdgcn_readfirstlane(threadIdx.x >> 6);
    const int wg = blockIdx.x;
    volatile unsigned* ctl = (volatile unsigned*)(lds + LDS_CTL);
    char* ws = p.ws[p.rank];
    if (threadIdx.x == 0) {
        const unsigned ticket = __hip_atomic_fetch_add((unsigned*)(ws + WS_TICKET), 1u, __ATOMIC_RELAXED, __HIP_MEMORY_SCOPE_AGENT);
        ctl[0] = 0u; ctl[1] = 0u; ctl[2] = ticket / NWG + 1u;
    }
    __syncthreads();
    Ctx cx;
    cx.call = __builtin_amdgcn_readfirstlane(ctl[2]);
    cx.tmo = (unsigned*)(ws + WS_TMO);
    cx.t0 = __builtin_amdgcn_s_memrealtime();
    const int xcd = wg & 7, idx = wg >> 3;
    const int ti = (idx & 1) * 8 + (xcd & 1) * 4 + ((idx >> 1) & 3), tj = (xcd >> 1) * 4 + (idx >> 3);
    if (wave < 8) role_gemm(p, cx, ti, tj, wave, lane, lds);
    else if (wave == 8) role_poller(p, cx, ti, tj, lane, lds);
    else role_worker(p, cx, wg, wave - 9, lane);
}

extern "C" void kernel_launch(int rank, int nranks, ncclComm_t comm, hipStream_t stream,
                              void* const* d_in, const int* in_sizes, int n_in,
                              void* d_out, int out_size,
                              const int* in_dtypes, int out_dtype,
                              void* const* d_out_all, void* const* d_ws_all, size_t ws_bytes) {
    if (nranks != NR || n_in < 2 || ws_bytes < WS_END || (unsigned)rank >= (unsigned)NR) {
        fprintf(stderr, "unsupported configuration\n");
        return;
    }
    static bool attr_set[NR] = {};
    if (!attr_set[rank]) {
        HIP_CHECK(hipFuncSetAttribute((const void*)fused_gemm_ar, hipFuncAttributeMaxDynamicSharedMemorySize, LDS_BYTES));
        attr_set[rank] = true;
    }
    Params p;
    p.x = (const char*)d_in[0];
    p.w = (const char*)d_in[1];
    p.out = (char*)d_out;
    for (int i = 0; i < NR; ++i) p.ws[i] = (char*)d_ws_all[i];
    p.rank = rank;
    fused_gemm_ar<<<NWG, NTHR, LDS_BYTES, stream>>>(p);
    HIP_CHECK(hipGetLastError());
}
```

```cpp
#include <hip/hip_runtime.h>
#include <rccl/rccl.h>
#include <cstdio>
#include <cstdint>

#define HIP_CHECK(expr) do { hipError_t _e = (expr); if (_e != hipSuccess) { \
    fprintf(stderr, "HIP error %s at %s:%d\n", hipGetErrorString(_e), __FILE__, __LINE__); \
    return; } } while (0)

typedef short bf16x8 __attribute__((ext_vector_type(8)));
typedef float f32x4 __attribute__((ext_vector_type(4)));
typedef unsigned u32x4 __attribute__((ext_vector_type(4)));
typedef unsigned u32x2 __attribute__((ext_vector_type(2)));
typedef unsigned short us2 __attribute__((ext_vector_type(2)));
typedef unsigned long long u64;
#define LAS __attribute__((address_space(3)))
#define GAS __attribute__((address_space(1)))

namespace {

constexpr int NR = 8;
constexpr int NG = 8192;
constexpr int KSH = 1024;
constexpr int NPOS = 128;
constexpr int NWG = 256;
constexpr int NTHR = 768;
#define CFG_LAG 6
#define CFG_NPI 27
#define CFG_NPO 12
constexpr int LAG = CFG_LAG;
constexpr int NPI = CFG_NPI;
constexpr int NPO = CFG_NPO;
static_assert(NPI <= 34 && NPO <= 18, "comm waves: two per CU of group 1, one per CU of group 0");
constexpr int U_IN = 1536;
constexpr int U_OUT = 1024;
#define CFG_PSLEEP 64
#define CFG_ESLEEP 32
#define CFG_ESLEEP2 127
#define CFG_ELAST 512
#define CFG_RSLEEP 32
#define CFG_EARLY 0
constexpr int NSEG = 2048;
constexpr int SEG_BYTES = 512 + 8192;
constexpr int CU_BYTES = 6144;
constexpr u64 DEADLINE_TICKS = 50000000ull;

constexpr size_t WS_TICKET = 0;
constexpr size_t WS_TMO = 128;
constexpr size_t WS_LINKCTR = 256;
constexpr size_t WS_XF = 8192;
constexpr size_t WS_WF = WS_XF + 16 * 128 * 16;
constexpr size_t WS_SEGRDY = WS_WF + 16 * 128 * 8;
constexpr size_t WS_RF = WS_SEGRDY + NSEG * 4;
constexpr size_t WS_MSEG = WS_RF + NR * NSEG * 4;
constexpr size_t WS_HW0 = WS_MSEG + 8 * NSEG * 4;
constexpr size_t WS_MW0 = WS_HW0 + 16 * 256 * 4;
constexpr size_t WS_HDR = WS_MW0 + 8 * 16 * 128 * 4;
constexpr size_t WS_DBG = WS_HDR + 16 * 256 * 64;
constexpr size_t WS_CTL_END = 2u << 20;
constexpr size_t WS_PKX = WS_CTL_END;
constexpr size_t WS_PKW = WS_PKX + (16u << 20);
constexpr size_t WS_CX = WS_PKW + (16u << 20);
constexpr size_t WS_CW = WS_CX + (12u << 20);
constexpr size_t WS_XA = WS_CW + (12u << 20);
constexpr size_t WS_WB = WS_XA + (size_t)NPOS * 16 * 32768;
constexpr size_t WS_OSTG = WS_WB + (size_t)NPOS * 16 * 16384;
constexpr size_t WS_ORECV = WS_OSTG + (size_t)NSEG * SEG_BYTES;
constexpr size_t WS_END = WS_ORECV + (size_t)NR * NSEG * SEG_BYTES;
static_assert(WS_DBG + 64 * 1024 <= WS_CTL_END, "control region");
static_assert(WS_END < 0x7FFF0000u, "32-bit workspace offsets");

constexpr int STAGE_BYTES = 49152;
constexpr int LDS_CTL = 3 * STAGE_BYTES;
constexpr int LDS_WRK = LDS_CTL + 64;
constexpr int LDS_BYTES = LDS_WRK + 3 * 4096;
constexpr int WAVE_LDS = 13312 + 256;
constexpr int WL_ESC = 8192 + 64;
constexpr int WL_EXP = WL_ESC + 4096 + 64;

struct Params {
    const char* x;
    const char* w;
    char* out;
    char* ws[NR];
    int rank;
};

constexpr int AUX_SYS = 17;
constexpr int AUX_AGT = 16;
constexpr unsigned OOB = 0x7FFFFF00u;

__device__ __forceinline__ __amdgpu_buffer_rsrc_t mk_rsrc(const void* p, unsigned bytes) {
    return __builtin_amdgcn_make_buffer_rsrc((void*)p, 0, bytes, 0x00020000);
}
__device__ __forceinline__ void st_sys(unsigned* p, unsigned v) {
    __hip_atomic_store(p, v, __ATOMIC_RELAXED, __HIP_MEMORY_SCOPE_SYSTEM);
}
__device__ __forceinline__ void st_agt(unsigned* p, unsigned v) {
    __hip_atomic_store(p, v, __ATOMIC_RELAXED, __HIP_MEMORY_SCOPE_AGENT);
}
__device__ __forceinline__ void wait_vm0() { asm volatile("s_waitcnt vmcnt(0)" ::: "memory"); __builtin_amdgcn_s_waitcnt(0x0F70); }
__device__ __forceinline__ void wait_vm4() { asm volatile("s_waitcnt vmcnt(4)" ::: "memory"); __builtin_amdgcn_s_waitcnt(0x0F74); }
__device__ __forceinline__ void wait_vm8() { asm volatile("s_waitcnt vmcnt(8)" ::: "memory"); __builtin_amdgcn_s_waitcnt(0x0F78); }
__device__ __forceinline__ unsigned mbcnt64(u64 m) {
    return __builtin_amdgcn_mbcnt_hi((unsigned)(m >> 32), __builtin_amdgcn_mbcnt_lo((unsigned)m, 0));
}
__device__ __forceinline__ unsigned pk_max(unsigned a, unsigned b) {
    return __builtin_bit_cast(unsigned, __builtin_elementwise_max(__builtin_bit_cast(us2, a), __builtin_bit_cast(us2, b)));
}
__device__ __forceinline__ unsigned pk_min(unsigned a, unsigned b) {
    return __builtin_bit_cast(unsigned, __builtin_elementwise_min(__builtin_bit_cast(us2, a), __builtin_bit_cast(us2, b)));
}
__device__ __forceinline__ unsigned bfi(unsigned m, unsigned a, unsigned b) { return (a & m) | (b & ~m); }
__device__ __forceinline__ unsigned pk_sub(unsigned a, unsigned b) {
    return __builtin_bit_cast(unsigned, (us2)(__builtin_bit_cast(us2, a) - __builtin_bit_cast(us2, b)));
}
__device__ __forceinline__ unsigned pk_sub_sat(unsigned a, unsigned b) {
    return __builtin_bit_cast(unsigned, __builtin_elementwise_sub_sat(__builtin_bit_cast(us2, a), __builtin_bit_cast(us2, b)));
}
__device__ __forceinline__ unsigned wave_pk_max(unsigned v) {
#pragma unroll
    for (int o = 32; o > 0; o >>= 1) v = pk_max(v, (unsigned)__shfl_xor((int)v, o, 64));
    return __builtin_amdgcn_readfirstlane(v);
}

struct Ctx {
    unsigned call;
    unsigned* tmo;
    u64 t0;
};
__device__ __forceinline__ bool give_up(const Ctx& c) {
    const u64 dt = __builtin_amdgcn_s_memrealtime() - c.t0;
    if (dt < 200000ull) return false;
    if (__hip_atomic_load(c.tmo, __ATOMIC_RELAXED, __HIP_MEMORY_SCOPE_AGENT) == c.call) return true;
    if (dt > DEADLINE_TICKS) {
        __hip_atomic_store(c.tmo, c.call, __ATOMIC_RELAXED, __HIP_MEMORY_SCOPE_AGENT);
        return true;
    }
    return false;
}
__device__ __forceinline__ bool timed_out(const Ctx& c) {
    return __hip_atomic_load(c.tmo, __ATOMIC_RELAXED, __HIP_MEMORY_SCOPE_AGENT) == c.call;
}

#define STAMP(p, wg, k, lane) do {} while (0)

__device__ __forceinline__ int seq_slot(int seq, bool g1) {
    if (g1) {
        if (seq < 16) return seq * 8;
        const int m = seq - 16;
        return (m / 7) * 8 + m % 7 + 1;
    }
    if (seq < 4) return seq * 8;
    if (seq < 58) {
        const int m = seq - 4, c = m / 9, r = m % 9;
        return r < 7 ? c * 8 + r + 1 : (4 + 2 * c + r - 7) * 8;
    }
    const int m = seq - 58;
    return (6 + m / 7) * 8 + m % 7 + 1;
}

__device__ __forceinline__ unsigned seg_tag(unsigned call) { return call & 63u; }
__device__ __forceinline__ bool seg_ok(unsigned f, unsigned tag) { return (f >> 26) == tag; }
__device__ __forceinline__ unsigned seg_len(unsigned f) {
    const unsigned lb = ((f >> 8) & 511u) * 16u;
    return (512u + (((f >> 25) & 1u) ? 2u * lb : lb + ((lb / 4u + 15u) & ~15u) + (f & 255u) * 16u) + 63u) & ~63u;
}
__device__ __forceinline__ unsigned pick_base(unsigned top, const unsigned (&ex)[8]) {
    unsigned t = top;
#pragma unroll 1
    for (int it = 0; it < 2 && t > 6u; ++it) {
        int diff = 0;
#pragma unroll
        for (int k = 0; k < 8; ++k)
            diff += __builtin_popcountll(__ballot(ex[k] == t)) - __builtin_popcountll(__ballot(ex[k] + 7u == t));
        if (diff >= 0) break;
        --t;
    }
    const unsigned b = t > 6u ? t - 6u : 0u;
    return b < 248u ? b : 248u;
}
__device__ __forceinline__ unsigned code4(unsigned d0, unsigned d1, unsigned base2) {
    const unsigned c0 = pk_min(pk_sub((d0 >> 7) & 0x00FF00FFu, base2), 0x00070007u);
    const unsigned c1 = pk_min(pk_sub((d1 >> 7) & 0x00FF00FFu, base2), 0x00070007u);
    return __builtin_amdgcn_perm(c1, c0, 0x06040200u);
}
template <int BITS>
__device__ __forceinline__ unsigned wave_scan(unsigned n, unsigned& sum) {
    unsigned pre = 0;
    sum = 0;
#pragma unroll
    for (int b = 0; b < BITS; ++b) {
        const u64 m = __ballot((n >> b) & 1u);
        pre += mbcnt64(m) << b;
        sum += (unsigned)__builtin_popcountll(m) << b;
    }
    return pre;
}

__device__ __forceinline__ unsigned hdr_tag(unsigned call) { return call & 0xFFFu; }

__device__ __forceinline__ void pack_load(const Params& p, int j, int lane, u32x4 (&v)[8]) {
    const int c = j >> 8, r = j & 255, un = r & 127;
    if (r < 128) {
        const char* src = p.w + (size_t)(c * 64 + (lane >> 3) * 8) * (NG * 2) + (size_t)(un * 64 + (lane & 7) * 8) * 2;
#pragma unroll
        for (int k = 0; k < 8; ++k) v[k] = *(const u32x4*)(src + (size_t)k * (NG * 2));
    } else {
        const int rr = lane >> 3, pp = lane & 7;
        const char* src = p.x + (size_t)(un * 64 + rr) * (KSH * 2) + c * 128;
        const int o0 = (pp ^ (rr >> 1)) * 16, o1 = (pp ^ (4 + (rr >> 1))) * 16;
#pragma unroll
        for (int k = 0; k < 8; ++k) v[k] = *(const u32x4*)(src + (size_t)k * 8 * (KSH * 2) + ((k & 1) ? o1 : o0));
    }
}

__device__ __forceinline__ void role_packer(const Params& p, const Ctx& cx, int h, int lane, u32x4 (&v)[8], char* wl) {
    const int rank = p.rank;
    char* ws = p.ws[rank];
    const unsigned tagw = hdr_tag(cx.call) << 20;
#pragma unroll 1
    for (int j = h; j < 4096;) {
        const int c = j >> 8, r = j & 255, un = r & 127;
        const int jn = j < 512 ? 4096 : j + 1024;
        unsigned* flag = nullptr;
        if (r < 128) {
            if ((un >> 5) == (rank & 3)) flag = (unsigned*)(ws + WS_WF) + (((un >> 1) & 15) * NPOS + c * 8) * 2 + (un & 1);
            u32x4 o[8];
#pragma unroll
            for (int cc = 0; cc < 8; ++cc)
#pragma unroll
                for (int d = 0; d < 4; ++d)
                    o[cc][d] = __builtin_amdgcn_perm(v[2 * d + 1][cc >> 1], v[2 * d][cc >> 1], (cc & 1) ? 0x07060302u : 0x05040100u);
#pragma unroll
            for (int cc = 0; cc < 8; ++cc) v[cc] = o[cc];
        } else {
            if ((un >> 6) == (rank >> 2)) flag = (unsigned*)(ws + WS_XF) + (((un >> 2) & 15) * NPOS + c * 8) * 4 + (un & 3);
        }
        unsigned mx = 0, mn = 0;
#pragma unroll
        for (int k = 0; k < 8; ++k)
#pragma unroll
            for (int d = 0; d < 4; ++d) {
                mx = pk_max(mx, v[k][d] & 0x7F807F80u);
                mn = pk_max(mn, pk_sub(0u, v[k][d] & 0x7F007F00u));
            }
        const unsigned red = wave_pk_max((max(mx >> 16, mx & 0xFFFFu) << 16) | max(mn >> 16, mn & 0xFFFFu));
        const __amdgpu_buffer_rsrc_t rc = mk_rsrc(ws + (r < 128 ? WS_CW : WS_CX) + (size_t)(c * 128 + un) * CU_BYTES, CU_BYTES);
        const __amdgpu_buffer_rsrc_t rh = mk_rsrc(ws + WS_HDR + (size_t)(c * 256 + r) * 64, 64);
        unsigned ex[8];
#pragma unroll
        for (int k = 0; k < 8; ++k) ex[k] = (v[k][k & 3] >> 7) & 0xFFu;
        const unsigned basea = pick_base(red >> 23, ex);
        unsigned C2[4] = {0u, 0u, 0u, 0u}, C1[2] = {0u, 0u};
#pragma unroll
        for (int hf = 0; hf < 2; ++hf)
#pragma unroll
            for (int g = 0; g < 8; ++g) {
                const u32x4 d = v[hf * 4 + (g >> 1)];
                const unsigned cd = code4(d[(g & 1) * 2], d[(g & 1) * 2 + 1], basea * 0x00010001u);
                C2[hf * 2 + (g >> 2)] |= ((cd >> 1) & 0x03030303u) << (2 * (g & 3));
                C1[hf] |= (cd & 0x01010101u) << (2 * (g & 3) + (g >> 2));
            }
        unsigned ne[2], eoff[2], E[4];
#pragma unroll
        for (int q = 0; q < 4; ++q) E[q] = C2[q] & (C2[q] >> 1) & (C1[q >> 1] >> (q & 1)) & 0x55555555u;
#pragma unroll
        for (int hf = 0; hf < 2; ++hf) eoff[hf] = wave_scan<6>(__builtin_popcount(E[2 * hf]) + __builtin_popcount(E[2 * hf + 1]), ne[hf]);
        unsigned hw0;
        bool raw = false;
        if (ne[0] <= 240u && ne[1] <= 240u) {
#pragma unroll
            for (int hf = 0; hf < 2; ++hf) {
                char* img = wl + 2048 + hf * 1024;
#pragma unroll
                for (int q = 0; q < 2; ++q) {
                    u32x4 B, X;
#pragma unroll
                    for (int g = 0; g < 4; ++g) {
                        const u32x4 d = v[hf * 4 + 2 * q + (g >> 1)];
                        const unsigned lo = __builtin_amdgcn_perm(d[(g & 1) * 2 + 1], d[(g & 1) * 2], 0x06040200u);
                        const unsigned hi = __builtin_amdgcn_perm(d[(g & 1) * 2 + 1], d[(g & 1) * 2], 0x07050301u);
                        B[g] = bfi(0x7F7F7F7Fu, lo, hi);
                        X[g] = bfi(0x01010101u, lo >> 7, hi << 1);
                    }
                    __builtin_amdgcn_raw_buffer_store_b128(B, rc, lane * 16 + hf * 3072 + q * 1024, 0, AUX_SYS);
                    *(u32x4*)(wl + lane * 32 + q * 16) = X;
                }
                *(unsigned*)(img + lane * 4) = C2[2 * hf];
                *(unsigned*)(img + 256 + lane * 4) = C2[2 * hf + 1];
                *(unsigned*)(img + 512 + lane * 4) = C1[hf];
                unsigned off = 768u + eoff[hf];
                u64 e = ((u64)E[2 * hf + 1] << 32) | E[2 * hf];
                while (e) {
                    const unsigned pb = __builtin_ctzll(e);
                    img[off] = wl[lane * 32 + (pb >> 5) * 16 + ((pb >> 1) & 3u) * 4 + ((pb >> 3) & 3u)];
                    ++off;
                    e &= e - 1ull;
                }
                __builtin_amdgcn_fence(__ATOMIC_RELEASE, "wavefront");
                __builtin_amdgcn_wave_barrier();
                __builtin_amdgcn_raw_buffer_store_b128(*(const u32x4*)(img + lane * 16), rc, hf * 3072 + 2048 + lane * 16, 0, AUX_SYS);
            }
            hw0 = basea | 0x4000u | (((ne[0] + 15u) >> 4) << 8) | (((ne[1] + 15u) >> 4) << 16);
        } else {
            const unsigned top = red >> 24;
            const unsigned base = top > 7u ? top - 7u : 0u;
            const unsigned lowest = ((0x10000u - (red & 0xFFFFu)) >> 8);
            const unsigned base2 = base * 0x01000100u;
#pragma unroll
            for (int hf = 0; hf < 2; ++hf) {
                u32x4 L[2], N;
#pragma unroll
                for (int jj = 0; jj < 4; ++jj) {
                    const u32x4 d = v[hf * 4 + jj];
                    unsigned nb[4];
#pragma unroll
                    for (int q = 0; q < 4; ++q) nb[q] = pk_sub_sat(d[q] & 0x7F007F00u, base2) | ((d[q] >> 4) & 0x08000800u);
                    L[jj >> 1][(jj & 1) * 2] = __builtin_amdgcn_perm(d[1], d[0], 0x06040200u);
                    L[jj >> 1][(jj & 1) * 2 + 1] = __builtin_amdgcn_perm(d[3], d[2], 0x06040200u);
                    N[jj] = __builtin_amdgcn_perm(nb[1], nb[0], 0x07050301u) | (__builtin_amdgcn_perm(nb[3], nb[2], 0x07050301u) << 4);
                }
                __builtin_amdgcn_raw_buffer_store_b128(L[0], rc, lane * 16 + hf * 3072, 0, AUX_SYS);
                __builtin_amdgcn_raw_buffer_store_b128(L[1], rc, lane * 16 + hf * 3072 + 1024, 0, AUX_SYS);
                __builtin_amdgcn_raw_buffer_store_b128(N, rc, lane * 16 + hf * 3072 + 2048, 0, AUX_SYS);
            }
            unsigned nesc = 0;
            if (lowest <= base) {
                unsigned r0 = 0, r1 = 0, n = 0;
#pragma unroll
                for (int k = 0; k < 8; ++k)
#pragma unroll
                    for (int e = 0; e < 8; ++e) {
                        const unsigned hb = (v[k][e >> 1] >> ((e & 1) ? 24 : 8)) & 0xFFu;
                        if ((hb & 0x7Fu) - 1u < base) { r1 = r0; r0 = ((unsigned)(k << 9 | e) << 8) | hb; ++n; }
                    }
                const u64 m1 = __ballot(n >= 1), m2 = __ballot(n >= 2);
                nesc = __builtin_popcountll(m1) + __builtin_popcountll(m2);
                raw = __ballot(n > 2) != 0ull || nesc > 15u;
                if (!raw) {
                    const unsigned lt = tagw | ((unsigned)lane << 11);
                    __builtin_amdgcn_raw_buffer_store_b32(r0 | lt, rh, n >= 1 ? 4 + mbcnt64(m1) * 4 : OOB, 0, AUX_SYS);
                    __builtin_amdgcn_raw_buffer_store_b32(r1 | lt, rh, n >= 2 ? 4 + ((unsigned)__builtin_popcountll(m1) + mbcnt64(m2)) * 4 : OOB, 0, AUX_SYS);
                } else nesc = 0;
            }
            hw0 = base | (nesc << 8) | (raw ? 0x8000u : 0u);
        }
        if (raw || flag) {
            const __amdgpu_buffer_rsrc_t rs = mk_rsrc(ws + (r < 128 ? WS_PKW : WS_PKX) + (size_t)(c * 128 + un) * 8192, 8192);
#pragma unroll
            for (int k = 0; k < 8; ++k) __builtin_amdgcn_raw_buffer_store_b128(v[k], rs, lane * 16 + k * 1024, 0, AUX_SYS);
        }
        asm volatile("" ::: "memory");
        if (jn < 4096) {
            pack_load(p, jn, lane, v);
            asm volatile("s_waitcnt vmcnt(8)" ::: "memory");
        } else wait_vm0();
        if (lane == 0) st_sys((unsigned*)(ws + WS_HW0) + c * 256 + r, tagw | hw0);
        if (flag && lane == 0) st_agt(flag, cx.call);
        j = jn;
    }
}

#define UNI(x) __builtin_amdgcn_readfirstlane(x)
#define LD_REMOTE(off, imm) __builtin_amdgcn_raw_buffer_load_b128(rrem, (off) + (imm), 0, AUX_SYS)
#define ST_LOCAL(v, off, imm) __builtin_amdgcn_raw_buffer_store_b128(v, rloc, (off) + (imm), 0, AUX_AGT)

struct InUnit { unsigned idx, dst, flag, info; bool end; };

__device__ __forceinline__ void decode_half(const u32x4 l0, const u32x4 l1, const u32x4 nn, unsigned hdr, unsigned h0, unsigned k,
                                            int lane, char* wl, u32x4 (&o)[4]) {
    const unsigned bs = (h0 & 0xFFu) * 0x01010101u;
    unsigned H[8];
    u32x4 la = l0, lb = l1;
    if (h0 & 0x4000u) {
        *(u32x4*)(wl + lane * 16) = nn;
        __builtin_amdgcn_fence(__ATOMIC_RELEASE, "wavefront");
        __builtin_amdgcn_wave_barrier();
        const unsigned c0 = *(const unsigned*)(wl + lane * 4), c1 = *(const unsigned*)(wl + 256 + lane * 4);
        const unsigned cb = *(const unsigned*)(wl + 512 + lane * 4);
        unsigned e0 = c0 & (c0 >> 1) & cb & 0x55555555u, e1 = c1 & (c1 >> 1) & (cb >> 1) & 0x55555555u, sum;
        unsigned off = 768u + wave_scan<6>(__builtin_popcount(e0) + __builtin_popcount(e1), sum);
#pragma unroll
        for (int g = 0; g < 8; ++g)
            H[g] = ((((g & 4 ? c1 : c0) >> (2 * (g & 3))) & 0x03030303u) << 1) + ((cb >> (2 * (g & 3) + (g >> 2))) & 0x01010101u) + bs;
        char* hp = wl + 1024 + lane * 32;
        *(u32x4*)hp = (u32x4){H[0], H[1], H[2], H[3]};
        *(u32x4*)(hp + 16) = (u32x4){H[4], H[5], H[6], H[7]};
        u64 e = ((u64)e1 << 32) | e0;
        while (e) {
            const unsigned w0 = *(const unsigned*)(wl + (off & ~3u)), w1 = *(const unsigned*)(wl + (off & ~3u) + 4);
            unsigned w = __builtin_amdgcn_alignbyte(w1, w0, off & 3u);
#pragma unroll
            for (int r = 0; r < 4; ++r)
                if (e) {
                    const unsigned pb = __builtin_ctzll(e);
                    hp[(pb >> 5) * 16 + ((pb >> 1) & 3u) * 4 + ((pb >> 3) & 3u)] = (char)w;
                    w >>= 8;
                    e &= e - 1ull;
                }
            off += 4;
        }
        __builtin_amdgcn_fence(__ATOMIC_RELEASE, "wavefront");
        __builtin_amdgcn_wave_barrier();
        const u32x4 xa = *(const u32x4*)hp, xb = *(const u32x4*)(hp + 16);
#pragma unroll
        for (int g = 0; g < 4; ++g) {
            H[g] = bfi(0x80808080u, l0[g], xa[g] >> 1);
            H[4 + g] = bfi(0x80808080u, l1[g], xb[g] >> 1);
            la[g] = bfi(0x7F7F7F7Fu, l0[g], xa[g] << 7);
            lb[g] = bfi(0x7F7F7F7Fu, l1[g], xb[g] << 7);
        }
    } else {
        const unsigned tlo = (bs + 0x03020100u) & 0xFFFFFF00u, thi = bs + 0x07060504u;
#pragma unroll
        for (int jj = 0; jj < 4; ++jj) {
            const unsigned n = nn[jj];
            H[2 * jj] = __builtin_amdgcn_perm(thi, tlo, n & 0x07070707u) | ((n << 4) & 0x80808080u);
            H[2 * jj + 1] = __builtin_amdgcn_perm(thi, tlo, (n >> 4) & 0x07070707u) | (n & 0x80808080u);
        }
        const int nesc = (h0 >> 8) & 15;
#pragma unroll 1
        for (int i = 1; i <= nesc; ++i) {
            const unsigned e = __builtin_amdgcn_readlane(hdr, i);
            if (((e >> 19) & 1u) != k) continue;
            const unsigned key = ((unsigned)lane == ((e >> 11) & 63u)) ? ((e >> 17) & 3u) * 2 + ((e >> 10) & 1u) : 8u;
            const unsigned sh = ((e >> 8) & 3u) * 8, msk = 0xFFu << sh, val = (e & 0xFFu) << sh;
#pragma unroll
            for (int q = 0; q < 8; ++q) H[q] = (key == (unsigned)q) ? ((H[q] & ~msk) | val) : H[q];
        }
    }
#pragma unroll
    for (int jj = 0; jj < 4; ++jj) {
        const unsigned p0 = (jj >> 1 ? lb : la)[(jj & 1) * 2], p1 = (jj >> 1 ? lb : la)[(jj & 1) * 2 + 1];
        o[jj][0] = __builtin_amdgcn_perm(H[2 * jj], p0, 0x05010400u);
        o[jj][1] = __builtin_amdgcn_perm(H[2 * jj], p0, 0x07030602u);
        o[jj][2] = __builtin_amdgcn_perm(H[2 * jj + 1], p1, 0x05010400u);
        o[jj][3] = __builtin_amdgcn_perm(H[2 * jj + 1], p1, 0x07030602u);
    }
}

__device__ __forceinline__ void role_pull_in(const Params& p, const Ctx& cx, int link, int lane, char* wl) {
    const int rank = p.rank;
    const __amdgpu_buffer_rsrc_t rloc = mk_rsrc(p.ws[rank], (unsigned)WS_END);
    const __amdgpu_buffer_rsrc_t rrem = mk_rsrc(p.ws[(rank + 1 + link) & 7], (unsigned)WS_END);
    const unsigned ctr_off = (lane == 0) ? (unsigned)WS_LINKCTR + link * 128 : OOB;
    const unsigned base = (cx.call - 1) * (unsigned)(U_IN + NPI);
    const unsigned l16 = lane * 16;
    const unsigned htag = hdr_tag(cx.call);

    auto ticket = [&]() -> unsigned { return (unsigned)__builtin_amdgcn_raw_ptr_buffer_atomic_add_i32(1, rloc, ctr_off, 0, AUX_AGT); };
    auto make = [&](unsigned tv) -> InUnit {
        const int n = (int)(UNI(tv) - base);
        InUnit u;
        int d, r;
        if (n < 64 * LAG) { d = n >> 6; r = n & 63; }
        else if (n < 64 * LAG + 96 * (16 - LAG)) { const int m = n - 64 * LAG; d = LAG + m / 96; r = m % 96; }
        else { const int m = n - 64 * LAG - 96 * (16 - LAG); d = 16 + (m >> 5); r = 64 + (m & 31); }
        const int c = (r >= 64) ? d - LAG : d;
        const int pos = c * 8 + 1 + link;
        const bool isw = r < 32;
        const int vv = isw ? r : r - 32;
        u.end = n >= U_IN;
        u.info = (unsigned)WS_MW0 + ((link * 16 + c) * 128 + r) * 4;
        u.idx = isw ? (unsigned)(c * 128 + (rank & 3) * 32 + vv) : (unsigned)(c * 128 + (rank >> 2) * 64 + vv) | 0x80000000u;
        u.dst = (isw ? (unsigned)WS_WB + pos * (16 * 16384) : (unsigned)WS_XA + pos * (16 * 32768)) + vv * 8192;
        u.flag = isw ? (unsigned)WS_WF + ((vv >> 1) * NPOS + pos) * 8 + (vv & 1) * 4
                     : (unsigned)WS_XF + ((vv >> 2) * NPOS + pos) * 16 + (vv & 3) * 4;
        return u;
    };
    auto load_hdr = [&](const InUnit& u) -> unsigned {
        return __builtin_amdgcn_raw_buffer_load_b32(rloc, (lane == 0 && !u.end) ? u.info : OOB, 0, AUX_AGT);
    };
    auto hdr_ok = [&](unsigned hv) -> bool {
        const unsigned w0 = UNI(hv);
        return (w0 >> 20) == htag && ((w0 & 0xC000u) != 0u || (w0 & 0xF00u) == 0u);
    };
    auto publish = [&](unsigned flag) {
        __builtin_amdgcn_raw_buffer_store_b32(cx.call, rloc, lane == 0 ? flag : OOB, 0, AUX_AGT);
    };
    auto store_half = [&](u32x4 (&C)[4], unsigned hdr, unsigned h0, unsigned dst, unsigned k) {
        if (!(h0 & 0x8000u)) {
            u32x4 o[4];
            decode_half(C[0], C[1], C[2], hdr, h0, k, lane, wl, o);
#pragma unroll
            for (int j = 0; j < 4; ++j) C[j] = o[j];
        }
        const unsigned a0 = dst + l16 + k * 4096u;
#pragma unroll
        for (int j = 0; j < 4; ++j) ST_LOCAL(C[j], a0, j * 1024);
    };
    auto src_of = [&](const InUnit& u, unsigned h0) -> unsigned {
        const bool x = (u.idx >> 31) != 0u;
        const unsigned ix = u.idx & 0x7FFFFFFFu;
        return ((h0 & 0x8000u) ? (x ? (unsigned)WS_PKX : (unsigned)WS_PKW) + ix * 8192u
                               : (x ? (unsigned)WS_CX : (unsigned)WS_CW) + ix * (unsigned)CU_BYTES) + l16;
    };
    auto fetch = [&](u32x4 (&N)[4], unsigned so0, unsigned h0, unsigned k) {
        const bool raw = (h0 & 0x8000u) != 0u;
        const unsigned so = so0 + k * (raw ? 4096u : 3072u);
        const unsigned third = (h0 & 0x4000u) ? 768u + ((h0 >> (8 + 8 * k)) & 15u) * 16u : 1024u;
        N[0] = LD_REMOTE(so, 0); N[1] = LD_REMOTE(so, 1024); N[2] = LD_REMOTE(l16 < third ? so : OOB, 2048);
        N[3] = LD_REMOTE(raw ? so : OOB, 3072);
    };
    auto wait_hdr = [&](const InUnit& u) -> unsigned {
        const unsigned ix = u.idx & 0x7FFFFFFFu;
        const unsigned hu = (ix >> 7) * 256 + (u.idx >> 31) * 128 + (ix & 127u);
        unsigned off = lane == 0 ? (unsigned)WS_HW0 + hu * 4 : OOB;
        for (;;) {
            const unsigned hv = __builtin_amdgcn_raw_buffer_load_b32(rrem, off, 0, AUX_SYS);
            const unsigned w0 = UNI(hv);
            const unsigned need = (w0 & 0xC000u) ? 1u : (2u << ((w0 >> 8) & 15u)) - 1u;
            if (((unsigned)__ballot((hv >> 20) == htag) & need) == need) return hv;
            if (give_up(cx)) return 0x8000u;
            if ((w0 >> 20) == htag) { if (lane > 0 && lane < 16) off = (unsigned)WS_HDR + hu * 64 + lane * 4; }
            else __builtin_amdgcn_s_sleep(8);
        }
    };

    InUnit u = make(ticket());
    if (u.end) return;
    unsigned t = ticket();
    unsigned hdr = wait_hdr(u);
    unsigned h0 = UNI(hdr);
    unsigned so = src_of(u, h0);
    u32x4 A[4], B[4];
    fetch(A, so, h0, 0);
    wait_vm0();
    unsigned pflag = OOB;
    for (;;) {
        asm volatile("" : "+v"(A[0]), "+v"(A[1]), "+v"(A[2]), "+v"(A[3]), "+v"(t));
        fetch(B, so, h0, 1);
        __builtin_amdgcn_sched_barrier(0);
        const InUnit un = make(t);
        unsigned info = load_hdr(un);
        store_half(A, hdr, h0, u.dst, 0);
        wait_vm4();
        asm volatile("" : "+v"(B[0]), "+v"(B[1]), "+v"(B[2]), "+v"(B[3]), "+v"(info));
        if (un.end || !hdr_ok(info)) {
            publish(pflag);
            store_half(B, hdr, h0, u.dst, 1);
            wait_vm0();
            publish(u.flag);
            pflag = OOB;
            if (un.end) return;
            t = ticket();
            hdr = wait_hdr(un);
            h0 = UNI(hdr);
            so = src_of(un, h0);
            fetch(A, so, h0, 0);
            wait_vm0();
        } else {
            const unsigned hn0 = UNI(info);
            so = src_of(un, hn0);
            fetch(A, so, hn0, 0);
            __builtin_amdgcn_sched_barrier(0);
            publish(pflag);
            t = ticket();
            store_half(B, hdr, h0, u.dst, 1);
            wait_vm4();
            pflag = u.flag;
            hdr = info; h0 = hn0;
        }
        u = un;
    }
}

__device__ __forceinline__ void role_pull_out(const Params& p, const Ctx& cx, int link, int lane) {
    const int rank = p.rank, src = (rank + 1 + link) & 7;
    const __amdgpu_buffer_rsrc_t rloc = mk_rsrc(p.ws[rank], (unsigned)WS_END);
    const __amdgpu_buffer_rsrc_t rrem = mk_rsrc(p.ws[src], (unsigned)WS_END);
    const unsigned ctr_off = (lane == 0) ? (unsigned)WS_LINKCTR + (7 + link) * 128 : OOB;
    const unsigned base = (cx.call - 1) * (unsigned)(U_OUT + NPO);
    const unsigned tag = seg_tag(cx.call);
    const unsigned l16 = lane * 16;
    bool dead = false;
    {
        const unsigned ibase = (cx.call - 1) * (unsigned)(U_IN + NPI);
        for (;;) {
            const unsigned v = UNI(__builtin_amdgcn_raw_buffer_load_b32(rloc, lane == 0 ? (unsigned)WS_LINKCTR + link * 128 : OOB, 0, AUX_AGT));
            const int rem = U_IN - CFG_EARLY - (int)(v - ibase);
            if (rem <= 0) break;
            if (give_up(cx)) { dead = true; break; }
            for (int k = rem >> 6; k > 0; --k) __builtin_amdgcn_s_sleep(100);
            __builtin_amdgcn_s_sleep(16);
        }
        STAMP(p, blockIdx.x, 1, lane);
    }
    auto ticket = [&]() -> unsigned { return (unsigned)__builtin_amdgcn_raw_ptr_buffer_atomic_add_i32(1, rloc, ctr_off, 0, AUX_AGT); };
    auto load_words = [&](unsigned n) -> unsigned {
        return __builtin_amdgcn_raw_buffer_load_b32(rloc, (lane < 2 && n < (unsigned)U_OUT) ? (unsigned)WS_MSEG + (link * NSEG + 2 * n + lane) * 4 : OOB, 0, AUX_AGT);
    };
    unsigned pn = U_OUT, qn = U_OUT, pw = 0, qw = 0;
    auto publish = [&]() {
        __builtin_amdgcn_raw_buffer_store_b32(pw, rloc, (lane < 2 && pn < (unsigned)U_OUT) ? (unsigned)WS_RF + (src * NSEG + 2 * pn + lane) * 4 : OOB, 0, AUX_AGT);
        pn = qn; pw = qw; qn = U_OUT;
    };
    auto store_unit = [&](u32x4 (&C)[8], unsigned n, unsigned w, unsigned len0, unsigned len1) {
        const unsigned d0 = (unsigned)WS_ORECV + (src * NSEG + 2 * n) * (unsigned)SEG_BYTES + l16, d1 = d0 + SEG_BYTES;
#pragma unroll
        for (int j = 0; j < 4; ++j) ST_LOCAL(C[j], l16 + j * 1024u < len0 ? d0 : OOB, j * 1024);
#pragma unroll
        for (int j = 0; j < 4; ++j) ST_LOCAL(C[4 + j], l16 + j * 1024u < len1 ? d1 : OOB, j * 1024);
        qn = n; qw = w;
    };
    auto rest = [&](unsigned sg, unsigned len) {
#pragma unroll 1
        for (unsigned o = 4096; o < len; o += 1024) {
            const bool in = o + l16 < len;
            const u32x4 v = LD_REMOTE(in ? (unsigned)WS_OSTG + sg * (unsigned)SEG_BYTES + o + l16 : OOB, 0);
            ST_LOCAL(v, in ? (unsigned)WS_ORECV + (src * NSEG + sg) * (unsigned)SEG_BYTES + o + l16 : OOB, 0);
        }
    };
    unsigned nn = UNI(ticket()) - base;
    if (nn >= (unsigned)U_OUT) return;
    unsigned t = ticket();
    unsigned info = load_words(nn), cn = U_OUT, cw = 0, cl0 = 0, cl1 = 0;
    wait_vm0();
    u32x4 A[8], B[8];
#pragma unroll
    for (int j = 0; j < 8; ++j) A[j] = (u32x4){0u, 0u, 0u, 0u};
#define OUT_STEP(C, N) do { \
        asm volatile("" : "+v"(C[0]), "+v"(C[1]), "+v"(C[2]), "+v"(C[3]), "+v"(C[4]), "+v"(C[5]), "+v"(C[6]), "+v"(C[7]), "+v"(t), "+v"(info)); \
        unsigned w0_ = UNI(info), w1_ = __builtin_amdgcn_readlane(info, 1); \
        if (!seg_ok(w0_, tag) || !seg_ok(w1_, tag) || nn >= (unsigned)U_OUT) {        \
            publish(); \
            store_unit(C, cn, cw, cl0, cl1); cn = U_OUT; cl0 = cl1 = 0; \
            wait_vm0(); publish(); publish(); \
            if (nn >= (unsigned)U_OUT) return; \
            for (;;) { \
                info = load_words(nn); \
                w0_ = UNI(info); w1_ = __builtin_amdgcn_readlane(info, 1); \
                if (seg_ok(w0_, tag) && seg_ok(w1_, tag)) break; \
                if (dead || give_up(cx)) { dead = true; w0_ = w1_ = info = (tag ^ 1u) << 26; break; } \
                __builtin_amdgcn_s_sleep(8); \
            } \
        } \
        const unsigned wv_ = info, l0_ = seg_len(w0_), l1_ = seg_len(w1_); \
        const unsigned s0_ = (unsigned)WS_OSTG + 2 * nn * (unsigned)SEG_BYTES + l16, s1_ = s0_ + SEG_BYTES; \
        _Pragma("unroll") for (int j = 0; j < 4; ++j) N[j] = LD_REMOTE(l16 + j * 1024u < l0_ ? s0_ : OOB, j * 1024); \
        _Pragma("unroll") for (int j = 0; j < 4; ++j) N[4 + j] = LD_REMOTE(l16 + j * 1024u < l1_ ? s1_ : OOB, j * 1024); \
        __builtin_amdgcn_sched_barrier(0); \
        publish(); \
        if (l0_ > 4096u) rest(2 * nn, l0_); \
        if (l1_ > 4096u) rest(2 * nn + 1, l1_); \
        const unsigned n2_ = UNI(t) - base; \
        if (n2_ < (unsigned)U_OUT) t = ticket(); \
        info = load_words(n2_); \
        store_unit(C, cn, cw, cl0, cl1); \
        wait_vm8(); \
        cn = nn; cw = wv_; cl0 = l0_; cl1 = l1_; nn = n2_; \
    } while (0)
    for (;;) {
        OUT_STEP(A, B);
        OUT_STEP(B, A);
    }
}

__device__ __forceinline__ void role_relay(const Params& p, const Ctx& cx, int link, int lane) {
    const int rank = p.rank;
    const int src = (rank + 1 + link) & 7;
    const __amdgpu_buffer_rsrc_t rctl = mk_rsrc(p.ws[rank], (unsigned)WS_CTL_END);
    const __amdgpu_buffer_rsrc_t rinfo = mk_rsrc(p.ws[src], (unsigned)WS_CTL_END);
    const unsigned tag = seg_tag(cx.call);
    {
        const int sub = lane & 31;
        const unsigned woff = sub < 8 ? (unsigned)((rank & 3) * 32 + sub * 4) : (unsigned)(128 + (rank >> 2) * 64 + (sub - 8) * 4);
        const unsigned ht = hdr_tag(cx.call);
#pragma unroll 1
        for (int c = 0; c < 16;) {
            const int cc = c + (lane >> 5);
            const bool act = sub < 24 && cc < 16;
            const u32x4 hd = __builtin_amdgcn_raw_buffer_load_b128(rinfo, act ? (unsigned)WS_HW0 + (cc * 256 + woff) * 4 : OOB, 0, AUX_SYS);
            __builtin_amdgcn_raw_buffer_store_b128(hd, rctl, act ? (unsigned)WS_MW0 + ((link * 16 + cc) * 128 + sub * 4) * 4 : OOB, 0, AUX_AGT);
            bool ok = true;
#pragma unroll
            for (int q = 0; q < 4; ++q) ok = ok && (((hd[q] >> 20) - ht) & 0xFFFu) <= 1u;
            const u64 bad = __ballot(act && !ok);
            if ((unsigned)bad == 0u) { c += (bad == 0ull) ? 2 : 1; continue; }
            if (give_up(cx)) return;
            __builtin_amdgcn_s_sleep(16);
        }
    }
    int pos = 0;
    while (pos < NSEG) {
        const unsigned q = pos + lane * 4;
        const u32x4 v = __builtin_amdgcn_raw_buffer_load_b128(rinfo, q < NSEG ? (unsigned)WS_SEGRDY + q * 4 : OOB, 0, AUX_SYS);
        const bool any = seg_ok(v[0], tag) || seg_ok(v[1], tag) || seg_ok(v[2], tag) || seg_ok(v[3], tag);
        const bool all = seg_ok(v[0], tag) && seg_ok(v[1], tag) && seg_ok(v[2], tag) && seg_ok(v[3], tag);
        __builtin_amdgcn_raw_buffer_store_b128(v, rctl, (any && q < NSEG) ? (unsigned)WS_MSEG + (link * NSEG + q) * 4 : OOB, 0, AUX_AGT);
        const u64 m = __ballot(all && q < NSEG);
        const int n = (m == ~0ull) ? 64 : __builtin_ctzll(~m);
        pos += n * 4;
        if (n < 64) {
            if (give_up(cx)) return;
            __builtin_amdgcn_s_sleep(CFG_RSLEEP);
        }
    }
}

__device__ __forceinline__ int packer_id(int wg, int wave) {
    const int gi = (wg >> 4) * 8 + (wg & 7);
    const bool g1 = (wg & 8) != 0;
    if (wave < 8) return g1 ? 512 + wave * 128 + gi : -1;
    const int k = wave - 9;
    return (k < 0 || (g1 && k < 2)) ? -1 : gi * 4 + (g1 ? 3 : k);
}
__device__ __forceinline__ void role_worker(const Params& p, const Ctx& cx, int wg, int k, int lane, char* wl) {
    const int gi = (wg >> 4) * 8 + (wg & 7);
    if (wg & 8) {
        if (k < 2 && gi < 119) { if (gi / 7 + 17 * k < NPI) role_pull_in(p, cx, gi % 7, lane, wl); }
        else if (gi >= 121 && k == 0) role_relay(p, cx, gi - 121, lane);
    } else {
        if (k == 0 && gi < NPO * 7) role_pull_out(p, cx, gi % 7, lane);
    }
    if (k == 0) STAMP(p, wg, 2, lane);
}

__device__ __forceinline__ void role_poller(const Params& p, const Ctx& cx, int ti, int tj, int lane, char* lds) {
    const char* ws = p.ws[p.rank];
    const __amdgpu_buffer_rsrc_t rx = mk_rsrc(ws + WS_XF + ti * (NPOS * 16), NPOS * 16);
    const __amdgpu_buffer_rsrc_t rw = mk_rsrc(ws + WS_WF + tj * (NPOS * 8), NPOS * 8);
    volatile LAS unsigned* ready = (volatile LAS unsigned*)(LAS char*)(lds + LDS_CTL + 4);
    int pos = 0;
    while (pos < NPOS) {
        const int sq = pos + (lane & 15);
        const int q = sq < NPOS ? seq_slot(sq, ti >= 8) : NPOS;
        bool ok = false;
        if (lane < 16) {
            const u32x4 a = __builtin_amdgcn_raw_buffer_load_b128(rx, q * 16, 0, AUX_AGT);
            const u32x2 b = __builtin_amdgcn_raw_buffer_load_b64(rw, q * 8, 0, AUX_AGT);
            ok = a[0] == cx.call && a[1] == cx.call && a[2] == cx.call && a[3] == cx.call && b[0] == cx.call && b[1] == cx.call;
        }
        const u64 m = __ballot(ok);
        const int n = __builtin_ctzll(~m);
        if (n > 0) {
            if (pos <= (ti >= 8 ? 16 : 4) && pos + n > (ti >= 8 ? 16 : 4)) STAMP(p, blockIdx.x, 8, lane);
            pos += n;
            if (lane == 0) *ready = pos;
        } else {
            if (give_up(cx)) { pos = NPOS; if (lane == 0) *ready = NPOS; break; }
            __builtin_amdgcn_s_sleep(CFG_PSLEEP);
        }
    }
    STAMP(p, blockIdx.x, 6, lane);
}

#define CFG_OUTAUX 1
__device__ __forceinline__ void st_out(char* o, u32x2 v) {
    __builtin_nontemporal_store(v, (u32x2*)o);
}
__device__ __forceinline__ void expand_segment(const Params& p, const Ctx& cx, int src, int seg, int lane, char* wl, bool last) {
    const char* ws = p.ws[p.rank];
    const unsigned* rf = (const unsigned*)(ws + WS_RF) + src * NSEG + seg;
    unsigned f;
    bool dead = false;
    for (;;) {
        f = __builtin_amdgcn_readfirstlane(__hip_atomic_load(rf, __ATOMIC_RELAXED, __HIP_MEMORY_SCOPE_AGENT));
        if (seg_ok(f, seg_tag(cx.call))) break;
        if (give_up(cx)) { dead = true; break; }
        if (last) __builtin_amdgcn_s_sleep(CFG_ESLEEP); else __builtin_amdgcn_s_sleep(CFG_ESLEEP2);
    }
    const int tile = seg >> 3, wv = seg & 7;
    const int row0 = (src >> 2) * 4096 + (tile >> 4) * 256 + (wv >> 1) * 64 + (lane >> 4) * 4;
    const int col0 = (src & 3) * 2048 + (tile & 15) * 128 + (wv & 1) * 64 + (lane & 7) * 8 + ((lane >> 3) & 1) * 4;
    char* o = p.out + ((size_t)row0 * NG + col0) * 2;
    if (dead) {
        const u32x2 nan2 = {0x7FC07FC0u, 0x7FC07FC0u};
#pragma unroll 1
        for (int i = 0; i < 16; ++i) *(u32x2*)(o + (size_t)((i >> 2) * 16 + (i & 3)) * (NG * 2)) = nan2;
        return;
    }
    const unsigned lb = ((f >> 8) & 511u) * 16u;
    const char* sp = ws + WS_ORECV + ((size_t)src * NSEG + seg) * SEG_BYTES;
    const __amdgpu_buffer_rsrc_t rs = mk_rsrc(sp, seg_len(f));
    const u32x2 mk = __builtin_amdgcn_raw_buffer_load_b64(rs, lane * 8, 0, AUX_AGT);
    if ((f >> 25) & 1u) {
        u32x4 r[8];
#pragma unroll
        for (int j = 0; j < 8; ++j) if (j * 512u < lb) r[j] = __builtin_amdgcn_raw_buffer_load_b128(rs, 512 + lane * 16 + j * 1024, 0, AUX_AGT);
#pragma unroll
        for (int j = 0; j < 8; ++j) if (j * 512u < lb) *(u32x4*)(wl + lane * 16 + j * 1024) = r[j];
    } else {
        u32x4 L[4], E[3];
        unsigned W[4];
        const unsigned oc = 512u + lb, oe = oc + ((lb / 4u + 15u) & ~15u), eb = (f & 255u) * 16u;
#pragma unroll
        for (int j = 0; j < 4; ++j)
            if (j * 1024u < lb) {
                L[j] = __builtin_amdgcn_raw_buffer_load_b128(rs, 512 + lane * 16 + j * 1024, 0, AUX_AGT);
                W[j] = __builtin_amdgcn_raw_buffer_load_b32(rs, j * 1024u + lane * 16u < lb ? oc + lane * 4 + j * 256 : OOB, 0, AUX_AGT);
            }
#pragma unroll
        for (int j = 0; j < 3; ++j) if (j * 1024u < eb) E[j] = __builtin_amdgcn_raw_buffer_load_b128(rs, oe + lane * 16 + j * 1024, 0, AUX_AGT);
#pragma unroll
        for (int j = 0; j < 3; ++j) if (j * 1024u < eb) *(u32x4*)(wl + WL_ESC + lane * 16 + j * 1024) = E[j];
        const unsigned bs = ((f >> 17) & 0xFFu) * 0x01010101u;
        unsigned ebase = 0;
        char* xp = wl + WL_EXP + lane * 16;
#pragma unroll
        for (int j = 0; j < 4; ++j)
            if (j * 1024u < lb) {
                u32x4 X;
                unsigned b0 = 0;
#pragma unroll
                for (int g = 0; g < 4; ++g) {
                    const unsigned c1 = (L[j][g] >> 7) & 0x01010101u;
                    X[g] = (((W[j] >> (2 * g)) & 0x03030303u) << 1) + c1 + bs;
                    b0 |= c1 << (2 * g);
                }
                unsigned e = W[j] & (W[j] >> 1) & b0, sum;
                unsigned off = ebase + wave_scan<5>(__builtin_popcount(e), sum);
                ebase += sum;
                *(u32x4*)xp = X;
                __builtin_amdgcn_fence(__ATOMIC_RELEASE, "wavefront");
                __builtin_amdgcn_wave_barrier();
                while (e) {
                    const unsigned pb = __builtin_ctz(e);
                    xp[((pb >> 1) & 3u) * 4 + (pb >> 3)] = wl[WL_ESC + off];
                    ++off;
                    e &= e - 1u;
                }
                X = *(const u32x4*)xp;
#pragma unroll
                for (int g2 = 0; g2 < 2; ++g2) {
                    u32x4 w4;
#pragma unroll
                    for (int q = 0; q < 2; ++q) {
                        const int g = 2 * g2 + q;
                        const unsigned lo = bfi(0x7F7F7F7Fu, L[j][g], X[g] << 7), hi = (X[g] >> 1) & 0x7F7F7F7Fu;
                        w4[2 * q] = __builtin_amdgcn_perm(hi, lo, 0x05010400u);
                        w4[2 * q + 1] = __builtin_amdgcn_perm(hi, lo, 0x07030602u);
                    }
                    *(u32x4*)(wl + j * 2048 + lane * 32 + g2 * 16) = w4;
                }
            }
    }
    __builtin_amdgcn_fence(__ATOMIC_RELEASE, "wavefront");
    __builtin_amdgcn_wave_barrier();
    const unsigned short* vals = (const unsigned short*)wl;
    unsigned base = 0;
#pragma unroll
    for (int i = 0; i < 16; ++i) {
        unsigned e[4];
#pragma unroll
        for (int f4 = 0; f4 < 4; ++f4) {
            const unsigned mlo = __builtin_amdgcn_readlane(mk[0], i * 4 + f4), mhi = __builtin_amdgcn_readlane(mk[1], i * 4 + f4);
            const unsigned idx = __builtin_amdgcn_mbcnt_hi(mhi, __builtin_amdgcn_mbcnt_lo(mlo, 0)) + base;
            const u64 m = ((u64)mhi << 32) | mlo;
            const unsigned val = vals[idx];
            e[f4] = __builtin_amdgcn_inverse_ballot_w64(m) ? val : 0u;
            base += __builtin_popcountll(m);
        }
        const u32x2 ov = {e[0] | (e[1] << 16), e[2] | (e[3] << 16)};
        st_out(o + (size_t)((i >> 2) * 16 + (i & 3)) * (NG * 2), ov);
    }
    __builtin_amdgcn_fence(__ATOMIC_RELEASE, "wavefront");
    __builtin_amdgcn_wave_barrier();
}

__device__ __forceinline__ unsigned bf16_rne(float f) {
    const unsigned u = __float_as_uint(f);
    return (u + 0x7FFFu + ((u >> 16) & 1u)) >> 16;
}

__device__ __forceinline__ void role_gemm(const Params& p, const Ctx& cx, int ti, int tj, int wv, int lane, char* lds) {
    const char* ws = p.ws[p.rank];
    const unsigned ldsbase = (unsigned)(uintptr_t)(LAS char*)lds;
    const unsigned ctl = ldsbase + LDS_CTL;
    const int q = wv >> 1, h = wv & 1;
    const unsigned lo_a = wv * 4096 + lane * 16, lo_b = wv * 2048 + lane * 16;
    const char* ga = ws + WS_XA + ti * 32768;
    const char* gb = ws + WS_WB + tj * 16384;
    const char* ga_own = ws + WS_PKX + (size_t)((p.rank >> 2) * 16 + ti) * 32768;
    const char* gb_own = ws + WS_PKW + (size_t)((p.rank & 3) * 16 + tj) * 16384;
    const int fr = lane & 15, kq = lane >> 4;
    const int offA0 = q * 8192 + fr * 128 + ((kq ^ (fr >> 1)) * 16);
    const int offA1 = q * 8192 + fr * 128 + (((4 + kq) ^ (fr >> 1)) * 16);
    const int offB = 32768 + h * 8192 + (fr >> 3) * 4096 + kq * 128 + (fr & 7) * 16;

    f32x4 acc[4][4];
#pragma unroll
    for (int m = 0; m < 4; ++m)
#pragma unroll
        for (int n = 0; n < 4; ++n) acc[m][n] = (f32x4){0.f, 0.f, 0.f, 0.f};

    int issued = 0, islot = 0;
    unsigned rdy = 0, cnt = 0;
    auto issue = [&]() {
        const int slot = seq_slot(issued, ti >= 8);
        const bool own = (slot & 7) == 0;
        const char* a = (own ? ga_own + (size_t)slot * (16 * 8192) : ga + (size_t)slot * (16 * 32768)) + lo_a;
        const char* b = (own ? gb_own + (size_t)slot * (16 * 8192) : gb + (size_t)slot * (16 * 16384)) + lo_b;
        char* la = lds + islot * STAGE_BYTES + wv * 4096;
        char* lb = lds + islot * STAGE_BYTES + 32768 + wv * 2048;
        __builtin_amdgcn_global_load_lds((const GAS void*)a, (LAS void*)la, 16, 0, AUX_AGT);
        __builtin_amdgcn_global_load_lds((const GAS void*)a, (LAS void*)la, 16, 1024, AUX_AGT);
        __builtin_amdgcn_global_load_lds((const GAS void*)a, (LAS void*)la, 16, 2048, AUX_AGT);
        __builtin_amdgcn_global_load_lds((const GAS void*)a, (LAS void*)la, 16, 3072, AUX_AGT);
        __builtin_amdgcn_global_load_lds((const GAS void*)b, (LAS void*)lb, 16, 0, AUX_AGT);
        __builtin_amdgcn_global_load_lds((const GAS void*)b, (LAS void*)lb, 16, 1024, AUX_AGT);
        ++issued;
        islot = (islot == 2) ? 0 : islot + 1;
    };
    auto poll = [&]() {
        u32x2 v;
        asm volatile("ds_read_b64 %0, %1\n\ts_waitcnt lgkmcnt(0)" : "=v"(v) : "v"(ctl) : "memory");
        cnt = __builtin_amdgcn_readfirstlane(v[0]);
        rdy = __builtin_amdgcn_readfirstlane(v[1]);
    };
    auto arrive = [&]() {
        if (lane == 0) asm volatile("ds_add_u32 %0, %1" :: "v"(ctl), "v"(1u) : "memory");
    };
    auto wait_bar = [&](unsigned target) {
        poll();
        while (cnt < target) { __builtin_amdgcn_s_sleep(1); poll(); }
    };
    auto need = [&](int upto) {
        while (issued < upto) {
            while ((int)rdy <= issued) { __builtin_amdgcn_s_sleep(2); poll(); }
            issue();
        }
    };

    need(1);
    if ((int)rdy > issued) issue();
    if (issued == 2) asm volatile("s_waitcnt vmcnt(6)" ::: "memory"); else wait_vm0();
    arrive();
    wait_bar(8);

    bf16x8 a0[4], b0[4], a1[4], b1[4];
    int cslot = 0;
    {
        const char* s = lds;
#pragma unroll
        for (int m = 0; m < 4; ++m) a0[m] = *(const bf16x8*)(s + offA0 + m * 2048);
#pragma unroll
        for (int n = 0; n < 4; ++n) b0[n] = *(const bf16x8*)(s + offB + n * 1024);
    }
#pragma unroll 1
    for (int t = 0; t < NPOS; ++t) {
        const char* s = lds + cslot * STAGE_BYTES;
        cslot = (cslot == 2) ? 0 : cslot + 1;
        const char* sn = lds + cslot * STAGE_BYTES;
        {
            const int lim = (t + 3 < NPOS) ? t + 3 : NPOS;
            if (issued < lim && issued < (int)rdy) issue();
            if (issued < lim && issued < (int)rdy) issue();
        }
        __builtin_amdgcn_s_setprio(1);
#pragma unroll
        for (int n = 0; n < 4; ++n) acc[0][n] = __builtin_amdgcn_mfma_f32_16x16x32_bf16(a0[0], b0[n], acc[0][n], 0, 0, 0);
        __builtin_amdgcn_sched_barrier(0);
#pragma unroll
        for (int m = 0; m < 4; ++m) a1[m] = *(const bf16x8*)(s + offA1 + m * 2048);
#pragma unroll
        for (int n = 0; n < 4; ++n) b1[n] = *(const bf16x8*)(s + offB + 512 + n * 1024);
        __builtin_amdgcn_sched_barrier(0);
#pragma unroll
        for (int m = 1; m < 4; ++m)
#pragma unroll
            for (int n = 0; n < 4; ++n) acc[m][n] = __builtin_amdgcn_mfma_f32_16x16x32_bf16(a0[m], b0[n], acc[m][n], 0, 0, 0);
        __builtin_amdgcn_s_setprio(0);
        __builtin_amdgcn_sched_barrier(0);
        if (t + 1 < NPOS) {
            need(t + 2);
            if (issued == t + 3) asm volatile("s_waitcnt vmcnt(6) lgkmcnt(0)" ::: "memory");
            else asm volatile("s_waitcnt vmcnt(0) lgkmcnt(0)" ::: "memory");
        } else {
            asm volatile("s_waitcnt lgkmcnt(0)" ::: "memory");
        }
        arrive();
        __builtin_amdgcn_sched_barrier(0);
        __builtin_amdgcn_s_setprio(1);
#pragma unroll
        for (int m = 0; m < 2; ++m)
#pragma unroll
            for (int n = 0; n < 4; ++n) acc[m][n] = __builtin_amdgcn_mfma_f32_16x16x32_bf16(a1[m], b1[n], acc[m][n], 0, 0, 0);
        __builtin_amdgcn_s_setprio(0);
        __builtin_amdgcn_sched_barrier(0);
        wait_bar(8u * (t + 2));
        if (t + 1 < NPOS) {
#pragma unroll
            for (int m = 0; m < 4; ++m) a0[m] = *(const bf16x8*)(sn + offA0 + m * 2048);
#pragma unroll
            for (int n = 0; n < 4; ++n) b0[n] = *(const bf16x8*)(sn + offB + n * 1024);
        }
        __builtin_amdgcn_s_setprio(1);
#pragma unroll
        for (int m = 2; m < 4; ++m)
#pragma unroll
            for (int n = 0; n < 4; ++n) acc[m][n] = __builtin_amdgcn_mfma_f32_16x16x32_bf16(a1[m], b1[n], acc[m][n], 0, 0, 0);
        __builtin_amdgcn_s_setprio(0);
        __builtin_amdgcn_sched_barrier(0);
    }
    if (wv == 0) STAMP(p, blockIdx.x, 3, lane);

    char* wl = lds + wv * WAVE_LDS;
    unsigned short* vals = (unsigned short*)wl;
    const bool bad = timed_out(cx);
    const int seg = ((ti * 16 + tj) << 3) + wv;
    u32x2 mk = {0u, 0u};
    u32x2 ov[16];
    unsigned base = 0, mx = 0;
#pragma unroll
    for (int i = 0; i < 16; ++i) {
        unsigned e[4];
#pragma unroll
        for (int f4 = 0; f4 < 4; ++f4) {
            const float v = acc[i >> 2][f4][i & 3];
            unsigned b = bf16_rne(v > 0.f ? v : 0.f);
            if (bad) b = 0x7FC0u;
            e[f4] = b;
            const u64 m = __ballot(b != 0u);
            const unsigned idx = mbcnt64(m) + base;
            if (b != 0u) vals[idx] = (unsigned short)b;
            base += __builtin_popcountll(m);
            if (lane == i * 4 + f4) { mk[0] = (unsigned)m; mk[1] = (unsigned)(m >> 32); }
        }
        ov[i] = (u32x2){e[0] | (e[1] << 16), e[2] | (e[3] << 16)};
        mx = pk_max(mx, pk_max(ov[i][0], ov[i][1]));
    }
    {
        unsigned ex[8];
#pragma unroll
        for (int i = 0; i < 8; ++i) { const unsigned w = ov[2 * i][i & 1] & 0xFFFFu; ex[i] = w ? w >> 7 : 999u; }
        const unsigned hb = pick_base(wave_pk_max(max(mx >> 16, mx & 0xFFFFu)) >> 7, ex);
        const unsigned lb = (base + 15u) & ~15u;
        if (lane < 16) vals[base + lane] = (unsigned short)(hb << 7);
        __builtin_amdgcn_fence(__ATOMIC_RELEASE, "wavefront");
        __builtin_amdgcn_wave_barrier();
        u32x4 L[4];
        unsigned W[4];
        unsigned ne = 0;
#pragma unroll
        for (int j = 0; j < 4; ++j)
            if (j * 1024u < lb) {
                unsigned w = 0, b0 = 0;
#pragma unroll
                for (int g = 0; g < 4; ++g) {
                    const u32x2 d = *(const u32x2*)(wl + j * 2048 + lane * 32 + g * 8);
                    const unsigned cd = code4(d[0], d[1], hb * 0x00010001u);
                    L[j][g] = bfi(0x7F7F7F7Fu, __builtin_amdgcn_perm(d[1], d[0], 0x06040200u), cd << 7);
                    w |= ((cd >> 1) & 0x03030303u) << (2 * g);
                    b0 |= (cd & 0x01010101u) << (2 * g);
                }
                if (j * 1024u + lane * 16u >= lb) w = 0;
                W[j] = w;
                unsigned e = w & (w >> 1) & b0, sum;
                unsigned off = ne + wave_scan<5>(__builtin_popcount(e), sum);
                ne += sum;
                while (e) {
                    const unsigned pb = __builtin_ctz(e);
                    wl[WL_ESC + off] = (char)(vals[j * 1024 + lane * 16 + ((pb >> 1) & 3u) * 4 + (pb >> 3)] >> 7);
                    ++off;
                    e &= e - 1u;
                }
            }
        __builtin_amdgcn_fence(__ATOMIC_RELEASE, "wavefront");
        __builtin_amdgcn_wave_barrier();
        const bool raw = ne * 4u > lb * 3u - (lb ? 128u : 0u) || ne > 3072u;
        const unsigned f = (seg_tag(cx.call) << 26) | (raw ? 1u << 25 : 0u) | (hb << 17) | ((lb >> 4) << 8) | (raw ? 0u : (ne + 15u) >> 4);
        const __amdgpu_buffer_rsrc_t rs = mk_rsrc(ws + WS_OSTG + (size_t)seg * SEG_BYTES, seg_len(f));
        __builtin_amdgcn_raw_buffer_store_b64(mk, rs, lane * 8, 0, AUX_SYS);
        if (raw) {
#pragma unroll
            for (int j = 0; j < 8; ++j)
                if (j * 512u < lb)
                    __builtin_amdgcn_raw_buffer_store_b128(*(const u32x4*)(wl + lane * 16 + j * 1024), rs, 512 + lane * 16 + j * 1024, 0, AUX_SYS);
        } else {
            const unsigned oc = 512u + lb, oe = oc + ((lb / 4u + 15u) & ~15u);
#pragma unroll
            for (int j = 0; j < 4; ++j)
                if (j * 1024u < lb) {
                    const bool in = j * 1024u + lane * 16u < lb;
                    __builtin_amdgcn_raw_buffer_store_b128(L[j], rs, in ? 512 + lane * 16 + j * 1024 : OOB, 0, AUX_SYS);
                    __builtin_amdgcn_raw_buffer_store_b32(W[j], rs, in ? oc + lane * 4 + j * 256 : OOB, 0, AUX_SYS);
                }
#pragma unroll
            for (int j = 0; j < 3; ++j)
                if (j * 1024u < ne)
                    __builtin_amdgcn_raw_buffer_store_b128(*(const u32x4*)(wl + WL_ESC + lane * 16 + j * 1024), rs,
                                                           j * 1024u + lane * 16u < ne ? oe + lane * 16 + j * 1024 : OOB, 0, AUX_SYS);
        }
        wait_vm0();
        if (lane == 0) st_sys((unsigned*)(ws + WS_SEGRDY) + seg, f);
    }
    {
        const int row0 = (p.rank >> 2) * 4096 + ti * 256 + q * 64 + (lane >> 4) * 4;
        const int col0 = (p.rank & 3) * 2048 + tj * 128 + h * 64 + (lane & 7) * 8 + ((lane >> 3) & 1) * 4;
        char* o = p.out + ((size_t)row0 * NG + col0) * 2;
#pragma unroll
        for (int i = 0; i < 16; ++i) st_out(o + (size_t)((i >> 2) * 16 + (i & 3)) * (NG * 2), ov[i]);
    }
    __builtin_amdgcn_fence(__ATOMIC_RELEASE, "wavefront");
    __builtin_amdgcn_wave_barrier();
    if (wv == 0) STAMP(p, blockIdx.x, 4, lane);
#pragma unroll 1
    for (int i = 0; i < 7; ++i) {
        const int e = seg + NSEG * i;
        expand_segment(p, cx, (p.rank + 1 + e % 7) & 7, e / 7, lane, wl, e >= NSEG * 7 - CFG_ELAST);
        if (wv == 0 && i == 0) STAMP(p, blockIdx.x, 9, lane);
    }
    if (wv == 7) STAMP(p, blockIdx.x, 5, lane);
}

}

__global__ void __launch_bounds__(NTHR) fused_gemm_ar(Params p) {
    extern __shared__ __attribute__((aligned(16))) char lds[];
    const int lane = threadIdx.x & 63;
    const int wave = __builtin_amdgcn_readfirstlane(threadIdx.x >> 6);
    const int wg = blockIdx.x;
    volatile LAS unsigned* ctl = (volatile LAS unsigned*)(LAS char*)(lds + LDS_CTL);
    char* ws = p.ws[p.rank];
    u32x4 v[8];
    if (wave >= 9) __builtin_amdgcn_s_setprio(3);
    const int pid = packer_id(wg, wave);
    if (pid >= 0) pack_load(p, pid, lane, v);
    if (threadIdx.x == 0) {
        const unsigned ticket = __hip_atomic_fetch_add((unsigned*)(ws + WS_TICKET), 1u, __ATOMIC_RELAXED, __HIP_MEMORY_SCOPE_AGENT);
        ctl[0] = 0u; ctl[1] = 0u; ctl[2] = ticket / NWG + 1u;
    }
    asm volatile("s_waitcnt lgkmcnt(0)" ::: "memory");
    __builtin_amdgcn_s_barrier();
    asm volatile("" ::: "memory");
    Ctx cx;
    cx.call = __builtin_amdgcn_readfirstlane(ctl[2]);
    cx.tmo = (unsigned*)(ws + WS_TMO);
    cx.t0 = __builtin_amdgcn_s_memrealtime();
    const int xcd = wg & 7, idx = wg >> 3;
    const int ti = (idx & 1) * 8 + (xcd & 1) * 4 + ((idx >> 1) & 3), tj = (xcd >> 1) * 4 + (idx >> 3);
    char* wl = lds + (wave < 8 ? wave * 4096 : LDS_WRK + (wave - 9) * 4096);
    if (wave == 9) STAMP(p, wg, 0, lane);
    if (pid >= 0) {
        role_packer(p, cx, pid, lane, v, wl);
        asm volatile("s_waitcnt lgkmcnt(0)" ::: "memory");
        if (wave == 0 || wave == 11) STAMP(p, wg, 7, lane);
    }
    if (wave < 8) role_gemm(p, cx, ti, tj, wave, lane, lds);
    else if (wave == 8) role_poller(p, cx, ti, tj, lane, lds);
    else role_worker(p, cx, wg, wave - 9, lane, wl);
}

extern "C" void kernel_launch(int rank, int nranks, ncclComm_t comm, hipStream_t stream,
                              void* const* d_in, const int* in_sizes, int n_in,
                              void* d_out, int out_size,
                              const int* in_dtypes, int out_dtype,
                              void* const* d_out_all, void* const* d_ws_all, size_t ws_bytes) {
    if (nranks != NR || n_in < 2 || ws_bytes < WS_END || (unsigned)rank >= (unsigned)NR) {
        fprintf(stderr, "unsupported configuration\n");
        return;
    }
    static bool attr_set[NR] = {};
    if (!attr_set[rank]) {
        HIP_CHECK(hipFuncSetAttribute((const void*)fused_gemm_ar, hipFuncAttributeMaxDynamicSharedMemorySize, LDS_BYTES));
        attr_set[rank] = true;
    }
    Params p;
    p.x = (const char*)d_in[0];
    p.w = (const char*)d_in[1];
    p.out = (char*)d_out;
    for (int i = 0; i < NR; ++i) p.ws[i] = (char*)d_ws_all[i];
    p.rank = rank;
    fused_gemm_ar<<<NWG, NTHR, LDS_BYTES, stream>>>(p);
    HIP_CHECK(hipGetLastError());
}
```

```cpp
#include <hip/hip_runtime.h>
#include <rccl/rccl.h>
#include <cstdio>
#include <cstdint>

#define HIP_CHECK(expr) do { hipError_t _e = (expr); if (_e != hipSuccess) { \
    fprintf(stderr, "HIP error %s at %s:%d\n", hipGetErrorString(_e), __FILE__, __LINE__); \
    return; } } while (0)

typedef short bf16x8 __attribute__((ext_vector_type(8)));
typedef float f32x4 __attribute__((ext_vector_type(4)));
typedef unsigned u32x4 __attribute__((ext_vector_type(4)));
typedef unsigned u32x2 __attribute__((ext_vector_type(2)));
typedef unsigned short us2 __attribute__((ext_vector_type(2)));
typedef unsigned long long u64;
#define LAS __attribute__((address_space(3)))
#define GAS __attribute__((address_space(1)))

namespace {

constexpr int NR = 8;
constexpr int NG = 8192;
constexpr int KSH = 1024;
constexpr int NPOS = 128;
constexpr int NWG = 256;
constexpr int NTHR = 768;
#define CFG_LAG 6
#define CFG_NPI 27
#define CFG_NPO 12
constexpr int LAG = CFG_LAG;
constexpr int NPI = CFG_NPI;
constexpr int NPO = CFG_NPO;
static_assert(NPI <= 34 && NPO <= 18, "comm waves: two per CU of group 1, one per CU of group 0");
constexpr int U_IN = 1536;
constexpr int U_OUT = 1024;
#define CFG_PSLEEP 64
#define CFG_ESLEEP 32
#define CFG_ESLEEP2 127
#define CFG_ELAST 512
#define CFG_RSLEEP 32
#define CFG_OWN0 6
#define CFG_EARLY 0
constexpr int NSEG = 2048;
constexpr int SEG_BYTES = 512 + 8192;
constexpr int CU_BYTES = 6144;
constexpr u64 DEADLINE_TICKS = 50000000ull;

constexpr size_t WS_TICKET = 0;
constexpr size_t WS_TMO = 128;
constexpr size_t WS_LINKCTR = 256;
constexpr size_t WS_XF = 8192;
constexpr size_t WS_WF = WS_XF + 16 * 128 * 16;
constexpr size_t WS_SEGRDY = WS_WF + 16 * 128 * 8;
constexpr size_t WS_RF = WS_SEGRDY + NSEG * 4;
constexpr size_t WS_MSEG = WS_RF + NR * NSEG * 4;
constexpr size_t WS_HW0 = WS_MSEG + 8 * NSEG * 4;
constexpr size_t WS_MW0 = WS_HW0 + 16 * 256 * 4;
constexpr size_t WS_HDR = WS_MW0 + 8 * 16 * 128 * 4;
constexpr size_t WS_DBG = WS_HDR + 16 * 256 * 64;
constexpr size_t WS_CTL_END = 2u << 20;
constexpr size_t WS_PKX = WS_CTL_END;
constexpr size_t WS_PKW = WS_PKX + (16u << 20);
constexpr size_t WS_CX = WS_PKW + (16u << 20);
constexpr size_t WS_CW = WS_CX + (12u << 20);
constexpr size_t WS_XA = WS_CW + (12u << 20);
constexpr size_t WS_WB = WS_XA + (size_t)NPOS * 16 * 32768;
constexpr size_t WS_OSTG = WS_WB + (size_t)NPOS * 16 * 16384;
constexpr size_t WS_ORECV = WS_OSTG + (size_t)NSEG * SEG_BYTES;
constexpr size_t WS_END = WS_ORECV + (size_t)NR * NSEG * SEG_BYTES;
static_assert(WS_DBG + 64 * 1024 <= WS_CTL_END, "control region");
static_assert(WS_END < 0x7FFF0000u, "32-bit workspace offsets");

constexpr int STAGE_BYTES = 49152;
constexpr int LDS_CTL = 3 * STAGE_BYTES;
constexpr int LDS_WRK = LDS_CTL + 64;
constexpr int LDS_BYTES = LDS_WRK + 3 * 4096;
constexpr int WAVE_LDS = 13312 + 256;
constexpr int WL_ESC = 8192 + 64;
constexpr int WL_EXP = WL_ESC + 4096 + 64;

struct Params {
    const char* x;
    const char* w;
    char* out;
    char* ws[NR];
    int rank;
};

constexpr int AUX_SYS = 17;
constexpr int AUX_AGT = 16;
constexpr unsigned OOB = 0x7FFFFF00u;

__device__ __forceinline__ __amdgpu_buffer_rsrc_t mk_rsrc(const void* p, unsigned bytes) {
    return __builtin_amdgcn_make_buffer_rsrc((void*)p, 0, bytes, 0x00020000);
}
__device__ __forceinline__ void st_sys(unsigned* p, unsigned v) {
    __hip_atomic_store(p, v, __ATOMIC_RELAXED, __HIP_MEMORY_SCOPE_SYSTEM);
}
__device__ __forceinline__ void st_agt(unsigned* p, unsigned v) {
    __hip_atomic_store(p, v, __ATOMIC_RELAXED, __HIP_MEMORY_SCOPE_AGENT);
}
__device__ __forceinline__ void wait_vm0() { asm volatile("s_waitcnt vmcnt(0)" ::: "memory"); __builtin_amdgcn_s_waitcnt(0x0F70); }
__device__ __forceinline__ void wait_vm4() { asm volatile("s_waitcnt vmcnt(4)" ::: "memory"); __builtin_amdgcn_s_waitcnt(0x0F74); }
__device__ __forceinline__ void wait_vm8() { asm volatile("s_waitcnt vmcnt(8)" ::: "memory"); __builtin_amdgcn_s_waitcnt(0x0F78); }
__device__ __forceinline__ unsigned mbcnt64(u64 m) {
    return __builtin_amdgcn_mbcnt_hi((unsigned)(m >> 32), __builtin_amdgcn_mbcnt_lo((unsigned)m, 0));
}
__device__ __forceinline__ unsigned pk_max(unsigned a, unsigned b) {
    return __builtin_bit_cast(unsigned, __builtin_elementwise_max(__builtin_bit_cast(us2, a), __builtin_bit_cast(us2, b)));
}
__device__ __forceinline__ unsigned pk_min(unsigned a, unsigned b) {
    return __builtin_bit_cast(unsigned, __builtin_elementwise_min(__builtin_bit_cast(us2, a), __builtin_bit_cast(us2, b)));
}
__device__ __forceinline__ unsigned bfi(unsigned m, unsigned a, unsigned b) { return (a & m) | (b & ~m); }
__device__ __forceinline__ unsigned pk_sub(unsigned a, unsigned b) {
    return __builtin_bit_cast(unsigned, (us2)(__builtin_bit_cast(us2, a) - __builtin_bit_cast(us2, b)));
}
__device__ __forceinline__ unsigned pk_sub_sat(unsigned a, unsigned b) {
    return __builtin_bit_cast(unsigned, __builtin_elementwise_sub_sat(__builtin_bit_cast(us2, a), __builtin_bit_cast(us2, b)));
}
__device__ __forceinline__ unsigned wave_pk_max(unsigned v) {
#pragma unroll
    for (int o = 32; o > 0; o >>= 1) v = pk_max(v, (unsigned)__shfl_xor((int)v, o, 64));
    return __builtin_amdgcn_readfirstlane(v);
}

struct Ctx {
    unsigned call;
    unsigned* tmo;
    u64 t0;
};
__device__ __forceinline__ bool give_up(const Ctx& c) {
    const u64 dt = __builtin_amdgcn_s_memrealtime() - c.t0;
    if (dt < 200000ull) return false;
    if (__hip_atomic_load(c.tmo, __ATOMIC_RELAXED, __HIP_MEMORY_SCOPE_AGENT) == c.call) return true;
    if (dt > DEADLINE_TICKS) {
        __hip_atomic_store(c.tmo, c.call, __ATOMIC_RELAXED, __HIP_MEMORY_SCOPE_AGENT);
        return true;
    }
    return false;
}
__device__ __forceinline__ bool timed_out(const Ctx& c) {
    return __hip_atomic_load(c.tmo, __ATOMIC_RELAXED, __HIP_MEMORY_SCOPE_AGENT) == c.call;
}

#define STAMP(p, wg, k, lane) do {} while (0)

__device__ __forceinline__ int seq_slot(int seq, bool g1) {
    if (g1) {
        if (seq < 16) return seq * 8;
        const int m = seq - 16;
        return (m / 7) * 8 + m % 7 + 1;
    }
    constexpr int F = CFG_OWN0, R = (16 - F) / 2;
    if (seq < F) return seq * 8;
    if (seq < F + 9 * R) {
        const int m = seq - F, c = m / 9, r = m % 9;
        return r < 7 ? c * 8 + r + 1 : (F + 2 * c + r - 7) * 8;
    }
    const int m = seq - F - 9 * R;
    return (R + m / 7) * 8 + m % 7 + 1;
}

__device__ __forceinline__ unsigned seg_tag(unsigned call) { return call & 63u; }
__device__ __forceinline__ bool seg_ok(unsigned f, unsigned tag) { return (f >> 26) == tag; }
__device__ __forceinline__ unsigned seg_len(unsigned f) {
    const unsigned lb = ((f >> 8) & 511u) * 16u;
    return (512u + (((f >> 25) & 1u) ? 2u * lb : lb + ((lb / 4u + 15u) & ~15u) + (f & 255u) * 16u) + 63u) & ~63u;
}
__device__ __forceinline__ unsigned pick_base(unsigned top, const unsigned (&ex)[8]) {
    unsigned t = top;
#pragma unroll 1
    for (int it = 0; it < 2 && t > 6u; ++it) {
        int diff = 0;
#pragma unroll
        for (int k = 0; k < 8; ++k)
            diff += __builtin_popcountll(__ballot(ex[k] == t)) - __builtin_popcountll(__ballot(ex[k] + 7u == t));
        if (diff >= 0) break;
        --t;
    }
    const unsigned b = t > 6u ? t - 6u : 0u;
    return b < 248u ? b : 248u;
}
__device__ __forceinline__ unsigned code4(unsigned d0, unsigned d1, unsigned base2) {
    const unsigned c0 = pk_min(pk_sub((d0 >> 7) & 0x00FF00FFu, base2), 0x00070007u);
    const unsigned c1 = pk_min(pk_sub((d1 >> 7) & 0x00FF00FFu, base2), 0x00070007u);
    return __builtin_amdgcn_perm(c1, c0, 0x06040200u);
}
template <int BITS>
__device__ __forceinline__ unsigned wave_scan(unsigned n, unsigned& sum) {
    unsigned pre = 0;
    sum = 0;
#pragma unroll
    for (int b = 0; b < BITS; ++b) {
        const u64 m = __ballot((n >> b) & 1u);
        pre += mbcnt64(m) << b;
        sum += (unsigned)__builtin_popcountll(m) << b;
    }
    return pre;
}

__device__ __forceinline__ unsigned hdr_tag(unsigned call) { return call & 0xFFFu; }

__device__ __forceinline__ void pack_load(const Params& p, int j, int lane, u32x4 (&v)[8]) {
    const int c = j >> 8, r = j & 255, un = r & 127;
    if (r < 128) {
        const char* src = p.w + (size_t)(c * 64 + (lane >> 3) * 8) * (NG * 2) + (size_t)(un * 64 + (lane & 7) * 8) * 2;
#pragma unroll
        for (int k = 0; k < 8; ++k) v[k] = *(const u32x4*)(src + (size_t)k * (NG * 2));
    } else {
        const int rr = lane >> 3, pp = lane & 7;
        const char* src = p.x + (size_t)(un * 64 + rr) * (KSH * 2) + c * 128;
        const int o0 = (pp ^ (rr >> 1)) * 16, o1 = (pp ^ (4 + (rr >> 1))) * 16;
#pragma unroll
        for (int k = 0; k < 8; ++k) v[k] = *(const u32x4*)(src + (size_t)k * 8 * (KSH * 2) + ((k & 1) ? o1 : o0));
    }
}

__device__ __forceinline__ void role_packer(const Params& p, const Ctx& cx, int h, int lane, u32x4 (&v)[8], char* wl) {
    const int rank = p.rank;
    char* ws = p.ws[rank];
    const unsigned tagw = hdr_tag(cx.call) << 20;
#pragma unroll 1
    for (int j = h; j < 4096;) {
        const int c = j >> 8, r = j & 255, un = r & 127;
        const int jn = j < 512 ? 4096 : j + 1024;
        unsigned* flag = nullptr;
        if (r < 128) {
            if ((un >> 5) == (rank & 3)) flag = (unsigned*)(ws + WS_WF) + (((un >> 1) & 15) * NPOS + c * 8) * 2 + (un & 1);
            u32x4 o[8];
#pragma unroll
            for (int cc = 0; cc < 8; ++cc)
#pragma unroll
                for (int d = 0; d < 4; ++d)
                    o[cc][d] = __builtin_amdgcn_perm(v[2 * d + 1][cc >> 1], v[2 * d][cc >> 1], (cc & 1) ? 0x07060302u : 0x05040100u);
#pragma unroll
            for (int cc = 0; cc < 8; ++cc) v[cc] = o[cc];
        } else {
            if ((un >> 6) == (rank >> 2)) flag = (unsigned*)(ws + WS_XF) + (((un >> 2) & 15) * NPOS + c * 8) * 4 + (un & 3);
        }
        unsigned mx = 0, mn = 0;
#pragma unroll
        for (int k = 0; k < 8; ++k)
#pragma unroll
            for (int d = 0; d < 4; ++d) {
                mx = pk_max(mx, v[k][d] & 0x7F807F80u);
                mn = pk_max(mn, pk_sub(0u, v[k][d] & 0x7F007F00u));
            }
        const unsigned red = wave_pk_max((max(mx >> 16, mx & 0xFFFFu) << 16) | max(mn >> 16, mn & 0xFFFFu));
        const __amdgpu_buffer_rsrc_t rc = mk_rsrc(ws + (r < 128 ? WS_CW : WS_CX) + (size_t)(c * 128 + un) * CU_BYTES, CU_BYTES);
        const __amdgpu_buffer_rsrc_t rh = mk_rsrc(ws + WS_HDR + (size_t)(c * 256 + r) * 64, 64);
        unsigned ex[8];
#pragma unroll
        for (int k = 0; k < 8; ++k) ex[k] = (v[k][k & 3] >> 7) & 0xFFu;
        const unsigned basea = pick_base(red >> 23, ex);
        unsigned C2[4] = {0u, 0u, 0u, 0u}, C1[2] = {0u, 0u};
#pragma unroll
        for (int hf = 0; hf < 2; ++hf)
#pragma unroll
            for (int g = 0; g < 8; ++g) {
                const u32x4 d = v[hf * 4 + (g >> 1)];
                const unsigned cd = code4(d[(g & 1) * 2], d[(g & 1) * 2 + 1], basea * 0x00010001u);
                C2[hf * 2 + (g >> 2)] |= ((cd >> 1) & 0x03030303u) << (2 * (g & 3));
                C1[hf] |= (cd & 0x01010101u) << (2 * (g & 3) + (g >> 2));
            }
        unsigned ne[2], eoff[2], E[4];
#pragma unroll
        for (int q = 0; q < 4; ++q) E[q] = C2[q] & (C2[q] >> 1) & (C1[q >> 1] >> (q & 1)) & 0x55555555u;
#pragma unroll
        for (int hf = 0; hf < 2; ++hf) eoff[hf] = wave_scan<6>(__builtin_popcount(E[2 * hf]) + __builtin_popcount(E[2 * hf + 1]), ne[hf]);
        unsigned hw0;
        bool raw = false;
        if (ne[0] <= 240u && ne[1] <= 240u) {
#pragma unroll
            for (int hf = 0; hf < 2; ++hf) {
                char* img = wl + 2048 + hf * 1024;
#pragma unroll
                for (int q = 0; q < 2; ++q) {
                    u32x4 B, X;
#pragma unroll
                    for (int g = 0; g < 4; ++g) {
                        const u32x4 d = v[hf * 4 + 2 * q + (g >> 1)];
                        const unsigned lo = __builtin_amdgcn_perm(d[(g & 1) * 2 + 1], d[(g & 1) * 2], 0x06040200u);
                        const unsigned hi = __builtin_amdgcn_perm(d[(g & 1) * 2 + 1], d[(g & 1) * 2], 0x07050301u);
                        B[g] = bfi(0x7F7F7F7Fu, lo, hi);
                        X[g] = bfi(0x01010101u, lo >> 7, hi << 1);
                    }
                    __builtin_amdgcn_raw_buffer_store_b128(B, rc, lane * 16 + hf * 3072 + q * 1024, 0, AUX_SYS);
                    *(u32x4*)(wl + lane * 32 + q * 16) = X;
                }
                *(unsigned*)(img + lane * 4) = C2[2 * hf];
                *(unsigned*)(img + 256 + lane * 4) = C2[2 * hf + 1];
                *(unsigned*)(img + 512 + lane * 4) = C1[hf];
                unsigned off = 768u + eoff[hf];
                u64 e = ((u64)E[2 * hf + 1] << 32) | E[2 * hf];
                while (e) {
                    const unsigned pb = __builtin_ctzll(e);
                    img[off] = wl[lane * 32 + (pb >> 5) * 16 + ((pb >> 1) & 3u) * 4 + ((pb >> 3) & 3u)];
                    ++off;
                    e &= e - 1ull;
                }
                __builtin_amdgcn_fence(__ATOMIC_RELEASE, "wavefront");
                __builtin_amdgcn_wave_barrier();
                __builtin_amdgcn_raw_buffer_store_b128(*(const u32x4*)(img + lane * 16), rc, hf * 3072 + 2048 + lane * 16, 0, AUX_SYS);
            }
            hw0 = basea | 0x4000u | (((ne[0] + 15u) >> 4) << 8) | (((ne[1] + 15u) >> 4) << 16);
        } else {
            const unsigned top = red >> 24;
            const unsigned base = top > 7u ? top - 7u : 0u;
            const unsigned lowest = ((0x10000u - (red & 0xFFFFu)) >> 8);
            const unsigned base2 = base * 0x01000100u;
#pragma unroll
            for (int hf = 0; hf < 2; ++hf) {
                u32x4 L[2], N;
#pragma unroll
                for (int jj = 0; jj < 4; ++jj) {
                    const u32x4 d = v[hf * 4 + jj];
                    unsigned nb[4];
#pragma unroll
                    for (int q = 0; q < 4; ++q) nb[q] = pk_sub_sat(d[q] & 0x7F007F00u, base2) | ((d[q] >> 4) & 0x08000800u);
                    L[jj >> 1][(jj & 1) * 2] = __builtin_amdgcn_perm(d[1], d[0], 0x06040200u);
                    L[jj >> 1][(jj & 1) * 2 + 1] = __builtin_amdgcn_perm(d[3], d[2], 0x06040200u);
                    N[jj] = __builtin_amdgcn_perm(nb[1], nb[0], 0x07050301u) | (__builtin_amdgcn_perm(nb[3], nb[2], 0x07050301u) << 4);
                }
                __builtin_amdgcn_raw_buffer_store_b128(L[0], rc, lane * 16 + hf * 3072, 0, AUX_SYS);
                __builtin_amdgcn_raw_buffer_store_b128(L[1], rc, lane * 16 + hf * 3072 + 1024, 0, AUX_SYS);
                __builtin_amdgcn_raw_buffer_store_b128(N, rc, lane * 16 + hf * 3072 + 2048, 0, AUX_SYS);
            }
            unsigned nesc = 0;
            if (lowest <= base) {
                unsigned r0 = 0, r1 = 0, n = 0;
#pragma unroll
                for (int k = 0; k < 8; ++k)
#pragma unroll
                    for (int e = 0; e < 8; ++e) {
                        const unsigned hb = (v[k][e >> 1] >> ((e & 1) ? 24 : 8)) & 0xFFu;
                        if ((hb & 0x7Fu) - 1u < base) { r1 = r0; r0 = ((unsigned)(k << 9 | e) << 8) | hb; ++n; }
                    }
                const u64 m1 = __ballot(n >= 1), m2 = __ballot(n >= 2);
                nesc = __builtin_popcountll(m1) + __builtin_popcountll(m2);
                raw = __ballot(n > 2) != 0ull || nesc > 15u;
                if (!raw) {
                    const unsigned lt = tagw | ((unsigned)lane << 11);
                    __builtin_amdgcn_raw_buffer_store_b32(r0 | lt, rh, n >= 1 ? 4 + mbcnt64(m1) * 4 : OOB, 0, AUX_SYS);
                    __builtin_amdgcn_raw_buffer_store_b32(r1 | lt, rh, n >= 2 ? 4 + ((unsigned)__builtin_popcountll(m1) + mbcnt64(m2)) * 4 : OOB, 0, AUX_SYS);
                } else nesc = 0;
            }
            hw0 = base | (nesc << 8) | (raw ? 0x8000u : 0u);
        }
        if (raw || flag) {
            const __amdgpu_buffer_rsrc_t rs = mk_rsrc(ws + (r < 128 ? WS_PKW : WS_PKX) + (size_t)(c * 128 + un) * 8192, 8192);
#pragma unroll
            for (int k = 0; k < 8; ++k) __builtin_amdgcn_raw_buffer_store_b128(v[k], rs, lane * 16 + k * 1024, 0, AUX_SYS);
        }
        asm volatile("" ::: "memory");
        if (jn < 4096) {
            pack_load(p, jn, lane, v);
            asm volatile("s_waitcnt vmcnt(8)" ::: "memory");
        } else wait_vm0();
        if (lane == 0) st_sys((unsigned*)(ws + WS_HW0) + c * 256 + r, tagw | hw0);
        if (flag && lane == 0) st_agt(flag, cx.call);
        j = jn;
    }
}

#define UNI(x) __builtin_amdgcn_readfirstlane(x)
#define LD_REMOTE(off, imm) __builtin_amdgcn_raw_buffer_load_b128(rrem, (off) + (imm), 0, AUX_SYS)
#define ST_LOCAL(v, off, imm) __builtin_amdgcn_raw_buffer_store_b128(v, rloc, (off) + (imm), 0, AUX_AGT)

struct InUnit { unsigned idx, dst, flag, info; bool end; };

__device__ __forceinline__ void decode_half(const u32x4 l0, const u32x4 l1, const u32x4 nn, unsigned hdr, unsigned h0, unsigned k,
                                            int lane, char* wl, u32x4 (&o)[4]) {
    const unsigned bs = (h0 & 0xFFu) * 0x01010101u;
    unsigned H[8];
    u32x4 la = l0, lb = l1;
    if (h0 & 0x4000u) {
        *(u32x4*)(wl + lane * 16) = nn;
        __builtin_amdgcn_fence(__ATOMIC_RELEASE, "wavefront");
        __builtin_amdgcn_wave_barrier();
        const unsigned c0 = *(const unsigned*)(wl + lane * 4), c1 = *(const unsigned*)(wl + 256 + lane * 4);
        const unsigned cb = *(const unsigned*)(wl + 512 + lane * 4);
        unsigned e0 = c0 & (c0 >> 1) & cb & 0x55555555u, e1 = c1 & (c1 >> 1) & (cb >> 1) & 0x55555555u, sum;
        unsigned off = 768u + wave_scan<6>(__builtin_popcount(e0) + __builtin_popcount(e1), sum);
#pragma unroll
        for (int g = 0; g < 8; ++g)
            H[g] = ((((g & 4 ? c1 : c0) >> (2 * (g & 3))) & 0x03030303u) << 1) + ((cb >> (2 * (g & 3) + (g >> 2))) & 0x01010101u) + bs;
        char* hp = wl + 1024 + lane * 32;
        *(u32x4*)hp = (u32x4){H[0], H[1], H[2], H[3]};
        *(u32x4*)(hp + 16) = (u32x4){H[4], H[5], H[6], H[7]};
        u64 e = ((u64)e1 << 32) | e0;
        while (e) {
            const unsigned w0 = *(const unsigned*)(wl + (off & ~3u)), w1 = *(const unsigned*)(wl + (off & ~3u) + 4);
            unsigned w = __builtin_amdgcn_alignbyte(w1, w0, off & 3u);
#pragma unroll
            for (int r = 0; r < 4; ++r)
                if (e) {
                    const unsigned pb = __builtin_ctzll(e);
                    hp[(pb >> 5) * 16 + ((pb >> 1) & 3u) * 4 + ((pb >> 3) & 3u)] = (char)w;
                    w >>= 8;
                    e &= e - 1ull;
                }
            off += 4;
        }
        __builtin_amdgcn_fence(__ATOMIC_RELEASE, "wavefront");
        __builtin_amdgcn_wave_barrier();
        const u32x4 xa = *(const u32x4*)hp, xb = *(const u32x4*)(hp + 16);
#pragma unroll
        for (int g = 0; g < 4; ++g) {
            H[g] = bfi(0x80808080u, l0[g], xa[g] >> 1);
            H[4 + g] = bfi(0x80808080u, l1[g], xb[g] >> 1);
            la[g] = bfi(0x7F7F7F7Fu, l0[g], xa[g] << 7);
            lb[g] = bfi(0x7F7F7F7Fu, l1[g], xb[g] << 7);
        }
    } else {
        const unsigned tlo = (bs + 0x03020100u) & 0xFFFFFF00u, thi = bs + 0x07060504u;
#pragma unroll
        for (int jj = 0; jj < 4; ++jj) {
            const unsigned n = nn[jj];
            H[2 * jj] = __builtin_amdgcn_perm(thi, tlo, n & 0x07070707u) | ((n << 4) & 0x80808080u);
            H[2 * jj + 1] = __builtin_amdgcn_perm(thi, tlo, (n >> 4) & 0x07070707u) | (n & 0x80808080u);
        }
        const int nesc = (h0 >> 8) & 15;
#pragma unroll 1
        for (int i = 1; i <= nesc; ++i) {
            const unsigned e = __builtin_amdgcn_readlane(hdr, i);
            if (((e >> 19) & 1u) != k) continue;
            const unsigned key = ((unsigned)lane == ((e >> 11) & 63u)) ? ((e >> 17) & 3u) * 2 + ((e >> 10) & 1u) : 8u;
            const unsigned sh = ((e >> 8) & 3u) * 8, msk = 0xFFu << sh, val = (e & 0xFFu) << sh;
#pragma unroll
            for (int q = 0; q < 8; ++q) H[q] = (key == (unsigned)q) ? ((H[q] & ~msk) | val) : H[q];
        }
    }
#pragma unroll
    for (int jj = 0; jj < 4; ++jj) {
        const unsigned p0 = (jj >> 1 ? lb : la)[(jj & 1) * 2], p1 = (jj >> 1 ? lb : la)[(jj & 1) * 2 + 1];
        o[jj][0] = __builtin_amdgcn_perm(H[2 * jj], p0, 0x05010400u);
        o[jj][1] = __builtin_amdgcn_perm(H[2 * jj], p0, 0x07030602u);
        o[jj][2] = __builtin_amdgcn_perm(H[2 * jj + 1], p1, 0x05010400u);
        o[jj][3] = __builtin_amdgcn_perm(H[2 * jj + 1], p1, 0x07030602u);
    }
}

__device__ __forceinline__ void role_pull_in(const Params& p, const Ctx& cx, int link, int lane, char* wl) {
    const int rank = p.rank;
    const __amdgpu_buffer_rsrc_t rloc = mk_rsrc(p.ws[rank], (unsigned)WS_END);
    const __amdgpu_buffer_rsrc_t rrem = mk_rsrc(p.ws[(rank + 1 + link) & 7], (unsigned)WS_END);
    const unsigned ctr_off = (lane == 0) ? (unsigned)WS_LINKCTR + link * 128 : OOB;
    const unsigned base = (cx.call - 1) * (unsigned)(U_IN + NPI);
    const unsigned l16 = lane * 16;
    const unsigned htag = hdr_tag(cx.call);

    auto ticket = [&]() -> unsigned { return (unsigned)__builtin_amdgcn_raw_ptr_buffer_atomic_add_i32(1, rloc, ctr_off, 0, AUX_AGT); };
    auto make = [&](unsigned tv) -> InUnit {
        const int n = (int)(UNI(tv) - base);
        InUnit u;
        int d, r;
        if (n < 64 * LAG) { d = n >> 6; r = n & 63; }
        else if (n < 64 * LAG + 96 * (16 - LAG)) { const int m = n - 64 * LAG; d = LAG + m / 96; r = m % 96; }
        else { const int m = n - 64 * LAG - 96 * (16 - LAG); d = 16 + (m >> 5); r = 64 + (m & 31); }
        const int c = (r >= 64) ? d - LAG : d;
        const int pos = c * 8 + 1 + link;
        const bool isw = r < 32;
        const int vv = isw ? r : r - 32;
        u.end = n >= U_IN;
        u.info = (unsigned)WS_MW0 + ((link * 16 + c) * 128 + r) * 4;
        u.idx = isw ? (unsigned)(c * 128 + (rank & 3) * 32 + vv) : (unsigned)(c * 128 + (rank >> 2) * 64 + vv) | 0x80000000u;
        u.dst = (isw ? (unsigned)WS_WB + pos * (16 * 16384) : (unsigned)WS_XA + pos * (16 * 32768)) + vv * 8192;
        u.flag = isw ? (unsigned)WS_WF + ((vv >> 1) * NPOS + pos) * 8 + (vv & 1) * 4
                     : (unsigned)WS_XF + ((vv >> 2) * NPOS + pos) * 16 + (vv & 3) * 4;
        return u;
    };
    auto load_hdr = [&](const InUnit& u) -> unsigned {
        return __builtin_amdgcn_raw_buffer_load_b32(rloc, (lane == 0 && !u.end) ? u.info : OOB, 0, AUX_AGT);
    };
    auto hdr_ok = [&](unsigned hv) -> bool {
        const unsigned w0 = UNI(hv);
        return (w0 >> 20) == htag && ((w0 & 0xC000u) != 0u || (w0 & 0xF00u) == 0u);
    };
    auto publish = [&](unsigned flag) {
        __builtin_amdgcn_raw_buffer_store_b32(cx.call, rloc, lane == 0 ? flag : OOB, 0, AUX_AGT);
    };
    auto store_half = [&](u32x4 (&C)[4], unsigned hdr, unsigned h0, unsigned dst, unsigned k) {
        if (!(h0 & 0x8000u)) {
            u32x4 o[4];
            decode_half(C[0], C[1], C[2], hdr, h0, k, lane, wl, o);
#pragma unroll
            for (int j = 0; j < 4; ++j) C[j] = o[j];
        }
        const unsigned a0 = dst + l16 + k * 4096u;
#pragma unroll
        for (int j = 0; j < 4; ++j) ST_LOCAL(C[j], a0, j * 1024);
    };
    auto src_of = [&](const InUnit& u, unsigned h0) -> unsigned {
        const bool x = (u.idx >> 31) != 0u;
        const unsigned ix = u.idx & 0x7FFFFFFFu;
        return ((h0 & 0x8000u) ? (x ? (unsigned)WS_PKX : (unsigned)WS_PKW) + ix * 8192u
                               : (x ? (unsigned)WS_CX : (unsigned)WS_CW) + ix * (unsigned)CU_BYTES) + l16;
    };
    auto fetch = [&](u32x4 (&N)[4], unsigned so0, unsigned h0, unsigned k) {
        const bool raw = (h0 & 0x8000u) != 0u;
        const unsigned so = so0 + k * (raw ? 4096u : 3072u);
        const unsigned third = (h0 & 0x4000u) ? 768u + ((h0 >> (8 + 8 * k)) & 15u) * 16u : 1024u;
        N[0] = LD_REMOTE(so, 0); N[1] = LD_REMOTE(so, 1024); N[2] = LD_REMOTE(l16 < third ? so : OOB, 2048);
        N[3] = LD_REMOTE(raw ? so : OOB, 3072);
    };
    auto wait_hdr = [&](const InUnit& u) -> unsigned {
        const unsigned ix = u.idx & 0x7FFFFFFFu;
        const unsigned hu = (ix >> 7) * 256 + (u.idx >> 31) * 128 + (ix & 127u);
        unsigned off = lane == 0 ? (unsigned)WS_HW0 + hu * 4 : OOB;
        for (;;) {
            const unsigned hv = __builtin_amdgcn_raw_buffer_load_b32(rrem, off, 0, AUX_SYS);
            const unsigned w0 = UNI(hv);
            const unsigned need = (w0 & 0xC000u) ? 1u : (2u << ((w0 >> 8) & 15u)) - 1u;
            if (((unsigned)__ballot((hv >> 20) == htag) & need) == need) return hv;
            if (give_up(cx)) return 0x8000u;
            if ((w0 >> 20) == htag) { if (lane > 0 && lane < 16) off = (unsigned)WS_HDR + hu * 64 + lane * 4; }
            else __builtin_amdgcn_s_sleep(8);
        }
    };

    InUnit u = make(ticket());
    if (u.end) return;
    unsigned t = ticket();
    unsigned hdr = wait_hdr(u);
    unsigned h0 = UNI(hdr);
    unsigned so = src_of(u, h0);
    u32x4 A[4], B[4];
    fetch(A, so, h0, 0);
    wait_vm0();
    unsigned pflag = OOB;
    for (;;) {
        asm volatile("" : "+v"(A[0]), "+v"(A[1]), "+v"(A[2]), "+v"(A[3]), "+v"(t));
        fetch(B, so, h0, 1);
        __builtin_amdgcn_sched_barrier(0);
        const InUnit un = make(t);
        unsigned info = load_hdr(un);
        store_half(A, hdr, h0, u.dst, 0);
        wait_vm4();
        asm volatile("" : "+v"(B[0]), "+v"(B[1]), "+v"(B[2]), "+v"(B[3]), "+v"(info));
        if (un.end || !hdr_ok(info)) {
            publish(pflag);
            store_half(B, hdr, h0, u.dst, 1);
            wait_vm0();
            publish(u.flag);
            pflag = OOB;
            if (un.end) return;
            t = ticket();
            hdr = wait_hdr(un);
            h0 = UNI(hdr);
            so = src_of(un, h0);
            fetch(A, so, h0, 0);
            wait_vm0();
        } else {
            const unsigned hn0 = UNI(info);
            so = src_of(un, hn0);
            fetch(A, so, hn0, 0);
            __builtin_amdgcn_sched_barrier(0);
            publish(pflag);
            t = ticket();
            store_half(B, hdr, h0, u.dst, 1);
            wait_vm4();
            pflag = u.flag;
            hdr = info; h0 = hn0;
        }
        u = un;
    }
}

__device__ __forceinline__ void role_pull_out(const Params& p, const Ctx& cx, int link, int lane) {
    const int rank = p.rank, src = (rank + 1 + link) & 7;
    const __amdgpu_buffer_rsrc_t rloc = mk_rsrc(p.ws[rank], (unsigned)WS_END);
    const __amdgpu_buffer_rsrc_t rrem = mk_rsrc(p.ws[src], (unsigned)WS_END);
    const unsigned ctr_off = (lane == 0) ? (unsigned)WS_LINKCTR + (7 + link) * 128 : OOB;
    const unsigned base = (cx.call - 1) * (unsigned)(U_OUT + NPO);
    const unsigned tag = seg_tag(cx.call);
    const unsigned l16 = lane * 16;
    bool dead = false;
    {
        const unsigned ibase = (cx.call - 1) * (unsigned)(U_IN + NPI);
        for (;;) {
            const unsigned v = UNI(__builtin_amdgcn_raw_buffer_load_b32(rloc, lane == 0 ? (unsigned)WS_LINKCTR + link * 128 : OOB, 0, AUX_AGT));
            const int rem = U_IN - CFG_EARLY - (int)(v - ibase);
            if (rem <= 0) break;
            if (give_up(cx)) { dead = true; break; }
            for (int k = rem >> 6; k > 0; --k) __builtin_amdgcn_s_sleep(100);
            __builtin_amdgcn_s_sleep(16);
        }
        STAMP(p, blockIdx.x, 1, lane);
    }
    auto ticket = [&]() -> unsigned { return (unsigned)__builtin_amdgcn_raw_ptr_buffer_atomic_add_i32(1, rloc, ctr_off, 0, AUX_AGT); };
    auto load_words = [&](unsigned n) -> unsigned {
        return __builtin_amdgcn_raw_buffer_load_b32(rloc, (lane < 2 && n < (unsigned)U_OUT) ? (unsigned)WS_MSEG + (link * NSEG + 2 * n + lane) * 4 : OOB, 0, AUX_AGT);
    };
    unsigned pn = U_OUT, qn = U_OUT, pw = 0, qw = 0;
    auto publish = [&]() {
        __builtin_amdgcn_raw_buffer_store_b32(pw, rloc, (lane < 2 && pn < (unsigned)U_OUT) ? (unsigned)WS_RF + (src * NSEG + 2 * pn + lane) * 4 : OOB, 0, AUX_AGT);
        pn = qn; pw = qw; qn = U_OUT;
    };
    auto store_unit = [&](u32x4 (&C)[8], unsigned n, unsigned w, unsigned len0, unsigned len1) {
        const unsigned d0 = (unsigned)WS_ORECV + (src * NSEG + 2 * n) * (unsigned)SEG_BYTES + l16, d1 = d0 + SEG_BYTES;
#pragma unroll
        for (int j = 0; j < 4; ++j) ST_LOCAL(C[j], l16 + j * 1024u < len0 ? d0 : OOB, j * 1024);
#pragma unroll
        for (int j = 0; j < 4; ++j) ST_LOCAL(C[4 + j], l16 + j * 1024u < len1 ? d1 : OOB, j * 1024);
        qn = n; qw = w;
    };
    auto rest = [&](unsigned sg, unsigned len) {
#pragma unroll 1
        for (unsigned o = 4096; o < len; o += 1024) {
            const bool in = o + l16 < len;
            const u32x4 v = LD_REMOTE(in ? (unsigned)WS_OSTG + sg * (unsigned)SEG_BYTES + o + l16 : OOB, 0);
            ST_LOCAL(v, in ? (unsigned)WS_ORECV + (src * NSEG + sg) * (unsigned)SEG_BYTES + o + l16 : OOB, 0);
        }
    };
    unsigned nn = UNI(ticket()) - base;
    if (nn >= (unsigned)U_OUT) return;
    unsigned t = ticket();
    unsigned info = load_words(nn), cn = U_OUT, cw = 0, cl0 = 0, cl1 = 0;
    wait_vm0();
    u32x4 A[8], B[8];
#pragma unroll
    for (int j = 0; j < 8; ++j) A[j] = (u32x4){0u, 0u, 0u, 0u};
#define OUT_STEP(C, N) do { \
        asm volatile("" : "+v"(C[0]), "+v"(C[1]), "+v"(C[2]), "+v"(C[3]), "+v"(C[4]), "+v"(C[5]), "+v"(C[6]), "+v"(C[7]), "+v"(t), "+v"(info)); \
        unsigned w0_ = UNI(info), w1_ = __builtin_amdgcn_readlane(info, 1); \
        if (!seg_ok(w0_, tag) || !seg_ok(w1_, tag) || nn >= (unsigned)U_OUT) {        \
            publish(); \
            store_unit(C, cn, cw, cl0, cl1); cn = U_OUT; cl0 = cl1 = 0; \
            wait_vm0(); publish(); publish(); \
            if (nn >= (unsigned)U_OUT) return; \
            for (;;) { \
                info = load_words(nn); \
                w0_ = UNI(info); w1_ = __builtin_amdgcn_readlane(info, 1); \
                if (seg_ok(w0_, tag) && seg_ok(w1_, tag)) break; \
                if (dead || give_up(cx)) { dead = true; w0_ = w1_ = info = (tag ^ 1u) << 26; break; } \
                __builtin_amdgcn_s_sleep(8); \
            } \
        } \
        const unsigned wv_ = info, l0_ = seg_len(w0_), l1_ = seg_len(w1_); \
        const unsigned s0_ = (unsigned)WS_OSTG + 2 * nn * (unsigned)SEG_BYTES + l16, s1_ = s0_ + SEG_BYTES; \
        _Pragma("unroll") for (int j = 0; j < 4; ++j) N[j] = LD_REMOTE(l16 + j * 1024u < l0_ ? s0_ : OOB, j * 1024); \
        _Pragma("unroll") for (int j = 0; j < 4; ++j) N[4 + j] = LD_REMOTE(l16 + j * 1024u < l1_ ? s1_ : OOB, j * 1024); \
        __builtin_amdgcn_sched_barrier(0); \
        publish(); \
        if (l0_ > 4096u) rest(2 * nn, l0_); \
        if (l1_ > 4096u) rest(2 * nn + 1, l1_); \
        const unsigned n2_ = UNI(t) - base; \
        if (n2_ < (unsigned)U_OUT) t = ticket(); \
        info = load_words(n2_); \
        store_unit(C, cn, cw, cl0, cl1); \
        wait_vm8(); \
        cn = nn; cw = wv_; cl0 = l0_; cl1 = l1_; nn = n2_; \
    } while (0)
    for (;;) {
        OUT_STEP(A, B);
        OUT_STEP(B, A);
    }
}

__device__ __forceinline__ void role_relay(const Params& p, const Ctx& cx, int link, int lane) {
    const int rank = p.rank;
    const int src = (rank + 1 + link) & 7;
    const __amdgpu_buffer_rsrc_t rctl = mk_rsrc(p.ws[rank], (unsigned)WS_CTL_END);
    const __amdgpu_buffer_rsrc_t rinfo = mk_rsrc(p.ws[src], (unsigned)WS_CTL_END);
    const unsigned tag = seg_tag(cx.call);
    {
        const int sub = lane & 31;
        const unsigned woff = sub < 8 ? (unsigned)((rank & 3) * 32 + sub * 4) : (unsigned)(128 + (rank >> 2) * 64 + (sub - 8) * 4);
        const unsigned ht = hdr_tag(cx.call);
#pragma unroll 1
        for (int c = 0; c < 16;) {
            const int cc = c + (lane >> 5);
            const bool act = sub < 24 && cc < 16;
            const u32x4 hd = __builtin_amdgcn_raw_buffer_load_b128(rinfo, act ? (unsigned)WS_HW0 + (cc * 256 + woff) * 4 : OOB, 0, AUX_SYS);
            __builtin_amdgcn_raw_buffer_store_b128(hd, rctl, act ? (unsigned)WS_MW0 + ((link * 16 + cc) * 128 + sub * 4) * 4 : OOB, 0, AUX_AGT);
            bool ok = true;
#pragma unroll
            for (int q = 0; q < 4; ++q) ok = ok && (((hd[q] >> 20) - ht) & 0xFFFu) <= 1u;
            const u64 bad = __ballot(act && !ok);
            if ((unsigned)bad == 0u) { c += (bad == 0ull) ? 2 : 1; continue; }
            if (give_up(cx)) return;
            __builtin_amdgcn_s_sleep(16);
        }
    }
    int pos = 0;
    while (pos < NSEG) {
        const unsigned q = pos + lane * 4;
        const u32x4 v = __builtin_amdgcn_raw_buffer_load_b128(rinfo, q < NSEG ? (unsigned)WS_SEGRDY + q * 4 : OOB, 0, AUX_SYS);
        const bool any = seg_ok(v[0], tag) || seg_ok(v[1], tag) || seg_ok(v[2], tag) || seg_ok(v[3], tag);
        const bool all = seg_ok(v[0], tag) && seg_ok(v[1], tag) && seg_ok(v[2], tag) && seg_ok(v[3], tag);
        __builtin_amdgcn_raw_buffer_store_b128(v, rctl, (any && q < NSEG) ? (unsigned)WS_MSEG + (link * NSEG + q) * 4 : OOB, 0, AUX_AGT);
        const u64 m = __ballot(all && q < NSEG);
        const int n = (m == ~0ull) ? 64 : __builtin_ctzll(~m);
        pos += n * 4;
        if (n < 64) {
            if (give_up(cx)) return;
            __builtin_amdgcn_s_sleep(CFG_RSLEEP);
        }
    }
}

__device__ __forceinline__ int packer_id(int wg, int wave) {
    const int gi = (wg >> 4) * 8 + (wg & 7);
    const bool g1 = (wg & 8) != 0;
    if (wave < 8) return g1 ? 512 + wave * 128 + gi : -1;
    const int k = wave - 9;
    return (k < 0 || (g1 && k < 2)) ? -1 : gi * 4 + (g1 ? 3 : k);
}
__device__ __forceinline__ void role_worker(const Params& p, const Ctx& cx, int wg, int k, int lane, char* wl) {
    const int gi = (wg >> 4) * 8 + (wg & 7);
    if (wg & 8) {
        if (k < 2 && gi < 119) { if (gi / 7 + 17 * k < NPI) role_pull_in(p, cx, gi % 7, lane, wl); }
        else if (gi >= 121 && k == 0) role_relay(p, cx, gi - 121, lane);
    } else {
        if (k == 0 && gi < NPO * 7) role_pull_out(p, cx, gi % 7, lane);
    }
    if (k == 0) STAMP(p, wg, 2, lane);
}

__device__ __forceinline__ void role_poller(const Params& p, const Ctx& cx, int ti, int tj, int lane, char* lds) {
    const char* ws = p.ws[p.rank];
    const __amdgpu_buffer_rsrc_t rx = mk_rsrc(ws + WS_XF + ti * (NPOS * 16), NPOS * 16);
    const __amdgpu_buffer_rsrc_t rw = mk_rsrc(ws + WS_WF + tj * (NPOS * 8), NPOS * 8);
    volatile LAS unsigned* ready = (volatile LAS unsigned*)(LAS char*)(lds + LDS_CTL + 4);
    int pos = 0;
    while (pos < NPOS) {
        const int sq = pos + (lane & 15);
        const int q = sq < NPOS ? seq_slot(sq, ti >= 8) : NPOS;
        bool ok = false;
        if (lane < 16) {
            const u32x4 a = __builtin_amdgcn_raw_buffer_load_b128(rx, q * 16, 0, AUX_AGT);
            const u32x2 b = __builtin_amdgcn_raw_buffer_load_b64(rw, q * 8, 0, AUX_AGT);
            ok = a[0] == cx.call && a[1] == cx.call && a[2] == cx.call && a[3] == cx.call && b[0] == cx.call && b[1] == cx.call;
        }
        const u64 m = __ballot(ok);
        const int n = __builtin_ctzll(~m);
        if (n > 0) {
            if (pos <= (ti >= 8 ? 16 : CFG_OWN0) && pos + n > (ti >= 8 ? 16 : CFG_OWN0)) STAMP(p, blockIdx.x, 8, lane);
            pos += n;
            if (lane == 0) *ready = pos;
        } else {
            if (give_up(cx)) { pos = NPOS; if (lane == 0) *ready = NPOS; break; }
            __builtin_amdgcn_s_sleep(CFG_PSLEEP);
        }
    }
    STAMP(p, blockIdx.x, 6, lane);
}

#define CFG_OUTAUX 1
__device__ __forceinline__ void st_out(char* o, u32x2 v) {
    __builtin_nontemporal_store(v, (u32x2*)o);
}
__device__ __forceinline__ void expand_segment(const Params& p, const Ctx& cx, int src, int seg, int lane, char* wl, bool last) {
    const char* ws = p.ws[p.rank];
    const unsigned* rf = (const unsigned*)(ws + WS_RF) + src * NSEG + seg;
    unsigned f;
    bool dead = false;
    for (;;) {
        f = __builtin_amdgcn_readfirstlane(__hip_atomic_load(rf, __ATOMIC_RELAXED, __HIP_MEMORY_SCOPE_AGENT));
        if (seg_ok(f, seg_tag(cx.call))) break;
        if (give_up(cx)) { dead = true; break; }
        if (last) __builtin_amdgcn_s_sleep(CFG_ESLEEP); else __builtin_amdgcn_s_sleep(CFG_ESLEEP2);
    }
    const int tile = seg >> 3, wv = seg & 7;
    const int row0 = (src >> 2) * 4096 + (tile >> 4) * 256 + (wv >> 1) * 64 + (lane >> 4) * 4;
    const int col0 = (src & 3) * 2048 + (tile & 15) * 128 + (wv & 1) * 64 + (lane & 7) * 8 + ((lane >> 3) & 1) * 4;
    char* o = p.out + ((size_t)row0 * NG + col0) * 2;
    if (dead) {
        const u32x2 nan2 = {0x7FC07FC0u, 0x7FC07FC0u};
#pragma unroll 1
        for (int i = 0; i < 16; ++i) *(u32x2*)(o + (size_t)((i >> 2) * 16 + (i & 3)) * (NG * 2)) = nan2;
        return;
    }
    const unsigned lb = ((f >> 8) & 511u) * 16u;
    const char* sp = ws + WS_ORECV + ((size_t)src * NSEG + seg) * SEG_BYTES;
    const __amdgpu_buffer_rsrc_t rs = mk_rsrc(sp, seg_len(f));
    const u32x2 mk = __builtin_amdgcn_raw_buffer_load_b64(rs, lane * 8, 0, AUX_AGT);
    if ((f >> 25) & 1u) {
        u32x4 r[8];
#pragma unroll
        for (int j = 0; j < 8; ++j) if (j * 512u < lb) r[j] = __builtin_amdgcn_raw_buffer_load_b128(rs, 512 + lane * 16 + j * 1024, 0, AUX_AGT);
#pragma unroll
        for (int j = 0; j < 8; ++j) if (j * 512u < lb) *(u32x4*)(wl + lane * 16 + j * 1024) = r[j];
    } else {
        u32x4 L[4], E[3];
        unsigned W[4];
        const unsigned oc = 512u + lb, oe = oc + ((lb / 4u + 15u) & ~15u), eb = (f & 255u) * 16u;
#pragma unroll
        for (int j = 0; j < 4; ++j)
            if (j * 1024u < lb) {
                L[j] = __builtin_amdgcn_raw_buffer_load_b128(rs, 512 + lane * 16 + j * 1024, 0, AUX_AGT);
                W[j] = __builtin_amdgcn_raw_buffer_load_b32(rs, j * 1024u + lane * 16u < lb ? oc + lane * 4 + j * 256 : OOB, 0, AUX_AGT);
            }
#pragma unroll
        for (int j = 0; j < 3; ++j) if (j * 1024u < eb) E[j] = __builtin_amdgcn_raw_buffer_load_b128(rs, oe + lane * 16 + j * 1024, 0, AUX_AGT);
#pragma unroll
        for (int j = 0; j < 3; ++j) if (j * 1024u < eb) *(u32x4*)(wl + WL_ESC + lane * 16 + j * 1024) = E[j];
        const unsigned bs = ((f >> 17) & 0xFFu) * 0x01010101u;
        unsigned ebase = 0;
        char* xp = wl + WL_EXP + lane * 16;
#pragma unroll
        for (int j = 0; j < 4; ++j)
            if (j * 1024u < lb) {
                u32x4 X;
                unsigned b0 = 0;
#pragma unroll
                for (int g = 0; g < 4; ++g) {
                    const unsigned c1 = (L[j][g] >> 7) & 0x01010101u;
                    X[g] = (((W[j] >> (2 * g)) & 0x03030303u) << 1) + c1 + bs;
                    b0 |= c1 << (2 * g);
                }
                unsigned e = W[j] & (W[j] >> 1) & b0, sum;
                unsigned off = ebase + wave_scan<5>(__builtin_popcount(e), sum);
                ebase += sum;
                *(u32x4*)xp = X;
                __builtin_amdgcn_fence(__ATOMIC_RELEASE, "wavefront");
                __builtin_amdgcn_wave_barrier();
                while (e) {
                    const unsigned pb = __builtin_ctz(e);
                    xp[((pb >> 1) & 3u) * 4 + (pb >> 3)] = wl[WL_ESC + off];
                    ++off;
                    e &= e - 1u;
                }
                X = *(const u32x4*)xp;
#pragma unroll
                for (int g2 = 0; g2 < 2; ++g2) {
                    u32x4 w4;
#pragma unroll
                    for (int q = 0; q < 2; ++q) {
                        const int g = 2 * g2 + q;
                        const unsigned lo = bfi(0x7F7F7F7Fu, L[j][g], X[g] << 7), hi = (X[g] >> 1) & 0x7F7F7F7Fu;
                        w4[2 * q] = __builtin_amdgcn_perm(hi, lo, 0x05010400u);
                        w4[2 * q + 1] = __builtin_amdgcn_perm(hi, lo, 0x07030602u);
                    }
                    *(u32x4*)(wl + j * 2048 + lane * 32 + g2 * 16) = w4;
                }
            }
    }
    __builtin_amdgcn_fence(__ATOMIC_RELEASE, "wavefront");
    __builtin_amdgcn_wave_barrier();
    const unsigned short* vals = (const unsigned short*)wl;
    unsigned base = 0;
#pragma unroll
    for (int i = 0; i < 16; ++i) {
        unsigned e[4];
#pragma unroll
        for (int f4 = 0; f4 < 4; ++f4) {
            const unsigned mlo = __builtin_amdgcn_readlane(mk[0], i * 4 + f4), mhi = __builtin_amdgcn_readlane(mk[1], i * 4 + f4);
            const unsigned idx = __builtin_amdgcn_mbcnt_hi(mhi, __builtin_amdgcn_mbcnt_lo(mlo, 0)) + base;
            const u64 m = ((u64)mhi << 32) | mlo;
            const unsigned val = vals[idx];
            e[f4] = __builtin_amdgcn_inverse_ballot_w64(m) ? val : 0u;
            base += __builtin_popcountll(m);
        }
        const u32x2 ov = {e[0] | (e[1] << 16), e[2] | (e[3] << 16)};
        st_out(o + (size_t)((i >> 2) * 16 + (i & 3)) * (NG * 2), ov);
    }
    __builtin_amdgcn_fence(__ATOMIC_RELEASE, "wavefront");
    __builtin_amdgcn_wave_barrier();
}

__device__ __forceinline__ unsigned bf16_rne(float f) {
    const unsigned u = __float_as_uint(f);
    return (u + 0x7FFFu + ((u >> 16) & 1u)) >> 16;
}

__device__ __forceinline__ void role_gemm(const Params& p, const Ctx& cx, int ti, int tj, int wv, int lane, char* lds) {
    const char* ws = p.ws[p.rank];
    const unsigned ldsbase = (unsigned)(uintptr_t)(LAS char*)lds;
    const unsigned ctl = ldsbase + LDS_CTL;
    const int q = wv >> 1, h = wv & 1;
    const unsigned lo_a = wv * 4096 + lane * 16, lo_b = wv * 2048 + lane * 16;
    const char* ga = ws + WS_XA + ti * 32768;
    const char* gb = ws + WS_WB + tj * 16384;
    const char* ga_own = ws + WS_PKX + (size_t)((p.rank >> 2) * 16 + ti) * 32768;
    const char* gb_own = ws + WS_PKW + (size_t)((p.rank & 3) * 16 + tj) * 16384;
    const int fr = lane & 15, kq = lane >> 4;
    const int offA0 = q * 8192 + fr * 128 + ((kq ^ (fr >> 1)) * 16);
    const int offA1 = q * 8192 + fr * 128 + (((4 + kq) ^ (fr >> 1)) * 16);
    const int offB = 32768 + h * 8192 + (fr >> 3) * 4096 + kq * 128 + (fr & 7) * 16;

    f32x4 acc[4][4];
#pragma unroll
    for (int m = 0; m < 4; ++m)
#pragma unroll
        for (int n = 0; n < 4; ++n) acc[m][n] = (f32x4){0.f, 0.f, 0.f, 0.f};

    int issued = 0, islot = 0;
    unsigned rdy = 0, cnt = 0;
    auto issue = [&]() {
        const int slot = seq_slot(issued, ti >= 8);
        const bool own = (slot & 7) == 0;
        const char* a = (own ? ga_own + (size_t)slot * (16 * 8192) : ga + (size_t)slot * (16 * 32768)) + lo_a;
        const char* b = (own ? gb_own + (size_t)slot * (16 * 8192) : gb + (size_t)slot * (16 * 16384)) + lo_b;
        char* la = lds + islot * STAGE_BYTES + wv * 4096;
        char* lb = lds + islot * STAGE_BYTES + 32768 + wv * 2048;
        __builtin_amdgcn_global_load_lds((const GAS void*)a, (LAS void*)la, 16, 0, AUX_AGT);
        __builtin_amdgcn_global_load_lds((const GAS void*)a, (LAS void*)la, 16, 1024, AUX_AGT);
        __builtin_amdgcn_global_load_lds((const GAS void*)a, (LAS void*)la, 16, 2048, AUX_AGT);
        __builtin_amdgcn_global_load_lds((const GAS void*)a, (LAS void*)la, 16, 3072, AUX_AGT);
        __builtin_amdgcn_global_load_lds((const GAS void*)b, (LAS void*)lb, 16, 0, AUX_AGT);
        __builtin_amdgcn_global_load_lds((const GAS void*)b, (LAS void*)lb, 16, 1024, AUX_AGT);
        ++issued;
        islot = (islot == 2) ? 0 : islot + 1;
    };
    auto poll = [&]() {
        u32x2 v;
        asm volatile("ds_read_b64 %0, %1\n\ts_waitcnt lgkmcnt(0)" : "=v"(v) : "v"(ctl) : "memory");
        cnt = __builtin_amdgcn_readfirstlane(v[0]);
        rdy = __builtin_amdgcn_readfirstlane(v[1]);
    };
    auto arrive = [&]() {
        if (lane == 0) asm volatile("ds_add_u32 %0, %1" :: "v"(ctl), "v"(1u) : "memory");
    };
    auto wait_bar = [&](unsigned target) {
        poll();
        while (cnt < target) { __builtin_amdgcn_s_sleep(1); poll(); }
    };
    auto need = [&](int upto) {
        while (issued < upto) {
            while ((int)rdy <= issued) { __builtin_amdgcn_s_sleep(2); poll(); }
            issue();
        }
    };

    need(1);
    if ((int)rdy > issued) issue();
    if (issued == 2) asm volatile("s_waitcnt vmcnt(6)" ::: "memory"); else wait_vm0();
    arrive();
    wait_bar(8);

    bf16x8 a0[4], b0[4], a1[4], b1[4];
    int cslot = 0;
    {
        const char* s = lds;
#pragma unroll
        for (int m = 0; m < 4; ++m) a0[m] = *(const bf16x8*)(s + offA0 + m * 2048);
#pragma unroll
        for (int n = 0; n < 4; ++n) b0[n] = *(const bf16x8*)(s + offB + n * 1024);
    }
#define MMA4(A_, B_, m_) do { _Pragma("unroll") for (int n = 0; n < 4; ++n) \
        acc[m_][n] = __builtin_amdgcn_mfma_f32_16x16x32_bf16(A_[m_], B_[n], acc[m_][n], 0, 0, 0); \
        __builtin_amdgcn_sched_barrier(0); } while (0)
#define GLDS(g_, l_, o_) do { if (spread) __builtin_amdgcn_global_load_lds((const GAS void*)(g_), (LAS void*)(l_), 16, o_, AUX_AGT); \
        __builtin_amdgcn_sched_barrier(0); } while (0)
#pragma unroll 1
    for (int t = 0; t < NPOS; ++t) {
        const char* s = lds + cslot * STAGE_BYTES;
        cslot = (cslot == 2) ? 0 : cslot + 1;
        const char* sn = lds + cslot * STAGE_BYTES;
        if (issued == t + 1 && issued < NPOS && issued < (int)rdy) issue();
        const bool spread = issued == t + 2 && issued < NPOS && issued < (int)rdy;
        const int slot = seq_slot(spread ? issued : 0, ti >= 8);
        const bool own = (slot & 7) == 0;
        const char* pa = (own ? ga_own + (size_t)slot * (16 * 8192) : ga + (size_t)slot * (16 * 32768)) + lo_a;
        const char* pb = (own ? gb_own + (size_t)slot * (16 * 8192) : gb + (size_t)slot * (16 * 16384)) + lo_b;
        char* la = lds + islot * STAGE_BYTES + wv * 4096;
        char* lb = lds + islot * STAGE_BYTES + 32768 + wv * 2048;
        __builtin_amdgcn_sched_barrier(0);
        MMA4(a0, b0, 0);
#pragma unroll
        for (int m = 0; m < 4; ++m) a1[m] = *(const bf16x8*)(s + offA1 + m * 2048);
#pragma unroll
        for (int n = 0; n < 4; ++n) b1[n] = *(const bf16x8*)(s + offB + 512 + n * 1024);
        __builtin_amdgcn_sched_barrier(0);
        GLDS(pa, la, 0);
        MMA4(a0, b0, 1);
        GLDS(pa, la, 1024);
        MMA4(a0, b0, 2);
        GLDS(pa, la, 2048);
        MMA4(a0, b0, 3);
        if (t + 1 < NPOS) {
            need(t + 2);
            if (spread) asm volatile("s_waitcnt vmcnt(3) lgkmcnt(0)" ::: "memory");
            else asm volatile("s_waitcnt vmcnt(0) lgkmcnt(0)" ::: "memory");
        } else {
            asm volatile("s_waitcnt lgkmcnt(0)" ::: "memory");
        }
        arrive();
        __builtin_amdgcn_sched_barrier(0);
        MMA4(a1, b1, 0);
        GLDS(pa, la, 3072);
        MMA4(a1, b1, 1);
        wait_bar(8u * (t + 2));
        if (t + 1 < NPOS) {
#pragma unroll
            for (int m = 0; m < 4; ++m) a0[m] = *(const bf16x8*)(sn + offA0 + m * 2048);
#pragma unroll
            for (int n = 0; n < 4; ++n) b0[n] = *(const bf16x8*)(sn + offB + n * 1024);
        }
        __builtin_amdgcn_sched_barrier(0);
        GLDS(pb, lb, 0);
        MMA4(a1, b1, 2);
        GLDS(pb, lb, 1024);
        MMA4(a1, b1, 3);
        if (spread) { ++issued; islot = (islot == 2) ? 0 : islot + 1; }
    }
    if (wv == 0) STAMP(p, blockIdx.x, 3, lane);

    char* wl = lds + wv * WAVE_LDS;
    unsigned short* vals = (unsigned short*)wl;
    const bool bad = timed_out(cx);
    const int seg = ((ti * 16 + tj) << 3) + wv;
    u32x2 mk = {0u, 0u};
    u32x2 ov[16];
    unsigned base = 0, mx = 0;
#pragma unroll
    for (int i = 0; i < 16; ++i) {
        unsigned e[4];
#pragma unroll
        for (int f4 = 0; f4 < 4; ++f4) {
            const float v = acc[i >> 2][f4][i & 3];
            unsigned b = bf16_rne(v > 0.f ? v : 0.f);
            if (bad) b = 0x7FC0u;
            e[f4] = b;
            const u64 m = __ballot(b != 0u);
            const unsigned idx = mbcnt64(m) + base;
            if (b != 0u) vals[idx] = (unsigned short)b;
            base += __builtin_popcountll(m);
            if (lane == i * 4 + f4) { mk[0] = (unsigned)m; mk[1] = (unsigned)(m >> 32); }
        }
        ov[i] = (u32x2){e[0] | (e[1] << 16), e[2] | (e[3] << 16)};
        mx = pk_max(mx, pk_max(ov[i][0], ov[i][1]));
    }
    {
        unsigned ex[8];
#pragma unroll
        for (int i = 0; i < 8; ++i) { const unsigned w = ov[2 * i][i & 1] & 0xFFFFu; ex[i] = w ? w >> 7 : 999u; }
        const unsigned hb = pick_base(wave_pk_max(max(mx >> 16, mx & 0xFFFFu)) >> 7, ex);
        const unsigned lb = (base + 15u) & ~15u;
        if (lane < 16) vals[base + lane] = (unsigned short)(hb << 7);
        __builtin_amdgcn_fence(__ATOMIC_RELEASE, "wavefront");
        __builtin_amdgcn_wave_barrier();
        u32x4 L[4];
        unsigned W[4];
        unsigned ne = 0;
#pragma unroll
        for (int j = 0; j < 4; ++j)
            if (j * 1024u < lb) {
                unsigned w = 0, b0 = 0;
#pragma unroll
                for (int g = 0; g < 4; ++g) {
                    const u32x2 d = *(const u32x2*)(wl + j * 2048 + lane * 32 + g * 8);
                    const unsigned cd = code4(d[0], d[1], hb * 0x00010001u);
                    L[j][g] = bfi(0x7F7F7F7Fu, __builtin_amdgcn_perm(d[1], d[0], 0x06040200u), cd << 7);
                    w |= ((cd >> 1) & 0x03030303u) << (2 * g);
                    b0 |= (cd & 0x01010101u) << (2 * g);
                }
                if (j * 1024u + lane * 16u >= lb) w = 0;
                W[j] = w;
                unsigned e = w & (w >> 1) & b0, sum;
                unsigned off = ne + wave_scan<5>(__builtin_popcount(e), sum);
                ne += sum;
                while (e) {
                    const unsigned pb = __builtin_ctz(e);
                    wl[WL_ESC + off] = (char)(vals[j * 1024 + lane * 16 + ((pb >> 1) & 3u) * 4 + (pb >> 3)] >> 7);
                    ++off;
                    e &= e - 1u;
                }
            }
        __builtin_amdgcn_fence(__ATOMIC_RELEASE, "wavefront");
        __builtin_amdgcn_wave_barrier();
        const bool raw = ne * 4u > lb * 3u - (lb ? 128u : 0u) || ne > 3072u;
        const unsigned f = (seg_tag(cx.call) << 26) | (raw ? 1u << 25 : 0u) | (hb << 17) | ((lb >> 4) << 8) | (raw ? 0u : (ne + 15u) >> 4);
        const __amdgpu_buffer_rsrc_t rs = mk_rsrc(ws + WS_OSTG + (size_t)seg * SEG_BYTES, seg_len(f));
        __builtin_amdgcn_raw_buffer_store_b64(mk, rs, lane * 8, 0, AUX_SYS);
        if (raw) {
#pragma unroll
            for (int j = 0; j < 8; ++j)
                if (j * 512u < lb)
                    __builtin_amdgcn_raw_buffer_store_b128(*(const u32x4*)(wl + lane * 16 + j * 1024), rs, 512 + lane * 16 + j * 1024, 0, AUX_SYS);
        } else {
            const unsigned oc = 512u + lb, oe = oc + ((lb / 4u + 15u) & ~15u);
#pragma unroll
            for (int j = 0; j < 4; ++j)
                if (j * 1024u < lb) {
                    const bool in = j * 1024u + lane * 16u < lb;
                    __builtin_amdgcn_raw_buffer_store_b128(L[j], rs, in ? 512 + lane * 16 + j * 1024 : OOB, 0, AUX_SYS);
                    __builtin_amdgcn_raw_buffer_store_b32(W[j], rs, in ? oc + lane * 4 + j * 256 : OOB, 0, AUX_SYS);
                }
#pragma unroll
            for (int j = 0; j < 3; ++j)
                if (j * 1024u < ne)
                    __builtin_amdgcn_raw_buffer_store_b128(*(const u32x4*)(wl + WL_ESC + lane * 16 + j * 1024), rs,
                                                           j * 1024u + lane * 16u < ne ? oe + lane * 16 + j * 1024 : OOB, 0, AUX_SYS);
        }
        wait_vm0();
        if (lane == 0) st_sys((unsigned*)(ws + WS_SEGRDY) + seg, f);
    }
    {
        const int row0 = (p.rank >> 2) * 4096 + ti * 256 + q * 64 + (lane >> 4) * 4;
        const int col0 = (p.rank & 3) * 2048 + tj * 128 + h * 64 + (lane & 7) * 8 + ((lane >> 3) & 1) * 4;
        char* o = p.out + ((size_t)row0 * NG + col0) * 2;
#pragma unroll
        for (int i = 0; i < 16; ++i) st_out(o + (size_t)((i >> 2) * 16 + (i & 3)) * (NG * 2), ov[i]);
    }
    __builtin_amdgcn_fence(__ATOMIC_RELEASE, "wavefront");
    __builtin_amdgcn_wave_barrier();
    if (wv == 0) STAMP(p, blockIdx.x, 4, lane);
#pragma unroll 1
    for (int i = 0; i < 7; ++i) {
        const int e = seg + NSEG * i;
        expand_segment(p, cx, (p.rank + 1 + e % 7) & 7, e / 7, lane, wl, e >= NSEG * 7 - CFG_ELAST);
        if (wv == 0 && i == 0) STAMP(p, blockIdx.x, 9, lane);
    }
    if (wv == 7) STAMP(p, blockIdx.x, 5, lane);
}

}

__global__ void __launch_bounds__(NTHR) fused_gemm_ar(Params p) {
    extern __shared__ __attribute__((aligned(16))) char lds[];
    const int lane = threadIdx.x & 63;
    const int wave = __builtin_amdgcn_readfirstlane(threadIdx.x >> 6);
    const int wg = blockIdx.x;
    volatile LAS unsigned* ctl = (volatile LAS unsigned*)(LAS char*)(lds + LDS_CTL);
    char* ws = p.ws[p.rank];
    u32x4 v[8];
    if (wave >= 9) __builtin_amdgcn_s_setprio(3);
    const int pid = packer_id(wg, wave);
    if (pid >= 0) pack_load(p, pid, lane, v);
    if (threadIdx.x == 0) {
        const unsigned ticket = __hip_atomic_fetch_add((unsigned*)(ws + WS_TICKET), 1u, __ATOMIC_RELAXED, __HIP_MEMORY_SCOPE_AGENT);
        ctl[0] = 0u; ctl[1] = 0u; ctl[2] = ticket / NWG + 1u;
    }
    asm volatile("s_waitcnt lgkmcnt(0)" ::: "memory");
    __builtin_amdgcn_s_barrier();
    asm volatile("" ::: "memory");
    Ctx cx;
    cx.call = __builtin_amdgcn_readfirstlane(ctl[2]);
    cx.tmo = (unsigned*)(ws + WS_TMO);
    cx.t0 = __builtin_amdgcn_s_memrealtime();
    const int xcd = wg & 7, idx = wg >> 3;
    const int ti = (idx & 1) * 8 + (xcd & 1) * 4 + ((idx >> 1) & 3), tj = (xcd >> 1) * 4 + (idx >> 3);
    char* wl = lds + (wave < 8 ? wave * 4096 : LDS_WRK + (wave - 9) * 4096);
    if (wave == 9) STAMP(p, wg, 0, lane);
    if (pid >= 0) {
        role_packer(p, cx, pid, lane, v, wl);
        asm volatile("s_waitcnt lgkmcnt(0)" ::: "memory");
        if (wave == 0 || wave == 11) STAMP(p, wg, 7, lane);
    }
    if (wave < 8) role_gemm(p, cx, ti, tj, wave, lane, lds);
    else if (wave == 8) role_poller(p, cx, ti, tj, lane, lds);
    else role_worker(p, cx, wg, wave - 9, lane, wl);
}

extern "C" void kernel_launch(int rank, int nranks, ncclComm_t comm, hipStream_t stream,
                              void* const* d_in, const int* in_sizes, int n_in,
                              void* d_out, int out_size,
                              const int* in_dtypes, int out_dtype,
                              void* const* d_out_all, void* const* d_ws_all, size_t ws_bytes) {
    if (nranks != NR || n_in < 2 || ws_bytes < WS_END || (unsigned)rank >= (unsigned)NR) {
        fprintf(stderr, "unsupported configuration\n");
        return;
    }
    static bool attr_set[NR] = {};
    if (!attr_set[rank]) {
        HIP_CHECK(hipFuncSetAttribute((const void*)fused_gemm_ar, hipFuncAttributeMaxDynamicSharedMemorySize, LDS_BYTES));
        attr_set[rank] = true;
    }
    Params p;
    p.x = (const char*)d_in[0];
    p.w = (const char*)d_in[1];
    p.out = (char*)d_out;
    for (int i = 0; i < NR; ++i) p.ws[i] = (char*)d_ws_all[i];
    p.rank = rank;
    fused_gemm_ar<<<NWG, NTHR, LDS_BYTES, stream>>>(p);
    HIP_CHECK(hipGetLastError());
}
```

```cpp
#include <hip/hip_runtime.h>
#include <rccl/rccl.h>
#include <cstdio>
#include <cstdint>

#define HIP_CHECK(expr) do { hipError_t _e = (expr); if (_e != hipSuccess) { \
    fprintf(stderr, "HIP error %s at %s:%d\n", hipGetErrorString(_e), __FILE__, __LINE__); \
    return; } } while (0)

typedef short bf16x8 __attribute__((ext_vector_type(8)));
typedef float f32x4 __attribute__((ext_vector_type(4)));
typedef unsigned u32x4 __attribute__((ext_vector_type(4)));
typedef unsigned u32x2 __attribute__((ext_vector_type(2)));
typedef unsigned short us2 __attribute__((ext_vector_type(2)));
typedef unsigned long long u64;
#define LAS __attribute__((address_space(3)))
#define GAS __attribute__((address_space(1)))

namespace {

constexpr int NR = 8;
constexpr int NG = 8192;
constexpr int KSH = 1024;
constexpr int NPOS = 128;
constexpr int NWG = 256;
constexpr int NTHR = 768;
constexpr int LAG = 6;
constexpr int NPI = 27;
constexpr int NPO = 12;
static_assert(NPI <= 34 && NPO <= 18, "comm waves: two per CU of group 1, one per CU of group 0");
constexpr int U_IN = 1536;
constexpr int U_OUT = 1024;
constexpr int OWN0 = 6;
constexpr int NSEG = 2048;
constexpr int SEG_BYTES = 512 + 8192;
constexpr int CU_BYTES = 6144;
constexpr u64 DEADLINE_TICKS = 50000000ull;

constexpr size_t WS_TICKET = 0;
constexpr size_t WS_TMO = 128;
constexpr size_t WS_LINKCTR = 256;
constexpr size_t WS_XF = 8192;
constexpr size_t WS_WF = WS_XF + 16 * 128 * 16;
constexpr size_t WS_SEGRDY = WS_WF + 16 * 128 * 8;
constexpr size_t WS_RF = WS_SEGRDY + NSEG * 4;
constexpr size_t WS_MSEG = WS_RF + NR * NSEG * 4;
constexpr size_t WS_HW0 = WS_MSEG + 8 * NSEG * 4;
constexpr size_t WS_MW0 = WS_HW0 + 16 * 256 * 4;
constexpr size_t WS_HDR = WS_MW0 + 8 * 16 * 128 * 4;
constexpr size_t WS_CTL_USED = WS_HDR + 16 * 256 * 64;
constexpr size_t WS_CTL_END = 2u << 20;
constexpr size_t WS_PKX = WS_CTL_END;
constexpr size_t WS_PKW = WS_PKX + (16u << 20);
constexpr size_t WS_CX = WS_PKW + (16u << 20);
constexpr size_t WS_CW = WS_CX + (12u << 20);
constexpr size_t WS_XA = WS_CW + (12u << 20);
constexpr size_t WS_WB = WS_XA + (size_t)NPOS * 16 * 32768;
constexpr size_t WS_OSTG = WS_WB + (size_t)NPOS * 16 * 16384;
constexpr size_t WS_ORECV = WS_OSTG + (size_t)NSEG * SEG_BYTES;
constexpr size_t WS_END = WS_ORECV + (size_t)NR * NSEG * SEG_BYTES;
static_assert(WS_CTL_USED <= WS_CTL_END, "control region");
static_assert(WS_END < 0x7FFF0000u, "32-bit workspace offsets");

constexpr int STAGE_BYTES = 49152;
constexpr int LDS_CTL = 3 * STAGE_BYTES;
constexpr int LDS_WRK = LDS_CTL + 64;
constexpr int LDS_BYTES = LDS_WRK + 3 * 4096;
constexpr int WAVE_LDS = 13312 + 256;
constexpr int WL_ESC = 8192 + 64;
constexpr int WL_EXP = WL_ESC + 4096 + 64;

struct Params {
    const char* x;
    const char* w;
    char* out;
    char* ws[NR];
    int rank;
};

constexpr int AUX_SYS = 17;
constexpr int AUX_AGT = 16;
constexpr unsigned OOB = 0x7FFFFF00u;

__device__ __forceinline__ __amdgpu_buffer_rsrc_t mk_rsrc(const void* p, unsigned bytes) {
    return __builtin_amdgcn_make_buffer_rsrc((void*)p, 0, bytes, 0x00020000);
}
__device__ __forceinline__ void st_sys(unsigned* p, unsigned v) {
    __hip_atomic_store(p, v, __ATOMIC_RELAXED, __HIP_MEMORY_SCOPE_SYSTEM);
}
__device__ __forceinline__ void st_agt(unsigned* p, unsigned v) {
    __hip_atomic_store(p, v, __ATOMIC_RELAXED, __HIP_MEMORY_SCOPE_AGENT);
}
__device__ __forceinline__ void wait_vm0() { asm volatile("s_waitcnt vmcnt(0)" ::: "memory"); __builtin_amdgcn_s_waitcnt(0x0F70); }
__device__ __forceinline__ void wait_vm4() { asm volatile("s_waitcnt vmcnt(4)" ::: "memory"); __builtin_amdgcn_s_waitcnt(0x0F74); }
__device__ __forceinline__ void wait_vm8() { asm volatile("s_waitcnt vmcnt(8)" ::: "memory"); __builtin_amdgcn_s_waitcnt(0x0F78); }
__device__ __forceinline__ unsigned mbcnt64(u64 m) {
    return __builtin_amdgcn_mbcnt_hi((unsigned)(m >> 32), __builtin_amdgcn_mbcnt_lo((unsigned)m, 0));
}
__device__ __forceinline__ unsigned pk_max(unsigned a, unsigned b) {
    return __builtin_bit_cast(unsigned, __builtin_elementwise_max(__builtin_bit_cast(us2, a), __builtin_bit_cast(us2, b)));
}
__device__ __forceinline__ unsigned pk_min(unsigned a, unsigned b) {
    return __builtin_bit_cast(unsigned, __builtin_elementwise_min(__builtin_bit_cast(us2, a), __builtin_bit_cast(us2, b)));
}
__device__ __forceinline__ unsigned bfi(unsigned m, unsigned a, unsigned b) { return (a & m) | (b & ~m); }
__device__ __forceinline__ unsigned pk_sub(unsigned a, unsigned b) {
    return __builtin_bit_cast(unsigned, (us2)(__builtin_bit_cast(us2, a) - __builtin_bit_cast(us2, b)));
}
__device__ __forceinline__ unsigned pk_sub_sat(unsigned a, unsigned b) {
    return __builtin_bit_cast(unsigned, __builtin_elementwise_sub_sat(__builtin_bit_cast(us2, a), __builtin_bit_cast(us2, b)));
}
__device__ __forceinline__ unsigned wave_pk_max(unsigned v) {
#pragma unroll
    for (int o = 32; o > 0; o >>= 1) v = pk_max(v, (unsigned)__shfl_xor((int)v, o, 64));
    return __builtin_amdgcn_readfirstlane(v);
}

struct Ctx {
    unsigned call;
    unsigned* tmo;
    u64 t0;
};
__device__ __forceinline__ bool give_up(const Ctx& c) {
    const u64 dt = __builtin_amdgcn_s_memrealtime() - c.t0;
    if (dt < 200000ull) return false;
    if (__hip_atomic_load(c.tmo, __ATOMIC_RELAXED, __HIP_MEMORY_SCOPE_AGENT) == c.call) return true;
    if (dt > DEADLINE_TICKS) {
        __hip_atomic_store(c.tmo, c.call, __ATOMIC_RELAXED, __HIP_MEMORY_SCOPE_AGENT);
        return true;
    }
    return false;
}
__device__ __forceinline__ bool timed_out(const Ctx& c) {
    return __hip_atomic_load(c.tmo, __ATOMIC_RELAXED, __HIP_MEMORY_SCOPE_AGENT) == c.call;
}


__device__ __forceinline__ int seq_slot(int seq, bool g1) {
    if (g1) {
        if (seq < 16) return seq * 8;
        const int m = seq - 16;
        return (m / 7) * 8 + m % 7 + 1;
    }
    constexpr int F = OWN0, R = (16 - F) / 2;
    if (seq < F) return seq * 8;
    if (seq < F + 9 * R) {
        const int m = seq - F, c = m / 9, r = m % 9;
        return r < 7 ? c * 8 + r + 1 : (F + 2 * c + r - 7) * 8;
    }
    const int m = seq - F - 9 * R;
    return (R + m / 7) * 8 + m % 7 + 1;
}

__device__ __forceinline__ unsigned seg_tag(unsigned call) { return call & 63u; }
__device__ __forceinline__ bool seg_ok(unsigned f, unsigned tag) { return (f >> 26) == tag; }
__device__ __forceinline__ unsigned seg_len(unsigned f) {
    const unsigned lb = ((f >> 8) & 511u) * 16u;
    return (512u + (((f >> 25) & 1u) ? 2u * lb : lb + ((lb / 4u + 15u) & ~15u) + (f & 255u) * 16u) + 63u) & ~63u;
}
__device__ __forceinline__ unsigned pick_base(unsigned top, const unsigned (&ex)[8]) {
    unsigned t = top;
#pragma unroll 1
    for (int it = 0; it < 2 && t > 6u; ++it) {
        int diff = 0;
#pragma unroll
        for (int k = 0; k < 8; ++k)
            diff += __builtin_popcountll(__ballot(ex[k] == t)) - __builtin_popcountll(__ballot(ex[k] + 7u == t));
        if (diff >= 0) break;
        --t;
    }
    const unsigned b = t > 6u ? t - 6u : 0u;
    return b < 248u ? b : 248u;
}
__device__ __forceinline__ unsigned code4(unsigned d0, unsigned d1, unsigned base2) {
    const unsigned c0 = pk_min(pk_sub((d0 >> 7) & 0x00FF00FFu, base2), 0x00070007u);
    const unsigned c1 = pk_min(pk_sub((d1 >> 7) & 0x00FF00FFu, base2), 0x00070007u);
    return __builtin_amdgcn_perm(c1, c0, 0x06040200u);
}
template <int BITS>
__device__ __forceinline__ unsigned wave_scan(unsigned n, unsigned& sum) {
    int v = (int)n;
    v += __builtin_amdgcn_update_dpp(0, v, 0x111, 0xF, 0xF, true);
    v += __builtin_amdgcn_update_dpp(0, v, 0x112, 0xF, 0xF, true);
    v += __builtin_amdgcn_update_dpp(0, v, 0x114, 0xF, 0xF, true);
    v += __builtin_amdgcn_update_dpp(0, v, 0x118, 0xF, 0xF, true);
    v += __builtin_amdgcn_update_dpp(0, v, 0x142, 0xA, 0xF, true);
    v += __builtin_amdgcn_update_dpp(0, v, 0x143, 0xC, 0xF, true);
    sum = (unsigned)__builtin_amdgcn_readlane(v, 63);
    return (unsigned)v - n;
}

__device__ __forceinline__ unsigned hdr_tag(unsigned call) { return call & 0xFFFu; }

__device__ __forceinline__ void pack_load(const Params& p, int j, int lane, u32x4 (&v)[8]) {
    const int c = j >> 8, r = j & 255, un = r & 127;
    if (r < 128) {
        const char* src = p.w + (size_t)(c * 64 + (lane >> 3) * 8) * (NG * 2) + (size_t)(un * 64 + (lane & 7) * 8) * 2;
#pragma unroll
        for (int k = 0; k < 8; ++k) v[k] = *(const u32x4*)(src + (size_t)k * (NG * 2));
    } else {
        const int rr = lane >> 3, pp = lane & 7;
        const char* src = p.x + (size_t)(un * 64 + rr) * (KSH * 2) + c * 128;
        const int o0 = (pp ^ (rr >> 1)) * 16, o1 = (pp ^ (4 + (rr >> 1))) * 16;
#pragma unroll
        for (int k = 0; k < 8; ++k) v[k] = *(const u32x4*)(src + (size_t)k * 8 * (KSH * 2) + ((k & 1) ? o1 : o0));
    }
}

__device__ __forceinline__ void role_packer(const Params& p, const Ctx& cx, int h, int lane, u32x4 (&v)[8], char* wl) {
    const int rank = p.rank;
    char* ws = p.ws[rank];
    const unsigned tagw = hdr_tag(cx.call) << 20;
#pragma unroll 1
    for (int j = h; j < 4096;) {
        const int c = j >> 8, r = j & 255, un = r & 127;
        const int jn = j < 512 ? 4096 : j + 1024;
        unsigned* flag = nullptr;
        if (r < 128) {
            if ((un >> 5) == (rank & 3)) flag = (unsigned*)(ws + WS_WF) + (((un >> 1) & 15) * NPOS + c * 8) * 2 + (un & 1);
            u32x4 o[8];
#pragma unroll
            for (int cc = 0; cc < 8; ++cc)
#pragma unroll
                for (int d = 0; d < 4; ++d)
                    o[cc][d] = __builtin_amdgcn_perm(v[2 * d + 1][cc >> 1], v[2 * d][cc >> 1], (cc & 1) ? 0x07060302u : 0x05040100u);
#pragma unroll
            for (int cc = 0; cc < 8; ++cc) v[cc] = o[cc];
        } else {
            if ((un >> 6) == (rank >> 2)) flag = (unsigned*)(ws + WS_XF) + (((un >> 2) & 15) * NPOS + c * 8) * 4 + (un & 3);
        }
        unsigned mx = 0, mn = 0;
#pragma unroll
        for (int k = 0; k < 8; ++k)
#pragma unroll
            for (int d = 0; d < 4; ++d) {
                mx = pk_max(mx, v[k][d] & 0x7F807F80u);
                mn = pk_max(mn, pk_sub(0u, v[k][d] & 0x7F007F00u));
            }
        const unsigned red = wave_pk_max((max(mx >> 16, mx & 0xFFFFu) << 16) | max(mn >> 16, mn & 0xFFFFu));
        const __amdgpu_buffer_rsrc_t rc = mk_rsrc(ws + (r < 128 ? WS_CW : WS_CX) + (size_t)(c * 128 + un) * CU_BYTES, CU_BYTES);
        const __amdgpu_buffer_rsrc_t rh = mk_rsrc(ws + WS_HDR + (size_t)(c * 256 + r) * 64, 64);
        unsigned ex[8];
#pragma unroll
        for (int k = 0; k < 8; ++k) ex[k] = (v[k][k & 3] >> 7) & 0xFFu;
        const unsigned basea = pick_base(red >> 23, ex);
        unsigned C2[4] = {0u, 0u, 0u, 0u}, C1[2] = {0u, 0u};
#pragma unroll
        for (int hf = 0; hf < 2; ++hf)
#pragma unroll
            for (int g = 0; g < 8; ++g) {
                const u32x4 d = v[hf * 4 + (g >> 1)];
                const unsigned cd = code4(d[(g & 1) * 2], d[(g & 1) * 2 + 1], basea * 0x00010001u);
                C2[hf * 2 + (g >> 2)] |= ((cd >> 1) & 0x03030303u) << (2 * (g & 3));
                C1[hf] |= (cd & 0x01010101u) << (2 * (g & 3) + (g >> 2));
            }
        unsigned ne[2], eoff[2], E[4];
#pragma unroll
        for (int q = 0; q < 4; ++q) E[q] = C2[q] & (C2[q] >> 1) & (C1[q >> 1] >> (q & 1)) & 0x55555555u;
#pragma unroll
        for (int hf = 0; hf < 2; ++hf) eoff[hf] = wave_scan<6>(__builtin_popcount(E[2 * hf]) + __builtin_popcount(E[2 * hf + 1]), ne[hf]);
        unsigned hw0;
        bool raw = false;
        if (ne[0] <= 240u && ne[1] <= 240u) {
#pragma unroll
            for (int hf = 0; hf < 2; ++hf) {
                char* img = wl + 2048 + hf * 1024;
#pragma unroll
                for (int q = 0; q < 2; ++q) {
                    u32x4 B, X;
#pragma unroll
                    for (int g = 0; g < 4; ++g) {
                        const u32x4 d = v[hf * 4 + 2 * q + (g >> 1)];
                        const unsigned lo = __builtin_amdgcn_perm(d[(g & 1) * 2 + 1], d[(g & 1) * 2], 0x06040200u);
                        const unsigned hi = __builtin_amdgcn_perm(d[(g & 1) * 2 + 1], d[(g & 1) * 2], 0x07050301u);
                        B[g] = bfi(0x7F7F7F7Fu, lo, hi);
                        X[g] = bfi(0x01010101u, lo >> 7, hi << 1);
                    }
                    __builtin_amdgcn_raw_buffer_store_b128(B, rc, lane * 16 + hf * 3072 + q * 1024, 0, AUX_SYS);
                    *(u32x4*)(wl + lane * 32 + q * 16) = X;
                }
                *(unsigned*)(img + lane * 12) = C2[2 * hf];
                *(unsigned*)(img + lane * 12 + 4) = C2[2 * hf + 1];
                *(unsigned*)(img + lane * 12 + 8) = C1[hf];
                unsigned off = 768u + eoff[hf];
                u64 e = ((u64)E[2 * hf + 1] << 32) | E[2 * hf];
                while (e) {
                    const unsigned pb = __builtin_ctzll(e);
                    img[off] = wl[lane * 32 + (pb >> 5) * 16 + ((pb >> 1) & 3u) * 4 + ((pb >> 3) & 3u)];
                    ++off;
                    e &= e - 1ull;
                }
                __builtin_amdgcn_fence(__ATOMIC_RELEASE, "wavefront");
                __builtin_amdgcn_wave_barrier();
                __builtin_amdgcn_raw_buffer_store_b128(*(const u32x4*)(img + lane * 16), rc, hf * 3072 + 2048 + lane * 16, 0, AUX_SYS);
            }
            hw0 = basea | 0x4000u | (((ne[0] + 15u) >> 4) << 8) | (((ne[1] + 15u) >> 4) << 16);
        } else {
            const unsigned top = red >> 24;
            const unsigned base = top > 7u ? top - 7u : 0u;
            const unsigned lowest = ((0x10000u - (red & 0xFFFFu)) >> 8);
            const unsigned base2 = base * 0x01000100u;
#pragma unroll
            for (int hf = 0; hf < 2; ++hf) {
                u32x4 L[2], N;
#pragma unroll
                for (int jj = 0; jj < 4; ++jj) {
                    const u32x4 d = v[hf * 4 + jj];
                    unsigned nb[4];
#pragma unroll
                    for (int q = 0; q < 4; ++q) nb[q] = pk_sub_sat(d[q] & 0x7F007F00u, base2) | ((d[q] >> 4) & 0x08000800u);
                    L[jj >> 1][(jj & 1) * 2] = __builtin_amdgcn_perm(d[1], d[0], 0x06040200u);
                    L[jj >> 1][(jj & 1) * 2 + 1] = __builtin_amdgcn_perm(d[3], d[2], 0x06040200u);
                    N[jj] = __builtin_amdgcn_perm(nb[1], nb[0], 0x07050301u) | (__builtin_amdgcn_perm(nb[3], nb[2], 0x07050301u) << 4);
                }
                __builtin_amdgcn_raw_buffer_store_b128(L[0], rc, lane * 16 + hf * 3072, 0, AUX_SYS);
                __builtin_amdgcn_raw_buffer_store_b128(L[1], rc, lane * 16 + hf * 3072 + 1024, 0, AUX_SYS);
                __builtin_amdgcn_raw_buffer_store_b128(N, rc, lane * 16 + hf * 3072 + 2048, 0, AUX_SYS);
            }
            unsigned nesc = 0;
            if (lowest <= base) {
                unsigned r0 = 0, r1 = 0, n = 0;
#pragma unroll
                for (int k = 0; k < 8; ++k)
#pragma unroll
                    for (int e = 0; e < 8; ++e) {
                        const unsigned hb = (v[k][e >> 1] >> ((e & 1) ? 24 : 8)) & 0xFFu;
                        if ((hb & 0x7Fu) - 1u < base) { r1 = r0; r0 = ((unsigned)(k << 9 | e) << 8) | hb; ++n; }
                    }
                const u64 m1 = __ballot(n >= 1), m2 = __ballot(n >= 2);
                nesc = __builtin_popcountll(m1) + __builtin_popcountll(m2);
                raw = __ballot(n > 2) != 0ull || nesc > 15u;
                if (!raw) {
                    const unsigned lt = tagw | ((unsigned)lane << 11);
                    __builtin_amdgcn_raw_buffer_store_b32(r0 | lt, rh, n >= 1 ? 4 + mbcnt64(m1) * 4 : OOB, 0, AUX_SYS);
                    __builtin_amdgcn_raw_buffer_store_b32(r1 | lt, rh, n >= 2 ? 4 + ((unsigned)__builtin_popcountll(m1) + mbcnt64(m2)) * 4 : OOB, 0, AUX_SYS);
                } else nesc = 0;
            }
            hw0 = base | (nesc << 8) | (raw ? 0x8000u : 0u);
        }
        if (raw || flag) {
            const __amdgpu_buffer_rsrc_t rs = mk_rsrc(ws + (r < 128 ? WS_PKW : WS_PKX) + (size_t)(c * 128 + un) * 8192, 8192);
#pragma unroll
            for (int k = 0; k < 8; ++k) __builtin_amdgcn_raw_buffer_store_b128(v[k], rs, lane * 16 + k * 1024, 0, AUX_SYS);
        }
        asm volatile("" ::: "memory");
        if (jn < 4096) {
            pack_load(p, jn, lane, v);
            asm volatile("s_waitcnt vmcnt(8)" ::: "memory");
        } else wait_vm0();
        if (lane == 0) st_sys((unsigned*)(ws + WS_HW0) + c * 256 + r, tagw | hw0);
        if (flag && lane == 0) st_agt(flag, cx.call);
        j = jn;
    }
}

#define UNI(x) __builtin_amdgcn_readfirstlane(x)
#define LD_REMOTE(off, imm) __builtin_amdgcn_raw_buffer_load_b128(rrem, (off) + (imm), 0, AUX_SYS)
#define ST_LOCAL(v, off, imm) __builtin_amdgcn_raw_buffer_store_b128(v, rloc, (off) + (imm), 0, AUX_AGT)

struct InUnit { unsigned idx, dst, flag, info; bool end; };

__device__ __forceinline__ void decode_half(const u32x4 l0, const u32x4 l1, const u32x4 nn, const u32x4 ee, unsigned hdr, unsigned h0, unsigned k,
                                            int lane, char* wl, u32x4 (&o)[4]) {
    const unsigned bs = (h0 & 0xFFu) * 0x01010101u;
    unsigned H[8];
    u32x4 la = l0, lb = l1;
    if (h0 & 0x4000u) {
        if (lane < 16) *(u32x4*)(wl + lane * 16) = ee;
        const unsigned c0 = nn[0], c1 = nn[1], cb = nn[2];
        unsigned e0 = c0 & (c0 >> 1) & cb & 0x55555555u, e1 = c1 & (c1 >> 1) & (cb >> 1) & 0x55555555u, sum;
        unsigned off = wave_scan<6>(__builtin_popcount(e0) + __builtin_popcount(e1), sum) & 1023u;
#pragma unroll
        for (int g = 0; g < 8; ++g)
            H[g] = ((((g & 4 ? c1 : c0) >> (2 * (g & 3))) & 0x03030303u) << 1) + ((cb >> (2 * (g & 3) + (g >> 2))) & 0x01010101u) + bs;
        char* hp = wl + 1024 + lane * 32;
        *(u32x4*)hp = (u32x4){H[0], H[1], H[2], H[3]};
        *(u32x4*)(hp + 16) = (u32x4){H[4], H[5], H[6], H[7]};
        u64 e = ((u64)e1 << 32) | e0;
        __builtin_amdgcn_fence(__ATOMIC_RELEASE, "wavefront");
        __builtin_amdgcn_wave_barrier();
        while (e) {
            const unsigned w0 = *(const unsigned*)(wl + (off & ~3u)), w1 = *(const unsigned*)(wl + (off & ~3u) + 4);
            unsigned w = __builtin_amdgcn_alignbyte(w1, w0, off & 3u);
#pragma unroll
            for (int r = 0; r < 4; ++r)
                if (e) {
                    const unsigned pb = __builtin_ctzll(e);
                    hp[(pb >> 5) * 16 + ((pb >> 1) & 3u) * 4 + ((pb >> 3) & 3u)] = (char)w;
                    w >>= 8;
                    e &= e - 1ull;
                }
            off += 4;
        }
        __builtin_amdgcn_fence(__ATOMIC_RELEASE, "wavefront");
        __builtin_amdgcn_wave_barrier();
        const u32x4 xa = *(const u32x4*)hp, xb = *(const u32x4*)(hp + 16);
#pragma unroll
        for (int g = 0; g < 4; ++g) {
            H[g] = bfi(0x80808080u, l0[g], xa[g] >> 1);
            H[4 + g] = bfi(0x80808080u, l1[g], xb[g] >> 1);
            la[g] = bfi(0x7F7F7F7Fu, l0[g], xa[g] << 7);
            lb[g] = bfi(0x7F7F7F7Fu, l1[g], xb[g] << 7);
        }
    } else {
        const unsigned tlo = (bs + 0x03020100u) & 0xFFFFFF00u, thi = bs + 0x07060504u;
#pragma unroll
        for (int jj = 0; jj < 4; ++jj) {
            const unsigned n = nn[jj];
            H[2 * jj] = __builtin_amdgcn_perm(thi, tlo, n & 0x07070707u) | ((n << 4) & 0x80808080u);
            H[2 * jj + 1] = __builtin_amdgcn_perm(thi, tlo, (n >> 4) & 0x07070707u) | (n & 0x80808080u);
        }
        const int nesc = (h0 >> 8) & 15;
#pragma unroll 1
        for (int i = 1; i <= nesc; ++i) {
            const unsigned e = __builtin_amdgcn_readlane(hdr, i);
            if (((e >> 19) & 1u) != k) continue;
            const unsigned key = ((unsigned)lane == ((e >> 11) & 63u)) ? ((e >> 17) & 3u) * 2 + ((e >> 10) & 1u) : 8u;
            const unsigned sh = ((e >> 8) & 3u) * 8, msk = 0xFFu << sh, val = (e & 0xFFu) << sh;
#pragma unroll
            for (int q = 0; q < 8; ++q) H[q] = (key == (unsigned)q) ? ((H[q] & ~msk) | val) : H[q];
        }
    }
#pragma unroll
    for (int jj = 0; jj < 4; ++jj) {
        const unsigned p0 = (jj >> 1 ? lb : la)[(jj & 1) * 2], p1 = (jj >> 1 ? lb : la)[(jj & 1) * 2 + 1];
        o[jj][0] = __builtin_amdgcn_perm(H[2 * jj], p0, 0x05010400u);
        o[jj][1] = __builtin_amdgcn_perm(H[2 * jj], p0, 0x07030602u);
        o[jj][2] = __builtin_amdgcn_perm(H[2 * jj + 1], p1, 0x05010400u);
        o[jj][3] = __builtin_amdgcn_perm(H[2 * jj + 1], p1, 0x07030602u);
    }
}

__device__ __forceinline__ void role_pull_in(const Params& p, const Ctx& cx, int link, int lane, char* wl) {
    const int rank = p.rank;
    const __amdgpu_buffer_rsrc_t rloc = mk_rsrc(p.ws[rank], (unsigned)WS_END);
    const __amdgpu_buffer_rsrc_t rrem = mk_rsrc(p.ws[(rank + 1 + link) & 7], (unsigned)WS_END);
    const unsigned ctr_off = (lane == 0) ? (unsigned)WS_LINKCTR + link * 128 : OOB;
    const unsigned base = (cx.call - 1) * (unsigned)(U_IN + NPI);
    const unsigned l16 = lane * 16;
    const unsigned htag = hdr_tag(cx.call);

    auto ticket = [&]() -> unsigned { return (unsigned)__builtin_amdgcn_raw_ptr_buffer_atomic_add_i32(1, rloc, ctr_off, 0, AUX_AGT); };
    auto make = [&](unsigned tv) -> InUnit {
        const int n = (int)(UNI(tv) - base);
        InUnit u;
        int d, r;
        if (n < 64 * LAG) { d = n >> 6; r = n & 63; }
        else if (n < 64 * LAG + 96 * (16 - LAG)) { const int m = n - 64 * LAG; d = LAG + m / 96; r = m % 96; }
        else { const int m = n - 64 * LAG - 96 * (16 - LAG); d = 16 + (m >> 5); r = 64 + (m & 31); }
        const int c = (r >= 64) ? d - LAG : d;
        const int pos = c * 8 + 1 + link;
        const bool isw = r < 32;
        const int vv = isw ? r : r - 32;
        u.end = n >= U_IN;
        u.info = (unsigned)WS_MW0 + ((link * 16 + c) * 128 + r) * 4;
        u.idx = isw ? (unsigned)(c * 128 + (rank & 3) * 32 + vv) : (unsigned)(c * 128 + (rank >> 2) * 64 + vv) | 0x80000000u;
        u.dst = (isw ? (unsigned)WS_WB + pos * (16 * 16384) : (unsigned)WS_XA + pos * (16 * 32768)) + vv * 8192;
        u.flag = isw ? (unsigned)WS_WF + ((vv >> 1) * NPOS + pos) * 8 + (vv & 1) * 4
                     : (unsigned)WS_XF + ((vv >> 2) * NPOS + pos) * 16 + (vv & 3) * 4;
        return u;
    };
    auto load_hdr = [&](const InUnit& u) -> unsigned {
        return __builtin_amdgcn_raw_buffer_load_b32(rloc, (lane == 0 && !u.end) ? u.info : OOB, 0, AUX_AGT);
    };
    auto hdr_ok = [&](unsigned hv) -> bool {
        const unsigned w0 = UNI(hv);
        return (w0 >> 20) == htag && ((w0 & 0xC000u) != 0u || (w0 & 0xF00u) == 0u);
    };
    auto publish = [&](unsigned flag) {
        __builtin_amdgcn_raw_buffer_store_b32(cx.call, rloc, lane == 0 ? flag : OOB, 0, AUX_AGT);
    };
    auto store_half = [&](u32x4 (&C)[4], unsigned hdr, unsigned h0, unsigned dst, unsigned k) {
        if (!(h0 & 0x8000u)) {
            u32x4 o[4];
            decode_half(C[0], C[1], C[2], C[3], hdr, h0, k, lane, wl, o);
#pragma unroll
            for (int j = 0; j < 4; ++j) C[j] = o[j];
        }
        const unsigned a0 = dst + l16 + k * 4096u;
#pragma unroll
        for (int j = 0; j < 4; ++j) ST_LOCAL(C[j], a0, j * 1024);
    };
    auto src_of = [&](const InUnit& u, unsigned h0) -> unsigned {
        const bool x = (u.idx >> 31) != 0u;
        const unsigned ix = u.idx & 0x7FFFFFFFu;
        return ((h0 & 0x8000u) ? (x ? (unsigned)WS_PKX : (unsigned)WS_PKW) + ix * 8192u
                               : (x ? (unsigned)WS_CX : (unsigned)WS_CW) + ix * (unsigned)CU_BYTES) + l16;
    };
    auto fetch = [&](u32x4 (&N)[4], unsigned so0, unsigned h0, unsigned k) {
        const bool raw = (h0 & 0x8000u) != 0u;
        const unsigned so = so0 + k * (raw ? 4096u : 3072u);
        N[0] = LD_REMOTE(so, 0); N[1] = LD_REMOTE(so, 1024);
        if (h0 & 0x4000u) {
            typedef unsigned u32x3 __attribute__((ext_vector_type(3)));
            const u32x3 c3 = __builtin_amdgcn_raw_buffer_load_b96(rrem, so - lane * 4 + 2048, 0, AUX_SYS);
            N[2] = (u32x4){c3[0], c3[1], c3[2], 0u};
            N[3] = LD_REMOTE((unsigned)lane < ((h0 >> (8 + 8 * k)) & 15u) ? so : OOB, 2816);
        } else {
            N[2] = LD_REMOTE(so, 2048);
            N[3] = LD_REMOTE(raw ? so : OOB, 3072);
        }
    };
    auto wait_hdr = [&](const InUnit& u) -> unsigned {
        const unsigned ix = u.idx & 0x7FFFFFFFu;
        const unsigned hu = (ix >> 7) * 256 + (u.idx >> 31) * 128 + (ix & 127u);
        unsigned off = lane == 0 ? (unsigned)WS_HW0 + hu * 4 : OOB;
        for (;;) {
            const unsigned hv = __builtin_amdgcn_raw_buffer_load_b32(rrem, off, 0, AUX_SYS);
            const unsigned w0 = UNI(hv);
            const unsigned need = (w0 & 0xC000u) ? 1u : (2u << ((w0 >> 8) & 15u)) - 1u;
            if (((unsigned)__ballot((hv >> 20) == htag) & need) == need) return hv;
            if (give_up(cx)) return 0x8000u;
            if ((w0 >> 20) == htag) { if (lane > 0 && lane < 16) off = (unsigned)WS_HDR + hu * 64 + lane * 4; }
            else __builtin_amdgcn_s_sleep(8);
        }
    };

    InUnit u = make(ticket());
    if (u.end) return;
    unsigned t = ticket();
    unsigned hdr = wait_hdr(u);
    unsigned h0 = UNI(hdr);
    unsigned so = src_of(u, h0);
    u32x4 A[4], B[4];
    fetch(A, so, h0, 0);
    wait_vm0();
    unsigned pflag = OOB;
    for (;;) {
        asm volatile("" : "+v"(A[0]), "+v"(A[1]), "+v"(A[2]), "+v"(A[3]), "+v"(t));
        fetch(B, so, h0, 1);
        __builtin_amdgcn_sched_barrier(0);
        const InUnit un = make(t);
        unsigned info = load_hdr(un);
        store_half(A, hdr, h0, u.dst, 0);
        wait_vm4();
        asm volatile("" : "+v"(B[0]), "+v"(B[1]), "+v"(B[2]), "+v"(B[3]), "+v"(info));
        if (un.end || !hdr_ok(info)) {
            publish(pflag);
            store_half(B, hdr, h0, u.dst, 1);
            wait_vm0();
            publish(u.flag);
            pflag = OOB;
            if (un.end) return;
            t = ticket();
            hdr = wait_hdr(un);
            h0 = UNI(hdr);
            so = src_of(un, h0);
            fetch(A, so, h0, 0);
            wait_vm0();
        } else {
            const unsigned hn0 = UNI(info);
            so = src_of(un, hn0);
            fetch(A, so, hn0, 0);
            __builtin_amdgcn_sched_barrier(0);
            publish(pflag);
            t = ticket();
            store_half(B, hdr, h0, u.dst, 1);
            wait_vm4();
            pflag = u.flag;
            hdr = info; h0 = hn0;
        }
        u = un;
    }
}

__device__ __forceinline__ void role_pull_out(const Params& p, const Ctx& cx, int link, int lane) {
    const int rank = p.rank, src = (rank + 1 + link) & 7;
    const __amdgpu_buffer_rsrc_t rloc = mk_rsrc(p.ws[rank], (unsigned)WS_END);
    const __amdgpu_buffer_rsrc_t rrem = mk_rsrc(p.ws[src], (unsigned)WS_END);
    const unsigned ctr_off = (lane == 0) ? (unsigned)WS_LINKCTR + (7 + link) * 128 : OOB;
    const unsigned base = (cx.call - 1) * (unsigned)(U_OUT + NPO);
    const unsigned tag = seg_tag(cx.call);
    const unsigned l16 = lane * 16;
    bool dead = false;
    {
        const unsigned ibase = (cx.call - 1) * (unsigned)(U_IN + NPI);
        for (;;) {
            const unsigned v = UNI(__builtin_amdgcn_raw_buffer_load_b32(rloc, lane == 0 ? (unsigned)WS_LINKCTR + link * 128 : OOB, 0, AUX_AGT));
            const int rem = U_IN - (int)(v - ibase);
            if (rem <= 0) break;
            if (give_up(cx)) { dead = true; break; }
            for (int k = rem >> 6; k > 0; --k) __builtin_amdgcn_s_sleep(100);
            __builtin_amdgcn_s_sleep(16);
        }
    }
    auto ticket = [&]() -> unsigned { return (unsigned)__builtin_amdgcn_raw_ptr_buffer_atomic_add_i32(1, rloc, ctr_off, 0, AUX_AGT); };
    auto load_words = [&](unsigned n) -> unsigned {
        return __builtin_amdgcn_raw_buffer_load_b32(rloc, (lane < 2 && n < (unsigned)U_OUT) ? (unsigned)WS_MSEG + (link * NSEG + 2 * n + lane) * 4 : OOB, 0, AUX_AGT);
    };
    unsigned pn = U_OUT, qn = U_OUT, pw = 0, qw = 0;
    auto publish = [&]() {
        __builtin_amdgcn_raw_buffer_store_b32(pw, rloc, (lane < 2 && pn < (unsigned)U_OUT) ? (unsigned)WS_RF + (src * NSEG + 2 * pn + lane) * 4 : OOB, 0, AUX_AGT);
        pn = qn; pw = qw; qn = U_OUT;
    };
    auto store_unit = [&](u32x4 (&C)[8], unsigned n, unsigned w, unsigned len0, unsigned len1) {
        const unsigned d0 = (unsigned)WS_ORECV + (src * NSEG + 2 * n) * (unsigned)SEG_BYTES + l16, d1 = d0 + SEG_BYTES;
#pragma unroll
        for (int j = 0; j < 4; ++j) ST_LOCAL(C[j], l16 + j * 1024u < len0 ? d0 : OOB, j * 1024);
#pragma unroll
        for (int j = 0; j < 4; ++j) ST_LOCAL(C[4 + j], l16 + j * 1024u < len1 ? d1 : OOB, j * 1024);
        qn = n; qw = w;
    };
    auto rest = [&](unsigned sg, unsigned len) {
#pragma unroll 1
        for (unsigned o = 4096; o < len; o += 1024) {
            const bool in = o + l16 < len;
            const u32x4 v = LD_REMOTE(in ? (unsigned)WS_OSTG + sg * (unsigned)SEG_BYTES + o + l16 : OOB, 0);
            ST_LOCAL(v, in ? (unsigned)WS_ORECV + (src * NSEG + sg) * (unsigned)SEG_BYTES + o + l16 : OOB, 0);
        }
    };
    unsigned nn = UNI(ticket()) - base;
    if (nn >= (unsigned)U_OUT) return;
    unsigned t = ticket();
    unsigned info = load_words(nn), cn = U_OUT, cw = 0, cl0 = 0, cl1 = 0;
    wait_vm0();
    u32x4 A[8], B[8];
#pragma unroll
    for (int j = 0; j < 8; ++j) A[j] = (u32x4){0u, 0u, 0u, 0u};
#define OUT_STEP(C, N) do { \
        asm volatile("" : "+v"(C[0]), "+v"(C[1]), "+v"(C[2]), "+v"(C[3]), "+v"(C[4]), "+v"(C[5]), "+v"(C[6]), "+v"(C[7]), "+v"(t), "+v"(info)); \
        unsigned w0_ = UNI(info), w1_ = __builtin_amdgcn_readlane(info, 1); \
        if (!seg_ok(w0_, tag) || !seg_ok(w1_, tag) || nn >= (unsigned)U_OUT) {        \
            publish(); \
            store_unit(C, cn, cw, cl0, cl1); cn = U_OUT; cl0 = cl1 = 0; \
            wait_vm0(); publish(); publish(); \
            if (nn >= (unsigned)U_OUT) return; \
            for (;;) { \
                info = load_words(nn); \
                w0_ = UNI(info); w1_ = __builtin_amdgcn_readlane(info, 1); \
                if (seg_ok(w0_, tag) && seg_ok(w1_, tag)) break; \
                if (dead || give_up(cx)) { dead = true; w0_ = w1_ = info = (tag ^ 1u) << 26; break; } \
                __builtin_amdgcn_s_sleep(8); \
            } \
        } \
        const unsigned wv_ = info, l0_ = seg_len(w0_), l1_ = seg_len(w1_); \
        const unsigned s0_ = (unsigned)WS_OSTG + 2 * nn * (unsigned)SEG_BYTES + l16, s1_ = s0_ + SEG_BYTES; \
        _Pragma("unroll") for (int j = 0; j < 4; ++j) N[j] = LD_REMOTE(l16 + j * 1024u < l0_ ? s0_ : OOB, j * 1024); \
        _Pragma("unroll") for (int j = 0; j < 4; ++j) N[4 + j] = LD_REMOTE(l16 + j * 1024u < l1_ ? s1_ : OOB, j * 1024); \
        __builtin_amdgcn_sched_barrier(0); \
        publish(); \
        if (l0_ > 4096u) rest(2 * nn, l0_); \
        if (l1_ > 4096u) rest(2 * nn + 1, l1_); \
        const unsigned n2_ = UNI(t) - base; \
        if (n2_ < (unsigned)U_OUT) t = ticket(); \
        info = load_words(n2_); \
        store_unit(C, cn, cw, cl0, cl1); \
        wait_vm8(); \
        cn = nn; cw = wv_; cl0 = l0_; cl1 = l1_; nn = n2_; \
    } while (0)
    for (;;) {
        OUT_STEP(A, B);
        OUT_STEP(B, A);
    }
}

__device__ __forceinline__ void role_relay(const Params& p, const Ctx& cx, int link, int lane) {
    const int rank = p.rank;
    const int src = (rank + 1 + link) & 7;
    const __amdgpu_buffer_rsrc_t rctl = mk_rsrc(p.ws[rank], (unsigned)WS_CTL_END);
    const __amdgpu_buffer_rsrc_t rinfo = mk_rsrc(p.ws[src], (unsigned)WS_CTL_END);
    const unsigned tag = seg_tag(cx.call);
    {
        const int sub = lane & 31;
        const unsigned woff = sub < 8 ? (unsigned)((rank & 3) * 32 + sub * 4) : (unsigned)(128 + (rank >> 2) * 64 + (sub - 8) * 4);
        const unsigned ht = hdr_tag(cx.call);
#pragma unroll 1
        for (int c = 0; c < 16;) {
            const int cc = c + (lane >> 5);
            const bool act = sub < 24 && cc < 16;
            const u32x4 hd = __builtin_amdgcn_raw_buffer_load_b128(rinfo, act ? (unsigned)WS_HW0 + (cc * 256 + woff) * 4 : OOB, 0, AUX_SYS);
            __builtin_amdgcn_raw_buffer_store_b128(hd, rctl, act ? (unsigned)WS_MW0 + ((link * 16 + cc) * 128 + sub * 4) * 4 : OOB, 0, AUX_AGT);
            bool ok = true;
#pragma unroll
            for (int q = 0; q < 4; ++q) ok = ok && (((hd[q] >> 20) - ht) & 0xFFFu) <= 1u;
            const u64 bad = __ballot(act && !ok);
            if ((unsigned)bad == 0u) { c += (bad == 0ull) ? 2 : 1; continue; }
            if (give_up(cx)) return;
            __builtin_amdgcn_s_sleep(16);
        }
    }
    for (;;) {
        const unsigned v = UNI(__builtin_amdgcn_raw_buffer_load_b32(rctl, lane == 0 ? (unsigned)WS_LINKCTR + link * 128 : OOB, 0, AUX_AGT));
        const int rem = U_IN - 160 - (int)(v - (cx.call - 1) * (unsigned)(U_IN + NPI));
        if (rem <= 0) break;
        if (give_up(cx)) return;
        for (int k = rem >> 6; k > 0; --k) __builtin_amdgcn_s_sleep(100);
        __builtin_amdgcn_s_sleep(16);
    }
    int pos = 0;
    while (pos < NSEG) {
        const unsigned q = pos + lane * 4;
        const u32x4 v = __builtin_amdgcn_raw_buffer_load_b128(rinfo, q < NSEG ? (unsigned)WS_SEGRDY + q * 4 : OOB, 0, AUX_SYS);
        const bool any = seg_ok(v[0], tag) || seg_ok(v[1], tag) || seg_ok(v[2], tag) || seg_ok(v[3], tag);
        const bool all = seg_ok(v[0], tag) && seg_ok(v[1], tag) && seg_ok(v[2], tag) && seg_ok(v[3], tag);
        __builtin_amdgcn_raw_buffer_store_b128(v, rctl, (any && q < NSEG) ? (unsigned)WS_MSEG + (link * NSEG + q) * 4 : OOB, 0, AUX_AGT);
        const u64 m = __ballot(all && q < NSEG);
        const int n = (m == ~0ull) ? 64 : __builtin_ctzll(~m);
        pos += n * 4;
        if (n < 64) {
            if (give_up(cx)) return;
            __builtin_amdgcn_s_sleep(32);
        }
    }
}

__device__ __forceinline__ int packer_id(int wg, int wave) {
    const int gi = (wg >> 4) * 8 + (wg & 7);
    const bool g1 = (wg & 8) != 0;
    if (wave < 8) return g1 ? 512 + wave * 128 + gi : -1;
    const int k = wave - 9;
    return (k < 0 || (g1 && k < 2)) ? -1 : gi * 4 + (g1 ? 3 : k);
}
__device__ __forceinline__ void role_worker(const Params& p, const Ctx& cx, int wg, int k, int lane, char* wl) {
    const int gi = (wg >> 4) * 8 + (wg & 7);
    if (wg & 8) {
        if (k < 2 && gi < 119) { if (gi / 7 + 17 * k < NPI) role_pull_in(p, cx, gi % 7, lane, wl); }
        else if (gi >= 121 && k == 0) role_relay(p, cx, gi - 121, lane);
    } else {
        if (k == 0 && gi < NPO * 7) role_pull_out(p, cx, gi % 7, lane);
    }
}

__device__ __forceinline__ void role_poller(const Params& p, const Ctx& cx, int ti, int tj, int lane, char* lds) {
    const char* ws = p.ws[p.rank];
    const __amdgpu_buffer_rsrc_t rx = mk_rsrc(ws + WS_XF + ti * (NPOS * 16), NPOS * 16);
    const __amdgpu_buffer_rsrc_t rw = mk_rsrc(ws + WS_WF + tj * (NPOS * 8), NPOS * 8);
    volatile LAS unsigned* ready = (volatile LAS unsigned*)(LAS char*)(lds + LDS_CTL + 4);
    int pos = 0;
    while (pos < NPOS) {
        const int sq = pos + (lane & 15);
        const int q = sq < NPOS ? seq_slot(sq, ti >= 8) : NPOS;
        bool ok = false;
        if (lane < 16) {
            const u32x4 a = __builtin_amdgcn_raw_buffer_load_b128(rx, q * 16, 0, AUX_AGT);
            const u32x2 b = __builtin_amdgcn_raw_buffer_load_b64(rw, q * 8, 0, AUX_AGT);
            ok = a[0] == cx.call && a[1] == cx.call && a[2] == cx.call && a[3] == cx.call && b[0] == cx.call && b[1] == cx.call;
        }
        const u64 m = __ballot(ok);
        const int n = __builtin_ctzll(~m);
        if (n > 0) {
            pos += n;
            if (lane == 0) *ready = pos;
        } else {
            if (give_up(cx)) { pos = NPOS; if (lane == 0) *ready = NPOS; break; }
            __builtin_amdgcn_s_sleep(64);
        }
    }
}

__device__ __forceinline__ void st_out(char* o, u32x2 v) { __builtin_nontemporal_store(v, (u32x2*)o); }
__device__ __forceinline__ void expand_segment(const Params& p, const Ctx& cx, int src, int seg, int lane, char* wl, bool last) {
    const char* ws = p.ws[p.rank];
    const unsigned* rf = (const unsigned*)(ws + WS_RF) + src * NSEG + seg;
    unsigned f;
    bool dead = false;
    for (;;) {
        f = __builtin_amdgcn_readfirstlane(__hip_atomic_load(rf, __ATOMIC_RELAXED, __HIP_MEMORY_SCOPE_AGENT));
        if (seg_ok(f, seg_tag(cx.call))) break;
        if (give_up(cx)) { dead = true; break; }
        if (last) __builtin_amdgcn_s_sleep(32); else __builtin_amdgcn_s_sleep(127);
    }
    const int tile = seg >> 3, wv = seg & 7;
    const int row0 = (src >> 2) * 4096 + (tile >> 4) * 256 + (wv >> 1) * 64 + (lane >> 4) * 4;
    const int col0 = (src & 3) * 2048 + (tile & 15) * 128 + (wv & 1) * 64 + (lane & 7) * 8 + ((lane >> 3) & 1) * 4;
    char* o = p.out + ((size_t)row0 * NG + col0) * 2;
    if (dead) {
        const u32x2 nan2 = {0x7FC07FC0u, 0x7FC07FC0u};
#pragma unroll 1
        for (int i = 0; i < 16; ++i) *(u32x2*)(o + (size_t)((i >> 2) * 16 + (i & 3)) * (NG * 2)) = nan2;
        return;
    }
    const unsigned lb = ((f >> 8) & 511u) * 16u;
    const char* sp = ws + WS_ORECV + ((size_t)src * NSEG + seg) * SEG_BYTES;
    const __amdgpu_buffer_rsrc_t rs = mk_rsrc(sp, seg_len(f));
    const u32x2 mk = __builtin_amdgcn_raw_buffer_load_b64(rs, lane * 8, 0, AUX_AGT);
    if ((f >> 25) & 1u) {
        u32x4 r[8];
#pragma unroll
        for (int j = 0; j < 8; ++j) if (j * 512u < lb) r[j] = __builtin_amdgcn_raw_buffer_load_b128(rs, 512 + lane * 16 + j * 1024, 0, AUX_AGT);
#pragma unroll
        for (int j = 0; j < 8; ++j) if (j * 512u < lb) *(u32x4*)(wl + lane * 16 + j * 1024) = r[j];
    } else {
        u32x4 L[4], E[3];
        unsigned W[4];
        const unsigned oc = 512u + lb, oe = oc + ((lb / 4u + 15u) & ~15u), eb = (f & 255u) * 16u;
#pragma unroll
        for (int j = 0; j < 4; ++j)
            if (j * 1024u < lb) {
                L[j] = __builtin_amdgcn_raw_buffer_load_b128(rs, 512 + lane * 16 + j * 1024, 0, AUX_AGT);
                W[j] = __builtin_amdgcn_raw_buffer_load_b32(rs, j * 1024u + lane * 16u < lb ? oc + lane * 4 + j * 256 : OOB, 0, AUX_AGT);
            }
#pragma unroll
        for (int j = 0; j < 3; ++j) if (j * 1024u < eb) E[j] = __builtin_amdgcn_raw_buffer_load_b128(rs, oe + lane * 16 + j * 1024, 0, AUX_AGT);
#pragma unroll
        for (int j = 0; j < 3; ++j) if (j * 1024u < eb) *(u32x4*)(wl + WL_ESC + lane * 16 + j * 1024) = E[j];
        const unsigned bs = ((f >> 17) & 0xFFu) * 0x01010101u;
        unsigned ebase = 0;
        char* xp = wl + WL_EXP + lane * 16;
#pragma unroll
        for (int j = 0; j < 4; ++j)
            if (j * 1024u < lb) {
                u32x4 X;
                unsigned b0 = 0;
#pragma unroll
                for (int g = 0; g < 4; ++g) {
                    const unsigned c1 = (L[j][g] >> 7) & 0x01010101u;
                    X[g] = (((W[j] >> (2 * g)) & 0x03030303u) << 1) + c1 + bs;
                    b0 |= c1 << (2 * g);
                }
                unsigned e = W[j] & (W[j] >> 1) & b0, sum;
                unsigned off = ebase + wave_scan<5>(__builtin_popcount(e), sum);
                ebase += sum;
                *(u32x4*)xp = X;
                __builtin_amdgcn_fence(__ATOMIC_RELEASE, "wavefront");
                __builtin_amdgcn_wave_barrier();
                while (e) {
                    const unsigned pb = __builtin_ctz(e);
                    xp[((pb >> 1) & 3u) * 4 + (pb >> 3)] = wl[WL_ESC + off];
                    ++off;
                    e &= e - 1u;
                }
                X = *(const u32x4*)xp;
#pragma unroll
                for (int g2 = 0; g2 < 2; ++g2) {
                    u32x4 w4;
#pragma unroll
                    for (int q = 0; q < 2; ++q) {
                        const int g = 2 * g2 + q;
                        const unsigned lo = bfi(0x7F7F7F7Fu, L[j][g], X[g] << 7), hi = (X[g] >> 1) & 0x7F7F7F7Fu;
                        w4[2 * q] = __builtin_amdgcn_perm(hi, lo, 0x05010400u);
                        w4[2 * q + 1] = __builtin_amdgcn_perm(hi, lo, 0x07030602u);
                    }
                    *(u32x4*)(wl + j * 2048 + lane * 32 + g2 * 16) = w4;
                }
            }
    }
    __builtin_amdgcn_fence(__ATOMIC_RELEASE, "wavefront");
    __builtin_amdgcn_wave_barrier();
    const unsigned short* vals = (const unsigned short*)wl;
    unsigned base = 0;
#pragma unroll
    for (int i = 0; i < 16; ++i) {
        unsigned e[4];
#pragma unroll
        for (int f4 = 0; f4 < 4; ++f4) {
            const unsigned mlo = __builtin_amdgcn_readlane(mk[0], i * 4 + f4), mhi = __builtin_amdgcn_readlane(mk[1], i * 4 + f4);
            const unsigned idx = __builtin_amdgcn_mbcnt_hi(mhi, __builtin_amdgcn_mbcnt_lo(mlo, 0)) + base;
            const u64 m = ((u64)mhi << 32) | mlo;
            const unsigned val = vals[idx];
            e[f4] = __builtin_amdgcn_inverse_ballot_w64(m) ? val : 0u;
            base += __builtin_popcountll(m);
        }
        const u32x2 ov = {e[0] | (e[1] << 16), e[2] | (e[3] << 16)};
        st_out(o + (size_t)((i >> 2) * 16 + (i & 3)) * (NG * 2), ov);
    }
    __builtin_amdgcn_fence(__ATOMIC_RELEASE, "wavefront");
    __builtin_amdgcn_wave_barrier();
}

__device__ __forceinline__ unsigned bf16_rne(float f) {
    const unsigned u = __float_as_uint(f);
    return (u + 0x7FFFu + ((u >> 16) & 1u)) >> 16;
}

__device__ __forceinline__ void role_gemm(const Params& p, const Ctx& cx, int ti, int tj, int wv, int lane, char* lds) {
    const char* ws = p.ws[p.rank];
    const unsigned ldsbase = (unsigned)(uintptr_t)(LAS char*)lds;
    const unsigned ctl = ldsbase + LDS_CTL;
    const int q = wv >> 1, h = wv & 1;
    unsigned tA[2], tB[2];
#pragma unroll
    for (int i = 0; i < 2; ++i) {
        const unsigned slot = (unsigned)seq_slot(lane + 64 * i, ti >= 8);
        const bool own = (slot & 7u) == 0u;
        tA[i] = own ? (unsigned)WS_PKX + ((p.rank >> 2) * 16 + ti) * 32768 + slot * (16 * 8192) : (unsigned)WS_XA + ti * 32768 + slot * (16 * 32768);
        tB[i] = own ? (unsigned)WS_PKW + ((p.rank & 3) * 16 + tj) * 16384 + slot * (16 * 8192) : (unsigned)WS_WB + tj * 16384 + slot * (16 * 16384);
    }
    const unsigned lo_a = wv * 4096 + lane * 16, lo_b = wv * 2048 + lane * 16;
    auto tile_a = [&](int s) -> const char* {
        const unsigned o0 = __builtin_amdgcn_readlane(tA[0], s & 63), o1 = __builtin_amdgcn_readlane(tA[1], s & 63);
        return ws + ((s < 64 ? o0 : o1) + lo_a);
    };
    auto tile_b = [&](int s) -> const char* {
        const unsigned o0 = __builtin_amdgcn_readlane(tB[0], s & 63), o1 = __builtin_amdgcn_readlane(tB[1], s & 63);
        return ws + ((s < 64 ? o0 : o1) + lo_b);
    };
    const int fr = lane & 15, kq = lane >> 4;
    const int offA0 = q * 8192 + fr * 128 + ((kq ^ (fr >> 1)) * 16);
    const int offA1 = q * 8192 + fr * 128 + (((4 + kq) ^ (fr >> 1)) * 16);
    const int offB = 32768 + h * 8192 + (fr >> 3) * 4096 + kq * 128 + (fr & 7) * 16;

    f32x4 acc[4][4];
#pragma unroll
    for (int m = 0; m < 4; ++m)
#pragma unroll
        for (int n = 0; n < 4; ++n) acc[m][n] = (f32x4){0.f, 0.f, 0.f, 0.f};

    int issued = 0, islot = 0;
    unsigned rdy = 0, cnt = 0;
    auto issue = [&]() {
        const char* a = tile_a(issued);
        const char* b = tile_b(issued);
        char* la = lds + islot * STAGE_BYTES + wv * 4096;
        char* lb = lds + islot * STAGE_BYTES + 32768 + wv * 2048;
        __builtin_amdgcn_global_load_lds((const GAS void*)a, (LAS void*)la, 16, 0, AUX_AGT);
        __builtin_amdgcn_global_load_lds((const GAS void*)a, (LAS void*)la, 16, 1024, AUX_AGT);
        __builtin_amdgcn_global_load_lds((const GAS void*)a, (LAS void*)la, 16, 2048, AUX_AGT);
        __builtin_amdgcn_global_load_lds((const GAS void*)a, (LAS void*)la, 16, 3072, AUX_AGT);
        __builtin_amdgcn_global_load_lds((const GAS void*)b, (LAS void*)lb, 16, 0, AUX_AGT);
        __builtin_amdgcn_global_load_lds((const GAS void*)b, (LAS void*)lb, 16, 1024, AUX_AGT);
        ++issued;
        islot = (islot == 2) ? 0 : islot + 1;
    };
    auto poll = [&]() {
        u32x2 v;
        asm volatile("ds_read_b64 %0, %1\n\ts_waitcnt lgkmcnt(0)" : "=v"(v) : "v"(ctl) : "memory");
        cnt = __builtin_amdgcn_readfirstlane(v[0]);
        rdy = __builtin_amdgcn_readfirstlane(v[1]);
    };
    auto arrive = [&]() {
        if (lane == 0) asm volatile("ds_add_u32 %0, %1" :: "v"(ctl), "v"(1u) : "memory");
    };
    auto wait_bar = [&](unsigned target) {
        poll();
        while (cnt < target) { __builtin_amdgcn_s_sleep(1); poll(); }
    };
    auto need = [&](int upto) {
        while (issued < upto) {
            while ((int)rdy <= issued) { __builtin_amdgcn_s_sleep(2); poll(); }
            issue();
        }
    };

    need(1);
    if ((int)rdy > issued) issue();
    if (issued == 2) asm volatile("s_waitcnt vmcnt(6)" ::: "memory"); else wait_vm0();
    arrive();
    wait_bar(8);

    bf16x8 a0[4], b0[4], a1[4], b1[4];
    int cslot = 0;
    {
        const char* s = lds;
#pragma unroll
        for (int m = 0; m < 4; ++m) a0[m] = *(const bf16x8*)(s + offA0 + m * 2048);
#pragma unroll
        for (int n = 0; n < 4; ++n) b0[n] = *(const bf16x8*)(s + offB + n * 1024);
    }
#define MMA4(A_, B_, m_) do { _Pragma("unroll") for (int n = 0; n < 4; ++n) \
        acc[m_][n] = __builtin_amdgcn_mfma_f32_16x16x32_bf16(A_[m_], B_[n], acc[m_][n], 0, 0, 0); \
        __builtin_amdgcn_sched_barrier(0); } while (0)
#define GLDS(g_, l_, o_) do { if (spread) __builtin_amdgcn_global_load_lds((const GAS void*)(g_), (LAS void*)(l_), 16, o_, AUX_AGT); \
        __builtin_amdgcn_sched_barrier(0); } while (0)
#pragma unroll 1
    for (int t = 0; t < NPOS; ++t) {
        const char* s = lds + cslot * STAGE_BYTES;
        cslot = (cslot == 2) ? 0 : cslot + 1;
        const char* sn = lds + cslot * STAGE_BYTES;
        if (issued == t + 1 && issued < NPOS && issued < (int)rdy) issue();
        const bool spread = issued == t + 2 && issued < NPOS && issued < (int)rdy;
        const char* pa = tile_a(issued);
        const char* pb = tile_b(issued);
        char* la = lds + islot * STAGE_BYTES + wv * 4096;
        char* lb = lds + islot * STAGE_BYTES + 32768 + wv * 2048;
        __builtin_amdgcn_sched_barrier(0);
        MMA4(a0, b0, 0);
#pragma unroll
        for (int m = 0; m < 4; ++m) a1[m] = *(const bf16x8*)(s + offA1 + m * 2048);
#pragma unroll
        for (int n = 0; n < 4; ++n) b1[n] = *(const bf16x8*)(s + offB + 512 + n * 1024);
        __builtin_amdgcn_sched_barrier(0);
        GLDS(pa, la, 0);
        MMA4(a0, b0, 1);
        GLDS(pa, la, 1024);
        MMA4(a0, b0, 2);
        GLDS(pa, la, 2048);
        MMA4(a0, b0, 3);
        if (t + 1 < NPOS) {
            need(t + 2);
            if (spread) asm volatile("s_waitcnt vmcnt(3) lgkmcnt(0)" ::: "memory");
            else asm volatile("s_waitcnt vmcnt(0) lgkmcnt(0)" ::: "memory");
        } else {
            asm volatile("s_waitcnt lgkmcnt(0)" ::: "memory");
        }
        __builtin_amdgcn_s_waitcnt(0xC07F);
        arrive();
        __builtin_amdgcn_sched_barrier(0);
        MMA4(a1, b1, 0);
        GLDS(pa, la, 3072);
        MMA4(a1, b1, 1);
        wait_bar(8u * (t + 2));
        if (t + 1 < NPOS) {
#pragma unroll
            for (int m = 0; m < 4; ++m) a0[m] = *(const bf16x8*)(sn + offA0 + m * 2048);
#pragma unroll
            for (int n = 0; n < 4; ++n) b0[n] = *(const bf16x8*)(sn + offB + n * 1024);
        }
        __builtin_amdgcn_sched_barrier(0);
        GLDS(pb, lb, 0);
        MMA4(a1, b1, 2);
        GLDS(pb, lb, 1024);
        MMA4(a1, b1, 3);
        if (spread) { ++issued; islot = (islot == 2) ? 0 : islot + 1; }
    }

    char* wl = lds + wv * WAVE_LDS;
    unsigned short* vals = (unsigned short*)wl;
    const bool bad = timed_out(cx);
    const int seg = ((ti * 16 + tj) << 3) + wv;
    u32x2 mk = {0u, 0u};
    u32x2 ov[16];
    unsigned base = 0, mx = 0;
#pragma unroll
    for (int i = 0; i < 16; ++i) {
        unsigned e[4];
#pragma unroll
        for (int f4 = 0; f4 < 4; ++f4) {
            const float v = acc[i >> 2][f4][i & 3];
            unsigned b = bf16_rne(v > 0.f ? v : 0.f);
            if (bad || v != v) b = 0x7FC0u;
            e[f4] = b;
            const u64 m = __ballot(b != 0u);
            const unsigned idx = mbcnt64(m) + base;
            if (b != 0u) vals[idx] = (unsigned short)b;
            base += __builtin_popcountll(m);
            if (lane == i * 4 + f4) { mk[0] = (unsigned)m; mk[1] = (unsigned)(m >> 32); }
        }
        ov[i] = (u32x2){e[0] | (e[1] << 16), e[2] | (e[3] << 16)};
        mx = pk_max(mx, pk_max(ov[i][0], ov[i][1]));
    }
    {
        unsigned ex[8];
#pragma unroll
        for (int i = 0; i < 8; ++i) { const unsigned w = ov[2 * i][i & 1] & 0xFFFFu; ex[i] = w ? w >> 7 : 999u; }
        const unsigned hb = pick_base(wave_pk_max(max(mx >> 16, mx & 0xFFFFu)) >> 7, ex);
        const unsigned lb = (base + 15u) & ~15u;
        if (lane < 16) vals[base + lane] = (unsigned short)(hb << 7);
        __builtin_amdgcn_fence(__ATOMIC_RELEASE, "wavefront");
        __builtin_amdgcn_wave_barrier();
        u32x4 L[4];
        unsigned W[4];
        unsigned ne = 0;
#pragma unroll
        for (int j = 0; j < 4; ++j)
            if (j * 1024u < lb) {
                unsigned w = 0, b0 = 0;
#pragma unroll
                for (int g = 0; g < 4; ++g) {
                    const u32x2 d = *(const u32x2*)(wl + j * 2048 + lane * 32 + g * 8);
                    const unsigned cd = code4(d[0], d[1], hb * 0x00010001u);
                    L[j][g] = bfi(0x7F7F7F7Fu, __builtin_amdgcn_perm(d[1], d[0], 0x06040200u), cd << 7);
                    w |= ((cd >> 1) & 0x03030303u) << (2 * g);
                    b0 |= (cd & 0x01010101u) << (2 * g);
                }
                if (j * 1024u + lane * 16u >= lb) w = 0;
                W[j] = w;
                unsigned e = w & (w >> 1) & b0, sum;
                unsigned off = ne + wave_scan<5>(__builtin_popcount(e), sum);
                ne += sum;
                while (e) {
                    const unsigned pb = __builtin_ctz(e);
                    wl[WL_ESC + off] = (char)(vals[j * 1024 + lane * 16 + ((pb >> 1) & 3u) * 4 + (pb >> 3)] >> 7);
                    ++off;
                    e &= e - 1u;
                }
            }
        __builtin_amdgcn_fence(__ATOMIC_RELEASE, "wavefront");
        __builtin_amdgcn_wave_barrier();
        const bool raw = ne * 4u > lb * 3u - (lb ? 128u : 0u) || ne > 3072u;
        const unsigned f = (seg_tag(cx.call) << 26) | (raw ? 1u << 25 : 0u) | (hb << 17) | ((lb >> 4) << 8) | (raw ? 0u : (ne + 15u) >> 4);
        const __amdgpu_buffer_rsrc_t rs = mk_rsrc(ws + WS_OSTG + (size_t)seg * SEG_BYTES, seg_len(f));
        __builtin_amdgcn_raw_buffer_store_b64(mk, rs, lane * 8, 0, AUX_SYS);
        if (raw) {
#pragma unroll
            for (int j = 0; j < 8; ++j)
                if (j * 512u < lb)
                    __builtin_amdgcn_raw_buffer_store_b128(*(const u32x4*)(wl + lane * 16 + j * 1024), rs, 512 + lane * 16 + j * 1024, 0, AUX_SYS);
        } else {
            const unsigned oc = 512u + lb, oe = oc + ((lb / 4u + 15u) & ~15u);
#pragma unroll
            for (int j = 0; j < 4; ++j)
                if (j * 1024u < lb) {
                    const bool in = j * 1024u + lane * 16u < lb;
                    __builtin_amdgcn_raw_buffer_store_b128(L[j], rs, in ? 512 + lane * 16 + j * 1024 : OOB, 0, AUX_SYS);
                    __builtin_amdgcn_raw_buffer_store_b32(W[j], rs, in ? oc + lane * 4 + j * 256 : OOB, 0, AUX_SYS);
                }
#pragma unroll
            for (int j = 0; j < 3; ++j)
                if (j * 1024u < ne)
                    __builtin_amdgcn_raw_buffer_store_b128(*(const u32x4*)(wl + WL_ESC + lane * 16 + j * 1024), rs,
                                                           j * 1024u + lane * 16u < ne ? oe + lane * 16 + j * 1024 : OOB, 0, AUX_SYS);
        }
        wait_vm0();
        if (lane == 0) st_sys((unsigned*)(ws + WS_SEGRDY) + seg, f);
    }
    {
        const int row0 = (p.rank >> 2) * 4096 + ti * 256 + q * 64 + (lane >> 4) * 4;
        const int col0 = (p.rank & 3) * 2048 + tj * 128 + h * 64 + (lane & 7) * 8 + ((lane >> 3) & 1) * 4;
        char* o = p.out + ((size_t)row0 * NG + col0) * 2;
#pragma unroll
        for (int i = 0; i < 16; ++i) st_out(o + (size_t)((i >> 2) * 16 + (i & 3)) * (NG * 2), ov[i]);
    }
    __builtin_amdgcn_fence(__ATOMIC_RELEASE, "wavefront");
    __builtin_amdgcn_wave_barrier();
#pragma unroll 1
    for (int i = 0; i < 7; ++i) {
        const int e = seg + NSEG * i;
        expand_segment(p, cx, (p.rank + 1 + e % 7) & 7, e / 7, lane, wl, e >= NSEG * 7 - 512);
    }
}

}

__global__ void __launch_bounds__(NTHR) fused_gemm_ar(Params p) {
    extern __shared__ __attribute__((aligned(16))) char lds[];
    const int lane = threadIdx.x & 63;
    const int wave = __builtin_amdgcn_readfirstlane(threadIdx.x >> 6);
    const int wg = blockIdx.x;
    volatile LAS unsigned* ctl = (volatile LAS unsigned*)(LAS char*)(lds + LDS_CTL);
    char* ws = p.ws[p.rank];
    u32x4 v[8];
    if (wave >= 9) __builtin_amdgcn_s_setprio(3);
    const int pid = packer_id(wg, wave);
    if (pid >= 0) pack_load(p, pid, lane, v);
    if (threadIdx.x == 0) {
        const unsigned ticket = __hip_atomic_fetch_add((unsigned*)(ws + WS_TICKET), 1u, __ATOMIC_RELAXED, __HIP_MEMORY_SCOPE_AGENT);
        ctl[0] = 0u; ctl[1] = 0u; ctl[2] = ticket / NWG + 1u;
    }
    asm volatile("s_waitcnt lgkmcnt(0)" ::: "memory");
    __builtin_amdgcn_s_barrier();
    asm volatile("" ::: "memory");
    Ctx cx;
    cx.call = __builtin_amdgcn_readfirstlane(ctl[2]);
    cx.tmo = (unsigned*)(ws + WS_TMO);
    cx.t0 = __builtin_amdgcn_s_memrealtime();
    const int xcd = wg & 7, idx = wg >> 3;
    const int ti = (idx & 1) * 8 + (xcd & 1) * 4 + ((idx >> 1) & 3), tj = (xcd >> 1) * 4 + (idx >> 3);
    char* wl = lds + (wave < 8 ? wave * 4096 : LDS_WRK + (wave - 9) * 4096);
    if (pid >= 0) {
        role_packer(p, cx, pid, lane, v, wl);
        asm volatile("s_waitcnt lgkmcnt(0)" ::: "memory");
    }
    if (wave < 8) role_gemm(p, cx, ti, tj, wave, lane, lds);
    else if (wave == 8) role_poller(p, cx, ti, tj, lane, lds);
    else role_worker(p, cx, wg, wave - 9, lane, wl);
}

extern "C" void kernel_launch(int rank, int nranks, ncclComm_t comm, hipStream_t stream,
                              void* const* d_in, const int* in_sizes, int n_in,
                              void* d_out, int out_size,
                              const int* in_dtypes, int out_dtype,
                              void* const* d_out_all, void* const* d_ws_all, size_t ws_bytes) {
    if (nranks != NR || n_in < 2 || ws_bytes < WS_END || (unsigned)rank >= (unsigned)NR) {
        fprintf(stderr, "unsupported configuration\n");
        return;
    }
    static bool attr_set[NR] = {};
    if (!attr_set[rank]) {
        HIP_CHECK(hipFuncSetAttribute((const void*)fused_gemm_ar, hipFuncAttributeMaxDynamicSharedMemorySize, LDS_BYTES));
        attr_set[rank] = true;
    }
    Params p;
    p.x = (const char*)d_in[0];
    p.w = (const char*)d_in[1];
    p.out = (char*)d_out;
    for (int i = 0; i < NR; ++i) p.ws[i] = (char*)d_ws_all[i];
    p.rank = rank;
    fused_gemm_ar<<<NWG, NTHR, LDS_BYTES, stream>>>(p);
    HIP_CHECK(hipGetLastError());
}
```

```cpp
#include <hip/hip_runtime.h>
#include <rccl/rccl.h>
#include <cstdio>
#include <cstdint>

#define HIP_CHECK(expr) do { hipError_t _e = (expr); if (_e != hipSuccess) { \
    fprintf(stderr, "HIP error %s at %s:%d\n", hipGetErrorString(_e), __FILE__, __LINE__); \
    return; } } while (0)

typedef short bf16x8 __attribute__((ext_vector_type(8)));
typedef float f32x4 __attribute__((ext_vector_type(4)));
typedef unsigned u32x4 __attribute__((ext_vector_type(4)));
typedef unsigned u32x2 __attribute__((ext_vector_type(2)));
typedef unsigned short us2 __attribute__((ext_vector_type(2)));
typedef unsigned long long u64;
#define LAS __attribute__((address_space(3)))
#define GAS __attribute__((address_space(1)))

namespace {

constexpr int NR = 8;
constexpr int NG = 8192;
constexpr int KSH = 1024;
constexpr int NPOS = 128;
constexpr int NWG = 256;
constexpr int NTHR = 768;
constexpr int LAG = 6;
constexpr int NPI = 27;
constexpr int NPO = 12;
static_assert(NPI <= 34 && NPO <= 18, "comm waves: two per CU of group 1, one per CU of group 0");
constexpr int U_IN = 1536;
constexpr int U_OUT = 1024;
constexpr int OWN0 = 6;
constexpr int NSEG = 2048;
constexpr int SEG_BYTES = 512 + 8192;
constexpr int CU_BYTES = 6144;
constexpr u64 DEADLINE_TICKS = 50000000ull;

constexpr size_t WS_TICKET = 0;
constexpr size_t WS_TMO = 128;
constexpr size_t WS_LINKCTR = 256;
constexpr size_t WS_XF = 8192;
constexpr size_t WS_WF = WS_XF + 16 * 128 * 16;
constexpr size_t WS_SEGRDY = WS_WF + 16 * 128 * 8;
constexpr size_t WS_RF = WS_SEGRDY + NSEG * 4;
constexpr size_t WS_MSEG = WS_RF + NR * NSEG * 4;
constexpr size_t WS_HW0 = WS_MSEG + 8 * NSEG * 4;
constexpr size_t WS_MW0 = WS_HW0 + 16 * 256 * 4;
constexpr size_t WS_HDR = WS_MW0 + 8 * 16 * 128 * 4;
constexpr size_t WS_CTL_USED = WS_HDR + 16 * 256 * 64;
constexpr size_t WS_CTL_END = 2u << 20;
constexpr size_t WS_PKX = WS_CTL_END;
constexpr size_t WS_PKW = WS_PKX + (16u << 20);
constexpr size_t WS_CX = WS_PKW + (16u << 20);
constexpr size_t WS_CW = WS_CX + (12u << 20);
constexpr size_t WS_XA = WS_CW + (12u << 20);
constexpr size_t WS_WB = WS_XA + (size_t)NPOS * 16 * 32768;
constexpr size_t WS_OSTG = WS_WB + (size_t)NPOS * 16 * 16384;
constexpr size_t WS_ORECV = WS_OSTG + (size_t)NSEG * SEG_BYTES;
constexpr size_t WS_END = WS_ORECV + (size_t)NR * NSEG * SEG_BYTES;
static_assert(WS_CTL_USED <= WS_CTL_END, "control region");
static_assert(WS_END < 0x7FFF0000u, "32-bit workspace offsets");

constexpr int STAGE_BYTES = 49152;
constexpr int LDS_CTL = 3 * STAGE_BYTES;
constexpr int LDS_WRK = LDS_CTL + 64;
constexpr int LDS_BYTES = LDS_WRK + 3 * 4096;
constexpr int WAVE_LDS = 13312 + 256;
constexpr int WL_ESC = 8192 + 64;
constexpr int WL_EXP = WL_ESC + 4096 + 64;

struct Params {
    const char* x;
    const char* w;
    char* out;
    char* ws[NR];
    int rank;
};

constexpr int AUX_SYS = 17;
constexpr int AUX_AGT = 16;
constexpr unsigned OOB = 0x7FFFFF00u;

__device__ __forceinline__ __amdgpu_buffer_rsrc_t mk_rsrc(const void* p, unsigned bytes) {
    return __builtin_amdgcn_make_buffer_rsrc((void*)p, 0, bytes, 0x00020000);
}
__device__ __forceinline__ void st_sys(unsigned* p, unsigned v) {
    __hip_atomic_store(p, v, __ATOMIC_RELAXED, __HIP_MEMORY_SCOPE_SYSTEM);
}
__device__ __forceinline__ void st_agt(unsigned* p, unsigned v) {
    __hip_atomic_store(p, v, __ATOMIC_RELAXED, __HIP_MEMORY_SCOPE_AGENT);
}
__device__ __forceinline__ void wait_vm0() { asm volatile("s_waitcnt vmcnt(0)" ::: "memory"); __builtin_amdgcn_s_waitcnt(0x0F70); }
__device__ __forceinline__ void wait_vm4() { asm volatile("s_waitcnt vmcnt(4)" ::: "memory"); __builtin_amdgcn_s_waitcnt(0x0F74); }
__device__ __forceinline__ void wait_vm8() { asm volatile("s_waitcnt vmcnt(8)" ::: "memory"); __builtin_amdgcn_s_waitcnt(0x0F78); }
__device__ __forceinline__ unsigned mbcnt64(u64 m) {
    return __builtin_amdgcn_mbcnt_hi((unsigned)(m >> 32), __builtin_amdgcn_mbcnt_lo((unsigned)m, 0));
}
__device__ __forceinline__ unsigned pk_max(unsigned a, unsigned b) {
    return __builtin_bit_cast(unsigned, __builtin_elementwise_max(__builtin_bit_cast(us2, a), __builtin_bit_cast(us2, b)));
}
__device__ __forceinline__ unsigned pk_min(unsigned a, unsigned b) {
    return __builtin_bit_cast(unsigned, __builtin_elementwise_min(__builtin_bit_cast(us2, a), __builtin_bit_cast(us2, b)));
}
__device__ __forceinline__ unsigned bfi(unsigned m, unsigned a, unsigned b) { return (a & m) | (b & ~m); }
__device__ __forceinline__ unsigned pk_sub(unsigned a, unsigned b) {
    return __builtin_bit_cast(unsigned, (us2)(__builtin_bit_cast(us2, a) - __builtin_bit_cast(us2, b)));
}
__device__ __forceinline__ unsigned pk_sub_sat(unsigned a, unsigned b) {
    return __builtin_bit_cast(unsigned, __builtin_elementwise_sub_sat(__builtin_bit_cast(us2, a), __builtin_bit_cast(us2, b)));
}
__device__ __forceinline__ unsigned wave_pk_max(unsigned v) {
#pragma unroll
    for (int o = 32; o > 0; o >>= 1) v = pk_max(v, (unsigned)__shfl_xor((int)v, o, 64));
    return __builtin_amdgcn_readfirstlane(v);
}

struct Ctx {
    unsigned call;
    unsigned* tmo;
    u64 t0;
};
__device__ __forceinline__ bool give_up(const Ctx& c) {
    const u64 dt = __builtin_amdgcn_s_memrealtime() - c.t0;
    if (dt < 200000ull) return false;
    if (__hip_atomic_load(c.tmo, __ATOMIC_RELAXED, __HIP_MEMORY_SCOPE_AGENT) == c.call) return true;
    if (dt > DEADLINE_TICKS) {
        __hip_atomic_store(c.tmo, c.call, __ATOMIC_RELAXED, __HIP_MEMORY_SCOPE_AGENT);
        return true;
    }
    return false;
}
__device__ __forceinline__ bool timed_out(const Ctx& c) {
    return __hip_atomic_load(c.tmo, __ATOMIC_RELAXED, __HIP_MEMORY_SCOPE_AGENT) == c.call;
}


__device__ __forceinline__ int seq_slot(int seq, bool g1) {
    if (g1) {
        if (seq < 16) return seq * 8;
        const int m = seq - 16;
        return (m / 7) * 8 + m % 7 + 1;
    }
    constexpr int F = OWN0, R = (16 - F) / 2;
    if (seq < F) return seq * 8;
    if (seq < F + 9 * R) {
        const int m = seq - F, c = m / 9, r = m % 9;
        return r < 7 ? c * 8 + r + 1 : (F + 2 * c + r - 7) * 8;
    }
    const int m = seq - F - 9 * R;
    return (R + m / 7) * 8 + m % 7 + 1;
}

__device__ __forceinline__ unsigned seg_tag(unsigned call) { return call % 63u + 1u; }
__device__ __forceinline__ bool seg_ok(unsigned f, unsigned tag) { return (f >> 26) == tag; }
__device__ __forceinline__ unsigned seg_len(unsigned f) {
    const unsigned lb = ((f >> 8) & 511u) * 16u;
    return (512u + (((f >> 25) & 1u) ? 2u * lb : lb + ((lb / 4u + 15u) & ~15u) + (f & 255u) * 16u) + 63u) & ~63u;
}
__device__ __forceinline__ unsigned pick_base(unsigned top, const unsigned (&ex)[8]) {
    unsigned t = top;
#pragma unroll 1
    for (int it = 0; it < 2 && t > 6u; ++it) {
        int diff = 0;
#pragma unroll
        for (int k = 0; k < 8; ++k)
            diff += __builtin_popcountll(__ballot(ex[k] == t)) - __builtin_popcountll(__ballot(ex[k] + 7u == t));
        if (diff >= 0) break;
        --t;
    }
    const unsigned b = t > 6u ? t - 6u : 0u;
    return b < 248u ? b : 248u;
}
__device__ __forceinline__ unsigned code4(unsigned d0, unsigned d1, unsigned base2) {
    const unsigned c0 = pk_min(pk_sub((d0 >> 7) & 0x00FF00FFu, base2), 0x00070007u);
    const unsigned c1 = pk_min(pk_sub((d1 >> 7) & 0x00FF00FFu, base2), 0x00070007u);
    return __builtin_amdgcn_perm(c1, c0, 0x06040200u);
}
template <int BITS>
__device__ __forceinline__ unsigned wave_scan(unsigned n, unsigned& sum) {
    int v = (int)n;
    v += __builtin_amdgcn_update_dpp(0, v, 0x111, 0xF, 0xF, true);
    v += __builtin_amdgcn_update_dpp(0, v, 0x112, 0xF, 0xF, true);
    v += __builtin_amdgcn_update_dpp(0, v, 0x114, 0xF, 0xF, true);
    v += __builtin_amdgcn_update_dpp(0, v, 0x118, 0xF, 0xF, true);
    v += __builtin_amdgcn_update_dpp(0, v, 0x142, 0xA, 0xF, true);
    v += __builtin_amdgcn_update_dpp(0, v, 0x143, 0xC, 0xF, true);
    sum = (unsigned)__builtin_amdgcn_readlane(v, 63);
    return (unsigned)v - n;
}

__device__ __forceinline__ unsigned hdr_tag(unsigned call) { return call % 4095u + 1u; }

__device__ __forceinline__ void pack_load(const Params& p, int j, int lane, u32x4 (&v)[8]) {
    const int c = j >> 8, r = j & 255, un = r & 127;
    if (r < 128) {
        const char* src = p.w + (size_t)(c * 64 + (lane >> 3) * 8) * (NG * 2) + (size_t)(un * 64 + (lane & 7) * 8) * 2;
#pragma unroll
        for (int k = 0; k < 8; ++k) v[k] = *(const u32x4*)(src + (size_t)k * (NG * 2));
    } else {
        const int rr = lane >> 3, pp = lane & 7;
        const char* src = p.x + (size_t)(un * 64 + rr) * (KSH * 2) + c * 128;
        const int o0 = (pp ^ (rr >> 1)) * 16, o1 = (pp ^ (4 + (rr >> 1))) * 16;
#pragma unroll
        for (int k = 0; k < 8; ++k) v[k] = *(const u32x4*)(src + (size_t)k * 8 * (KSH * 2) + ((k & 1) ? o1 : o0));
    }
}

__device__ __forceinline__ void role_packer(const Params& p, const Ctx& cx, int h, int lane, u32x4 (&v)[8], char* wl) {
    const int rank = p.rank;
    char* ws = p.ws[rank];
    const unsigned tagw = hdr_tag(cx.call) << 20;
#pragma unroll 1
    for (int j = h; j < 4096;) {
        const int c = j >> 8, r = j & 255, un = r & 127;
        const int jn = j < 512 ? 4096 : j + 1024;
        unsigned* flag = nullptr;
        if (r < 128) {
            if ((un >> 5) == (rank & 3)) flag = (unsigned*)(ws + WS_WF) + (((un >> 1) & 15) * NPOS + c * 8) * 2 + (un & 1);
            u32x4 o[8];
#pragma unroll
            for (int cc = 0; cc < 8; ++cc)
#pragma unroll
                for (int d = 0; d < 4; ++d)
                    o[cc][d] = __builtin_amdgcn_perm(v[2 * d + 1][cc >> 1], v[2 * d][cc >> 1], (cc & 1) ? 0x07060302u : 0x05040100u);
#pragma unroll
            for (int cc = 0; cc < 8; ++cc) v[cc] = o[cc];
        } else {
            if ((un >> 6) == (rank >> 2)) flag = (unsigned*)(ws + WS_XF) + (((un >> 2) & 15) * NPOS + c * 8) * 4 + (un & 3);
        }
        unsigned mx = 0, mn = 0;
#pragma unroll
        for (int k = 0; k < 8; ++k)
#pragma unroll
            for (int d = 0; d < 4; ++d) {
                mx = pk_max(mx, v[k][d] & 0x7F807F80u);
                mn = pk_max(mn, pk_sub(0u, v[k][d] & 0x7F007F00u));
            }
        const unsigned red = wave_pk_max((max(mx >> 16, mx & 0xFFFFu) << 16) | max(mn >> 16, mn & 0xFFFFu));
        const __amdgpu_buffer_rsrc_t rc = mk_rsrc(ws + (r < 128 ? WS_CW : WS_CX) + (size_t)(c * 128 + un) * CU_BYTES, CU_BYTES);
        const __amdgpu_buffer_rsrc_t rh = mk_rsrc(ws + WS_HDR + (size_t)(c * 256 + r) * 64, 64);
        unsigned ex[8];
#pragma unroll
        for (int k = 0; k < 8; ++k) ex[k] = (v[k][k & 3] >> 7) & 0xFFu;
        const unsigned basea = pick_base(red >> 23, ex);
        unsigned C2[4] = {0u, 0u, 0u, 0u}, C1[2] = {0u, 0u};
#pragma unroll
        for (int hf = 0; hf < 2; ++hf)
#pragma unroll
            for (int g = 0; g < 8; ++g) {
                const u32x4 d = v[hf * 4 + (g >> 1)];
                const unsigned cd = code4(d[(g & 1) * 2], d[(g & 1) * 2 + 1], basea * 0x00010001u);
                C2[hf * 2 + (g >> 2)] |= ((cd >> 1) & 0x03030303u) << (2 * (g & 3));
                C1[hf] |= (cd & 0x01010101u) << (2 * (g & 3) + (g >> 2));
            }
        unsigned ne[2], eoff[2], E[4];
#pragma unroll
        for (int q = 0; q < 4; ++q) E[q] = C2[q] & (C2[q] >> 1) & (C1[q >> 1] >> (q & 1)) & 0x55555555u;
#pragma unroll
        for (int hf = 0; hf < 2; ++hf) eoff[hf] = wave_scan<6>(__builtin_popcount(E[2 * hf]) + __builtin_popcount(E[2 * hf + 1]), ne[hf]);
        unsigned hw0;
        bool raw = false;
        if (ne[0] <= 240u && ne[1] <= 240u) {
#pragma unroll
            for (int hf = 0; hf < 2; ++hf) {
                char* img = wl + 2048 + hf * 1024;
#pragma unroll
                for (int q = 0; q < 2; ++q) {
                    u32x4 B, X;
#pragma unroll
                    for (int g = 0; g < 4; ++g) {
                        const u32x4 d = v[hf * 4 + 2 * q + (g >> 1)];
                        const unsigned lo = __builtin_amdgcn_perm(d[(g & 1) * 2 + 1], d[(g & 1) * 2], 0x06040200u);
                        const unsigned hi = __builtin_amdgcn_perm(d[(g & 1) * 2 + 1], d[(g & 1) * 2], 0x07050301u);
                        B[g] = bfi(0x7F7F7F7Fu, lo, hi);
                        X[g] = bfi(0x01010101u, lo >> 7, hi << 1);
                    }
                    __builtin_amdgcn_raw_buffer_store_b128(B, rc, lane * 16 + hf * 3072 + q * 1024, 0, AUX_SYS);
                    *(u32x4*)(wl + lane * 32 + q * 16) = X;
                }
                *(unsigned*)(img + lane * 12) = C2[2 * hf];
                *(unsigned*)(img + lane * 12 + 4) = C2[2 * hf + 1];
                *(unsigned*)(img + lane * 12 + 8) = C1[hf];
                unsigned off = 768u + eoff[hf];
                u64 e = ((u64)E[2 * hf + 1] << 32) | E[2 * hf];
                while (e) {
                    const unsigned pb = __builtin_ctzll(e);
                    img[off] = wl[lane * 32 + (pb >> 5) * 16 + ((pb >> 1) & 3u) * 4 + ((pb >> 3) & 3u)];
                    ++off;
                    e &= e - 1ull;
                }
                __builtin_amdgcn_fence(__ATOMIC_RELEASE, "wavefront");
                __builtin_amdgcn_wave_barrier();
                __builtin_amdgcn_raw_buffer_store_b128(*(const u32x4*)(img + lane * 16), rc, hf * 3072 + 2048 + lane * 16, 0, AUX_SYS);
            }
            hw0 = basea | 0x4000u | (((ne[0] + 15u) >> 4) << 8) | (((ne[1] + 15u) >> 4) << 16);
        } else {
            const unsigned top = red >> 24;
            const unsigned base = top > 7u ? top - 7u : 0u;
            const unsigned lowest = ((0x10000u - (red & 0xFFFFu)) >> 8);
            const unsigned base2 = base * 0x01000100u;
#pragma unroll
            for (int hf = 0; hf < 2; ++hf) {
                u32x4 L[2], N;
#pragma unroll
                for (int jj = 0; jj < 4; ++jj) {
                    const u32x4 d = v[hf * 4 + jj];
                    unsigned nb[4];
#pragma unroll
                    for (int q = 0; q < 4; ++q) nb[q] = pk_sub_sat(d[q] & 0x7F007F00u, base2) | ((d[q] >> 4) & 0x08000800u);
                    L[jj >> 1][(jj & 1) * 2] = __builtin_amdgcn_perm(d[1], d[0], 0x06040200u);
                    L[jj >> 1][(jj & 1) * 2 + 1] = __builtin_amdgcn_perm(d[3], d[2], 0x06040200u);
                    N[jj] = __builtin_amdgcn_perm(nb[1], nb[0], 0x07050301u) | (__builtin_amdgcn_perm(nb[3], nb[2], 0x07050301u) << 4);
                }
                __builtin_amdgcn_raw_buffer_store_b128(L[0], rc, lane * 16 + hf * 3072, 0, AUX_SYS);
                __builtin_amdgcn_raw_buffer_store_b128(L[1], rc, lane * 16 + hf * 3072 + 1024, 0, AUX_SYS);
                __builtin_amdgcn_raw_buffer_store_b128(N, rc, lane * 16 + hf * 3072 + 2048, 0, AUX_SYS);
            }
            unsigned nesc = 0;
            if (lowest <= base) {
                unsigned r0 = 0, r1 = 0, n = 0;
#pragma unroll
                for (int k = 0; k < 8; ++k)
#pragma unroll
                    for (int e = 0; e < 8; ++e) {
                        const unsigned hb = (v[k][e >> 1] >> ((e & 1) ? 24 : 8)) & 0xFFu;
                        if ((hb & 0x7Fu) - 1u < base) { r1 = r0; r0 = ((unsigned)(k << 9 | e) << 8) | hb; ++n; }
                    }
                const u64 m1 = __ballot(n >= 1), m2 = __ballot(n >= 2);
                nesc = __builtin_popcountll(m1) + __builtin_popcountll(m2);
                raw = __ballot(n > 2) != 0ull || nesc > 15u;
                if (!raw) {
                    const unsigned lt = tagw | ((unsigned)lane << 11);
                    __builtin_amdgcn_raw_buffer_store_b32(r0 | lt, rh, n >= 1 ? 4 + mbcnt64(m1) * 4 : OOB, 0, AUX_SYS);
                    __builtin_amdgcn_raw_buffer_store_b32(r1 | lt, rh, n >= 2 ? 4 + ((unsigned)__builtin_popcountll(m1) + mbcnt64(m2)) * 4 : OOB, 0, AUX_SYS);
                } else nesc = 0;
            }
            hw0 = base | (nesc << 8) | (raw ? 0x8000u : 0u);
        }
        if (raw || flag) {
            const __amdgpu_buffer_rsrc_t rs = mk_rsrc(ws + (r < 128 ? WS_PKW : WS_PKX) + (size_t)(c * 128 + un) * 8192, 8192);
#pragma unroll
            for (int k = 0; k < 8; ++k) __builtin_amdgcn_raw_buffer_store_b128(v[k], rs, lane * 16 + k * 1024, 0, AUX_SYS);
        }
        asm volatile("" ::: "memory");
        if (jn < 4096) {
            pack_load(p, jn, lane, v);
            asm volatile("s_waitcnt vmcnt(8)" ::: "memory");
        } else wait_vm0();
        if (lane == 0) st_sys((unsigned*)(ws + WS_HW0) + c * 256 + r, tagw | hw0);
        if (flag && lane == 0) st_agt(flag, cx.call);
        j = jn;
    }
}

#define UNI(x) __builtin_amdgcn_readfirstlane(x)
#define LD_REMOTE(off, imm) __builtin_amdgcn_raw_buffer_load_b128(rrem, (off) + (imm), 0, AUX_SYS)
#define ST_LOCAL(v, off, imm) __builtin_amdgcn_raw_buffer_store_b128(v, rloc, (off) + (imm), 0, AUX_AGT)

struct InUnit { unsigned idx, dst, flag, info; bool end; };

__device__ __forceinline__ void decode_half(const u32x4 l0, const u32x4 l1, const u32x4 nn, const u32x4 ee, unsigned hdr, unsigned h0, unsigned k,
                                            int lane, char* wl, u32x4 (&o)[4]) {
    const unsigned bs = (h0 & 0xFFu) * 0x01010101u;
    unsigned H[8];
    u32x4 la = l0, lb = l1;
    if (h0 & 0x4000u) {
        if (lane < 16) *(u32x4*)(wl + lane * 16) = ee;
        const unsigned c0 = nn[0], c1 = nn[1], cb = nn[2];
        unsigned e0 = c0 & (c0 >> 1) & cb & 0x55555555u, e1 = c1 & (c1 >> 1) & (cb >> 1) & 0x55555555u, sum;
        unsigned off = wave_scan<6>(__builtin_popcount(e0) + __builtin_popcount(e1), sum) & 1023u;
#pragma unroll
        for (int g = 0; g < 8; ++g)
            H[g] = ((((g & 4 ? c1 : c0) >> (2 * (g & 3))) & 0x03030303u) << 1) + ((cb >> (2 * (g & 3) + (g >> 2))) & 0x01010101u) + bs;
        char* hp = wl + 1024 + lane * 32;
        *(u32x4*)hp = (u32x4){H[0], H[1], H[2], H[3]};
        *(u32x4*)(hp + 16) = (u32x4){H[4], H[5], H[6], H[7]};
        u64 e = ((u64)e1 << 32) | e0;
        __builtin_amdgcn_fence(__ATOMIC_RELEASE, "wavefront");
        __builtin_amdgcn_wave_barrier();
        while (e) {
            const unsigned w0 = *(const unsigned*)(wl + (off & ~3u)), w1 = *(const unsigned*)(wl + (off & ~3u) + 4);
            unsigned w = __builtin_amdgcn_alignbyte(w1, w0, off & 3u);
#pragma unroll
            for (int r = 0; r < 4; ++r)
                if (e) {
                    const unsigned pb = __builtin_ctzll(e);
                    hp[(pb >> 5) * 16 + ((pb >> 1) & 3u) * 4 + ((pb >> 3) & 3u)] = (char)w;
                    w >>= 8;
                    e &= e - 1ull;
                }
            off += 4;
        }
        __builtin_amdgcn_fence(__ATOMIC_RELEASE, "wavefront");
        __builtin_amdgcn_wave_barrier();
        const u32x4 xa = *(const u32x4*)hp, xb = *(const u32x4*)(hp + 16);
#pragma unroll
        for (int g = 0; g < 4; ++g) {
            H[g] = bfi(0x80808080u, l0[g], xa[g] >> 1);
            H[4 + g] = bfi(0x80808080u, l1[g], xb[g] >> 1);
            la[g] = bfi(0x7F7F7F7Fu, l0[g], xa[g] << 7);
            lb[g] = bfi(0x7F7F7F7Fu, l1[g], xb[g] << 7);
        }
    } else {
        const unsigned tlo = (bs + 0x03020100u) & 0xFFFFFF00u, thi = bs + 0x07060504u;
#pragma unroll
        for (int jj = 0; jj < 4; ++jj) {
            const unsigned n = nn[jj];
            H[2 * jj] = __builtin_amdgcn_perm(thi, tlo, n & 0x07070707u) | ((n << 4) & 0x80808080u);
            H[2 * jj + 1] = __builtin_amdgcn_perm(thi, tlo, (n >> 4) & 0x07070707u) | (n & 0x80808080u);
        }
        const int nesc = (h0 >> 8) & 15;
#pragma unroll 1
        for (int i = 1; i <= nesc; ++i) {
            const unsigned e = __builtin_amdgcn_readlane(hdr, i);
            if (((e >> 19) & 1u) != k) continue;
            const unsigned key = ((unsigned)lane == ((e >> 11) & 63u)) ? ((e >> 17) & 3u) * 2 + ((e >> 10) & 1u) : 8u;
            const unsigned sh = ((e >> 8) & 3u) * 8, msk = 0xFFu << sh, val = (e & 0xFFu) << sh;
#pragma unroll
            for (int q = 0; q < 8; ++q) H[q] = (key == (unsigned)q) ? ((H[q] & ~msk) | val) : H[q];
        }
    }
#pragma unroll
    for (int jj = 0; jj < 4; ++jj) {
        const unsigned p0 = (jj >> 1 ? lb : la)[(jj & 1) * 2], p1 = (jj >> 1 ? lb : la)[(jj & 1) * 2 + 1];
        o[jj][0] = __builtin_amdgcn_perm(H[2 * jj], p0, 0x05010400u);
        o[jj][1] = __builtin_amdgcn_perm(H[2 * jj], p0, 0x07030602u);
        o[jj][2] = __builtin_amdgcn_perm(H[2 * jj + 1], p1, 0x05010400u);
        o[jj][3] = __builtin_amdgcn_perm(H[2 * jj + 1], p1, 0x07030602u);
    }
}

__device__ __forceinline__ void role_pull_in(const Params& p, const Ctx& cx, int link, int lane, char* wl) {
    const int rank = p.rank;
    const __amdgpu_buffer_rsrc_t rloc = mk_rsrc(p.ws[rank], (unsigned)WS_END);
    const __amdgpu_buffer_rsrc_t rrem = mk_rsrc(p.ws[(rank + 1 + link) & 7], (unsigned)WS_END);
    const unsigned ctr_off = (lane == 0) ? (unsigned)WS_LINKCTR + link * 128 : OOB;
    const unsigned base = (cx.call - 1) * (unsigned)(U_IN + NPI);
    const unsigned l16 = lane * 16;
    const unsigned htag = hdr_tag(cx.call);

    auto ticket = [&]() -> unsigned { return (unsigned)__builtin_amdgcn_raw_ptr_buffer_atomic_add_i32(1, rloc, ctr_off, 0, AUX_AGT); };
    auto make = [&](unsigned tv) -> InUnit {
        const int n = (int)(UNI(tv) - base);
        InUnit u;
        int d, r;
        if (n < 64 * LAG) { d = n >> 6; r = n & 63; }
        else if (n < 64 * LAG + 96 * (16 - LAG)) { const int m = n - 64 * LAG; d = LAG + m / 96; r = m % 96; }
        else { const int m = n - 64 * LAG - 96 * (16 - LAG); d = 16 + (m >> 5); r = 64 + (m & 31); }
        const int c = (r >= 64) ? d - LAG : d;
        const int pos = c * 8 + 1 + link;
        const bool isw = r < 32;
        const int vv = isw ? r : r - 32;
        u.end = n >= U_IN;
        u.info = (unsigned)WS_MW0 + ((link * 16 + c) * 128 + r) * 4;
        u.idx = isw ? (unsigned)(c * 128 + (rank & 3) * 32 + vv) : (unsigned)(c * 128 + (rank >> 2) * 64 + vv) | 0x80000000u;
        u.dst = (isw ? (unsigned)WS_WB + pos * (16 * 16384) : (unsigned)WS_XA + pos * (16 * 32768)) + vv * 8192;
        u.flag = isw ? (unsigned)WS_WF + ((vv >> 1) * NPOS + pos) * 8 + (vv & 1) * 4
                     : (unsigned)WS_XF + ((vv >> 2) * NPOS + pos) * 16 + (vv & 3) * 4;
        return u;
    };
    auto load_hdr = [&](const InUnit& u) -> unsigned {
        return __builtin_amdgcn_raw_buffer_load_b32(rloc, (lane == 0 && !u.end) ? u.info : OOB, 0, AUX_AGT);
    };
    auto hdr_ok = [&](unsigned hv) -> bool {
        const unsigned w0 = UNI(hv);
        return (w0 >> 20) == htag && ((w0 & 0xC000u) != 0u || (w0 & 0xF00u) == 0u);
    };
    auto publish = [&](unsigned flag) {
        __builtin_amdgcn_raw_buffer_store_b32(cx.call, rloc, lane == 0 ? flag : OOB, 0, AUX_AGT);
    };
    auto store_half = [&](u32x4 (&C)[4], unsigned hdr, unsigned h0, unsigned dst, unsigned k) {
        if (!(h0 & 0x8000u)) {
            u32x4 o[4];
            decode_half(C[0], C[1], C[2], C[3], hdr, h0, k, lane, wl, o);
#pragma unroll
            for (int j = 0; j < 4; ++j) C[j] = o[j];
        }
        const unsigned a0 = dst + l16 + k * 4096u;
#pragma unroll
        for (int j = 0; j < 4; ++j) ST_LOCAL(C[j], a0, j * 1024);
    };
    auto src_of = [&](const InUnit& u, unsigned h0) -> unsigned {
        const bool x = (u.idx >> 31) != 0u;
        const unsigned ix = u.idx & 0x7FFFFFFFu;
        return ((h0 & 0x8000u) ? (x ? (unsigned)WS_PKX : (unsigned)WS_PKW) + ix * 8192u
                               : (x ? (unsigned)WS_CX : (unsigned)WS_CW) + ix * (unsigned)CU_BYTES) + l16;
    };
    auto fetch = [&](u32x4 (&N)[4], unsigned so0, unsigned h0, unsigned k) {
        const bool raw = (h0 & 0x8000u) != 0u;
        const unsigned so = so0 + k * (raw ? 4096u : 3072u);
        N[0] = LD_REMOTE(so, 0); N[1] = LD_REMOTE(so, 1024);
        if (h0 & 0x4000u) {
            typedef unsigned u32x3 __attribute__((ext_vector_type(3)));
            const u32x3 c3 = __builtin_amdgcn_raw_buffer_load_b96(rrem, so - lane * 4 + 2048, 0, AUX_SYS);
            N[2] = (u32x4){c3[0], c3[1], c3[2], 0u};
            N[3] = LD_REMOTE((unsigned)lane < ((h0 >> (8 + 8 * k)) & 15u) ? so : OOB, 2816);
        } else {
            N[2] = LD_REMOTE(so, 2048);
            N[3] = LD_REMOTE(raw ? so : OOB, 3072);
        }
    };
    auto wait_hdr = [&](const InUnit& u) -> unsigned {
        const unsigned ix = u.idx & 0x7FFFFFFFu;
        const unsigned hu = (ix >> 7) * 256 + (u.idx >> 31) * 128 + (ix & 127u);
        unsigned off = lane == 0 ? (unsigned)WS_HW0 + hu * 4 : OOB;
        for (;;) {
            const unsigned hv = __builtin_amdgcn_raw_buffer_load_b32(rrem, off, 0, AUX_SYS);
            const unsigned w0 = UNI(hv);
            const unsigned need = (w0 & 0xC000u) ? 1u : (2u << ((w0 >> 8) & 15u)) - 1u;
            if (((unsigned)__ballot((hv >> 20) == htag) & need) == need) return hv;
            if (give_up(cx)) return 0x8000u;
            if ((w0 >> 20) == htag) { if (lane > 0 && lane < 16) off = (unsigned)WS_HDR + hu * 64 + lane * 4; }
            else __builtin_amdgcn_s_sleep(8);
        }
    };

    InUnit u = make(ticket());
    if (u.end) return;
    unsigned t = ticket();
    unsigned hdr = wait_hdr(u);
    unsigned h0 = UNI(hdr);
    unsigned so = src_of(u, h0);
    u32x4 A[4], B[4];
    fetch(A, so, h0, 0);
    wait_vm0();
    unsigned pflag = OOB;
    for (;;) {
        asm volatile("" : "+v"(A[0]), "+v"(A[1]), "+v"(A[2]), "+v"(A[3]), "+v"(t));
        fetch(B, so, h0, 1);
        __builtin_amdgcn_sched_barrier(0);
        const InUnit un = make(t);
        unsigned info = load_hdr(un);
        store_half(A, hdr, h0, u.dst, 0);
        wait_vm4();
        asm volatile("" : "+v"(B[0]), "+v"(B[1]), "+v"(B[2]), "+v"(B[3]), "+v"(info));
        if (un.end || !hdr_ok(info)) {
            publish(pflag);
            store_half(B, hdr, h0, u.dst, 1);
            wait_vm0();
            publish(u.flag);
            pflag = OOB;
            if (un.end) return;
            t = ticket();
            hdr = wait_hdr(un);
            h0 = UNI(hdr);
            so = src_of(un, h0);
            fetch(A, so, h0, 0);
            wait_vm0();
        } else {
            const unsigned hn0 = UNI(info);
            so = src_of(un, hn0);
            fetch(A, so, hn0, 0);
            __builtin_amdgcn_sched_barrier(0);
            publish(pflag);
            t = ticket();
            store_half(B, hdr, h0, u.dst, 1);
            wait_vm4();
            pflag = u.flag;
            hdr = info; h0 = hn0;
        }
        u = un;
    }
}

__device__ __forceinline__ void role_pull_out(const Params& p, const Ctx& cx, int link, int lane) {
    const int rank = p.rank, src = (rank + 1 + link) & 7;
    const __amdgpu_buffer_rsrc_t rloc = mk_rsrc(p.ws[rank], (unsigned)WS_END);
    const __amdgpu_buffer_rsrc_t rrem = mk_rsrc(p.ws[src], (unsigned)WS_END);
    const unsigned ctr_off = (lane == 0) ? (unsigned)WS_LINKCTR + (7 + link) * 128 : OOB;
    const unsigned base = (cx.call - 1) * (unsigned)(U_OUT + NPO);
    const unsigned tag = seg_tag(cx.call);
    const unsigned l16 = lane * 16;
    bool dead = false;
    {
        const unsigned ibase = (cx.call - 1) * (unsigned)(U_IN + NPI);
        for (;;) {
            const unsigned v = UNI(__builtin_amdgcn_raw_buffer_load_b32(rloc, lane == 0 ? (unsigned)WS_LINKCTR + link * 128 : OOB, 0, AUX_AGT));
            const int rem = U_IN - (int)(v - ibase);
            if (rem <= 0) break;
            if (give_up(cx)) { dead = true; break; }
            for (int k = rem >> 6; k > 0; --k) __builtin_amdgcn_s_sleep(100);
            __builtin_amdgcn_s_sleep(16);
        }
    }
    auto ticket = [&]() -> unsigned { return (unsigned)__builtin_amdgcn_raw_ptr_buffer_atomic_add_i32(1, rloc, ctr_off, 0, AUX_AGT); };
    auto load_words = [&](unsigned n) -> unsigned {
        return __builtin_amdgcn_raw_buffer_load_b32(rloc, (lane < 2 && n < (unsigned)U_OUT) ? (unsigned)WS_MSEG + (link * NSEG + 2 * n + lane) * 4 : OOB, 0, AUX_AGT);
    };
    unsigned pn = U_OUT, qn = U_OUT, pw = 0, qw = 0;
    auto publish = [&]() {
        __builtin_amdgcn_raw_buffer_store_b32(pw, rloc, (lane < 2 && pn < (unsigned)U_OUT) ? (unsigned)WS_RF + (src * NSEG + 2 * pn + lane) * 4 : OOB, 0, AUX_AGT);
        pn = qn; pw = qw; qn = U_OUT;
    };
    auto store_unit = [&](u32x4 (&C)[8], unsigned n, unsigned w, unsigned len0, unsigned len1) {
        const unsigned d0 = (unsigned)WS_ORECV + (src * NSEG + 2 * n) * (unsigned)SEG_BYTES + l16, d1 = d0 + SEG_BYTES;
#pragma unroll
        for (int j = 0; j < 4; ++j) ST_LOCAL(C[j], l16 + j * 1024u < len0 ? d0 : OOB, j * 1024);
#pragma unroll
        for (int j = 0; j < 4; ++j) ST_LOCAL(C[4 + j], l16 + j * 1024u < len1 ? d1 : OOB, j * 1024);
        qn = n; qw = w;
    };
    auto rest = [&](unsigned sg, unsigned len) {
#pragma unroll 1
        for (unsigned o = 4096; o < len; o += 1024) {
            const bool in = o + l16 < len;
            const u32x4 v = LD_REMOTE(in ? (unsigned)WS_OSTG + sg * (unsigned)SEG_BYTES + o + l16 : OOB, 0);
            ST_LOCAL(v, in ? (unsigned)WS_ORECV + (src * NSEG + sg) * (unsigned)SEG_BYTES + o + l16 : OOB, 0);
        }
    };
    unsigned nn = UNI(ticket()) - base;
    if (nn >= (unsigned)U_OUT) return;
    unsigned t = ticket();
    unsigned info = load_words(nn), cn = U_OUT, cw = 0, cl0 = 0, cl1 = 0;
    wait_vm0();
    u32x4 A[8], B[8];
#pragma unroll
    for (int j = 0; j < 8; ++j) A[j] = (u32x4){0u, 0u, 0u, 0u};
#define OUT_STEP(C, N) do { \
        asm volatile("" : "+v"(C[0]), "+v"(C[1]), "+v"(C[2]), "+v"(C[3]), "+v"(C[4]), "+v"(C[5]), "+v"(C[6]), "+v"(C[7]), "+v"(t), "+v"(info)); \
        unsigned w0_ = UNI(info), w1_ = __builtin_amdgcn_readlane(info, 1); \
        if (!seg_ok(w0_, tag) || !seg_ok(w1_, tag) || nn >= (unsigned)U_OUT) {        \
            publish(); \
            store_unit(C, cn, cw, cl0, cl1); cn = U_OUT; cl0 = cl1 = 0; \
            wait_vm0(); publish(); publish(); \
            if (nn >= (unsigned)U_OUT) return; \
            for (;;) { \
                info = load_words(nn); \
                w0_ = UNI(info); w1_ = __builtin_amdgcn_readlane(info, 1); \
                if (seg_ok(w0_, tag) && seg_ok(w1_, tag)) break; \
                if (dead || give_up(cx)) { dead = true; w0_ = w1_ = info = 0u; break; } \
                __builtin_amdgcn_s_sleep(8); \
            } \
        } \
        const unsigned wv_ = info, l0_ = seg_len(w0_), l1_ = seg_len(w1_); \
        const unsigned s0_ = (unsigned)WS_OSTG + 2 * nn * (unsigned)SEG_BYTES + l16, s1_ = s0_ + SEG_BYTES; \
        _Pragma("unroll") for (int j = 0; j < 4; ++j) N[j] = LD_REMOTE(l16 + j * 1024u < l0_ ? s0_ : OOB, j * 1024); \
        _Pragma("unroll") for (int j = 0; j < 4; ++j) N[4 + j] = LD_REMOTE(l16 + j * 1024u < l1_ ? s1_ : OOB, j * 1024); \
        __builtin_amdgcn_sched_barrier(0); \
        publish(); \
        if (l0_ > 4096u) rest(2 * nn, l0_); \
        if (l1_ > 4096u) rest(2 * nn + 1, l1_); \
        const unsigned n2_ = UNI(t) - base; \
        if (n2_ < (unsigned)U_OUT) t = ticket(); \
        info = load_words(n2_); \
        store_unit(C, cn, cw, cl0, cl1); \
        wait_vm8(); \
        cn = nn; cw = wv_; cl0 = l0_; cl1 = l1_; nn = n2_; \
    } while (0)
    for (;;) {
        OUT_STEP(A, B);
        OUT_STEP(B, A);
    }
}

__device__ __forceinline__ void role_relay(const Params& p, const Ctx& cx, int link, int lane) {
    const int rank = p.rank;
    const int src = (rank + 1 + link) & 7;
    const __amdgpu_buffer_rsrc_t rctl = mk_rsrc(p.ws[rank], (unsigned)WS_CTL_END);
    const __amdgpu_buffer_rsrc_t rinfo = mk_rsrc(p.ws[src], (unsigned)WS_CTL_END);
    const unsigned tag = seg_tag(cx.call);
    {
        const int sub = lane & 31;
        const unsigned woff = sub < 8 ? (unsigned)((rank & 3) * 32 + sub * 4) : (unsigned)(128 + (rank >> 2) * 64 + (sub - 8) * 4);
        const unsigned ht = hdr_tag(cx.call), ht1 = hdr_tag(cx.call + 1u);
#pragma unroll 1
        for (int c = 0; c < 16;) {
            const int cc = c + (lane >> 5);
            const bool act = sub < 24 && cc < 16;
            const u32x4 hd = __builtin_amdgcn_raw_buffer_load_b128(rinfo, act ? (unsigned)WS_HW0 + (cc * 256 + woff) * 4 : OOB, 0, AUX_SYS);
            __builtin_amdgcn_raw_buffer_store_b128(hd, rctl, act ? (unsigned)WS_MW0 + ((link * 16 + cc) * 128 + sub * 4) * 4 : OOB, 0, AUX_AGT);
            bool ok = true;
#pragma unroll
            for (int q = 0; q < 4; ++q) ok = ok && ((hd[q] >> 20) == ht || (hd[q] >> 20) == ht1);
            const u64 bad = __ballot(act && !ok);
            if ((unsigned)bad == 0u) { c += (bad == 0ull) ? 2 : 1; continue; }
            if (give_up(cx)) return;
            __builtin_amdgcn_s_sleep(16);
        }
    }
    for (;;) {
        const unsigned v = UNI(__builtin_amdgcn_raw_buffer_load_b32(rctl, lane == 0 ? (unsigned)WS_LINKCTR + link * 128 : OOB, 0, AUX_AGT));
        const int rem = U_IN - 160 - (int)(v - (cx.call - 1) * (unsigned)(U_IN + NPI));
        if (rem <= 0) break;
        if (give_up(cx)) return;
        for (int k = rem >> 6; k > 0; --k) __builtin_amdgcn_s_sleep(100);
        __builtin_amdgcn_s_sleep(16);
    }
    int pos = 0;
    while (pos < NSEG) {
        const unsigned q = pos + lane * 4;
        const u32x4 v = __builtin_amdgcn_raw_buffer_load_b128(rinfo, q < NSEG ? (unsigned)WS_SEGRDY + q * 4 : OOB, 0, AUX_SYS);
        const bool any = seg_ok(v[0], tag) || seg_ok(v[1], tag) || seg_ok(v[2], tag) || seg_ok(v[3], tag);
        const bool all = seg_ok(v[0], tag) && seg_ok(v[1], tag) && seg_ok(v[2], tag) && seg_ok(v[3], tag);
        __builtin_amdgcn_raw_buffer_store_b128(v, rctl, (any && q < NSEG) ? (unsigned)WS_MSEG + (link * NSEG + q) * 4 : OOB, 0, AUX_AGT);
        const u64 m = __ballot(all && q < NSEG);
        const int n = (m == ~0ull) ? 64 : __builtin_ctzll(~m);
        pos += n * 4;
        if (n < 64) {
            if (give_up(cx)) return;
            __builtin_amdgcn_s_sleep(32);
        }
    }
}

__device__ __forceinline__ int packer_id(int wg, int wave) {
    const int gi = (wg >> 4) * 8 + (wg & 7);
    const bool g1 = (wg & 8) != 0;
    if (wave < 8) return g1 ? 512 + wave * 128 + gi : -1;
    const int k = wave - 9;
    return (k < 0 || (g1 && k < 2)) ? -1 : gi * 4 + (g1 ? 3 : k);
}
__device__ __forceinline__ void role_worker(const Params& p, const Ctx& cx, int wg, int k, int lane, char* wl) {
    const int gi = (wg >> 4) * 8 + (wg & 7);
    if (wg & 8) {
        if (k < 2 && gi < 119) { if (gi / 7 + 17 * k < NPI) role_pull_in(p, cx, gi % 7, lane, wl); }
        else if (gi >= 121 && k == 0) role_relay(p, cx, gi - 121, lane);
    } else {
        if (k == 0 && gi < NPO * 7) role_pull_out(p, cx, gi % 7, lane);
    }
}

__device__ __forceinline__ void role_poller(const Params& p, const Ctx& cx, int ti, int tj, int lane, char* lds) {
    const char* ws = p.ws[p.rank];
    const __amdgpu_buffer_rsrc_t rx = mk_rsrc(ws + WS_XF + ti * (NPOS * 16), NPOS * 16);
    const __amdgpu_buffer_rsrc_t rw = mk_rsrc(ws + WS_WF + tj * (NPOS * 8), NPOS * 8);
    volatile LAS unsigned* ready = (volatile LAS unsigned*)(LAS char*)(lds + LDS_CTL + 4);
    int pos = 0;
    while (pos < NPOS) {
        const int sq = pos + (lane & 15);
        const int q = sq < NPOS ? seq_slot(sq, ti >= 8) : NPOS;
        bool ok = false;
        if (lane < 16) {
            const u32x4 a = __builtin_amdgcn_raw_buffer_load_b128(rx, q * 16, 0, AUX_AGT);
            const u32x2 b = __builtin_amdgcn_raw_buffer_load_b64(rw, q * 8, 0, AUX_AGT);
            ok = a[0] == cx.call && a[1] == cx.call && a[2] == cx.call && a[3] == cx.call && b[0] == cx.call && b[1] == cx.call;
        }
        const u64 m = __ballot(ok);
        const int n = __builtin_ctzll(~m);
        if (n > 0) {
            pos += n;
            if (lane == 0) *ready = pos;
        } else {
            if (give_up(cx)) { pos = NPOS; if (lane == 0) *ready = NPOS; break; }
            __builtin_amdgcn_s_sleep(64);
        }
    }
}

__device__ __forceinline__ void st_out(char* o, u32x2 v) { __builtin_nontemporal_store(v, (u32x2*)o); }
__device__ __forceinline__ void expand_segment(const Params& p, const Ctx& cx, int src, int seg, int lane, char* wl, bool last) {
    const char* ws = p.ws[p.rank];
    const unsigned* rf = (const unsigned*)(ws + WS_RF) + src * NSEG + seg;
    unsigned f;
    bool dead = false;
    for (;;) {
        f = __builtin_amdgcn_readfirstlane(__hip_atomic_load(rf, __ATOMIC_RELAXED, __HIP_MEMORY_SCOPE_AGENT));
        if (seg_ok(f, seg_tag(cx.call))) break;
        if (give_up(cx)) { dead = true; break; }
        if (last) __builtin_amdgcn_s_sleep(32); else __builtin_amdgcn_s_sleep(127);
    }
    const int tile = seg >> 3, wv = seg & 7;
    const int row0 = (src >> 2) * 4096 + (tile >> 4) * 256 + (wv >> 1) * 64 + (lane >> 4) * 4;
    const int col0 = (src & 3) * 2048 + (tile & 15) * 128 + (wv & 1) * 64 + (lane & 7) * 8 + ((lane >> 3) & 1) * 4;
    char* o = p.out + ((size_t)row0 * NG + col0) * 2;
    if (dead) {
        const u32x2 nan2 = {0x7FC07FC0u, 0x7FC07FC0u};
#pragma unroll 1
        for (int i = 0; i < 16; ++i) *(u32x2*)(o + (size_t)((i >> 2) * 16 + (i & 3)) * (NG * 2)) = nan2;
        return;
    }
    const unsigned lb = ((f >> 8) & 511u) * 16u;
    const char* sp = ws + WS_ORECV + ((size_t)src * NSEG + seg) * SEG_BYTES;
    const __amdgpu_buffer_rsrc_t rs = mk_rsrc(sp, seg_len(f));
    const u32x2 mk = __builtin_amdgcn_raw_buffer_load_b64(rs, lane * 8, 0, AUX_AGT);
    if ((f >> 25) & 1u) {
        u32x4 r[8];
#pragma unroll
        for (int j = 0; j < 8; ++j) if (j * 512u < lb) r[j] = __builtin_amdgcn_raw_buffer_load_b128(rs, 512 + lane * 16 + j * 1024, 0, AUX_AGT);
#pragma unroll
        for (int j = 0; j < 8; ++j) if (j * 512u < lb) *(u32x4*)(wl + lane * 16 + j * 1024) = r[j];
    } else {
        u32x4 L[4], E[3];
        unsigned W[4];
        const unsigned oc = 512u + lb, oe = oc + ((lb / 4u + 15u) & ~15u), eb = (f & 255u) * 16u;
#pragma unroll
        for (int j = 0; j < 4; ++j)
            if (j * 1024u < lb) {
                L[j] = __builtin_amdgcn_raw_buffer_load_b128(rs, 512 + lane * 16 + j * 1024, 0, AUX_AGT);
                W[j] = __builtin_amdgcn_raw_buffer_load_b32(rs, j * 1024u + lane * 16u < lb ? oc + lane * 4 + j * 256 : OOB, 0, AUX_AGT);
            }
#pragma unroll
        for (int j = 0; j < 3; ++j) if (j * 1024u < eb) E[j] = __builtin_amdgcn_raw_buffer_load_b128(rs, oe + lane * 16 + j * 1024, 0, AUX_AGT);
#pragma unroll
        for (int j = 0; j < 3; ++j) if (j * 1024u < eb) *(u32x4*)(wl + WL_ESC + lane * 16 + j * 1024) = E[j];
        const unsigned bs = ((f >> 17) & 0xFFu) * 0x01010101u;
        unsigned ebase = 0;
        char* xp = wl + WL_EXP + lane * 16;
#pragma unroll
        for (int j = 0; j < 4; ++j)
            if (j * 1024u < lb) {
                u32x4 X;
                unsigned b0 = 0;
#pragma unroll
                for (int g = 0; g < 4; ++g) {
                    const unsigned c1 = (L[j][g] >> 7) & 0x01010101u;
                    X[g] = (((W[j] >> (2 * g)) & 0x03030303u) << 1) + c1 + bs;
                    b0 |= c1 << (2 * g);
                }
                unsigned e = W[j] & (W[j] >> 1) & b0, sum;
                unsigned off = ebase + wave_scan<5>(__builtin_popcount(e), sum);
                ebase += sum;
                *(u32x4*)xp = X;
                __builtin_amdgcn_fence(__ATOMIC_RELEASE, "wavefront");
                __builtin_amdgcn_wave_barrier();
                while (e) {
                    const unsigned pb = __builtin_ctz(e);
                    xp[((pb >> 1) & 3u) * 4 + (pb >> 3)] = wl[WL_ESC + off];
                    ++off;
                    e &= e - 1u;
                }
                X = *(const u32x4*)xp;
#pragma unroll
                for (int g2 = 0; g2 < 2; ++g2) {
                    u32x4 w4;
#pragma unroll
                    for (int q = 0; q < 2; ++q) {
                        const int g = 2 * g2 + q;
                        const unsigned lo = bfi(0x7F7F7F7Fu, L[j][g], X[g] << 7), hi = (X[g] >> 1) & 0x7F7F7F7Fu;
                        w4[2 * q] = __builtin_amdgcn_perm(hi, lo, 0x05010400u);
                        w4[2 * q + 1] = __builtin_amdgcn_perm(hi, lo, 0x07030602u);
                    }
                    *(u32x4*)(wl + j * 2048 + lane * 32 + g2 * 16) = w4;
                }
            }
    }
    __builtin_amdgcn_fence(__ATOMIC_RELEASE, "wavefront");
    __builtin_amdgcn_wave_barrier();
    const unsigned short* vals = (const unsigned short*)wl;
    unsigned base = 0;
#pragma unroll
    for (int i = 0; i < 16; ++i) {
        unsigned e[4];
#pragma unroll
        for (int f4 = 0; f4 < 4; ++f4) {
            const unsigned mlo = __builtin_amdgcn_readlane(mk[0], i * 4 + f4), mhi = __builtin_amdgcn_readlane(mk[1], i * 4 + f4);
            const unsigned idx = __builtin_amdgcn_mbcnt_hi(mhi, __builtin_amdgcn_mbcnt_lo(mlo, 0)) + base;
            const u64 m = ((u64)mhi << 32) | mlo;
            const unsigned val = vals[idx];
            e[f4] = __builtin_amdgcn_inverse_ballot_w64(m) ? val : 0u;
            base += __builtin_popcountll(m);
        }
        const u32x2 ov = {e[0] | (e[1] << 16), e[2] | (e[3] << 16)};
        st_out(o + (size_t)((i >> 2) * 16 + (i & 3)) * (NG * 2), ov);
    }
    __builtin_amdgcn_fence(__ATOMIC_RELEASE, "wavefront");
    __builtin_amdgcn_wave_barrier();
}

__device__ __forceinline__ unsigned bf16_rne(float f) {
    const unsigned u = __float_as_uint(f);
    return (u + 0x7FFFu + ((u >> 16) & 1u)) >> 16;
}

__device__ __forceinline__ void role_gemm(const Params& p, const Ctx& cx, int ti, int tj, int wv, int lane, char* lds) {
    const char* ws = p.ws[p.rank];
    const unsigned ldsbase = (unsigned)(uintptr_t)(LAS char*)lds;
    const unsigned ctl = ldsbase + LDS_CTL;
    const int q = wv >> 1, h = wv & 1;
    unsigned tA[2], tB[2];
#pragma unroll
    for (int i = 0; i < 2; ++i) {
        const unsigned slot = (unsigned)seq_slot(lane + 64 * i, ti >= 8);
        const bool own = (slot & 7u) == 0u;
        tA[i] = own ? (unsigned)WS_PKX + ((p.rank >> 2) * 16 + ti) * 32768 + slot * (16 * 8192) : (unsigned)WS_XA + ti * 32768 + slot * (16 * 32768);
        tB[i] = own ? (unsigned)WS_PKW + ((p.rank & 3) * 16 + tj) * 16384 + slot * (16 * 8192) : (unsigned)WS_WB + tj * 16384 + slot * (16 * 16384);
    }
    const unsigned lo_a = wv * 4096 + lane * 16, lo_b = wv * 2048 + lane * 16;
    auto tile_a = [&](int s) -> const char* {
        const unsigned o0 = __builtin_amdgcn_readlane(tA[0], s & 63), o1 = __builtin_amdgcn_readlane(tA[1], s & 63);
        return ws + ((s < 64 ? o0 : o1) + lo_a);
    };
    auto tile_b = [&](int s) -> const char* {
        const unsigned o0 = __builtin_amdgcn_readlane(tB[0], s & 63), o1 = __builtin_amdgcn_readlane(tB[1], s & 63);
        return ws + ((s < 64 ? o0 : o1) + lo_b);
    };
    const int fr = lane & 15, kq = lane >> 4;
    const int offA0 = q * 8192 + fr * 128 + ((kq ^ (fr >> 1)) * 16);
    const int offA1 = q * 8192 + fr * 128 + (((4 + kq) ^ (fr >> 1)) * 16);
    const int offB = 32768 + h * 8192 + (fr >> 3) * 4096 + kq * 128 + (fr & 7) * 16;

    f32x4 acc[4][4];
#pragma unroll
    for (int m = 0; m < 4; ++m)
#pragma unroll
        for (int n = 0; n < 4; ++n) acc[m][n] = (f32x4){0.f, 0.f, 0.f, 0.f};

    int issued = 0, islot = 0;
    unsigned rdy = 0, cnt = 0;
    auto issue = [&]() {
        const char* a = tile_a(issued);
        const char* b = tile_b(issued);
        char* la = lds + islot * STAGE_BYTES + wv * 4096;
        char* lb = lds + islot * STAGE_BYTES + 32768 + wv * 2048;
        __builtin_amdgcn_global_load_lds((const GAS void*)a, (LAS void*)la, 16, 0, AUX_AGT);
        __builtin_amdgcn_global_load_lds((const GAS void*)a, (LAS void*)la, 16, 1024, AUX_AGT);
        __builtin_amdgcn_global_load_lds((const GAS void*)a, (LAS void*)la, 16, 2048, AUX_AGT);
        __builtin_amdgcn_global_load_lds((const GAS void*)a, (LAS void*)la, 16, 3072, AUX_AGT);
        __builtin_amdgcn_global_load_lds((const GAS void*)b, (LAS void*)lb, 16, 0, AUX_AGT);
        __builtin_amdgcn_global_load_lds((const GAS void*)b, (LAS void*)lb, 16, 1024, AUX_AGT);
        ++issued;
        islot = (islot == 2) ? 0 : islot + 1;
    };
    auto poll = [&]() {
        u32x2 v;
        asm volatile("ds_read_b64 %0, %1\n\ts_waitcnt lgkmcnt(0)" : "=v"(v) : "v"(ctl) : "memory");
        cnt = __builtin_amdgcn_readfirstlane(v[0]);
        rdy = __builtin_amdgcn_readfirstlane(v[1]);
    };
    auto arrive = [&]() {
        if (lane == 0) asm volatile("ds_add_u32 %0, %1" :: "v"(ctl), "v"(1u) : "memory");
    };
    auto wait_bar = [&](unsigned target) {
        poll();
        while (cnt < target) { __builtin_amdgcn_s_sleep(1); poll(); }
    };
    auto need = [&](int upto) {
        while (issued < upto) {
            while ((int)rdy <= issued) { __builtin_amdgcn_s_sleep(2); poll(); }
            issue();
        }
    };

    need(1);
    if ((int)rdy > issued) issue();
    if (issued == 2) asm volatile("s_waitcnt vmcnt(6)" ::: "memory"); else wait_vm0();
    arrive();
    wait_bar(8);

    bf16x8 a0[4], b0[4], a1[4], b1[4];
    int cslot = 0;
    {
        const char* s = lds;
#pragma unroll
        for (int m = 0; m < 4; ++m) a0[m] = *(const bf16x8*)(s + offA0 + m * 2048);
#pragma unroll
        for (int n = 0; n < 4; ++n) b0[n] = *(const bf16x8*)(s + offB + n * 1024);
    }
#define MMA4(A_, B_, m_) do { _Pragma("unroll") for (int n = 0; n < 4; ++n) \
        acc[m_][n] = __builtin_amdgcn_mfma_f32_16x16x32_bf16(A_[m_], B_[n], acc[m_][n], 0, 0, 0); \
        __builtin_amdgcn_sched_barrier(0); } while (0)
#define GLDS(g_, l_, o_) do { if (spread) __builtin_amdgcn_global_load_lds((const GAS void*)(g_), (LAS void*)(l_), 16, o_, AUX_AGT); \
        __builtin_amdgcn_sched_barrier(0); } while (0)
#pragma unroll 1
    for (int t = 0; t < NPOS; ++t) {
        const char* s = lds + cslot * STAGE_BYTES;
        cslot = (cslot == 2) ? 0 : cslot + 1;
        const char* sn = lds + cslot * STAGE_BYTES;
        if (issued == t + 1 && issued < NPOS && issued < (int)rdy) issue();
        const bool spread = issued == t + 2 && issued < NPOS && issued < (int)rdy;
        const char* pa = tile_a(issued);
        const char* pb = tile_b(issued);
        char* la = lds + islot * STAGE_BYTES + wv * 4096;
        char* lb = lds + islot * STAGE_BYTES + 32768 + wv * 2048;
        __builtin_amdgcn_sched_barrier(0);
        MMA4(a0, b0, 0);
#pragma unroll
        for (int m = 0; m < 4; ++m) a1[m] = *(const bf16x8*)(s + offA1 + m * 2048);
#pragma unroll
        for (int n = 0; n < 4; ++n) b1[n] = *(const bf16x8*)(s + offB + 512 + n * 1024);
        __builtin_amdgcn_sched_barrier(0);
        GLDS(pa, la, 0);
        MMA4(a0, b0, 1);
        GLDS(pa, la, 1024);
        MMA4(a0, b0, 2);
        GLDS(pa, la, 2048);
        MMA4(a0, b0, 3);
        if (t + 1 < NPOS) {
            need(t + 2);
            if (spread) asm volatile("s_waitcnt vmcnt(3) lgkmcnt(0)" ::: "memory");
            else asm volatile("s_waitcnt vmcnt(0) lgkmcnt(0)" ::: "memory");
        } else {
            asm volatile("s_waitcnt lgkmcnt(0)" ::: "memory");
        }
        __builtin_amdgcn_s_waitcnt(0xC07F);
        arrive();
        __builtin_amdgcn_sched_barrier(0);
        MMA4(a1, b1, 0);
        GLDS(pa, la, 3072);
        MMA4(a1, b1, 1);
        wait_bar(8u * (t + 2));
        if (t + 1 < NPOS) {
#pragma unroll
            for (int m = 0; m < 4; ++m) a0[m] = *(const bf16x8*)(sn + offA0 + m * 2048);
#pragma unroll
            for (int n = 0; n < 4; ++n) b0[n] = *(const bf16x8*)(sn + offB + n * 1024);
        }
        __builtin_amdgcn_sched_barrier(0);
        GLDS(pb, lb, 0);
        MMA4(a1, b1, 2);
        GLDS(pb, lb, 1024);
        MMA4(a1, b1, 3);
        if (spread) { ++issued; islot = (islot == 2) ? 0 : islot + 1; }
    }

    char* wl = lds + wv * WAVE_LDS;
    unsigned short* vals = (unsigned short*)wl;
    const bool bad = timed_out(cx);
    const int seg = ((ti * 16 + tj) << 3) + wv;
    u32x2 mk = {0u, 0u};
    u32x2 ov[16];
    unsigned base = 0, mx = 0;
#pragma unroll
    for (int i = 0; i < 16; ++i) {
        unsigned e[4];
#pragma unroll
        for (int f4 = 0; f4 < 4; ++f4) {
            const float v = acc[i >> 2][f4][i & 3];
            unsigned b = bf16_rne(v > 0.f ? v : 0.f);
            if (bad || v != v) b = 0x7FC0u;
            e[f4] = b;
            const u64 m = __ballot(b != 0u);
            const unsigned idx = mbcnt64(m) + base;
            if (b != 0u) vals[idx] = (unsigned short)b;
            base += __builtin_popcountll(m);
            if (lane == i * 4 + f4) { mk[0] = (unsigned)m; mk[1] = (unsigned)(m >> 32); }
        }
        ov[i] = (u32x2){e[0] | (e[1] << 16), e[2] | (e[3] << 16)};
        mx = pk_max(mx, pk_max(ov[i][0], ov[i][1]));
    }
    {
        unsigned ex[8];
#pragma unroll
        for (int i = 0; i < 8; ++i) { const unsigned w = ov[2 * i][i & 1] & 0xFFFFu; ex[i] = w ? w >> 7 : 999u; }
        const unsigned hb = pick_base(wave_pk_max(max(mx >> 16, mx & 0xFFFFu)) >> 7, ex);
        const unsigned lb = (base + 15u) & ~15u;
        if (lane < 16) vals[base + lane] = (unsigned short)(hb << 7);
        __builtin_amdgcn_fence(__ATOMIC_RELEASE, "wavefront");
        __builtin_amdgcn_wave_barrier();
        u32x4 L[4];
        unsigned W[4];
        unsigned ne = 0;
#pragma unroll
        for (int j = 0; j < 4; ++j)
            if (j * 1024u < lb) {
                unsigned w = 0, b0 = 0;
#pragma unroll
                for (int g = 0; g < 4; ++g) {
                    const u32x2 d = *(const u32x2*)(wl + j * 2048 + lane * 32 + g * 8);
                    const unsigned cd = code4(d[0], d[1], hb * 0x00010001u);
                    L[j][g] = bfi(0x7F7F7F7Fu, __builtin_amdgcn_perm(d[1], d[0], 0x06040200u), cd << 7);
                    w |= ((cd >> 1) & 0x03030303u) << (2 * g);
                    b0 |= (cd & 0x01010101u) << (2 * g);
                }
                if (j * 1024u + lane * 16u >= lb) w = 0;
                W[j] = w;
                unsigned e = w & (w >> 1) & b0, sum;
                unsigned off = ne + wave_scan<5>(__builtin_popcount(e), sum);
                ne += sum;
                while (e) {
                    const unsigned pb = __builtin_ctz(e);
                    wl[WL_ESC + off] = (char)(vals[j * 1024 + lane * 16 + ((pb >> 1) & 3u) * 4 + (pb >> 3)] >> 7);
                    ++off;
                    e &= e - 1u;
                }
            }
        __builtin_amdgcn_fence(__ATOMIC_RELEASE, "wavefront");
        __builtin_amdgcn_wave_barrier();
        const bool raw = ne * 4u > lb * 3u - (lb ? 128u : 0u) || ne > 3072u;
        const unsigned f = (seg_tag(cx.call) << 26) | (raw ? 1u << 25 : 0u) | (hb << 17) | ((lb >> 4) << 8) | (raw ? 0u : (ne + 15u) >> 4);
        const __amdgpu_buffer_rsrc_t rs = mk_rsrc(ws + WS_OSTG + (size_t)seg * SEG_BYTES, seg_len(f));
        __builtin_amdgcn_raw_buffer_store_b64(mk, rs, lane * 8, 0, AUX_SYS);
        if (raw) {
#pragma unroll
            for (int j = 0; j < 8; ++j)
                if (j * 512u < lb)
                    __builtin_amdgcn_raw_buffer_store_b128(*(const u32x4*)(wl + lane * 16 + j * 1024), rs, 512 + lane * 16 + j * 1024, 0, AUX_SYS);
        } else {
            const unsigned oc = 512u + lb, oe = oc + ((lb / 4u + 15u) & ~15u);
#pragma unroll
            for (int j = 0; j < 4; ++j)
                if (j * 1024u < lb) {
                    const bool in = j * 1024u + lane * 16u < lb;
                    __builtin_amdgcn_raw_buffer_store_b128(L[j], rs, in ? 512 + lane * 16 + j * 1024 : OOB, 0, AUX_SYS);
                    __builtin_amdgcn_raw_buffer_store_b32(W[j], rs, in ? oc + lane * 4 + j * 256 : OOB, 0, AUX_SYS);
                }
#pragma unroll
            for (int j = 0; j < 3; ++j)
                if (j * 1024u < ne)
                    __builtin_amdgcn_raw_buffer_store_b128(*(const u32x4*)(wl + WL_ESC + lane * 16 + j * 1024), rs,
                                                           j * 1024u + lane * 16u < ne ? oe + lane * 16 + j * 1024 : OOB, 0, AUX_SYS);
        }
        wait_vm0();
        if (lane == 0) st_sys((unsigned*)(ws + WS_SEGRDY) + seg, f);
    }
    {
        const int row0 = (p.rank >> 2) * 4096 + ti * 256 + q * 64 + (lane >> 4) * 4;
        const int col0 = (p.rank & 3) * 2048 + tj * 128 + h * 64 + (lane & 7) * 8 + ((lane >> 3) & 1) * 4;
        char* o = p.out + ((size_t)row0 * NG + col0) * 2;
#pragma unroll
        for (int i = 0; i < 16; ++i) st_out(o + (size_t)((i >> 2) * 16 + (i & 3)) * (NG * 2), ov[i]);
    }
    __builtin_amdgcn_fence(__ATOMIC_RELEASE, "wavefront");
    __builtin_amdgcn_wave_barrier();
#pragma unroll 1
    for (int i = 0; i < 7; ++i) {
        const int e = seg + NSEG * i;
        expand_segment(p, cx, (p.rank + 1 + e % 7) & 7, e / 7, lane, wl, e >= NSEG * 7 - 512);
    }
}

}

__global__ void __launch_bounds__(NTHR) fused_gemm_ar(Params p) {
    extern __shared__ __attribute__((aligned(16))) char lds[];
    const int lane = threadIdx.x & 63;
    const int wave = __builtin_amdgcn_readfirstlane(threadIdx.x >> 6);
    const int wg = blockIdx.x;
    volatile LAS unsigned* ctl = (volatile LAS unsigned*)(LAS char*)(lds + LDS_CTL);
    char* ws = p.ws[p.rank];
    u32x4 v[8];
    if (wave >= 9) __builtin_amdgcn_s_setprio(3);
    const int pid = packer_id(wg, wave);
    if (pid >= 0) pack_load(p, pid, lane, v);
    if (threadIdx.x == 0) {
        const unsigned ticket = __hip_atomic_fetch_add((unsigned*)(ws + WS_TICKET), 1u, __ATOMIC_RELAXED, __HIP_MEMORY_SCOPE_AGENT);
        ctl[0] = 0u; ctl[1] = 0u; ctl[2] = ticket / NWG + 1u;
    }
    asm volatile("s_waitcnt lgkmcnt(0)" ::: "memory");
    __builtin_amdgcn_s_barrier();
    asm volatile("" ::: "memory");
    Ctx cx;
    cx.call = __builtin_amdgcn_readfirstlane(ctl[2]);
    cx.tmo = (unsigned*)(ws + WS_TMO);
    cx.t0 = __builtin_amdgcn_s_memrealtime();
    const int xcd = wg & 7, idx = wg >> 3;
    const int ti = (idx & 1) * 8 + (xcd & 1) * 4 + ((idx >> 1) & 3), tj = (xcd >> 1) * 4 + (idx >> 3);
    char* wl = lds + (wave < 8 ? wave * 4096 : LDS_WRK + (wave - 9) * 4096);
    if (pid >= 0) {
        role_packer(p, cx, pid, lane, v, wl);
        asm volatile("s_waitcnt lgkmcnt(0)" ::: "memory");
    }
    if (wave < 8) role_gemm(p, cx, ti, tj, wave, lane, lds);
    else if (wave == 8) role_poller(p, cx, ti, tj, lane, lds);
    else role_worker(p, cx, wg, wave - 9, lane, wl);
}

extern "C" void kernel_launch(int rank, int nranks, ncclComm_t comm, hipStream_t stream,
                              void* const* d_in, const int* in_sizes, int n_in,
                              void* d_out, int out_size,
                              const int* in_dtypes, int out_dtype,
                              void* const* d_out_all, void* const* d_ws_all, size_t ws_bytes) {
    if (nranks != NR || n_in < 2 || ws_bytes < WS_END || (unsigned)rank >= (unsigned)NR) {
        fprintf(stderr, "unsupported configuration\n");
        return;
    }
    static bool attr_set[NR] = {};
    if (!attr_set[rank]) {
        HIP_CHECK(hipFuncSetAttribute((const void*)fused_gemm_ar, hipFuncAttributeMaxDynamicSharedMemorySize, LDS_BYTES));
        attr_set[rank] = true;
    }
    Params p;
    p.x = (const char*)d_in[0];
    p.w = (const char*)d_in[1];
    p.out = (char*)d_out;
    for (int i = 0; i < NR; ++i) p.ws[i] = (char*)d_ws_all[i];
    p.rank = rank;
    fused_gemm_ar<<<NWG, NTHR, LDS_BYTES, stream>>>(p);
    HIP_CHECK(hipGetLastError());
}
```
